```python
import numpy as np

kernel_name = "gpu_gemm_ar_m8192_k8192_n8192_bf16_gelu"


ABSMAX_RTOL = 0.008

COMPUTE_DTYPE = "bfloat16"
EPILOGUE = "gelu"
IN_QUANT = None
OUT_QUANT = None
IN_DTYPES = {"x": "bfloat16", "w_mat": "bfloat16"}
WORLD_SIZE = 8

def _to_bf16(x):
    u = np.ascontiguousarray(x, dtype=np.float32).view(np.uint32)
    return ((u + 0x7FFF + ((u >> 16) & 1)) & 0xFFFF0000).view(np.float32)

def setup_inputs():

    x = _to_bf16(np.random.randn(8192, 8192).astype(np.float32))
    w_mat = _to_bf16((np.random.randn(8192, 8192) / np.sqrt(8192)).astype(np.float32))
    return {"x": x, "w_mat": w_mat}

def _epilogue(y):
    c = 0.7978845608028654
    return (0.5 * y * (1.0 + np.tanh(c * (y + 0.044715 * y ** 3)))).astype(np.float32)

def reference(x, w_mat):
    y = (x @ w_mat).astype(np.float32)
    return _epilogue(y)
```

```cpp
#include <hip/hip_runtime.h>
#include <rccl/rccl.h>
#include <cstdio>

namespace fk {

#define LAS __attribute__((address_space(3)))
#define GAS __attribute__((address_space(1)))
typedef unsigned short bf16_t;
typedef short bf16x8 __attribute__((ext_vector_type(8)));
typedef float f32x4 __attribute__((ext_vector_type(4)));
typedef unsigned u32x4 __attribute__((ext_vector_type(4)));
typedef unsigned u32x2 __attribute__((ext_vector_type(2)));
typedef unsigned short u16x2 __attribute__((ext_vector_type(2)));
typedef GAS unsigned gu32;
typedef GAS unsigned long long gu64;
typedef __amdgpu_buffer_rsrc_t rsrc_t;

constexpr int NRANK = 8;
constexpr int MD = 8192, ND = 8192, KD = 8192, KS = KD / NRANK;
constexpr int NT = KD / 64;
constexpr unsigned OUT_BYTES = (unsigned)MD * ND * 2u;
constexpr unsigned IN_BYTES = (unsigned)MD * KS * 2u;

constexpr int GRID = 256, NTHR = 512;
#define CFG_SLEEP 2
#define CFG_PJ 4
#define CFG_PMAP 1
constexpr int PJ = CFG_PJ;
constexpr int N_P = 7 * PJ, N_D = 128 - N_P;
constexpr int AUX_SYS = 17;

constexpr int NIMG = 3072;
constexpr unsigned IMG = 32768, PKT = 24576;
constexpr int NIT = 1024;
constexpr int NOT = 512;
constexpr int NPI = 384 + NOT;
constexpr int HCAP = 13;

constexpr unsigned WS_CALL = 0;
constexpr unsigned WS_TMO  = WS_CALL + 256 * 128;
constexpr unsigned WS_TK   = WS_TMO + 1024;
constexpr unsigned WS_CNT  = WS_TMO + 4096;
constexpr unsigned WS_PCNT = WS_CNT + NIMG * 128;
constexpr unsigned WS_HIN  = WS_PCNT + 32768;
constexpr unsigned WS_HOUT = WS_HIN + NIMG * 128;
constexpr unsigned WS_HOBI = WS_HOUT + NRANK * NOT * 128;
constexpr unsigned WS_HOBO = WS_HOBI + NIT * 128;
constexpr unsigned WS_DBG  = WS_HOBO + NOT * 128;
constexpr unsigned WS_PIN  = 4u << 20;
constexpr unsigned WS_POUT = WS_PIN + NIMG * PKT;
constexpr unsigned WS_OBI  = WS_POUT + NRANK * NOT * PKT;
constexpr unsigned WS_OBR  = WS_OBI + NIT * PKT;
constexpr unsigned WS_OBO  = WS_OBR + NIT * IMG;
constexpr unsigned WS_IMG  = WS_OBO + NOT * PKT;
constexpr unsigned WS_END  = WS_IMG + NIMG * IMG;
static_assert(WS_DBG + 256 * 64 * 8 <= WS_PIN, "control region overflow");
static_assert((size_t)WS_IMG + (size_t)NIMG * IMG < (1ull << 31), "workspace offsets must fit 31 bits");

constexpr unsigned long long DEADLINE = 40000000ull;

constexpr int HTB = 16384, STAGE_BYTES = 8 * HTB;
constexpr int LDS_FLAG = STAGE_BYTES;
constexpr int LDS_CALL = STAGE_BYTES + 64;
constexpr int LDS_TK = STAGE_BYTES + 96;
constexpr int LDS_DEAD = STAGE_BYTES + 80;
constexpr int LDS_GHDR = STAGE_BYTES + 128;
constexpr int LDS_HDR = 0;
constexpr int LDS_BYTES = STAGE_BYTES + 128 + 256;

struct Params {
  const char* x; const char* w;
  char* ws[NRANK]; char* out[NRANK];
  int rank; int pad;
};

__device__ __forceinline__ rsrc_t mkr(const void* p, unsigned bytes) { return __builtin_amdgcn_make_buffer_rsrc((void*)p, 0, (int)bytes, 0x00020000); }
__device__ __forceinline__ u32x4 ld128s(rsrc_t r, unsigned off) { return __builtin_amdgcn_raw_buffer_load_b128(r, (int)off, 0, AUX_SYS); }
__device__ __forceinline__ void st128s(u32x4 v, rsrc_t r, unsigned off) { __builtin_amdgcn_raw_buffer_store_b128(v, r, (int)off, 0, AUX_SYS); }
__device__ __forceinline__ unsigned long long rt() { return __builtin_amdgcn_s_memrealtime(); }
__device__ __forceinline__ unsigned rfl(unsigned v) { return (unsigned)__builtin_amdgcn_readfirstlane((int)v); }
__device__ __forceinline__ unsigned rdl(unsigned v, int l) { return (unsigned)__builtin_amdgcn_readlane((int)v, l); }
#define VM0() asm volatile("s_waitcnt vmcnt(0)" ::: "memory")
#define BARRIER() do { asm volatile("" ::: "memory"); __builtin_amdgcn_s_barrier(); asm volatile("" ::: "memory"); } while (0)

#define STAMP(p, idx) do { } while (0)

struct Ctx {
  gu32* tmo; unsigned call;
  __device__ __forceinline__ bool timed_out() const { return rfl(__hip_atomic_load(tmo, __ATOMIC_RELAXED, __HIP_MEMORY_SCOPE_AGENT)) == call; }
  __device__ __forceinline__ bool give_up(unsigned long long t0) const {
    if (timed_out() || rt() - t0 > DEADLINE) { __hip_atomic_store(tmo, call, __ATOMIC_RELAXED, __HIP_MEMORY_SCOPE_AGENT); return true; }
    __builtin_amdgcn_s_sleep(CFG_SLEEP);
    return false;
  }
};

__device__ __forceinline__ unsigned wave_max_u32(unsigned x) {
  x = max(x, (unsigned)__builtin_amdgcn_update_dpp(0, (int)x, 0x111, 0xf, 0xf, false));
  x = max(x, (unsigned)__builtin_amdgcn_update_dpp(0, (int)x, 0x112, 0xf, 0xf, false));
  x = max(x, (unsigned)__builtin_amdgcn_update_dpp(0, (int)x, 0x114, 0xf, 0xf, false));
  x = max(x, (unsigned)__builtin_amdgcn_update_dpp(0, (int)x, 0x118, 0xf, 0xf, false));
  x = max(x, (unsigned)__builtin_amdgcn_update_dpp(0, (int)x, 0x142, 0xa, 0xf, false));
  x = max(x, (unsigned)__builtin_amdgcn_update_dpp(0, (int)x, 0x143, 0xc, 0xf, false));
  return rdl(x, 63);
}

__device__ __forceinline__ bool encode_chunk(const unsigned (&v)[16], LAS unsigned* hdr, int wid, int lane, u32x4& L0, u32x4& L1, u32x4& C) {
  u16x2 mx = __builtin_bit_cast(u16x2, v[0] & 0x7FFF7FFFu);
#pragma unroll
  for (int i = 1; i < 16; ++i) mx = __builtin_elementwise_max(mx, __builtin_bit_cast(u16x2, v[i] & 0x7FFF7FFFu));
  const unsigned m1 = max((unsigned)mx.x, (unsigned)mx.y);
  const unsigned base = wave_max_u32(m1) >> 8;
  const unsigned B4 = base * 0x01010101u;
  unsigned Lw[8], D[8], H[8], acc = 0;
#pragma unroll
  for (int g = 0; g < 8; ++g) {
    H[g] = __builtin_amdgcn_perm(v[2 * g + 1], v[2 * g], 0x07050301u);
    Lw[g] = __builtin_amdgcn_perm(v[2 * g + 1], v[2 * g], 0x06040200u);
    D[g] = B4 - (H[g] & 0x7F7F7F7Fu);
    acc |= D[g];
  }
  unsigned Cw[4];
#pragma unroll
  for (int i = 0; i < 4; ++i) {
    const unsigned ca = ((D[2 * i] << 4) & 0x70707070u) | (H[2 * i] & 0x80808080u);
    const unsigned cb = (D[2 * i + 1] & 0x07070707u) | ((H[2 * i + 1] >> 4) & 0x08080808u);
    Cw[i] = ca | cb;
  }
  L0 = (u32x4){Lw[0], Lw[1], Lw[2], Lw[3]}; L1 = (u32x4){Lw[4], Lw[5], Lw[6], Lw[7]}; C = (u32x4){Cw[0], Cw[1], Cw[2], Cw[3]};
  bool raw = false;
  if (__any((acc & 0x78787878u) != 0u)) {
    bool over = false;
#pragma unroll
    for (int g = 0; g < 8; ++g) {
      if (__any((D[g] & 0x78787878u) != 0u)) {
#pragma unroll
        for (int b = 0; b < 4; ++b) {
          if (((D[g] >> (8 * b)) & 0x78u) != 0u) {
            const unsigned slot = __hip_atomic_fetch_add(hdr + 2, 1u, __ATOMIC_RELAXED, __HIP_MEMORY_SCOPE_WORKGROUP);
            const unsigned val = (((H[g] >> (8 * b)) & 0xFFu) << 8) | ((Lw[g] >> (8 * b)) & 0xFFu);
            if (slot < (unsigned)HCAP) hdr[3 + slot] = ((unsigned)wid << 27) | ((unsigned)lane << 21) | ((unsigned)(4 * g + b) << 16) | val;
            else over = true;
          }
        }
      }
    }
    raw = __any(over);
  }
  if (lane == 0) ((LAS unsigned char*)hdr)[wid] = (unsigned char)(base | (raw ? 0x80u : 0u));
  return raw;
}

__device__ __forceinline__ void decode_chunk(u32x4 L0, u32x4 L1, u32x4 C, unsigned base, unsigned (&v)[16]) {
  const unsigned B4 = base * 0x01010101u;
  const unsigned Lw[8] = {L0.x, L0.y, L0.z, L0.w, L1.x, L1.y, L1.z, L1.w};
  const unsigned Cw[4] = {C.x, C.y, C.z, C.w};
#pragma unroll
  for (int i = 0; i < 4; ++i) {
    const unsigned Ha = (B4 - ((Cw[i] >> 4) & 0x07070707u)) | (Cw[i] & 0x80808080u);
    const unsigned Hb = (B4 - (Cw[i] & 0x07070707u)) | ((Cw[i] << 4) & 0x80808080u);
    v[4 * i + 0] = __builtin_amdgcn_perm(Ha, Lw[2 * i], 0x05010400u);
    v[4 * i + 1] = __builtin_amdgcn_perm(Ha, Lw[2 * i], 0x07030602u);
    v[4 * i + 2] = __builtin_amdgcn_perm(Hb, Lw[2 * i + 1], 0x05010400u);
    v[4 * i + 3] = __builtin_amdgcn_perm(Hb, Lw[2 * i + 1], 0x07030602u);
  }
}

struct MapX { unsigned base; __device__ __forceinline__ unsigned operator()(int w, int j, int l) const { return base + (unsigned)(w * 4096 + j * 1024 + l * 16); } };
struct MapW { unsigned base; __device__ __forceinline__ unsigned operator()(int w, int j, int l) const { return base + (unsigned)((j >> 1) * 16384 + ((w >> 1) * 2 + (j & 1)) * 2048 + (w & 1) * 1024 + l * 16); } };
struct MapO { unsigned base; __device__ __forceinline__ unsigned operator()(int w, int j, int l) const { return base + (unsigned)((64 * (w >> 2) + 16 * (j >> 1) + (l & 15)) * (ND * 2) + (64 * (w & 3) + 32 * (j & 1) + 8 * (l >> 4)) * 2); } };
__device__ __forceinline__ unsigned out_task_base(int rank, int task) {
  const int tile = task >> 2, q = task & 3, pm = tile >> 3, pn = tile & 7;
  return (unsigned)(((rank >> 2) * 4096 + pm * 256 + (q >> 1) * 128 + (q & 1) * 32) * (ND * 2) + ((rank & 3) * 2048 + pn * 256) * 2);
}

#define DRAIN_ALL() do { asm volatile("s_waitcnt vmcnt(0) lgkmcnt(0)" ::: "memory"); __builtin_amdgcn_s_waitcnt(0x0070); } while (0)

__device__ __forceinline__ unsigned long long hdr_granule(LAS unsigned* hdr, int l16, unsigned call) {
  unsigned pl = hdr[l16];
  if (l16 == 2) pl = min(pl, (unsigned)HCAP);
  return ((unsigned long long)call << 32) | pl;
}
__device__ __forceinline__ unsigned long long ld_granule(const char* ws, unsigned line, int lane) {
  return __hip_atomic_load((const gu64*)(ws + line) + (lane & 15), __ATOMIC_RELAXED, __HIP_MEMORY_SCOPE_SYSTEM);
}
__device__ __forceinline__ bool granule_ok(unsigned long long g, unsigned call) { return __all((unsigned)(g >> 32) == call); }
__device__ __forceinline__ bool poll_granule(const Ctx& cx, const char* ws, unsigned line, int lane, unsigned long long& g) {
  const unsigned long long t0 = rt();
  while (!granule_ok(g, cx.call)) {
    if (cx.give_up(t0)) return false;
    g = ld_granule(ws, line, lane);
  }
  asm volatile("" ::: "memory");
  return true;
}
__device__ __forceinline__ unsigned chunk_byte(unsigned pl, int wid) { return ((wid < 4 ? rdl(pl, 0) : rdl(pl, 1)) >> ((wid & 3) * 8)) & 0xFFu; }

template <class Map>
__device__ __forceinline__ bool decode_task(const Ctx& cx, const char* ws, rsrc_t lw, unsigned hline, unsigned pkoff, rsrc_t fin, const Map& map, int wid, int lane) {
  unsigned long long g = ld_granule(ws, hline, lane);
  if (!poll_granule(cx, ws, hline, lane, g)) return false;
  const unsigned pl = (unsigned)g;
  const unsigned bb = chunk_byte(pl, wid), base = bb & 0x7Fu, raw = bb & 0x80u, cnt = rdl(pl, 2);
  if (!raw) {
    const unsigned o = pkoff + (unsigned)(wid * 3072 + lane * 16);
    const u32x4 L0 = ld128s(lw, o), L1 = ld128s(lw, o + 1024), C = ld128s(lw, o + 2048);
    unsigned v[16];
    decode_chunk(L0, L1, C, base, v);
#pragma unroll
    for (int j = 0; j < 4; ++j) st128s((u32x4){v[4 * j], v[4 * j + 1], v[4 * j + 2], v[4 * j + 3]}, fin, map(wid, j, lane));
    if (cnt != 0u) {
      VM0();
      if (lane >= 3 && lane < 3 + (int)cnt && lane < 16 && (int)(pl >> 27) == wid) {
        const int l2 = (int)(pl >> 21) & 63, e = (int)(pl >> 16) & 31;
        __builtin_amdgcn_raw_buffer_store_b16((unsigned short)(pl & 0xFFFFu), fin, (int)(map(wid, e >> 3, l2) + (unsigned)(e & 7) * 2u), 0, AUX_SYS);
      }
    }
  }
  return true;
}
template <class Map>
__device__ __forceinline__ void nan_chunk(rsrc_t fin, const Map& map, int wid, int lane) {
#pragma unroll
  for (int j = 0; j < 4; ++j) st128s((u32x4){0x7FC07FC0u, 0x7FC07FC0u, 0x7FC07FC0u, 0x7FC07FC0u}, fin, map(wid, j, lane));
}

struct InTask { int kt, isw, idx; };
__device__ __forceinline__ InTask in_task(int i) {
  InTask t;
  if (i < 768) { t.kt = i / 48; const int u = i - t.kt * 48, u3 = u / 3, r3 = u - u3 * 3;
    if (r3 == 0) { t.isw = 0; t.idx = (u3 & 1) * 16 + (u3 >> 1); } else { t.isw = 1; const int c = u3 * 2 + r3 - 1; t.idx = (c & 3) * 8 + (c >> 2); } }
  else { const int u = i - 768; t.kt = (u >> 4) & 15; t.isw = 0; t.idx = (u & 1) * 16 + 8 + ((u & 15) >> 1); }
  return t;
}
__device__ __forceinline__ int otask_of(int isw, int idx, int kt) { return isw * 512 + idx * 16 + kt; }
__device__ __forceinline__ void in_load(const Params& p, int i, u32x4 (&r)[4], int wid, int lane) {
  const InTask t = in_task(i);
  const int rr = lane >> 2, cp = (lane & 3) ^ ((rr >> 3) << 1);
  if (!t.isw) {
    const rsrc_t rx = mkr(p.x, IN_BYTES);
#pragma unroll
    for (int j = 0; j < 4; ++j) {
      const unsigned off = (unsigned)((256 * t.idx + 32 * wid + 16 * (j >> 1) + rr) * (KS * 2) + (64 * t.kt + 32 * (j & 1) + 8 * cp) * 2);
      r[j] = __builtin_amdgcn_raw_buffer_load_b128(rx, (int)off, 0, 0);
    }
  } else {
    const rsrc_t rw = mkr(p.w, IN_BYTES);
    const unsigned off0 = (unsigned)((64 * t.kt + 32 * (wid & 1) + 8 * cp) * (ND * 2) + (256 * t.idx + 64 * (wid >> 1) + 4 * rr) * 2);
#pragma unroll
    for (int d = 0; d < 4; ++d) {
      const u32x2 a = __builtin_amdgcn_raw_buffer_load_b64(rw, (int)(off0 + (unsigned)(2 * d) * (ND * 2)), 0, 0);
      const u32x2 b = __builtin_amdgcn_raw_buffer_load_b64(rw, (int)(off0 + (unsigned)(2 * d + 1) * (ND * 2)), 0, 0);
      r[d] = __builtin_shufflevector(a, b, 0, 1, 2, 3);
    }
  }
}
__device__ __forceinline__ void in_prep(int isw, const u32x4 (&r)[4], unsigned (&v)[16]) {
  if (isw) {
#pragma unroll
    for (int d = 0; d < 4; ++d) {
      v[0 + d] = __builtin_amdgcn_perm(r[d].z, r[d].x, 0x05040100u);
      v[4 + d] = __builtin_amdgcn_perm(r[d].z, r[d].x, 0x07060302u);
      v[8 + d] = __builtin_amdgcn_perm(r[d].w, r[d].y, 0x05040100u);
      v[12 + d] = __builtin_amdgcn_perm(r[d].w, r[d].y, 0x07060302u);
    }
  } else {
#pragma unroll
    for (int j = 0; j < 4; ++j) { v[4 * j] = r[j].x; v[4 * j + 1] = r[j].y; v[4 * j + 2] = r[j].z; v[4 * j + 3] = r[j].w; }
  }
}

__device__ __forceinline__ void role_encode(const Params& p, const Ctx& cx, LAS unsigned char* lds, int e, int ne) {
  const int tid = threadIdx.x, wid = (int)rfl(tid >> 6), lane = tid & 63;
  char* ws = p.ws[p.rank];
  const rsrc_t lw = mkr(ws, WS_END);
  if (tid < 48) ((LAS unsigned*)(lds + LDS_HDR))[tid] = 0u;
  u32x4 A[4];
  in_load(p, e, A, wid, lane);
  DRAIN_ALL(); BARRIER();
  int it = 0;
  for (int i = e; i < NIT; i += ne, ++it) {
    const InTask t = in_task(i);
    unsigned v[16];
    in_prep(t.isw, A, v);
    if (i + ne < NIT) in_load(p, i + ne, A, wid, lane);
    LAS unsigned* hdr = (LAS unsigned*)(lds + LDS_HDR + (it % 3) * 64);
    u32x4 L0, L1, C;
    const bool raw = encode_chunk(v, hdr, wid, lane, L0, L1, C);
    const int ot = otask_of(t.isw, t.idx, t.kt);
    if (!raw) {
      const unsigned o = WS_OBI + (unsigned)ot * PKT + (unsigned)(wid * 3072 + lane * 16);
      st128s(L0, lw, o); st128s(L1, lw, o + 1024); st128s(C, lw, o + 2048);
    } else {
      const bool mine = t.isw ? ((t.idx >> 3) == (p.rank & 3)) : ((t.idx >> 4) == (p.rank >> 2));
      const int T = t.kt * 8 + p.rank;
      const unsigned ib = WS_IMG + (unsigned)(t.isw ? 2048 + (t.idx & 7) * 128 + T : (t.idx & 15) * 128 + T) * IMG;
#pragma unroll
      for (int j = 0; j < 4; ++j) {
        const u32x4 q = (u32x4){v[4 * j], v[4 * j + 1], v[4 * j + 2], v[4 * j + 3]};
        st128s(q, lw, WS_OBR + (unsigned)ot * IMG + (unsigned)(wid * 4096 + j * 1024 + lane * 16));
        if (mine) st128s(q, lw, t.isw ? MapW{ib}(wid, j, lane) : MapX{ib}(wid, j, lane));
      }
    }
    DRAIN_ALL(); BARRIER();
    if (wid == 0) {
      const unsigned long long g = hdr_granule(hdr, lane & 15, cx.call);
      if (lane == 2) hdr[2] = 0u;
      if (lane < 16) __hip_atomic_store((gu64*)(ws + WS_HOBI + (unsigned)ot * 128u) + lane, g, __ATOMIC_RELAXED, __HIP_MEMORY_SCOPE_SYSTEM);
      asm volatile("s_waitcnt lgkmcnt(0)" ::: "memory");
    }
  }
  VM0(); BARRIER();
}

struct PItem { unsigned hsrc, dsrc, hdst, ddst, rs, rd; int kind; };
__device__ __forceinline__ PItem p_item(int rank, int d, int i) {
  PItem r;
  if (i < 384) {
    int kt, isw, k;
    if (i < 256) { kt = i >> 4; isw = i & 1; k = (i & 15) >> 1; } else { kt = (i - 256) >> 3; isw = 0; k = 8 + ((i - 256) & 7); }
    const int idx = isw ? (d & 3) * 8 + k : (d >> 2) * 16 + k;
    const int ot = otask_of(isw, idx, kt), T = kt * 8 + rank, img = isw ? 2048 + k * 128 + T : k * 128 + T;
    r.hsrc = WS_HOBI + (unsigned)ot * 128u; r.dsrc = WS_OBI + (unsigned)ot * PKT;
    r.hdst = WS_HIN + (unsigned)img * 128u; r.ddst = WS_PIN + (unsigned)img * PKT;
    r.rs = WS_OBR + (unsigned)ot * IMG; r.rd = WS_IMG + (unsigned)img * IMG; r.kind = isw;
  } else {
    const int task = (i - 384) & (NOT - 1);
    r.hsrc = WS_HOBO + (unsigned)task * 128u; r.dsrc = WS_OBO + (unsigned)task * PKT;
    r.hdst = WS_HOUT + (unsigned)(rank * NOT + task) * 128u; r.ddst = WS_POUT + (unsigned)(rank * NOT + task) * PKT;
    r.rs = r.rd = out_task_base(rank, task); r.kind = 2;
  }
  return r;
}

#define P_LDG(ws, line, lane) ld_granule(ws, line, lane)
__device__ __forceinline__ unsigned atomic_add_lane0_nowait(gu32* ptr) {
  unsigned r; unsigned long long sv; const unsigned one = 1u;
  asm volatile("s_mov_b64 %1, exec\n\ts_mov_b64 exec, 1\n\tglobal_atomic_add %0, %2, %3, off sc0\n\ts_mov_b64 exec, %1"
               : "=&v"(r), "=&s"(sv) : "v"(ptr), "v"(one) : "memory");
  return r;
}
#define DRAIN_PIN2(a, b) do { asm volatile("s_waitcnt vmcnt(0) lgkmcnt(0)" : "+v"(a), "+v"(b) :: "memory"); __builtin_amdgcn_s_waitcnt(0x0070); } while (0)

__device__ __forceinline__ void role_push(const Params& p, const Ctx& cx, int link, int v0, int nv) {
  constexpr int NC = NPI * 8;
  const int lane = threadIdx.x & 63;
  int d = p.rank + 1 + link; if (d >= NRANK) d -= NRANK;
  if ((unsigned)d >= (unsigned)NRANK) return;
  char* ws = p.ws[p.rank]; char* wd = p.ws[d];
  const rsrc_t lw = mkr(ws, WS_END), dw = mkr(wd, WS_END);
  gu32* tkc = (gu32*)(ws + WS_TK) + link * 32;
  gu32* pcnt = (gu32*)(ws + WS_PCNT) + link * NPI;
  const unsigned tkbase = (cx.call - 1u) * (unsigned)NC - 3u * (unsigned)nv;
  const unsigned lo = (unsigned)lane * 16u;
  int it = v0, nx = v0 + nv, n2 = v0 + 2 * nv;
  PItem cur = p_item(p.rank, d, it >> 3);
  unsigned long long gc = P_LDG(ws, cur.hsrc, lane), gn, gp = 0;
  if (!poll_granule(cx, ws, cur.hsrc, lane, gc)) return;
  u32x4 c0 = ld128s(lw, cur.dsrc + (it & 7) * 3072 + lo), c1 = ld128s(lw, cur.dsrc + (it & 7) * 3072 + lo + 1024), c2 = ld128s(lw, cur.dsrc + (it & 7) * 3072 + lo + 2048);
  gn = P_LDG(ws, p_item(p.rank, d, nx >> 3).hsrc, lane);
  DRAIN_ALL();
  int prev = -1; unsigned prev_hdst = 0;
  while (it < NC) {
    const int w = it & 7;
    const unsigned pl = (unsigned)gc;
    const unsigned bb = (w < 4 ? rdl(pl, 0) : rdl(pl, 1)) >> ((w & 3) * 8);
    if (!(bb & 0x80u)) {
      const unsigned o = cur.ddst + (unsigned)w * 3072u + lo;
      st128s(c0, dw, o); st128s(c1, dw, o + 1024); st128s(c2, dw, o + 2048);
    } else if (cur.kind == 2) {
      const rsrc_t so = mkr(p.out[p.rank], OUT_BYTES), dd = mkr(p.out[d], OUT_BYTES); const MapO map{cur.rs};
#pragma unroll
      for (int j = 0; j < 4; ++j) st128s(ld128s(so, map(w, j, lane)), dd, map(w, j, lane));
    } else {
#pragma unroll
      for (int j = 0; j < 4; ++j) { const u32x4 q = ld128s(lw, cur.rs + (unsigned)(w * 4096 + j * 1024) + lo);
        st128s(q, dw, cur.kind ? MapW{cur.rd}(w, j, lane) : MapX{cur.rd}(w, j, lane)); }
    }
    PItem nxt = cur; u32x4 n0 = c0, n1 = c1, n2v = c2; unsigned long long g2 = 0;
    bool have = true;
    if (nx < NC) {
      nxt = p_item(p.rank, d, nx >> 3);
      have = granule_ok(gn, cx.call);
      if (have) { const unsigned o = nxt.dsrc + (unsigned)(nx & 7) * 3072u + lo; n0 = ld128s(lw, o); n1 = ld128s(lw, o + 1024); n2v = ld128s(lw, o + 2048); }
      if (n2 < NC) g2 = P_LDG(ws, p_item(p.rank, d, n2 >> 3).hsrc, lane);
    }
    unsigned tk = atomic_add_lane0_nowait(tkc);
    unsigned pc = 0;
    if (prev >= 0) pc = atomic_add_lane0_nowait(pcnt + (prev >> 3));
    DRAIN_PIN2(tk, pc);
    if (prev >= 0 && rfl(pc) == 8u * cx.call - 1u && lane < 16)
      __hip_atomic_store((gu64*)(wd + prev_hdst) + lane, gp, __ATOMIC_RELAXED, __HIP_MEMORY_SCOPE_SYSTEM);
    const int n3 = (int)(rfl(tk) - tkbase);
    if (!have) {
      unsigned pc2 = atomic_add_lane0_nowait(pcnt + (it >> 3)), dummy = 0;
      DRAIN_PIN2(pc2, dummy);
      if (rfl(pc2) == 8u * cx.call - 1u && lane < 16)
        __hip_atomic_store((gu64*)(wd + cur.hdst) + lane, gc, __ATOMIC_RELAXED, __HIP_MEMORY_SCOPE_SYSTEM);
      prev = -1;
      if (!poll_granule(cx, ws, nxt.hsrc, lane, gn)) return;
      const unsigned o = nxt.dsrc + (unsigned)(nx & 7) * 3072u + lo; n0 = ld128s(lw, o); n1 = ld128s(lw, o + 1024); n2v = ld128s(lw, o + 2048);
      DRAIN_ALL();
    } else {
      prev = it; prev_hdst = cur.hdst; gp = gc;
    }
    cur = nxt; gc = gn; gn = g2; it = nx; nx = n2; n2 = n3;
    c0 = n0; c1 = n1; c2 = n2v;
  }
  if (prev >= 0) {
    unsigned pc = atomic_add_lane0_nowait(pcnt + (prev >> 3)), dummy = 0;
    DRAIN_PIN2(pc, dummy);
    if (rfl(pc) == 8u * cx.call - 1u && lane < 16)
      __hip_atomic_store((gu64*)(wd + prev_hdst) + lane, gp, __ATOMIC_RELAXED, __HIP_MEMORY_SCOPE_SYSTEM);
  }
  STAMP(p, 2);
}

__device__ __forceinline__ void role_din(const Params& p, const Ctx& cx, int dw) {
  const int tid = threadIdx.x, wid = (int)rfl(tid >> 6), lane = tid & 63;
  char* ws = p.ws[p.rank];
  const rsrc_t lw = mkr(ws, WS_END);
  gu32* cnt = (gu32*)(ws + WS_CNT);
  for (int i = dw; i < NIMG; i += N_D) {
    int k, T; bool isw = false;
    if (i < 2048) { T = i >> 4; isw = (i & 1); k = (i & 15) >> 1; } else { const int u = i - 2048; T = u >> 3; k = 8 + (u & 7); }
    const int img = isw ? 2048 + k * 128 + T : k * 128 + T;
    unsigned hl = WS_HIN + (unsigned)img * 128u, pk = WS_PIN + (unsigned)img * PKT;
    if ((T & 7) == p.rank) {
      const int ot = otask_of(isw, isw ? (p.rank & 3) * 8 + k : (p.rank >> 2) * 16 + k, T >> 3);
      hl = WS_HOBI + (unsigned)ot * 128u; pk = WS_OBI + (unsigned)ot * PKT;
    }
    const unsigned ib = WS_IMG + (unsigned)img * IMG;
    bool ok;
    if (isw) ok = decode_task(cx, ws, lw, hl, pk, lw, MapW{ib}, wid, lane);
    else ok = decode_task(cx, ws, lw, hl, pk, lw, MapX{ib}, wid, lane);
    if (!ok) return;
    VM0();
    if (lane == 0) __hip_atomic_fetch_add(cnt + img * 32, 1u, __ATOMIC_RELAXED, __HIP_MEMORY_SCOPE_AGENT);
  }
}

__device__ __forceinline__ void role_dout(const Params& p, const Ctx& cx, int dw, int nw) {
  const int tid = threadIdx.x, wid = (int)rfl(tid >> 6), lane = tid & 63;
  char* ws = p.ws[p.rank];
  const rsrc_t lw = mkr(ws, WS_END), fin = mkr(p.out[p.rank], OUT_BYTES);
  bool dead = false;
  for (int i = dw; i < NOT * (NRANK - 1); i += nw) {
    const int task = i / (NRANK - 1), k = i - task * (NRANK - 1);
    int s = p.rank + 1 + k; if (s >= NRANK) s -= NRANK;
    const unsigned tidx = (unsigned)(s * NOT + task);
    const MapO map{out_task_base(s, task)};
    if (!dead) dead = !decode_task(cx, ws, lw, WS_HOUT + tidx * 128u, WS_POUT + tidx * PKT, fin, map, wid, lane);
    if (dead) nan_chunk(fin, map, wid, lane);
  }
}

__host__ __device__ __forceinline__ int lds_byte(int r, int c) { const int st = (r >> 4) * 2 + (c >> 5), rr = r & 15, cc = c & 31, ob = rr * 64 + cc * 2; return st * 1024 + (ob ^ (((ob >> 9) & 1) << 5)); }
__device__ __forceinline__ unsigned cvt_pk_bf16(float lo, float hi) { unsigned r; asm volatile("v_cvt_pk_bf16_f32 %0, %1, %2" : "=v"(r) : "v"(lo), "v"(hi)); return r; }
__device__ __forceinline__ float gelu1(float y) {
  const float k1 = -2.0f * 0.7978845608028654f * 1.4426950408889634f, k3 = k1 * 0.044715f;
  const float t = y * (k3 * (y * y) + k1);
  return y * __builtin_amdgcn_rcpf(1.0f + __builtin_amdgcn_exp2f(t));
}

__device__ __forceinline__ void role_gemm(const Params& p, const Ctx& cx, LAS unsigned char* lds, int pm, int pn) {
  const int tid = threadIdx.x, wid = (int)rfl(tid >> 6), lane = tid & 63, wr = wid >> 2, wc = wid & 3, fr = lane & 15, fq = lane >> 4;
  char* ws = p.ws[p.rank];
  const char* cA = ws + WS_IMG + (size_t)(pm * 128) * IMG;
  const char* cB = ws + WS_IMG + (size_t)(2048 + pn * 128) * IMG;
  const gu32* fl = (const gu32*)(ws + WS_CNT) + (size_t)(((lane & 2) ? 2048 + pn * 128 : pm * 128) + (lane & 1)) * 32;
  const unsigned need = 8u * cx.call;
  const unsigned voff = (unsigned)tid * 16u;
  const unsigned ldsw = (unsigned)wid * 1024u;
  const int aoff = lds_byte(wr * 64 + fr, fq * 8), boff = lds_byte(wc * 32 + fr, fq * 8);
  bool dead = false;
  auto wait_tiles = [&](int T) {
    if (wid == 0 && !dead) {
      if (!__all(__hip_atomic_load(fl + T * 32, __ATOMIC_RELAXED, __HIP_MEMORY_SCOPE_AGENT) >= need)) {
        const unsigned long long t0 = rt();
        while (!__all(__hip_atomic_load(fl + T * 32, __ATOMIC_RELAXED, __HIP_MEMORY_SCOPE_AGENT) >= need)) { if (cx.give_up(t0)) { dead = true; break; } }
      }
    }
  };
#define SA(b, h) (((b) * 2 + (h)) * HTB)
#define SB(b, h) ((4 + (b) * 2 + (h)) * HTB)
#define STAGE(bufoff, gbase) do { _Pragma("unroll") for (int _i = 0; _i < 2; ++_i) \
    __builtin_amdgcn_global_load_lds((const GAS unsigned*)((const char*)(gbase) + voff + _i * 8192), (LAS unsigned*)(lds + (bufoff) + ldsw + _i * 8192), 16, 0, AUX_SYS); } while (0)
#define LDA(dst, b, h) do { _Pragma("unroll") for (int m = 0; m < 4; ++m) _Pragma("unroll") for (int k = 0; k < 2; ++k) dst[m][k] = *(const LAS bf16x8*)(lds + SA(b, h) + aoff + m * 2048 + k * 1024); } while (0)
#define LDB(dst, b, h) do { _Pragma("unroll") for (int n = 0; n < 2; ++n) _Pragma("unroll") for (int k = 0; k < 2; ++k) dst[n][k] = *(const LAS bf16x8*)(lds + SB(b, h) + boff + n * 2048 + k * 1024); } while (0)
#define MMA(ai, bj, At, Bt) do { __builtin_amdgcn_s_setprio(1); _Pragma("unroll") for (int m = 0; m < 4; ++m) _Pragma("unroll") for (int n = 0; n < 2; ++n) _Pragma("unroll") for (int k = 0; k < 2; ++k) \
    acc[ai][bj][m][n] = __builtin_amdgcn_mfma_f32_16x16x32_bf16(Bt[n][k], At[m][k], acc[ai][bj][m][n], 0, 0, 0); __builtin_amdgcn_s_setprio(0); } while (0)
#define WAIT_V(n) asm volatile("s_waitcnt vmcnt(" #n ")" ::: "memory")
#define WAIT_L(n) asm volatile("s_waitcnt lgkmcnt(" #n ")" ::: "memory")
#define BAR __builtin_amdgcn_s_barrier()
#define SCHED __builtin_amdgcn_sched_barrier(0)
  f32x4 acc[2][2][4][2];
#pragma unroll
  for (int a = 0; a < 2; ++a)
#pragma unroll
    for (int b = 0; b < 2; ++b)
#pragma unroll
      for (int m = 0; m < 4; ++m)
#pragma unroll
        for (int n = 0; n < 2; ++n) acc[a][b][m][n] = (f32x4){0.f, 0.f, 0.f, 0.f};
  bf16x8 At[4][2], B0[2][2], B1[2][2];
  wait_tiles(0); wait_tiles(2);
  BARRIER();
  STAMP(p, 1);
  STAGE(SB(0, 0), cB); STAGE(SA(0, 0), cA); STAGE(SB(0, 1), cB + HTB); STAGE(SA(0, 1), cA + HTB);
  if (wr == 1) BAR;
  WAIT_V(4); BAR;
  STAGE(SB(1, 0), cB + IMG); STAGE(SA(1, 0), cA + IMG); STAGE(SB(1, 1), cB + IMG + HTB);
  WAIT_V(6); BAR;
  for (int t = 0; t < NT; t += 2) {
    const bool last = (t == NT - 2);
    const char* a1 = cA + (size_t)(t + 1) * IMG;
    const char* a2 = last ? cA : cA + (size_t)(t + 2) * IMG; const char* b2 = last ? cB : cB + (size_t)(t + 2) * IMG;
    const char* a3 = a2 + IMG; const char* b3 = b2 + IMG;
    LAS unsigned* fslot = (LAS unsigned*)(lds + LDS_FLAG + ((t >> 1) & 1) * 16);
    const bool chk = (t + 4 < NT);
    if (chk && wid == 0 && lane < 4) __builtin_amdgcn_global_load_lds(fl + (t + 4) * 32, fslot, 4, 0, AUX_SYS);
    LDB(B0, 0, 0); SCHED; LDA(At, 0, 0); STAGE(SA(1, 1), a1 + HTB);
    WAIT_L(8); BAR; WAIT_L(0); MMA(0, 0, At, B0); BAR; SCHED;
    LDB(B1, 0, 1); STAGE(SB(0, 0), b2);
    BAR; WAIT_L(0); MMA(0, 1, At, B1); BAR;
    LDA(At, 0, 1); STAGE(SA(0, 0), a2);
    BAR; WAIT_L(0); MMA(1, 0, At, B0); BAR; SCHED;
    STAGE(SB(0, 1), b2 + HTB);
    WAIT_V(6); BAR; MMA(1, 1, At, B1); BAR;
    LDB(B0, 1, 0); SCHED; LDA(At, 1, 0); STAGE(SA(0, 1), a2 + HTB);
    WAIT_L(8); BAR; WAIT_L(0); MMA(0, 0, At, B0); BAR; SCHED;
    LDB(B1, 1, 1); STAGE(SB(1, 0), b3);
    BAR; WAIT_L(0); MMA(0, 1, At, B1); BAR;
    LDA(At, 1, 1); STAGE(SA(1, 0), a3);
    BAR; WAIT_L(0); MMA(1, 0, At, B0); BAR; SCHED;
    const u32x4 f = *(const LAS u32x4*)fslot;
    STAGE(SB(1, 1), b3 + HTB);
    WAIT_V(6); BAR; MMA(1, 1, At, B1); BAR;
    if (chk) {
      const unsigned fm = rfl(min(min(f.x, f.y), min(f.z, f.w)));
      if (fm < need) { wait_tiles(t + 4); BARRIER(); }
    }
  }
  if (wr == 0) BAR;
  STAMP(p, 2);
  const bool bad = cx.timed_out();
  {
    int tz = threadIdx.x; asm volatile("" : "+v"(tz));
    const int ln = tz & 63;
    const rsrc_t ro = mkr(p.out[p.rank], OUT_BYTES), lw = mkr(ws, WS_END);
    const int tile = pm * 8 + pn;
#pragma unroll
    for (int q = 0; q < 4; ++q) {
      unsigned v[16];
#pragma unroll
      for (int mm = 0; mm < 2; ++mm) {
        const int ai = q >> 1, m = (q & 1) * 2 + mm;
        unsigned d[8];
#pragma unroll
        for (int c = 0; c < 8; ++c) { d[c] = cvt_pk_bf16(gelu1(acc[ai][c & 1][m][0][c >> 1]), gelu1(acc[ai][c & 1][m][1][c >> 1])); if (bad) d[c] = 0x7FC07FC0u; }
#pragma unroll
        for (int c = 0; c < 4; ++c) {
          auto r = __builtin_amdgcn_permlane16_swap(d[c], d[c + 4], false, false); d[c] = r[0]; d[c + 4] = r[1];
          auto s2 = __builtin_amdgcn_permlane32_swap(d[c], d[c + 4], false, false); d[c] = s2[0]; d[c + 4] = s2[1];
        }
#pragma unroll
        for (int c = 0; c < 8; ++c) v[8 * mm + c] = d[c];
      }
      const MapO map{out_task_base(p.rank, tile * 4 + q)};
#pragma unroll
      for (int j = 0; j < 4; ++j) st128s((u32x4){v[4 * j], v[4 * j + 1], v[4 * j + 2], v[4 * j + 3]}, ro, map(wid, j, ln));
      u32x4 L0, L1, C;
      const bool raw = encode_chunk(v, (LAS unsigned*)(lds + LDS_GHDR + q * 64), wid, ln, L0, L1, C);
      if (!raw) {
        const unsigned o = WS_OBO + (unsigned)(tile * 4 + q) * PKT + (unsigned)(wid * 3072 + ln * 16);
        st128s(L0, lw, o); st128s(L1, lw, o + 1024); st128s(C, lw, o + 2048);
      }
    }
    DRAIN_ALL(); BARRIER();
    if (!bad && wid == 0) {
      const unsigned long long g = hdr_granule((LAS unsigned*)(lds + LDS_GHDR + (ln >> 4) * 64), ln & 15, cx.call);
      __hip_atomic_store((gu64*)(ws + WS_HOBO + (unsigned)(tile * 4) * 128u) + ln, g, __ATOMIC_RELAXED, __HIP_MEMORY_SCOPE_SYSTEM);
    }
  }
  STAMP(p, 3);
#undef SA
#undef SB
#undef STAGE
#undef LDA
#undef LDB
#undef MMA
#undef WAIT_V
#undef WAIT_L
#undef BAR
#undef SCHED
}

__global__ __launch_bounds__(NTHR, 2) void fused_kernel(Params p) {
  extern __shared__ __attribute__((aligned(16))) unsigned char shm[];
  LAS unsigned char* lds = (LAS unsigned char*)shm;
  const int b = blockIdx.x, xcd = b & 7, slot = b >> 3;
  char* ws = p.ws[p.rank];
  if (threadIdx.x == 0) {
    const unsigned c = __hip_atomic_fetch_add((gu32*)(ws + WS_CALL) + b * 32, 1u, __ATOMIC_RELAXED, __HIP_MEMORY_SCOPE_AGENT) + 1u;
    *(LAS unsigned*)(lds + LDS_CALL) = c;
    *(LAS unsigned*)(lds + LDS_DEAD) = 0u;
  }
  if (threadIdx.x >= 64 && threadIdx.x < 128) ((LAS unsigned*)(lds + LDS_GHDR))[threadIdx.x - 64] = 0u;
  __syncthreads();
  Ctx cx; cx.tmo = (gu32*)(ws + WS_TMO); cx.call = rfl(*(LAS unsigned*)(lds + LDS_CALL));
  STAMP(p, 0);
  if (b < N_P) {
    { const int gw = b * 8 + (int)rfl(threadIdx.x >> 6); role_push(p, cx, gw % 7, gw / 7, PJ * 8); }
  } else if (slot < 16) {
    const int dw = b - N_P;
    role_din(p, cx, dw);
    STAMP(p, 1);
    role_dout(p, cx, dw, N_D + 64);
    STAMP(p, 2);
  } else {
    const int s = (slot - 16) & 7, grp = (slot - 16) >> 3;
    const int pm = grp * 8 + (xcd >> 1) * 2 + (s & 1), pn = (xcd & 1) * 4 + (s >> 1);
    if (grp == 1) { role_encode(p, cx, lds, b - 192, 64); STAMP(p, 5); }
    role_gemm(p, cx, lds, pm, pn);
    if (grp == 0) { role_dout(p, cx, N_D + (slot - 16) * 8 + xcd, N_D + 64); STAMP(p, 4); }
  }
}

}

extern "C" void kernel_launch(int rank, int nranks, ncclComm_t comm, hipStream_t stream,
                              void* const* d_in, const int* in_sizes, int n_in,
                              void* d_out, int out_size,
                              const int* in_dtypes, int out_dtype,
                              void* const* d_out_all, void* const* d_ws_all, size_t ws_bytes) {
  using namespace fk;
  (void)comm; (void)in_dtypes; (void)out_dtype;
  if (nranks != NRANK || n_in < 2 || rank < 0 || rank >= NRANK || ws_bytes < (size_t)WS_END ||
      in_sizes[0] != MD * KS || in_sizes[1] != KS * ND || out_size != MD * ND) {
    fprintf(stderr, "kernel_launch: unexpected problem shape\n");
    return;
  }
  static bool attr_set[64] = {};
  int dev = 0;
  if (hipGetDevice(&dev) != hipSuccess) return;
  if (dev >= 0 && dev < 64 && !attr_set[dev]) {
    hipError_t e = hipFuncSetAttribute((const void*)fused_kernel, hipFuncAttributeMaxDynamicSharedMemorySize, LDS_BYTES);
    if (e != hipSuccess) { fprintf(stderr, "HIP error %s\n", hipGetErrorString(e)); return; }
    attr_set[dev] = true;
  }
  Params p;
  p.x = (const char*)d_in[0]; p.w = (const char*)d_in[1];
  for (int i = 0; i < NRANK; ++i) { p.ws[i] = (char*)d_ws_all[i]; p.out[i] = (char*)d_out_all[i]; }
  p.out[rank] = (char*)d_out;
  p.rank = rank; p.pad = 0;
  hipLaunchKernelGGL(fused_kernel, dim3(GRID), dim3(NTHR), LDS_BYTES, stream, p);
  hipError_t e = hipGetLastError();
  if (e != hipSuccess) fprintf(stderr, "HIP launch error %s\n", hipGetErrorString(e));
}
```

```cpp
#include <hip/hip_runtime.h>
#include <rccl/rccl.h>
#include <cstdio>

namespace fk {

#define LAS __attribute__((address_space(3)))
#define GAS __attribute__((address_space(1)))
typedef unsigned short bf16_t;
typedef short bf16x8 __attribute__((ext_vector_type(8)));
typedef float f32x4 __attribute__((ext_vector_type(4)));
typedef unsigned u32x4 __attribute__((ext_vector_type(4)));
typedef unsigned u32x2 __attribute__((ext_vector_type(2)));
typedef unsigned short u16x2 __attribute__((ext_vector_type(2)));
typedef GAS unsigned gu32;
typedef GAS unsigned long long gu64;
typedef __amdgpu_buffer_rsrc_t rsrc_t;

constexpr int NRANK = 8;
constexpr int MD = 8192, ND = 8192, KD = 8192, KS = KD / NRANK;
constexpr int NT = KD / 64;
constexpr unsigned OUT_BYTES = (unsigned)MD * ND * 2u;
constexpr unsigned IN_BYTES = (unsigned)MD * KS * 2u;

constexpr int GRID = 256, NTHR = 512;
#define CFG_SLEEP 2
#define CFG_PJ 5
#define CFG_PMAP 1
constexpr int PJ = CFG_PJ;
constexpr int N_P = 7 * PJ, N_D = 128 - N_P;
constexpr int AUX_SYS = 17;

constexpr int NIMG = 3072;
constexpr unsigned IMG = 32768, PKT = 24576;
constexpr int NIT = 1024;
constexpr int NOT = 512;
constexpr int NPI = 384 + NOT;
constexpr int HCAP = 13;

constexpr unsigned WS_CALL = 0;
constexpr unsigned WS_TMO  = WS_CALL + 256 * 128;
constexpr unsigned WS_TK   = WS_TMO + 1024;
constexpr unsigned WS_CNT  = WS_TMO + 4096;
constexpr unsigned WS_PCNT = WS_CNT + NIMG * 128;
constexpr unsigned WS_HIN  = WS_PCNT + 32768;
constexpr unsigned WS_HOUT = WS_HIN + NIMG * 128;
constexpr unsigned WS_HOBI = WS_HOUT + NRANK * NOT * 128;
constexpr unsigned WS_HOBO = WS_HOBI + NIT * 128;
constexpr unsigned WS_PFLG = WS_HOBO + NOT * 128;
constexpr unsigned WS_DBG  = WS_PFLG + 2048 * 128;
constexpr unsigned WS_PIN  = 4u << 20;
constexpr unsigned WS_POUT = WS_PIN + NIMG * PKT;
constexpr unsigned WS_OBI  = WS_POUT + NRANK * NOT * PKT;
constexpr unsigned WS_OBR  = WS_OBI + NIT * PKT;
constexpr unsigned WS_OBO  = WS_OBR + NIT * IMG;
constexpr unsigned WS_PART = WS_OBO + NOT * PKT;
constexpr unsigned WS_IMG  = WS_PART + 256 * 131072;
constexpr unsigned WS_END  = WS_IMG + NIMG * IMG;
static_assert(WS_DBG + 256 * 64 * 8 <= WS_PIN, "control region overflow");
static_assert((size_t)WS_IMG + (size_t)NIMG * IMG < (1ull << 31), "workspace offsets must fit 31 bits");

constexpr unsigned long long DEADLINE = 40000000ull;

constexpr int HTB = 16384, STAGE_BYTES = 8 * HTB;
constexpr int LDS_FLAG = STAGE_BYTES;
constexpr int LDS_CALL = STAGE_BYTES + 64;
constexpr int LDS_TK = STAGE_BYTES + 96;
constexpr int LDS_DEAD = STAGE_BYTES + 80;
constexpr int LDS_GHDR = STAGE_BYTES + 128;
constexpr int LDS_HDR = 0;
constexpr int LDS_SCR = STAGE_BYTES + 512;
constexpr int LDS_BYTES = STAGE_BYTES + 512 + 8 * 1152;

struct Params {
  const char* x; const char* w;
  char* ws[NRANK]; char* out[NRANK];
  int rank; int pad;
};

__device__ __forceinline__ rsrc_t mkr(const void* p, unsigned bytes) { return __builtin_amdgcn_make_buffer_rsrc((void*)p, 0, (int)bytes, 0x00020000); }
__device__ __forceinline__ u32x4 ld128s(rsrc_t r, unsigned off) { return __builtin_amdgcn_raw_buffer_load_b128(r, (int)off, 0, AUX_SYS); }
__device__ __forceinline__ void st128s(u32x4 v, rsrc_t r, unsigned off) { __builtin_amdgcn_raw_buffer_store_b128(v, r, (int)off, 0, AUX_SYS); }
__device__ __forceinline__ unsigned long long rt() { return __builtin_amdgcn_s_memrealtime(); }
__device__ __forceinline__ unsigned rfl(unsigned v) { return (unsigned)__builtin_amdgcn_readfirstlane((int)v); }
__device__ __forceinline__ unsigned rdl(unsigned v, int l) { return (unsigned)__builtin_amdgcn_readlane((int)v, l); }
#define VM0() asm volatile("s_waitcnt vmcnt(0)" ::: "memory")
#define BARRIER() do { asm volatile("" ::: "memory"); __builtin_amdgcn_s_barrier(); asm volatile("" ::: "memory"); } while (0)

#define STAMP(p, idx) do { } while (0)

struct Ctx {
  gu32* tmo; unsigned call;
  __device__ __forceinline__ bool timed_out() const { return rfl(__hip_atomic_load(tmo, __ATOMIC_RELAXED, __HIP_MEMORY_SCOPE_AGENT)) == call; }
  __device__ __forceinline__ bool give_up(unsigned long long t0) const {
    if (timed_out() || rt() - t0 > DEADLINE) { __hip_atomic_store(tmo, call, __ATOMIC_RELAXED, __HIP_MEMORY_SCOPE_AGENT); return true; }
    __builtin_amdgcn_s_sleep(CFG_SLEEP);
    return false;
  }
};

__device__ __forceinline__ unsigned dpp_shr_add(unsigned x) {
  x += (unsigned)__builtin_amdgcn_update_dpp(0, (int)x, 0x111, 0xf, 0xf, false);
  x += (unsigned)__builtin_amdgcn_update_dpp(0, (int)x, 0x112, 0xf, 0xf, false);
  x += (unsigned)__builtin_amdgcn_update_dpp(0, (int)x, 0x114, 0xf, 0xf, false);
  x += (unsigned)__builtin_amdgcn_update_dpp(0, (int)x, 0x118, 0xf, 0xf, false);
  x += (unsigned)__builtin_amdgcn_update_dpp(0, (int)x, 0x142, 0xa, 0xf, false);
  x += (unsigned)__builtin_amdgcn_update_dpp(0, (int)x, 0x143, 0xc, 0xf, false);
  return x;
}
__device__ __forceinline__ unsigned row_max(unsigned x) {
  x = max(x, (unsigned)__builtin_amdgcn_update_dpp(0, (int)x, 0x111, 0xf, 0xf, false));
  x = max(x, (unsigned)__builtin_amdgcn_update_dpp(0, (int)x, 0x112, 0xf, 0xf, false));
  x = max(x, (unsigned)__builtin_amdgcn_update_dpp(0, (int)x, 0x114, 0xf, 0xf, false));
  x = max(x, (unsigned)__builtin_amdgcn_update_dpp(0, (int)x, 0x118, 0xf, 0xf, false));
  return x;
}
__device__ __forceinline__ unsigned wave_max_u32(unsigned x) {
  x = row_max(x);
  x = max(x, (unsigned)__builtin_amdgcn_update_dpp(0, (int)x, 0x142, 0xa, 0xf, false));
  x = max(x, (unsigned)__builtin_amdgcn_update_dpp(0, (int)x, 0x143, 0xc, 0xf, false));
  return rdl(x, 63);
}
__device__ __forceinline__ unsigned vperm(unsigned a, unsigned b, unsigned sel) { return __builtin_amdgcn_perm(a, b, sel); }
__device__ __forceinline__ unsigned lut4(unsigned tab, unsigned sel) { return __builtin_amdgcn_perm(0u, tab, sel); }
__device__ __forceinline__ unsigned bmask(unsigned e) { return __builtin_amdgcn_perm(0u, 0u, e | 0x0C0C0C0Cu); }
__device__ __forceinline__ unsigned bfi(unsigned m, unsigned a, unsigned b) { return (m & a) | (~m & b); }
__device__ __forceinline__ unsigned rank4(unsigned e) { return ((e & 0x010101u) * 0x010101u) << 8; }
__device__ __forceinline__ unsigned cnt4(unsigned e, unsigned acc) { return __builtin_popcount(e) + acc; }

constexpr int SCR_BYTES = 1152;
constexpr int SCR_HIST = 1056;

template <int W> __device__ __forceinline__ void place_bits(LAS unsigned* S, unsigned bitoff, unsigned c, unsigned e) {
  const unsigned pre = rank4(e), em = bmask(e);
  unsigned bits;
  if (W == 2) bits = __builtin_amdgcn_udot4(c, lut4(0x40100401u, pre) & em, 0u, false);
  else bits = __builtin_amdgcn_udot4(c, lut4(0x00001001u, pre) & em, 0u, false) | (__builtin_amdgcn_udot4(c, lut4(0x10010000u, pre) & em, 0u, false) << 8);
  const unsigned long long x = (unsigned long long)bits << (bitoff & 31u);
  LAS unsigned* q = S + (bitoff >> 5);
  __hip_atomic_fetch_or(q, (unsigned)x, __ATOMIC_RELAXED, __HIP_MEMORY_SCOPE_WAVEFRONT);
  __hip_atomic_fetch_or(q + 1, (unsigned)(x >> 32), __ATOMIC_RELAXED, __HIP_MEMORY_SCOPE_WAVEFRONT);
}
__device__ __forceinline__ unsigned fetch_bits(const LAS unsigned* S, unsigned bitoff) {
  const LAS unsigned* q = S + (bitoff >> 5);
  return __builtin_amdgcn_alignbit(q[1], q[0], bitoff & 31u);
}
template <int W> __device__ __forceinline__ unsigned expand_bits(unsigned w, unsigned e) {
  unsigned s;
  if (W == 2) { w &= 0xFFu; const unsigned t = (w | (w << 12)) & 0x000F000Fu; s = (t | (t << 6)) & 0x03030303u; }
  else if (W == 4) { w &= 0xFFFFu; const unsigned t = (w | (w << 8)) & 0x00FF00FFu; s = (t | (t << 4)) & 0x0F0F0F0Fu; }
  else s = w;
  return lut4(s, rank4(e)) & bmask(e);
}
__device__ __forceinline__ unsigned sym_of(bool out, unsigned base, unsigned t) { return out ? ((base - (t & 7u)) | ((t & 8u) << 4)) & 0xFFu : (base - t) & 0xFFu; }

template <bool OUT>
__device__ __forceinline__ unsigned long long encode_chunk(const unsigned (&v)[16], LAS unsigned* S, int lane, u32x4& P0, u32x4& P1, u32x2& P2, u32x2& F) {
  u16x2 mx = __builtin_bit_cast(u16x2, v[0] & 0x7FFF7FFFu);
#pragma unroll
  for (int i = 1; i < 16; ++i) mx = __builtin_elementwise_max(mx, __builtin_bit_cast(u16x2, v[i] & 0x7FFF7FFFu));
  const unsigned base = wave_max_u32(max((unsigned)mx.x, (unsigned)mx.y)) >> (OUT ? 8 : 7);
  const unsigned B4 = base * 0x01010101u;
  LAS unsigned* hist = S + SCR_HIST / 4;
  *(LAS u32x4*)(S + lane * 4) = (u32x4){0u, 0u, 0u, 0u};
  if (lane < 20) S[256 + lane] = 0u;
  if (lane >= 20 && lane < 32) S[256 + lane] = 0u;
  unsigned H[8], L[8], R[8], oor[8], anyo = 0;
#pragma unroll
  for (int g = 0; g < 8; ++g) {
    const unsigned Hs = vperm(v[2 * g + 1], v[2 * g], 0x07050301u), Ls = vperm(v[2 * g + 1], v[2 * g], 0x06040200u);
    if (OUT) { H[g] = Hs; L[g] = Ls; const unsigned D = B4 - (Hs & 0x7F7F7F7Fu); R[g] = (D & 0x07070707u) | ((Hs >> 4) & 0x08080808u); oor[g] = D & 0x78787878u; }
    else { H[g] = bfi(0x01010101u, Ls >> 7, Hs << 1); L[g] = bfi(0x80808080u, Hs, Ls); const unsigned D = B4 - H[g]; R[g] = D & 0x0F0F0F0Fu; oor[g] = D & 0xF0F0F0F0u; }
    anyo |= oor[g];
  }
#pragma unroll
  for (int g = 0; g < 2; ++g)
#pragma unroll
    for (int b = 0; b < 4; ++b) __hip_atomic_fetch_add(hist + ((R[g] >> (8 * b)) & 15u), 1u, __ATOMIC_RELAXED, __HIP_MEMORY_SCOPE_WAVEFRONT);
  unsigned key = (lane < 16) ? (((hist[lane & 15] + 1u) << 4) | (unsigned)lane) : 0u;
  unsigned t[6], cl = 0x0Fu;
#pragma unroll
  for (int i = 0; i < 6; ++i) {
    const unsigned m = rdl(row_max(key), 15);
    t[i] = m & 15u;
    if (key == m) { key = 0u; cl = (i < 3) ? (unsigned)i : (3u | ((unsigned)(i - 3) << 2)); }
  }
  cl <<= 8 * (lane & 3);
  cl |= (unsigned)__builtin_amdgcn_update_dpp(0, (int)cl, 0xB1, 0xf, 0xf, false);
  cl |= (unsigned)__builtin_amdgcn_update_dpp(0, (int)cl, 0x4E, 0xf, 0xf, false);
  const unsigned tb0 = rdl(cl, 0), tb1 = rdl(cl, 4), tb2 = rdl(cl, 8), tb3 = rdl(cl, 12);
  unsigned cls[8];
#pragma unroll
  for (int g = 0; g < 8; ++g) {
    const unsigned r7 = R[g] & 0x07070707u;
    cls[g] = bfi(bmask((R[g] >> 3) & 0x01010101u), vperm(tb3, tb2, r7), vperm(tb1, tb0, r7));
  }
  const bool has_oor = __any(anyo != 0u);
  unsigned e4[8];
#pragma unroll
  for (int g = 0; g < 8; ++g) e4[g] = 0u;
  if (has_oor) {
    const unsigned T1 = t[1] * 0x01010101u;
#pragma unroll
    for (int g = 0; g < 8; ++g) {
      const unsigned nz = OUT ? (oor[g] >> 3) : (oor[g] >> 4);
      e4[g] = ((nz + 0x0F0F0F0Fu) >> 4) & 0x01010101u;
      const unsigned m = bmask(e4[g]);
      cls[g] |= m & 0x0F0F0F0Fu; R[g] = bfi(m, T1, R[g]);
    }
  }
  bool modeB = false;
  if (OUT) {
    unsigned viol[8], nv = 0;
#pragma unroll
    for (int g = 0; g < 8; ++g) { viol[g] = lut4(0x000000FFu, cls[g] & 0x03030303u) & lut4(0xFFFFFF00u, (L[g] >> 6) & 0x03030303u); nv = cnt4(viol[g] & 0x01010101u, nv); }
    modeB = rdl(dpp_shr_add(nv), 63) < 24u;
    if (modeB) {
#pragma unroll
      for (int g = 0; g < 8; ++g) cls[g] |= viol[g] & 0x0F0F0F0Fu;
    }
  }
  unsigned e2[8], e3[8], eL[8], n2 = 0, n3 = 0, nL = 0, n4 = 0;
#pragma unroll
  for (int g = 0; g < 8; ++g) {
    const unsigned c1 = cls[g] & 0x03030303u;
    e2[g] = lut4(0x01000000u, c1); e3[g] = lut4(0x01000000u, (cls[g] >> 2) & 0x03030303u);
    n2 = cnt4(e2[g], n2); n3 = cnt4(e3[g], n3); n4 = cnt4(e4[g], n4);
    if (OUT) { eL[g] = modeB ? lut4(0x01010100u, c1) : 0x01010101u; nL = cnt4(eL[g], nL); } else eL[g] = 0u;
  }
  const unsigned x1 = nL | (n2 << 16), x2 = n3 | (n4 << 16);
  const unsigned s1 = dpp_shr_add(x1), s2 = dpp_shr_add(x2);
  const unsigned t1 = rdl(s1, 63), t2 = rdl(s2, 63);
  const unsigned NL = t1 & 0xFFFFu, N2 = t1 >> 16, N3 = t2 & 0xFFFFu, N4 = t2 >> 16;
  const unsigned bits = 2u * NL + 2u * N2 + 4u * N3 + 8u * N4;
  const bool raw = bits > (OUT ? 8192u : 4096u);
  const unsigned len3 = ((OUT ? 0u : 4096u) + bits + 255u) >> 8;
  if (!raw) {
    const unsigned p1 = s1 - x1, p2 = s2 - x2;
    unsigned oL = 2u * (p1 & 0xFFFFu), o2 = 2u * NL + 2u * (p1 >> 16), o3 = 2u * NL + 2u * N2 + 4u * (p2 & 0xFFFFu), o4 = 2u * NL + 2u * N2 + 4u * N3 + 8u * (p2 >> 16);
#pragma unroll
    for (int g = 0; g < 8; ++g) {
      if (OUT) { place_bits<2>(S, oL, (L[g] >> 6) & 0x03030303u, eL[g]); oL += 2u * (unsigned)__builtin_popcount(eL[g]); }
      place_bits<2>(S, o2, (cls[g] >> 2) & 0x03030303u, e2[g]); o2 += 2u * (unsigned)__builtin_popcount(e2[g]);
      place_bits<4>(S, o3, R[g], e3[g]); o3 += 4u * (unsigned)__builtin_popcount(e3[g]);
    }
    if (has_oor) {
#pragma unroll
      for (int g = 0; g < 8; ++g)
#pragma unroll
        for (int b = 0; b < 4; ++b)
          if ((e4[g] >> (8 * b)) & 1u) {
            const unsigned long long x = (unsigned long long)((H[g] >> (8 * b)) & 0xFFu) << (o4 & 31u);
            __hip_atomic_fetch_or(S + (o4 >> 5), (unsigned)x, __ATOMIC_RELAXED, __HIP_MEMORY_SCOPE_WAVEFRONT);
            __hip_atomic_fetch_or(S + (o4 >> 5) + 1, (unsigned)(x >> 32), __ATOMIC_RELAXED, __HIP_MEMORY_SCOPE_WAVEFRONT);
            o4 += 8u;
          }
    }
  }
  unsigned Fw[2];
#pragma unroll
  for (int i = 0; i < 2; ++i)
    Fw[i] = (cls[4 * i] & 0x03030303u) | ((cls[4 * i + 1] & 0x03030303u) << 2) | ((cls[4 * i + 2] & 0x03030303u) << 4) | ((cls[4 * i + 3] & 0x03030303u) << 6);
  F = (u32x2){Fw[0], Fw[1]};
  if (OUT) {
    unsigned p4[4], p2[2];
#pragma unroll
    for (int i = 0; i < 4; ++i) p4[i] = (L[2 * i] & 0x0F0F0F0Fu) | ((L[2 * i + 1] & 0x0F0F0F0Fu) << 4);
#pragma unroll
    for (int i = 0; i < 2; ++i)
      p2[i] = ((L[4 * i] >> 4) & 0x03030303u) | (((L[4 * i + 1] >> 4) & 0x03030303u) << 2) | (((L[4 * i + 2] >> 4) & 0x03030303u) << 4) | (((L[4 * i + 3] >> 4) & 0x03030303u) << 6);
    P0 = (u32x4){p4[0], p4[1], p4[2], p4[3]}; P1 = P0; P2 = (u32x2){p2[0], p2[1]};
  } else {
    P0 = (u32x4){L[0], L[1], L[2], L[3]}; P1 = (u32x4){L[4], L[5], L[6], L[7]}; P2 = F;
  }
  const unsigned tt = t[0] | (t[1] << 4) | (t[2] << 8) | (t[3] << 12) | (t[4] << 16) | (t[5] << 20);
  return (unsigned long long)(base | (raw ? 0x4000u : (len3 << 8)) | (modeB ? 0x8000u : 0u)) | ((unsigned long long)tt << 16);
}

template <bool OUT>
__device__ __forceinline__ void store_chunk(rsrc_t r, unsigned o, const LAS unsigned* S, unsigned len3, int lane, u32x4 P0, u32x4 P1, u32x2 P2, u32x2 F) {
  st128s(P0, r, o + lane * 16);
  const u32x4 sv = *(const LAS u32x4*)(S + lane * 4);
  if (OUT) {
    __builtin_amdgcn_raw_buffer_store_b64(P2, r, (int)(o + 1024 + lane * 8), 0, AUX_SYS);
    __builtin_amdgcn_raw_buffer_store_b64(F, r, (int)(o + 1536 + lane * 8), 0, AUX_SYS);
    if ((unsigned)lane < 2u * len3) st128s(sv, r, o + 2048 + lane * 16);
  } else {
    st128s(P1, r, o + 1024 + lane * 16);
    __builtin_amdgcn_raw_buffer_store_b64(F, r, (int)(o + 2048 + lane * 8), 0, AUX_SYS);
    if ((unsigned)lane + 32u < 2u * len3) st128s(sv, r, o + 2560 + lane * 16);
  }
}

template <bool OUT>
__device__ __forceinline__ void decode_chunk(rsrc_t r, unsigned o, unsigned long long hp, LAS unsigned* S, int lane, unsigned (&v)[16]) {
  const unsigned h0 = (unsigned)hp, base = h0 & 0xFFu, len3 = (h0 >> 8) & 63u, tt = (unsigned)(hp >> 16) & 0xFFFFFFu;
  const bool modeB = (h0 & 0x8000u) != 0u;
  const u32x4 P0 = ld128s(r, o + lane * 16);
  u32x4 P1 = P0, sv = (u32x4){0u, 0u, 0u, 0u}; u32x2 P2, F;
  if (OUT) {
    P2 = __builtin_amdgcn_raw_buffer_load_b64(r, (int)(o + 1024 + lane * 8), 0, AUX_SYS);
    F = __builtin_amdgcn_raw_buffer_load_b64(r, (int)(o + 1536 + lane * 8), 0, AUX_SYS);
    sv = ld128s(r, ((unsigned)lane < 2u * len3) ? o + 2048 + lane * 16 : 0xFFFFFF00u);
  } else {
    P1 = ld128s(r, o + 1024 + lane * 16);
    F = __builtin_amdgcn_raw_buffer_load_b64(r, (int)(o + 2048 + lane * 8), 0, AUX_SYS); P2 = F;
    sv = ld128s(r, ((unsigned)lane + 32u < 2u * len3) ? o + 2560 + lane * 16 : 0xFFFFFF00u);
  }
  unsigned tab1 = 0, tab2 = 0;
#pragma unroll
  for (int i = 0; i < 3; ++i) { tab1 |= sym_of(OUT, base, (tt >> (4 * i)) & 15u) << (8 * i); tab2 |= sym_of(OUT, base, (tt >> (12 + 4 * i)) & 15u) << (8 * i); }
  const unsigned B4 = base * 0x01010101u, T1 = ((tt >> 4) & 15u) * 0x01010101u;
  *(LAS u32x4*)(S + lane * 4) = sv;
  unsigned c1[8], e2[8], eL[8], n2 = 0, nL = 0;
#pragma unroll
  for (int g = 0; g < 8; ++g) {
    c1[g] = (((g < 4) ? F.x : F.y) >> (2 * (g & 3))) & 0x03030303u;
    e2[g] = lut4(0x01000000u, c1[g]); n2 = cnt4(e2[g], n2);
    if (OUT) { eL[g] = modeB ? lut4(0x01010100u, c1[g]) : 0x01010101u; nL = cnt4(eL[g], nL); } else eL[g] = 0u;
  }
  const unsigned x1 = nL | (n2 << 16), s1 = dpp_shr_add(x1), t1 = rdl(s1, 63), p1 = s1 - x1;
  const unsigned NL = t1 & 0xFFFFu, N2 = t1 >> 16;
  unsigned oL = 2u * (p1 & 0xFFFFu), o2 = 2u * NL + 2u * (p1 >> 16);
  unsigned H[8], L[8], e3[8], n3 = 0;
#pragma unroll
  for (int g = 0; g < 8; ++g) {
    const unsigned c2 = expand_bits<2>(fetch_bits(S, o2), e2[g]); o2 += 2u * (unsigned)__builtin_popcount(e2[g]);
    e3[g] = lut4(0x01000000u, c2); n3 = cnt4(e3[g], n3);
    H[g] = bfi(bmask(e2[g]), lut4(tab2, c2), lut4(tab1, c1[g]));
    if (OUT) {
      const unsigned top = expand_bits<2>(fetch_bits(S, oL), eL[g]); oL += 2u * (unsigned)__builtin_popcount(eL[g]);
      const unsigned p4 = (g >> 1) == 0 ? P0.x : (g >> 1) == 1 ? P0.y : (g >> 1) == 2 ? P0.z : P0.w;
      L[g] = ((p4 >> (4 * (g & 1))) & 0x0F0F0F0Fu) | (((((g < 4) ? P2.x : P2.y) >> (2 * (g & 3))) & 0x03030303u) << 4) | (top << 6);
    } else {
      L[g] = g == 0 ? P0.x : g == 1 ? P0.y : g == 2 ? P0.z : g == 3 ? P0.w : g == 4 ? P1.x : g == 5 ? P1.y : g == 6 ? P1.z : P1.w;
    }
  }
  const unsigned s2 = dpp_shr_add(n3), N3 = rdl(s2, 63);
  unsigned o3 = 2u * NL + 2u * N2 + 4u * (s2 - n3);
  unsigned e4[8], any4 = 0;
#pragma unroll
  for (int g = 0; g < 8; ++g) {
    unsigned R = expand_bits<4>(fetch_bits(S, o3), e3[g]); o3 += 4u * (unsigned)__builtin_popcount(e3[g]);
    e4[g] = e3[g] & ~((((R ^ T1) + 0x0F0F0F0Fu) >> 4) & 0x01010101u);
    R &= ~bmask(e4[g]);
    const unsigned H3 = OUT ? ((B4 - (R & 0x07070707u)) | ((R & 0x08080808u) << 4)) : (B4 - R);
    H[g] = bfi(bmask(e3[g]), H3, H[g]);
    any4 |= e4[g];
  }
  if (__any(any4 != 0u)) {
    unsigned n4 = 0;
#pragma unroll
    for (int g = 0; g < 8; ++g) n4 = cnt4(e4[g], n4);
    unsigned o4 = 2u * NL + 2u * N2 + 4u * N3 + 8u * (dpp_shr_add(n4) - n4);
#pragma unroll
    for (int g = 0; g < 8; ++g) {
      const unsigned Rr = expand_bits<8>(fetch_bits(S, o4), e4[g]); o4 += 8u * (unsigned)__builtin_popcount(e4[g]);
      H[g] = bfi(bmask(e4[g]), Rr, H[g]);
    }
  }
#pragma unroll
  for (int g = 0; g < 8; ++g) {
    const unsigned Hs = OUT ? H[g] : bfi(0x80808080u, L[g], H[g] >> 1), Ls = OUT ? L[g] : bfi(0x80808080u, H[g] << 7, L[g]);
    v[2 * g] = vperm(Hs, Ls, 0x05010400u);
    v[2 * g + 1] = vperm(Hs, Ls, 0x07030602u);
  }
}

struct MapX { unsigned base; __device__ __forceinline__ unsigned operator()(int w, int j, int l) const { return base + (unsigned)(w * 4096 + j * 1024 + l * 16); } };
struct MapW { unsigned base; __device__ __forceinline__ unsigned operator()(int w, int j, int l) const { return base + (unsigned)((j >> 1) * 16384 + ((w >> 1) * 2 + (j & 1)) * 2048 + (w & 1) * 1024 + l * 16); } };
struct MapO { unsigned base; __device__ __forceinline__ unsigned operator()(int w, int j, int l) const { return base + (unsigned)((64 * (w >> 2) + 16 * (j >> 1) + (l & 15)) * (ND * 2) + (64 * (w & 3) + 32 * (j & 1) + 8 * (l >> 4)) * 2); } };
__device__ __forceinline__ unsigned out_task_base(int rank, int task) {
  const int tile = task >> 2, q = task & 3, pm = tile >> 3, pn = tile & 7;
  return (unsigned)(((rank >> 2) * 4096 + pm * 256 + (q >> 1) * 128 + (q & 1) * 32) * (ND * 2) + ((rank & 3) * 2048 + pn * 256) * 2);
}

#define DRAIN_ALL() do { asm volatile("s_waitcnt vmcnt(0) lgkmcnt(0)" ::: "memory"); __builtin_amdgcn_s_waitcnt(0x0070); } while (0)

__device__ __forceinline__ unsigned long long mk_granule(unsigned long long payload, unsigned call) { return payload | ((unsigned long long)(call & 0xFFFFu) << 48); }
__device__ __forceinline__ unsigned long long uni64(unsigned long long g) { return ((unsigned long long)rfl((unsigned)(g >> 32)) << 32) | rfl((unsigned)g); }
__device__ __forceinline__ unsigned long long ld_granule(const char* ws, unsigned off) {
  return __hip_atomic_load((const gu64*)(ws + off), __ATOMIC_RELAXED, __HIP_MEMORY_SCOPE_SYSTEM);
}
__device__ __forceinline__ bool tag_ok(unsigned long long g, unsigned call) { return (unsigned)(g >> 48) == (call & 0xFFFFu); }
__device__ __forceinline__ bool poll_granule(const Ctx& cx, const char* ws, unsigned off, unsigned long long& g) {
  g = uni64(g);
  if (!tag_ok(g, cx.call)) {
    const unsigned long long t0 = rt();
    do {
      if (cx.give_up(t0)) return false;
      g = uni64(ld_granule(ws, off));
    } while (!tag_ok(g, cx.call));
  }
  asm volatile("" ::: "memory");
  return true;
}

template <bool OUT, class Map>
__device__ __forceinline__ bool decode_task(const Ctx& cx, const char* ws, rsrc_t lw, unsigned hline, unsigned pkoff, rsrc_t fin, const Map& map, LAS unsigned* S, int wid, int lane) {
  unsigned long long g = ld_granule(ws, hline + wid * 8);
  if (!poll_granule(cx, ws, hline + wid * 8, g)) return false;
  if (!((unsigned)g & 0x4000u)) {
    unsigned v[16];
    decode_chunk<OUT>(lw, pkoff + (unsigned)wid * 3072u, g, S, lane, v);
#pragma unroll
    for (int j = 0; j < 4; ++j) st128s((u32x4){v[4 * j], v[4 * j + 1], v[4 * j + 2], v[4 * j + 3]}, fin, map(wid, j, lane));
  }
  return true;
}
template <class Map>
__device__ __forceinline__ void nan_chunk(rsrc_t fin, const Map& map, int wid, int lane) {
#pragma unroll
  for (int j = 0; j < 4; ++j) st128s((u32x4){0x7FC07FC0u, 0x7FC07FC0u, 0x7FC07FC0u, 0x7FC07FC0u}, fin, map(wid, j, lane));
}

struct InTask { int kt, isw, idx; };
__device__ __forceinline__ InTask in_task(int i) {
  InTask t;
  if (i < 768) { t.kt = i / 48; const int u = i - t.kt * 48, u3 = u / 3, r3 = u - u3 * 3;
    if (r3 == 0) { t.isw = 0; t.idx = (u3 & 1) * 16 + (u3 >> 1); } else { t.isw = 1; const int c = u3 * 2 + r3 - 1; t.idx = (c & 3) * 8 + (c >> 2); } }
  else { const int u = i - 768; t.kt = (u >> 4) & 15; t.isw = 0; t.idx = (u & 1) * 16 + 8 + ((u & 15) >> 1); }
  return t;
}
__device__ __forceinline__ int otask_of(int isw, int idx, int kt) { return isw * 512 + idx * 16 + kt; }
__device__ __forceinline__ void in_load(const Params& p, int i, u32x4 (&r)[4], int wid, int lane) {
  const InTask t = in_task(i);
  const int rr = lane >> 2, cp = (lane & 3) ^ ((rr >> 3) << 1);
  if (!t.isw) {
    const rsrc_t rx = mkr(p.x, IN_BYTES);
#pragma unroll
    for (int j = 0; j < 4; ++j) {
      const unsigned off = (unsigned)((256 * t.idx + 32 * wid + 16 * (j >> 1) + rr) * (KS * 2) + (64 * t.kt + 32 * (j & 1) + 8 * cp) * 2);
      r[j] = __builtin_amdgcn_raw_buffer_load_b128(rx, (int)off, 0, 0);
    }
  } else {
    const rsrc_t rw = mkr(p.w, IN_BYTES);
    const unsigned off0 = (unsigned)((64 * t.kt + 32 * (wid & 1) + 8 * cp) * (ND * 2) + (256 * t.idx + 64 * (wid >> 1) + 4 * rr) * 2);
#pragma unroll
    for (int d = 0; d < 4; ++d) {
      const u32x2 a = __builtin_amdgcn_raw_buffer_load_b64(rw, (int)(off0 + (unsigned)(2 * d) * (ND * 2)), 0, 0);
      const u32x2 b = __builtin_amdgcn_raw_buffer_load_b64(rw, (int)(off0 + (unsigned)(2 * d + 1) * (ND * 2)), 0, 0);
      r[d] = __builtin_shufflevector(a, b, 0, 1, 2, 3);
    }
  }
}
__device__ __forceinline__ void in_prep(int isw, const u32x4 (&r)[4], unsigned (&v)[16]) {
  if (isw) {
#pragma unroll
    for (int d = 0; d < 4; ++d) {
      v[0 + d] = __builtin_amdgcn_perm(r[d].z, r[d].x, 0x05040100u);
      v[4 + d] = __builtin_amdgcn_perm(r[d].z, r[d].x, 0x07060302u);
      v[8 + d] = __builtin_amdgcn_perm(r[d].w, r[d].y, 0x05040100u);
      v[12 + d] = __builtin_amdgcn_perm(r[d].w, r[d].y, 0x07060302u);
    }
  } else {
#pragma unroll
    for (int j = 0; j < 4; ++j) { v[4 * j] = r[j].x; v[4 * j + 1] = r[j].y; v[4 * j + 2] = r[j].z; v[4 * j + 3] = r[j].w; }
  }
}

__device__ __forceinline__ void role_encode(const Params& p, const Ctx& cx, LAS unsigned* S, int e, int ne) {
  const int tid = threadIdx.x, wid = (int)rfl(tid >> 6), lane = tid & 63;
  char* ws = p.ws[p.rank];
  const rsrc_t lw = mkr(ws, WS_END);
  u32x4 A[4];
  in_load(p, e, A, wid, lane);
  DRAIN_ALL();
  for (int i = e; i < NIT; i += ne) {
    const InTask t = in_task(i);
    unsigned v[16];
    in_prep(t.isw, A, v);
    if (i + ne < NIT) in_load(p, i + ne, A, wid, lane);
    u32x4 P0, P1; u32x2 P2, F;
    const unsigned long long hp = encode_chunk<false>(v, S, lane, P0, P1, P2, F);
    const int ot = otask_of(t.isw, t.idx, t.kt);
    if (!((unsigned)hp & 0x4000u)) {
      store_chunk<false>(lw, WS_OBI + (unsigned)ot * PKT + (unsigned)wid * 3072u, S, ((unsigned)hp >> 8) & 63u, lane, P0, P1, P2, F);
    } else {
      const bool mine = t.isw ? ((t.idx >> 3) == (p.rank & 3)) : ((t.idx >> 4) == (p.rank >> 2));
      const int T = t.kt * 8 + p.rank;
      const unsigned ib = WS_IMG + (unsigned)(t.isw ? 2048 + (t.idx & 7) * 128 + T : (t.idx & 15) * 128 + T) * IMG;
#pragma unroll
      for (int j = 0; j < 4; ++j) {
        const u32x4 q = (u32x4){v[4 * j], v[4 * j + 1], v[4 * j + 2], v[4 * j + 3]};
        st128s(q, lw, WS_OBR + (unsigned)ot * IMG + (unsigned)(wid * 4096 + j * 1024 + lane * 16));
        if (mine) st128s(q, lw, t.isw ? MapW{ib}(wid, j, lane) : MapX{ib}(wid, j, lane));
      }
    }
    DRAIN_ALL();
    if (lane == 0) __hip_atomic_store((gu64*)(ws + WS_HOBI + (unsigned)ot * 64u) + wid, mk_granule(hp, cx.call), __ATOMIC_RELAXED, __HIP_MEMORY_SCOPE_SYSTEM);
  }
}

struct PItem { unsigned hsrc, dsrc, hdst, ddst, rs, rd; int kind; };
__device__ __forceinline__ PItem p_item(int rank, int d, int i) {
  PItem r;
  if (i < 384) {
    int kt, isw, k;
    if (i < 256) { kt = i >> 4; isw = i & 1; k = (i & 15) >> 1; } else { kt = (i - 256) >> 3; isw = 0; k = 8 + ((i - 256) & 7); }
    const int idx = isw ? (d & 3) * 8 + k : (d >> 2) * 16 + k;
    const int ot = otask_of(isw, idx, kt), T = kt * 8 + rank, img = isw ? 2048 + k * 128 + T : k * 128 + T;
    r.hsrc = WS_HOBI + (unsigned)ot * 64u; r.dsrc = WS_OBI + (unsigned)ot * PKT;
    r.hdst = WS_HIN + (unsigned)img * 64u; r.ddst = WS_PIN + (unsigned)img * PKT;
    r.rs = WS_OBR + (unsigned)ot * IMG; r.rd = WS_IMG + (unsigned)img * IMG; r.kind = isw;
  } else {
    const int task = (i - 384) & (NOT - 1);
    r.hsrc = WS_HOBO + (unsigned)task * 64u; r.dsrc = WS_OBO + (unsigned)task * PKT;
    r.hdst = WS_HOUT + (unsigned)(rank * NOT + task) * 64u; r.ddst = WS_POUT + (unsigned)(rank * NOT + task) * PKT;
    r.rs = r.rd = out_task_base(rank, task); r.kind = 2;
  }
  return r;
}

#define P_LDG(ws, off) ld_granule(ws, off)
__device__ __forceinline__ unsigned atomic_add_lane0_nowait(gu32* ptr) {
  unsigned r; unsigned long long sv; const unsigned one = 1u;
  asm volatile("s_mov_b64 %1, exec\n\ts_mov_b64 exec, 1\n\tglobal_atomic_add %0, %2, %3, off sc0\n\ts_mov_b64 exec, %1"
               : "=&v"(r), "=&s"(sv) : "v"(ptr), "v"(one) : "memory");
  return r;
}
#define DRAIN_PIN2(a, b) do { asm volatile("s_waitcnt vmcnt(0) lgkmcnt(0)" : "+v"(a), "+v"(b) :: "memory"); __builtin_amdgcn_s_waitcnt(0x0070); } while (0)

__device__ __forceinline__ void p_publish(const Ctx& cx, const char* ws, char* wd, unsigned hsrc, unsigned hdst, unsigned long long hl, int lane) {
  const gu64* src = (const gu64*)(ws + hsrc) + (lane & 7);
  while (!__all(tag_ok(hl, cx.call))) hl = __hip_atomic_load(src, __ATOMIC_RELAXED, __HIP_MEMORY_SCOPE_SYSTEM);
  if (lane < 8) __hip_atomic_store((gu64*)(wd + hdst) + lane, hl, __ATOMIC_RELAXED, __HIP_MEMORY_SCOPE_SYSTEM);
}

__device__ __forceinline__ void role_push(const Params& p, const Ctx& cx, int link, int v0, int nv) {
  constexpr int NC = NPI * 8;
  const int lane = threadIdx.x & 63;
  int d = p.rank + 1 + link; if (d >= NRANK) d -= NRANK;
  if ((unsigned)d >= (unsigned)NRANK) return;
  char* ws = p.ws[p.rank]; char* wd = p.ws[d];
  const rsrc_t lw = mkr(ws, WS_END), dw = mkr(wd, WS_END);
  gu32* tkc = (gu32*)(ws + WS_TK) + link * 32;
  gu32* pcnt = (gu32*)(ws + WS_PCNT) + link * NPI;
  const unsigned tkbase = (cx.call - 1u) * (unsigned)NC - 3u * (unsigned)nv;
  const unsigned lo = (unsigned)lane * 16u;
  int it = v0, nx = v0 + nv, n2 = v0 + 2 * nv;
  PItem cur = p_item(p.rank, d, it >> 3);
  unsigned long long gc = P_LDG(ws, cur.hsrc + (it & 7) * 8), gn;
  if (!poll_granule(cx, ws, cur.hsrc + (it & 7) * 8, gc)) return;
  auto load3 = [&](const PItem& t, int u, unsigned long long g, u32x4& a, u32x4& b, u32x4& c) {
    const unsigned o = t.dsrc + (unsigned)(u & 7) * 3072u + lo, n3 = 2u * (((unsigned)g >> 8) & 63u);
    a = ld128s(lw, o); b = ld128s(lw, o + 1024); c = ld128s(lw, (unsigned)lane < n3 ? o + 2048 : 0xFFFFFF00u);
  };
  u32x4 c0, c1, c2;
  load3(cur, it, gc, c0, c1, c2);
  gn = P_LDG(ws, p_item(p.rank, d, nx >> 3).hsrc + (nx & 7) * 8);
  DRAIN_ALL();
  int prev = -1; unsigned prev_hsrc = 0, prev_hdst = 0;
  while (it < NC) {
    const int w = it & 7;
    if (!((unsigned)gc & 0x4000u)) {
      const unsigned o = cur.ddst + (unsigned)w * 3072u + lo;
      st128s(c0, dw, o); st128s(c1, dw, o + 1024);
      if ((unsigned)lane < 2u * (((unsigned)gc >> 8) & 63u)) st128s(c2, dw, o + 2048);
    } else if (cur.kind == 2) {
      const rsrc_t so = mkr(p.out[p.rank], OUT_BYTES), dd = mkr(p.out[d], OUT_BYTES); const MapO map{cur.rs};
#pragma unroll
      for (int j = 0; j < 4; ++j) st128s(ld128s(so, map(w, j, lane)), dd, map(w, j, lane));
    } else {
#pragma unroll
      for (int j = 0; j < 4; ++j) { const u32x4 q = ld128s(lw, cur.rs + (unsigned)(w * 4096 + j * 1024) + lo);
        st128s(q, dw, cur.kind ? MapW{cur.rd}(w, j, lane) : MapX{cur.rd}(w, j, lane)); }
    }
    PItem nxt = cur; u32x4 n0 = c0, n1 = c1, n2v = c2; unsigned long long g2 = 0;
    bool have = true;
    if (nx < NC) {
      nxt = p_item(p.rank, d, nx >> 3);
      gn = uni64(gn);
      have = tag_ok(gn, cx.call);
      if (have) load3(nxt, nx, gn, n0, n1, n2v);
      if (n2 < NC) g2 = P_LDG(ws, p_item(p.rank, d, n2 >> 3).hsrc + (n2 & 7) * 8);
    }
    unsigned tk = atomic_add_lane0_nowait(tkc);
    unsigned pc = 0; unsigned long long hl = 0;
    if (prev >= 0) {
      pc = atomic_add_lane0_nowait(pcnt + (prev >> 3));
      hl = __hip_atomic_load((const gu64*)(ws + prev_hsrc) + (lane & 7), __ATOMIC_RELAXED, __HIP_MEMORY_SCOPE_SYSTEM);
    }
    DRAIN_PIN2(tk, pc);
    if (prev >= 0 && rfl(pc) == 8u * cx.call - 1u) p_publish(cx, ws, wd, prev_hsrc, prev_hdst, hl, lane);
    const int n3 = (int)(rfl(tk) - tkbase);
    if (!have) {
      unsigned pc2 = atomic_add_lane0_nowait(pcnt + (it >> 3)), dummy = 0;
      hl = __hip_atomic_load((const gu64*)(ws + cur.hsrc) + (lane & 7), __ATOMIC_RELAXED, __HIP_MEMORY_SCOPE_SYSTEM);
      DRAIN_PIN2(pc2, dummy);
      if (rfl(pc2) == 8u * cx.call - 1u) p_publish(cx, ws, wd, cur.hsrc, cur.hdst, hl, lane);
      prev = -1;
      if (!poll_granule(cx, ws, nxt.hsrc + (nx & 7) * 8, gn)) return;
      load3(nxt, nx, gn, n0, n1, n2v);
      DRAIN_ALL();
    } else {
      prev = it; prev_hsrc = cur.hsrc; prev_hdst = cur.hdst;
    }
    cur = nxt; gc = gn; gn = g2; it = nx; nx = n2; n2 = n3;
    c0 = n0; c1 = n1; c2 = n2v;
  }
  if (prev >= 0) {
    unsigned pc = atomic_add_lane0_nowait(pcnt + (prev >> 3)), dummy = 0;
    const unsigned long long hl = __hip_atomic_load((const gu64*)(ws + prev_hsrc) + (lane & 7), __ATOMIC_RELAXED, __HIP_MEMORY_SCOPE_SYSTEM);
    DRAIN_PIN2(pc, dummy);
    if (rfl(pc) == 8u * cx.call - 1u) p_publish(cx, ws, wd, prev_hsrc, prev_hdst, hl, lane);
  }
  STAMP(p, 2);
}

__device__ __forceinline__ void role_din(const Params& p, const Ctx& cx, LAS unsigned* S, int dw) {
  const int tid = threadIdx.x, wid = (int)rfl(tid >> 6), lane = tid & 63;
  char* ws = p.ws[p.rank];
  const rsrc_t lw = mkr(ws, WS_END);
  gu32* cnt = (gu32*)(ws + WS_CNT);
  for (int i = dw; i < NIMG; i += N_D) {
    int k, T; bool isw = false;
    if (i < 2048) { T = i >> 4; isw = (i & 1); k = (i & 15) >> 1; } else { const int u = i - 2048; T = u >> 3; k = 8 + (u & 7); }
    const int img = isw ? 2048 + k * 128 + T : k * 128 + T;
    unsigned hl = WS_HIN + (unsigned)img * 64u, pk = WS_PIN + (unsigned)img * PKT;
    if ((T & 7) == p.rank) {
      const int ot = otask_of(isw, isw ? (p.rank & 3) * 8 + k : (p.rank >> 2) * 16 + k, T >> 3);
      hl = WS_HOBI + (unsigned)ot * 64u; pk = WS_OBI + (unsigned)ot * PKT;
    }
    const unsigned ib = WS_IMG + (unsigned)img * IMG;
    bool ok;
    if (isw) ok = decode_task<false>(cx, ws, lw, hl, pk, lw, MapW{ib}, S, wid, lane);
    else ok = decode_task<false>(cx, ws, lw, hl, pk, lw, MapX{ib}, S, wid, lane);
    if (!ok) return;
    VM0();
    if (lane == 0) __hip_atomic_fetch_add(cnt + img * 32, 1u, __ATOMIC_RELAXED, __HIP_MEMORY_SCOPE_AGENT);
  }
}

__device__ __forceinline__ void role_dout(const Params& p, const Ctx& cx, LAS unsigned* S, int dw, int nw) {
  const int tid = threadIdx.x, wid = (int)rfl(tid >> 6), lane = tid & 63;
  char* ws = p.ws[p.rank];
  const rsrc_t lw = mkr(ws, WS_END), fin = mkr(p.out[p.rank], OUT_BYTES);
  bool dead = false;
  for (int i = dw; i < NOT * (NRANK - 1); i += nw) {
    const int task = i / (NRANK - 1), k = i - task * (NRANK - 1);
    int s = p.rank + 1 + k; if (s >= NRANK) s -= NRANK;
    const unsigned tidx = (unsigned)(s * NOT + task);
    const MapO map{out_task_base(s, task)};
    if (!dead) dead = !decode_task<true>(cx, ws, lw, WS_HOUT + tidx * 64u, WS_POUT + tidx * PKT, fin, map, S, wid, lane);
    if (dead) nan_chunk(fin, map, wid, lane);
  }
}

__host__ __device__ __forceinline__ int lds_byte(int r, int c) { const int st = (r >> 4) * 2 + (c >> 5), rr = r & 15, cc = c & 31, ob = rr * 64 + cc * 2; return st * 1024 + (ob ^ (((ob >> 9) & 1) << 5)); }
__device__ __forceinline__ unsigned cvt_pk_bf16(float lo, float hi) { unsigned r; asm volatile("v_cvt_pk_bf16_f32 %0, %1, %2" : "=v"(r) : "v"(lo), "v"(hi)); return r; }
__device__ __forceinline__ float gelu1(float y) {
  const float k1 = -2.0f * 0.7978845608028654f * 1.4426950408889634f, k3 = k1 * 0.044715f;
  const float t = y * (k3 * (y * y) + k1);
  return y * __builtin_amdgcn_rcpf(1.0f + __builtin_amdgcn_exp2f(t));
}

__device__ __forceinline__ void gemm_unit(const Params& p, const Ctx& cx, LAS unsigned char* lds, int pm, int pn, int h, int ph) {
  constexpr int NTH = NT / 2;
  const int tid = threadIdx.x, wid = (int)rfl(tid >> 6), lane = tid & 63, wr = wid >> 2, wc = wid & 3, fr = lane & 15, fq = lane >> 4;
  char* ws = p.ws[p.rank];
  const char* cA = ws + WS_IMG + (size_t)(pm * 128 + 8 * h) * IMG;
  const char* cB = ws + WS_IMG + (size_t)(2048 + pn * 128 + 8 * h) * IMG;
  const gu32* fl = (const gu32*)(ws + WS_CNT) + (size_t)(((lane & 2) ? 2048 + pn * 128 : pm * 128) + 8 * h + (lane & 1)) * 32;
  const unsigned need = 8u * cx.call;
  const unsigned voff = (unsigned)tid * 16u;
  const unsigned ldsw = (unsigned)wid * 1024u;
  const int aoff = lds_byte(wr * 64 + fr, fq * 8), boff = lds_byte(wc * 32 + fr, fq * 8);
  bool dead = false;
#define KT(t) ((((t) >> 3) << 4) + ((t) & 7))
  auto wait_tiles = [&](int t) {
    if (wid == 0 && !dead) {
      const gu32* f = fl + KT(t) * 32;
      if (!__all(__hip_atomic_load(f, __ATOMIC_RELAXED, __HIP_MEMORY_SCOPE_AGENT) >= need)) {
        const unsigned long long t0 = rt();
        while (!__all(__hip_atomic_load(f, __ATOMIC_RELAXED, __HIP_MEMORY_SCOPE_AGENT) >= need)) { if (cx.give_up(t0)) { dead = true; break; } }
      }
    }
  };
#define SA(b, h) (((b) * 2 + (h)) * HTB)
#define SB(b, h) ((4 + (b) * 2 + (h)) * HTB)
#define STAGE(bufoff, gbase) do { _Pragma("unroll") for (int _i = 0; _i < 2; ++_i) \
    __builtin_amdgcn_global_load_lds((const GAS unsigned*)((const char*)(gbase) + voff + _i * 8192), (LAS unsigned*)(lds + (bufoff) + ldsw + _i * 8192), 16, 0, AUX_SYS); } while (0)
#define LDA(dst, b, h) do { _Pragma("unroll") for (int m = 0; m < 4; ++m) _Pragma("unroll") for (int k = 0; k < 2; ++k) dst[m][k] = *(const LAS bf16x8*)(lds + SA(b, h) + aoff + m * 2048 + k * 1024); } while (0)
#define LDB(dst, b, h) do { _Pragma("unroll") for (int n = 0; n < 2; ++n) _Pragma("unroll") for (int k = 0; k < 2; ++k) dst[n][k] = *(const LAS bf16x8*)(lds + SB(b, h) + boff + n * 2048 + k * 1024); } while (0)
#define MMA(ai, bj, At, Bt) do { __builtin_amdgcn_s_setprio(1); _Pragma("unroll") for (int m = 0; m < 4; ++m) _Pragma("unroll") for (int n = 0; n < 2; ++n) _Pragma("unroll") for (int k = 0; k < 2; ++k) \
    acc[ai][bj][m][n] = __builtin_amdgcn_mfma_f32_16x16x32_bf16(Bt[n][k], At[m][k], acc[ai][bj][m][n], 0, 0, 0); __builtin_amdgcn_s_setprio(0); } while (0)
#define WAIT_V(n) asm volatile("s_waitcnt vmcnt(" #n ")" ::: "memory")
#define WAIT_L(n) asm volatile("s_waitcnt lgkmcnt(" #n ")" ::: "memory")
#define BAR __builtin_amdgcn_s_barrier()
#define SCHED __builtin_amdgcn_sched_barrier(0)
  f32x4 acc[2][2][4][2];
#pragma unroll
  for (int a = 0; a < 2; ++a)
#pragma unroll
    for (int b = 0; b < 2; ++b)
#pragma unroll
      for (int m = 0; m < 4; ++m)
#pragma unroll
        for (int n = 0; n < 2; ++n) acc[a][b][m][n] = (f32x4){0.f, 0.f, 0.f, 0.f};
  bf16x8 At[4][2], B0[2][2], B1[2][2];
  wait_tiles(0); wait_tiles(2);
  BARRIER();
  STAMP(p, 1 + 4 * ph);
  STAGE(SB(0, 0), cB); STAGE(SA(0, 0), cA); STAGE(SB(0, 1), cB + HTB); STAGE(SA(0, 1), cA + HTB);
  if (wr == 1) BAR;
  WAIT_V(4); BAR;
  STAGE(SB(1, 0), cB + IMG); STAGE(SA(1, 0), cA + IMG); STAGE(SB(1, 1), cB + IMG + HTB);
  WAIT_V(6); BAR;
  for (int t = 0; t < NTH; t += 2) {
    const bool last = (t == NTH - 2);
    const char* a1 = cA + (size_t)(KT(t) + 1) * IMG;
    const size_t o2 = last ? 0 : (size_t)KT(t + 2) * IMG;
    const char* a2 = cA + o2; const char* b2 = cB + o2;
    const char* a3 = a2 + IMG; const char* b3 = b2 + IMG;
    LAS unsigned* fslot = (LAS unsigned*)(lds + LDS_FLAG + ((t >> 1) & 1) * 16);
    const bool chk = (t + 4 < NTH);
    if (chk && wid == 0 && lane < 4) __builtin_amdgcn_global_load_lds(fl + KT(t + 4) * 32, fslot, 4, 0, AUX_SYS);
    LDB(B0, 0, 0); SCHED; LDA(At, 0, 0); STAGE(SA(1, 1), a1 + HTB);
    WAIT_L(8); BAR; WAIT_L(0); MMA(0, 0, At, B0); BAR; SCHED;
    LDB(B1, 0, 1); STAGE(SB(0, 0), b2);
    BAR; WAIT_L(0); MMA(0, 1, At, B1); BAR;
    LDA(At, 0, 1); STAGE(SA(0, 0), a2);
    BAR; WAIT_L(0); MMA(1, 0, At, B0); BAR; SCHED;
    STAGE(SB(0, 1), b2 + HTB);
    WAIT_V(6); BAR; MMA(1, 1, At, B1); BAR;
    LDB(B0, 1, 0); SCHED; LDA(At, 1, 0); STAGE(SA(0, 1), a2 + HTB);
    WAIT_L(8); BAR; WAIT_L(0); MMA(0, 0, At, B0); BAR; SCHED;
    LDB(B1, 1, 1); STAGE(SB(1, 0), b3);
    BAR; WAIT_L(0); MMA(0, 1, At, B1); BAR;
    LDA(At, 1, 1); STAGE(SA(1, 0), a3);
    BAR; WAIT_L(0); MMA(1, 0, At, B0); BAR; SCHED;
    const u32x4 f = *(const LAS u32x4*)fslot;
    STAGE(SB(1, 1), b3 + HTB);
    WAIT_V(6); BAR; MMA(1, 1, At, B1); BAR;
    if (chk) {
      const unsigned fm = rfl(min(min(f.x, f.y), min(f.z, f.w)));
      if (fm < need) { wait_tiles(t + 4); BARRIER(); }
    }
  }
  if (wr == 0) BAR;
  STAMP(p, 2 + 4 * ph);
  const int tile = pm * 8 + pn;
  {
    int tz = threadIdx.x; asm volatile("" : "+v"(tz));
    const int ln = tz & 63;
    const rsrc_t ro = mkr(p.out[p.rank], OUT_BYTES), lw = mkr(ws, WS_END);
    {
      const unsigned po = WS_PART + (unsigned)(tile * 2 + h) * 131072u + (unsigned)(wid * 16384 + ln * 16);
      const unsigned pi = WS_PART + (unsigned)(tile * 2 + 1 - h) * 131072u + (unsigned)(wid * 16384 + ln * 16);
#pragma unroll
      for (int i = 0; i < 16; ++i) {
        const f32x4 q = h ? acc[0][i >> 3][(i >> 1) & 3][i & 1] : acc[1][i >> 3][(i >> 1) & 3][i & 1];
        st128s(__builtin_bit_cast(u32x4, q), lw, po + i * 1024);
      }
      VM0();
      gu32* fo = (gu32*)(ws + WS_PFLG) + ((tile * 2 + h) * 8 + wid) * 32;
      const gu32* fi = (const gu32*)(ws + WS_PFLG) + ((tile * 2 + 1 - h) * 8 + wid) * 32;
      if (!dead && !cx.timed_out() && ln == 0) __hip_atomic_store(fo, cx.call, __ATOMIC_RELAXED, __HIP_MEMORY_SCOPE_AGENT);
      if (rfl(__hip_atomic_load(fi, __ATOMIC_RELAXED, __HIP_MEMORY_SCOPE_AGENT)) < cx.call) {
        const unsigned long long t0 = rt();
        while (rfl(__hip_atomic_load(fi, __ATOMIC_RELAXED, __HIP_MEMORY_SCOPE_AGENT)) < cx.call) { if (cx.give_up(t0)) break; }
      }
      asm volatile("" ::: "memory");
#pragma unroll
      for (int i = 0; i < 16; ++i) {
        const f32x4 q = __builtin_bit_cast(f32x4, ld128s(lw, pi + i * 1024));
        if (h) acc[1][i >> 3][(i >> 1) & 3][i & 1] += q; else acc[0][i >> 3][(i >> 1) & 3][i & 1] += q;
      }
    }
    const bool bad = cx.timed_out();
    unsigned long long hp[2];
#pragma unroll
    for (int qq = 0; qq < 2; ++qq) {
      unsigned v[16];
#pragma unroll
      for (int mm = 0; mm < 2; ++mm) {
        const int m = qq * 2 + mm;
        unsigned d[8];
#pragma unroll
        for (int c = 0; c < 8; ++c) {
          const float y0 = h ? acc[1][c & 1][m][0][c >> 1] : acc[0][c & 1][m][0][c >> 1], y1 = h ? acc[1][c & 1][m][1][c >> 1] : acc[0][c & 1][m][1][c >> 1];
          d[c] = cvt_pk_bf16(gelu1(y0), gelu1(y1)); if (bad) d[c] = 0x7FC07FC0u; }
#pragma unroll
        for (int c = 0; c < 4; ++c) {
          auto r = __builtin_amdgcn_permlane16_swap(d[c], d[c + 4], false, false); d[c] = r[0]; d[c + 4] = r[1];
          auto s2 = __builtin_amdgcn_permlane32_swap(d[c], d[c + 4], false, false); d[c] = s2[0]; d[c + 4] = s2[1];
        }
#pragma unroll
        for (int c = 0; c < 8; ++c) v[8 * mm + c] = d[c];
      }
      const int task = tile * 4 + h * 2 + qq;
      const MapO map{out_task_base(p.rank, task)};
#pragma unroll
      for (int j = 0; j < 4; ++j) st128s((u32x4){v[4 * j], v[4 * j + 1], v[4 * j + 2], v[4 * j + 3]}, ro, map(wid, j, ln));
      u32x4 P0, P1; u32x2 P2, F;
      LAS unsigned* S = (LAS unsigned*)(lds + LDS_SCR + wid * SCR_BYTES);
      hp[qq] = encode_chunk<true>(v, S, ln, P0, P1, P2, F);
      if (!((unsigned)hp[qq] & 0x4000u)) store_chunk<true>(lw, WS_OBO + (unsigned)task * PKT + (unsigned)wid * 3072u, S, ((unsigned)hp[qq] >> 8) & 63u, ln, P0, P1, P2, F);
    }
    DRAIN_ALL();
    if (!bad && ln < 2)
      __hip_atomic_store((gu64*)(ws + WS_HOBO + (unsigned)(tile * 4 + h * 2 + ln) * 64u) + wid, mk_granule(ln ? hp[1] : hp[0], cx.call), __ATOMIC_RELAXED, __HIP_MEMORY_SCOPE_SYSTEM);
  }
  STAMP(p, 3 + 4 * ph);
#undef KT
#undef SA
#undef SB
#undef STAGE
#undef LDA
#undef LDB
#undef MMA
#undef WAIT_V
#undef WAIT_L
#undef BAR
#undef SCHED
}

__global__ __launch_bounds__(NTHR, 2) void fused_kernel(Params p) {
  extern __shared__ __attribute__((aligned(16))) unsigned char shm[];
  LAS unsigned char* lds = (LAS unsigned char*)shm;
  const int b = blockIdx.x, xcd = b & 7, slot = b >> 3;
  char* ws = p.ws[p.rank];
  if (threadIdx.x == 0) {
    const unsigned c = __hip_atomic_fetch_add((gu32*)(ws + WS_CALL) + b * 32, 1u, __ATOMIC_RELAXED, __HIP_MEMORY_SCOPE_AGENT) + 1u;
    *(LAS unsigned*)(lds + LDS_CALL) = c;
    *(LAS unsigned*)(lds + LDS_DEAD) = 0u;
  }
  __syncthreads();
  Ctx cx; cx.tmo = (gu32*)(ws + WS_TMO); cx.call = rfl(*(LAS unsigned*)(lds + LDS_CALL));
  STAMP(p, 0);
  LAS unsigned* S = (LAS unsigned*)(lds + LDS_SCR + (int)rfl(threadIdx.x >> 6) * SCR_BYTES);
  if (b < N_P) {
    { const int gw = b * 8 + (int)rfl(threadIdx.x >> 6); role_push(p, cx, gw % 7, gw / 7, PJ * 8); }
  } else if (slot < 16) {
    const int dw = b - N_P;
    role_din(p, cx, S, dw);
    STAMP(p, 1);
    role_dout(p, cx, S, dw, N_D + 128);
    STAMP(p, 2);
  } else {
    const int g = slot - 16, h = g & 1, s = g >> 1;
    const int pm = (xcd >> 1) * 2 + (s & 1), pn = (xcd & 1) * 4 + (s >> 1);
    role_encode(p, cx, S, b - 128, 128);
    STAMP(p, 9);
    for (int ph = 0; ph < 2; ++ph) gemm_unit(p, cx, lds, pm + 8 * ph, pn, h, ph);
    role_dout(p, cx, S, N_D + (b - 128), N_D + 128);
    STAMP(p, 10);
  }
}

}

extern "C" void kernel_launch(int rank, int nranks, ncclComm_t comm, hipStream_t stream,
                              void* const* d_in, const int* in_sizes, int n_in,
                              void* d_out, int out_size,
                              const int* in_dtypes, int out_dtype,
                              void* const* d_out_all, void* const* d_ws_all, size_t ws_bytes) {
  using namespace fk;
  (void)comm; (void)in_dtypes; (void)out_dtype;
  if (nranks != NRANK || n_in < 2 || rank < 0 || rank >= NRANK || ws_bytes < (size_t)WS_END ||
      in_sizes[0] != MD * KS || in_sizes[1] != KS * ND || out_size != MD * ND) {
    fprintf(stderr, "kernel_launch: unexpected problem shape\n");
    return;
  }
  static bool attr_set[64] = {};
  int dev = 0;
  if (hipGetDevice(&dev) != hipSuccess) return;
  if (dev >= 0 && dev < 64 && !attr_set[dev]) {
    hipError_t e = hipFuncSetAttribute((const void*)fused_kernel, hipFuncAttributeMaxDynamicSharedMemorySize, LDS_BYTES);
    if (e != hipSuccess) { fprintf(stderr, "HIP error %s\n", hipGetErrorString(e)); return; }
    attr_set[dev] = true;
  }
  Params p;
  p.x = (const char*)d_in[0]; p.w = (const char*)d_in[1];
  for (int i = 0; i < NRANK; ++i) { p.ws[i] = (char*)d_ws_all[i]; p.out[i] = (char*)d_out_all[i]; }
  p.out[rank] = (char*)d_out;
  p.rank = rank; p.pad = 0;
  hipLaunchKernelGGL(fused_kernel, dim3(GRID), dim3(NTHR), LDS_BYTES, stream, p);
  hipError_t e = hipGetLastError();
  if (e != hipSuccess) fprintf(stderr, "HIP launch error %s\n", hipGetErrorString(e));
}
```

```cpp
#include <hip/hip_runtime.h>
#include <rccl/rccl.h>
#include <cstdio>

namespace fk {

#define LAS __attribute__((address_space(3)))
#define GAS __attribute__((address_space(1)))
typedef unsigned short bf16_t;
typedef short bf16x8 __attribute__((ext_vector_type(8)));
typedef float f32x4 __attribute__((ext_vector_type(4)));
typedef unsigned u32x4 __attribute__((ext_vector_type(4)));
typedef unsigned u32x2 __attribute__((ext_vector_type(2)));
typedef unsigned short u16x2 __attribute__((ext_vector_type(2)));
typedef GAS unsigned gu32;
typedef GAS unsigned long long gu64;
typedef __amdgpu_buffer_rsrc_t rsrc_t;

constexpr int NRANK = 8;
constexpr int MD = 8192, ND = 8192, KD = 8192, KS = KD / NRANK;
constexpr int NT = KD / 64;
constexpr unsigned OUT_BYTES = (unsigned)MD * ND * 2u;
constexpr unsigned IN_BYTES = (unsigned)MD * KS * 2u;

constexpr int GRID = 256, NTHR = 512;
#define CFG_SLEEP 2
#define CFG_PJ 5
#define CFG_AUX_DIN 17
#define CFG_AUX_DOUT 17
#define CFG_PMAP 1
constexpr int PJ = CFG_PJ;
#define CFG_NEO 28
constexpr int N_EO = CFG_NEO;
constexpr int N_P = 7 * PJ, N_D = 128 - N_P - N_EO;
constexpr int AUX_SYS = 17;

constexpr int NIMG = 3072;
constexpr unsigned IMG = 32768, PKT = 24576;
constexpr int NIT = 1024;
constexpr int NOT = 512;
constexpr int NPI = 384 + NOT;
constexpr int HCAP = 13;

constexpr unsigned WS_CALL = 0;
constexpr unsigned WS_TMO  = WS_CALL + 256 * 128;
constexpr unsigned WS_TK   = WS_TMO + 1024;
constexpr unsigned WS_CNT  = WS_TMO + 4096;
constexpr unsigned WS_PCNT = WS_CNT + NIMG * 128;
constexpr unsigned WS_HIN  = WS_PCNT + 32768;
constexpr unsigned WS_HOUT = WS_HIN + NIMG * 128;
constexpr unsigned WS_HOBI = WS_HOUT + NRANK * NOT * 128;
constexpr unsigned WS_HOBO = WS_HOBI + NIT * 128;
constexpr unsigned WS_PFLG = WS_HOBO + NOT * 128;
constexpr unsigned WS_RFLG = WS_PFLG + 2048 * 128;
constexpr unsigned WS_DBG  = WS_RFLG + NOT * 8 * 16;
constexpr unsigned WS_PIN  = 4u << 20;
constexpr unsigned WS_POUT = WS_PIN + NIMG * PKT;
constexpr unsigned WS_OBI  = WS_POUT + NRANK * NOT * PKT;
constexpr unsigned WS_OBR  = WS_OBI + NIT * PKT;
constexpr unsigned WS_OBO  = WS_OBR + NIT * IMG;
constexpr unsigned WS_PART = WS_OBO + NOT * PKT;
constexpr unsigned WS_IMG  = WS_PART + 256 * 131072;
constexpr unsigned WS_END  = WS_IMG + NIMG * IMG;
static_assert(WS_DBG + 256 * 64 * 8 <= WS_PIN, "control region overflow");
static_assert((size_t)WS_IMG + (size_t)NIMG * IMG < (1ull << 31), "workspace offsets must fit 31 bits");

constexpr unsigned long long DEADLINE = 40000000ull;

constexpr int HTB = 16384, STAGE_BYTES = 8 * HTB;
constexpr int LDS_FLAG = STAGE_BYTES;
constexpr int LDS_CALL = STAGE_BYTES + 64;
constexpr int LDS_TK = STAGE_BYTES + 96;
constexpr int LDS_DEAD = STAGE_BYTES + 80;
constexpr int LDS_GHDR = STAGE_BYTES + 128;
constexpr int LDS_HDR = 0;
constexpr int LDS_SCR = STAGE_BYTES + 512;
constexpr int LDS_BYTES = STAGE_BYTES + 512 + 8 * 1664;

struct Params {
  const char* x; const char* w;
  char* ws[NRANK]; char* out[NRANK];
  int rank; int pad;
};

__device__ __forceinline__ rsrc_t mkr(const void* p, unsigned bytes) { return __builtin_amdgcn_make_buffer_rsrc((void*)p, 0, (int)bytes, 0x00020000); }
__device__ __forceinline__ u32x4 ld128s(rsrc_t r, unsigned off) { return __builtin_amdgcn_raw_buffer_load_b128(r, (int)off, 0, AUX_SYS); }
__device__ __forceinline__ void st128s(u32x4 v, rsrc_t r, unsigned off) { __builtin_amdgcn_raw_buffer_store_b128(v, r, (int)off, 0, AUX_SYS); }
__device__ __forceinline__ unsigned long long rt() { return __builtin_amdgcn_s_memrealtime(); }
__device__ __forceinline__ unsigned rfl(unsigned v) { return (unsigned)__builtin_amdgcn_readfirstlane((int)v); }
__device__ __forceinline__ unsigned rdl(unsigned v, int l) { return (unsigned)__builtin_amdgcn_readlane((int)v, l); }
#define VM0() asm volatile("s_waitcnt vmcnt(0)" ::: "memory")
#define DRAIN_ALL() do { asm volatile("s_waitcnt vmcnt(0) lgkmcnt(0)" ::: "memory"); __builtin_amdgcn_s_waitcnt(0x0070); } while (0)
#define DRAIN_PIN2(a, b) do { asm volatile("s_waitcnt vmcnt(0) lgkmcnt(0)" : "+v"(a), "+v"(b) :: "memory"); __builtin_amdgcn_s_waitcnt(0x0070); } while (0)
#define BARRIER() do { asm volatile("" ::: "memory"); __builtin_amdgcn_s_barrier(); asm volatile("" ::: "memory"); } while (0)

#define STAMP(p, idx) do { } while (0)

__device__ __forceinline__ unsigned atomic_add_lane0_nowait(gu32* ptr) {
  unsigned r; unsigned long long sv; const unsigned one = 1u;
  asm volatile("s_mov_b64 %1, exec\n\ts_mov_b64 exec, 1\n\tglobal_atomic_add %0, %2, %3, off sc0\n\ts_mov_b64 exec, %1"
               : "=&v"(r), "=&s"(sv) : "v"(ptr), "v"(one) : "memory");
  return r;
}

struct Ctx {
  gu32* tmo; unsigned call;
  __device__ __forceinline__ bool timed_out() const { return rfl(__hip_atomic_load(tmo, __ATOMIC_RELAXED, __HIP_MEMORY_SCOPE_AGENT)) == call; }
  __device__ __forceinline__ bool give_up(unsigned long long t0) const {
    if (timed_out() || rt() - t0 > DEADLINE) { __hip_atomic_store(tmo, call, __ATOMIC_RELAXED, __HIP_MEMORY_SCOPE_AGENT); return true; }
    __builtin_amdgcn_s_sleep(CFG_SLEEP);
    return false;
  }
};

__device__ __forceinline__ unsigned dpp_shr_add(unsigned x) {
  x += (unsigned)__builtin_amdgcn_update_dpp(0, (int)x, 0x111, 0xf, 0xf, false);
  x += (unsigned)__builtin_amdgcn_update_dpp(0, (int)x, 0x112, 0xf, 0xf, false);
  x += (unsigned)__builtin_amdgcn_update_dpp(0, (int)x, 0x114, 0xf, 0xf, false);
  x += (unsigned)__builtin_amdgcn_update_dpp(0, (int)x, 0x118, 0xf, 0xf, false);
  x += (unsigned)__builtin_amdgcn_update_dpp(0, (int)x, 0x142, 0xa, 0xf, false);
  x += (unsigned)__builtin_amdgcn_update_dpp(0, (int)x, 0x143, 0xc, 0xf, false);
  return x;
}
__device__ __forceinline__ unsigned row_max(unsigned x) {
  x = max(x, (unsigned)__builtin_amdgcn_update_dpp(0, (int)x, 0x111, 0xf, 0xf, false));
  x = max(x, (unsigned)__builtin_amdgcn_update_dpp(0, (int)x, 0x112, 0xf, 0xf, false));
  x = max(x, (unsigned)__builtin_amdgcn_update_dpp(0, (int)x, 0x114, 0xf, 0xf, false));
  x = max(x, (unsigned)__builtin_amdgcn_update_dpp(0, (int)x, 0x118, 0xf, 0xf, false));
  return x;
}
__device__ __forceinline__ unsigned wave_max_u32(unsigned x) {
  x = row_max(x);
  x = max(x, (unsigned)__builtin_amdgcn_update_dpp(0, (int)x, 0x142, 0xa, 0xf, false));
  x = max(x, (unsigned)__builtin_amdgcn_update_dpp(0, (int)x, 0x143, 0xc, 0xf, false));
  return rdl(x, 63);
}
__device__ __forceinline__ unsigned vperm(unsigned a, unsigned b, unsigned sel) { return __builtin_amdgcn_perm(a, b, sel); }
__device__ __forceinline__ unsigned lut4(unsigned tab, unsigned sel) { return __builtin_amdgcn_perm(0u, tab, sel); }
__device__ __forceinline__ unsigned bmask(unsigned e) { return __builtin_amdgcn_perm(0u, 0u, e | 0x0C0C0C0Cu); }
__device__ __forceinline__ unsigned bfi(unsigned m, unsigned a, unsigned b) { return (m & a) | (~m & b); }
__device__ __forceinline__ unsigned mul24(unsigned a, unsigned k) { unsigned r; asm("v_mul_u32_u24 %0, %1, %2" : "=v"(r) : "s"(k), "v"(a)); return r; }
__device__ __forceinline__ unsigned rank4(unsigned e) { return mul24(e, 0x010101u) << 8; }
__device__ __forceinline__ unsigned pc(unsigned e) { return (unsigned)__builtin_popcount(e); }
__device__ __forceinline__ unsigned nz4(unsigned x) { return ((x + 0x0F0F0F0Fu) >> 4) & 0x01010101u; }

constexpr int SCR_BYTES = 1664;

template <int W> __device__ __forceinline__ unsigned pack_bits(unsigned c, unsigned e) {
  const unsigned pre = rank4(e), em = bmask(e);
  if (W == 1) return __builtin_amdgcn_udot4(c, lut4(0x08040201u, pre) & em, 0u, false);
  if (W == 2) return __builtin_amdgcn_udot4(c, lut4(0x40100401u, pre) & em, 0u, false);
  return __builtin_amdgcn_udot4(c, lut4(0x00001001u, pre) & em, 0u, false) | (__builtin_amdgcn_udot4(c, lut4(0x10010000u, pre) & em, 0u, false) << 8);
}
template <int W> __device__ __forceinline__ unsigned unpack_bits(unsigned w, unsigned e) {
  unsigned s;
  if (W == 1) s = mul24(w & 0xFu, 0x00204081u) & 0x01010101u;
  else if (W == 2) { w &= 0xFFu; const unsigned t = (w | (w << 12)) & 0x000F000Fu; s = (t | (t << 6)) & 0x03030303u; }
  else if (W == 4) { w &= 0xFFFFu; const unsigned t = (w | (w << 8)) & 0x00FF00FFu; s = (t | (t << 4)) & 0x0F0F0F0Fu; }
  else s = w;
  return lut4(s, rank4(e)) & bmask(e);
}
__device__ __forceinline__ void lds_or(LAS unsigned* q, unsigned x) { if (x != 0u) __hip_atomic_fetch_or(q, x, __ATOMIC_RELAXED, __HIP_MEMORY_SCOPE_WAVEFRONT); }
__device__ __forceinline__ void flush32(LAS unsigned* S, unsigned off, unsigned acc) {
  const unsigned long long x = (unsigned long long)acc << (off & 31u);
  LAS unsigned* q = S + (off >> 5);
  lds_or(q, (unsigned)x); lds_or(q + 1, (unsigned)(x >> 32));
}
__device__ __forceinline__ void flush64(LAS unsigned* S, unsigned off, unsigned long long acc) {
  const unsigned sh = off & 31u;
  const unsigned long long x = acc << sh;
  LAS unsigned* q = S + (off >> 5);
  lds_or(q, (unsigned)x); lds_or(q + 1, (unsigned)(x >> 32)); lds_or(q + 2, ((unsigned)(acc >> 32) >> 1) >> (31u - sh));
}
__device__ __forceinline__ unsigned fetch32(const LAS unsigned* S, unsigned off) {
  const LAS unsigned* q = S + (off >> 5);
  return __builtin_amdgcn_alignbit(q[1], q[0], off & 31u);
}
__device__ __forceinline__ unsigned long long fetch64(const LAS unsigned* S, unsigned off) {
  const LAS unsigned* q = S + (off >> 5);
  const unsigned a = q[0], b = q[1], c = q[2];
  return ((unsigned long long)__builtin_amdgcn_alignbit(c, b, off & 31u) << 32) | __builtin_amdgcn_alignbit(b, a, off & 31u);
}
__device__ __forceinline__ unsigned wave_sum_lane63(unsigned x) { return dpp_shr_add(x); }
__device__ __forceinline__ unsigned sym_of(bool out, unsigned base, unsigned t) { return out ? ((base - (t & 7u)) | ((t & 8u) << 4)) & 0xFFu : (base - t) & 0xFFu; }

template <bool OUT>
__device__ __forceinline__ unsigned long long encode_chunk(const unsigned (&v)[16], LAS unsigned* S, int lane, u32x4& P0, u32x4& P1, u32x2& F) {
  u16x2 mx = __builtin_bit_cast(u16x2, v[0] & 0x7FFF7FFFu);
#pragma unroll
  for (int i = 1; i < 16; ++i) mx = __builtin_elementwise_max(mx, __builtin_bit_cast(u16x2, v[i] & 0x7FFF7FFFu));
  const unsigned base = wave_max_u32(max((unsigned)mx.x, (unsigned)mx.y)) >> (OUT ? 8 : 7);
  const unsigned B4 = base * 0x01010101u;
  *(LAS u32x4*)(S + lane * 4) = (u32x4){0u, 0u, 0u, 0u};
  if (lane < (OUT ? 34 : 1)) *(LAS u32x4*)(S + 256 + lane * 4) = (u32x4){0u, 0u, 0u, 0u};
  unsigned H[8], L[8], R[8], oor[8], anyo = 0;
#pragma unroll
  for (int g = 0; g < 8; ++g) {
    const unsigned Hs = vperm(v[2 * g + 1], v[2 * g], 0x07050301u), Ls = vperm(v[2 * g + 1], v[2 * g], 0x06040200u);
    if (OUT) { H[g] = Hs; L[g] = Ls; const unsigned D = B4 - (Hs & 0x7F7F7F7Fu); R[g] = (D & 0x07070707u) | ((Hs >> 4) & 0x08080808u); oor[g] = D & 0x78787878u; }
    else { H[g] = bfi(0x01010101u, Ls >> 7, Hs << 1); L[g] = bfi(0x80808080u, Hs, Ls); const unsigned D = B4 - H[g]; R[g] = D & 0x0F0F0F0Fu; oor[g] = D & 0xF0F0F0F0u; }
    anyo |= oor[g];
  }
  unsigned key;
  {
    unsigned long long hc = 0;
#pragma unroll
    for (int b = 0; b < 8; ++b) hc += 1ull << (((R[b >> 2] >> (8 * (b & 3))) & 15u) * 4u);
    const unsigned lo = (unsigned)hc, hi = (unsigned)(hc >> 32);
    unsigned q[8];
#pragma unroll
    for (int k = 0; k < 8; ++k) q[k] = rdl(dpp_shr_add(((lo >> (4 * k)) & 15u) | (((hi >> (4 * k)) & 15u) << 16)), 63);
    const int k7 = lane & 7;
    const unsigned qa = (k7 & 1) ? ((k7 & 2) ? q[3] : q[1]) : ((k7 & 2) ? q[2] : q[0]), qb = (k7 & 1) ? ((k7 & 2) ? q[7] : q[5]) : ((k7 & 2) ? q[6] : q[4]);
    const unsigned cnt = (((k7 & 4) ? qb : qa) >> ((lane & 8) * 2)) & 0xFFFFu;
    key = (lane < 16) ? (((cnt + 1u) << 4) | (unsigned)lane) : 0u;
  }
  unsigned tt = 0, cl = 0x1Fu;
#pragma unroll
  for (int i = 0; i < 7; ++i) {
    const unsigned bias = (OUT && i == 0 && lane >= 8 && lane < 16) ? (32u << 4) : 0u;
    const unsigned m = rdl(row_max(key + bias), 15);
    tt |= (m & 15u) << (4 * i);
    if (key + bias == m) { key = 0u; cl = (i < 3) ? (unsigned)i : (i == 3) ? 3u : (7u | ((unsigned)(i - 4) << 3)); }
  }
  cl <<= 8 * (lane & 3);
  cl |= (unsigned)__builtin_amdgcn_update_dpp(0, (int)cl, 0xB1, 0xf, 0xf, false);
  cl |= (unsigned)__builtin_amdgcn_update_dpp(0, (int)cl, 0x4E, 0xf, 0xf, false);
  const unsigned tb0 = rdl(cl, 0), tb1 = rdl(cl, 4), tb2 = rdl(cl, 8), tb3 = rdl(cl, 12);
  unsigned cls[8];
#pragma unroll
  for (int g = 0; g < 8; ++g) {
    const unsigned r7 = R[g] & 0x07070707u;
    cls[g] = bfi(bmask((R[g] >> 3) & 0x01010101u), vperm(tb3, tb2, r7), vperm(tb1, tb0, r7));
  }
  const bool has_oor = __any(anyo != 0u);
  unsigned e4[8];
#pragma unroll
  for (int g = 0; g < 8; ++g) e4[g] = 0u;
  if (has_oor) {
    const unsigned T1 = ((tt >> 4) & 15u) * 0x01010101u;
#pragma unroll
    for (int g = 0; g < 8; ++g) {
      e4[g] = nz4(OUT ? (oor[g] >> 3) : (oor[g] >> 4));
      const unsigned m = bmask(e4[g]);
      cls[g] |= m & 0x1F1F1F1Fu; R[g] = bfi(m, T1, R[g]);
    }
  }
  bool modeB = false; unsigned n0 = 0;
  if (OUT) {
    unsigned z[8], mxn = 0;
#pragma unroll
    for (int g = 0; g < 8; ++g) z[g] = lut4(0x000000FFu, cls[g] & 0x03030303u);
#pragma unroll
    for (int g = 0; g < 2; ++g) {
      const unsigned hn = (L[g] >> 4) & z[g] & 0x0F0F0F0Fu;
      mxn = max(mxn, max(max(hn & 0xFFu, (hn >> 8) & 0xFFu), max((hn >> 16) & 0xFFu, hn >> 24)));
    }
    n0 = wave_max_u32(mxn);
    unsigned viol[8], nv = 0;
    const unsigned N4 = (n0 | 0x80u) * 0x01010101u;
#pragma unroll
    for (int g = 0; g < 8; ++g) {
      const unsigned dn = (N4 - ((L[g] >> 4) & 0x0F0F0F0Fu)) & 0x7F7F7F7Fu;
      viol[g] = z[g] & bmask(((dn + 0x7D7D7D7Du) >> 7) & 0x01010101u);
      nv += pc(viol[g] & 0x01010101u);
    }
    modeB = n0 < 8u && rdl(dpp_shr_add(nv), 63) < 24u;
    if (modeB) {
#pragma unroll
      for (int g = 0; g < 8; ++g) cls[g] |= viol[g] & 0x1F1F1F1Fu;
    }
  }
  unsigned e2[8], e2b[8], e3[8], e0[8], m2[8], n2 = 0, n2b = 0, n3 = 0, n4 = 0, c0 = 0, cm2 = 0;
  const unsigned N0 = n0 * 0x01010101u;
#pragma unroll
  for (int g = 0; g < 8; ++g) {
    const unsigned c1 = cls[g] & 0x03030303u;
    e2[g] = lut4(0x01000000u, c1); e2b[g] = (cls[g] >> 2) & 0x01010101u; e3[g] = lut4(0x01000000u, (cls[g] >> 3) & 0x03030303u);
    n2 += pc(e2[g]); n2b += pc(e2b[g]); n3 += pc(e3[g]); n4 += pc(e4[g]);
    if (OUT) {
      e0[g] = modeB ? lut4(0x00000001u, c1) : 0u;
      m2[g] = e0[g] & nz4(((L[g] >> 4) & 0x0F0F0F0Fu) ^ N0);
      c0 += pc(e0[g]); cm2 += pc(m2[g]);
    } else { e0[g] = 0u; m2[g] = 0u; }
  }
  const unsigned x1 = c0 | (n2 << 16), x2 = cm2 | (n2b << 16), x3 = n3 | (n4 << 16);
  const unsigned s1 = dpp_shr_add(x1), s2 = dpp_shr_add(x2), s3 = dpp_shr_add(x3);
  const unsigned t1 = rdl(s1, 63), t2 = rdl(s2, 63), t3 = rdl(s3, 63);
  const unsigned C0 = t1 & 0xFFFFu, N2 = t1 >> 16, CM2 = t2 & 0xFFFFu, N2B = t2 >> 16, N3 = t3 & 0xFFFFu, N4c = t3 >> 16;
  const unsigned bSN = OUT ? 4u * (2048u - C0) : 0u;
  const unsigned aM1 = bSN, a2a = aM1 + C0, aM2 = a2a + N2, a2b = aM2 + CM2, a3 = a2b + 2u * N2B, a4 = a3 + 4u * N3, bits = a4 + 8u * N4c;
  const bool raw = bits > (OUT ? 12288u : 4096u);
  const unsigned len = ((OUT ? 12288u : 20480u) + bits + 255u) >> 8;
  if (!raw) {
    const unsigned p1 = s1 - x1, p2 = s2 - x2, p3 = s3 - x3;
    const unsigned oN = 4u * (32u * (unsigned)lane - (p1 & 0xFFFFu)), oM1 = aM1 + (p1 & 0xFFFFu), o2a = a2a + (p1 >> 16), oM2 = aM2 + (p2 & 0xFFFFu),
                   o2b = a2b + 2u * (p2 >> 16), o3 = a3 + 4u * (p3 & 0xFFFFu);
    unsigned o4 = a4 + 8u * (p3 >> 16);
    unsigned wM1 = 0, wM2 = 0, w2a = 0, kM1 = 0, kM2 = 0, k2a = 0, k2b = 0, kN = 0, k3 = 0, kNa = 0, k3a = 0;
    unsigned long long w2b = 0, wN = 0, w3 = 0;
#pragma unroll
    for (int g = 0; g < 8; ++g) {
      if (g == 4) {
        if (OUT) { flush64(S, oN, wN); wN = 0; kNa = kN; kN = 0; }
        flush64(S, o3, w3); w3 = 0; k3a = k3; k3 = 0;
      }
      if (OUT) {
        const unsigned hn = (L[g] >> 4) & 0x0F0F0F0Fu, eN = e0[g] ^ 0x01010101u;
        wN |= (unsigned long long)pack_bits<4>(hn, eN) << kN; kN += 4u * pc(eN);
        wM1 |= pack_bits<1>(m2[g], e0[g]) << kM1; kM1 += pc(e0[g]);
        wM2 |= pack_bits<1>(nz4(((N0 | 0x10101010u) - 0x01010101u - hn) & 0x0F0F0F0Fu), m2[g]) << kM2; kM2 += pc(m2[g]);
      }
      w2a |= pack_bits<1>(e2b[g], e2[g]) << k2a; k2a += pc(e2[g]);
      w2b |= (unsigned long long)pack_bits<2>((cls[g] >> 3) & 0x03030303u, e2b[g]) << k2b; k2b += 2u * pc(e2b[g]);
      w3 |= (unsigned long long)pack_bits<4>(R[g], e3[g]) << k3; k3 += 4u * pc(e3[g]);
    }
    if (OUT) { flush64(S, oN + kNa, wN); flush32(S, oM1, wM1); flush32(S, oM2, wM2); }
    flush32(S, o2a, w2a); flush64(S, o2b, w2b); flush64(S, o3 + k3a, w3);
    if (has_oor) {
#pragma unroll
      for (int g = 0; g < 8; ++g)
#pragma unroll
        for (int b = 0; b < 4; ++b)
          if ((e4[g] >> (8 * b)) & 1u) {
            const unsigned long long x = (unsigned long long)((H[g] >> (8 * b)) & 0xFFu) << (o4 & 31u);
            __hip_atomic_fetch_or(S + (o4 >> 5), (unsigned)x, __ATOMIC_RELAXED, __HIP_MEMORY_SCOPE_WAVEFRONT);
            __hip_atomic_fetch_or(S + (o4 >> 5) + 1, (unsigned)(x >> 32), __ATOMIC_RELAXED, __HIP_MEMORY_SCOPE_WAVEFRONT);
            o4 += 8u;
          }
    }
  }
  unsigned Fw[2];
#pragma unroll
  for (int i = 0; i < 2; ++i)
    Fw[i] = (cls[4 * i] & 0x03030303u) | ((cls[4 * i + 1] & 0x03030303u) << 2) | ((cls[4 * i + 2] & 0x03030303u) << 4) | ((cls[4 * i + 3] & 0x03030303u) << 6);
  F = (u32x2){Fw[0], Fw[1]};
  if (OUT) {
    unsigned p4[4];
#pragma unroll
    for (int i = 0; i < 4; ++i) p4[i] = (L[2 * i] & 0x0F0F0F0Fu) | ((L[2 * i + 1] & 0x0F0F0F0Fu) << 4);
    P0 = (u32x4){p4[0], p4[1], p4[2], p4[3]}; P1 = P0;
  } else {
    P0 = (u32x4){L[0], L[1], L[2], L[3]}; P1 = (u32x4){L[4], L[5], L[6], L[7]};
  }
  return (unsigned long long)(base | (raw ? 0x8000u : (len << 8))) | ((unsigned long long)tt << 16) | (modeB ? (1ull << 44) | ((unsigned long long)n0 << 45) : 0ull);
}
__device__ __forceinline__ bool hp_raw(unsigned long long hp) { return ((unsigned)hp & 0x8000u) != 0u; }
__device__ __forceinline__ unsigned hp_len(unsigned long long hp) { return (((unsigned)hp >> 8) & 127u) * 32u; }

template <bool OUT>
__device__ __forceinline__ void store_chunk(rsrc_t r, unsigned o, const LAS unsigned* S, unsigned len, int lane, u32x4 P0, u32x4 P1, u32x2 F) {
  const unsigned l16 = (unsigned)lane * 16u;
  st128s(P0, r, o + l16);
  const u32x4 sv = *(const LAS u32x4*)(S + lane * 4);
  if (OUT) {
    const u32x4 sw = *(const LAS u32x4*)(S + 256 + (lane & 31) * 4);
    __builtin_amdgcn_raw_buffer_store_b64(F, r, (int)(o + 1024 + lane * 8), 0, AUX_SYS);
    if (1536u + l16 < len) st128s(sv, r, o + 1536 + l16);
    if (lane < 32 && 2560u + l16 < len) st128s(sw, r, o + 2560 + l16);
  } else {
    st128s(P1, r, o + 1024 + l16);
    __builtin_amdgcn_raw_buffer_store_b64(F, r, (int)(o + 2048 + lane * 8), 0, AUX_SYS);
    if (lane < 32 && 2560u + l16 < len) st128s(sv, r, o + 2560 + l16);
  }
}

struct Rec { u32x4 P0, P1, sv; u32x2 F; };
template <bool OUT>
__device__ __forceinline__ Rec load_rec(rsrc_t r, unsigned o, unsigned long long hp, int lane) {
  const unsigned len = hp_len(hp), l16 = (unsigned)lane * 16u;
  Rec c;
  c.P0 = ld128s(r, o + l16);
  if (OUT) {
    c.F = __builtin_amdgcn_raw_buffer_load_b64(r, (int)(o + 1024 + lane * 8), 0, AUX_SYS);
    c.sv = ld128s(r, (1536u + l16 < len) ? o + 1536 + l16 : 0xFFFFFF00u);
    c.P1 = ld128s(r, (lane < 32 && 2560u + l16 < len) ? o + 2560 + l16 : 0xFFFFFF00u);
  } else {
    c.P1 = ld128s(r, o + 1024 + l16);
    c.F = __builtin_amdgcn_raw_buffer_load_b64(r, (int)(o + 2048 + lane * 8), 0, AUX_SYS);
    c.sv = ld128s(r, (lane < 32 && 2560u + l16 < len) ? o + 2560 + l16 : 0xFFFFFF00u);
  }
  return c;
}
template <bool OUT>
__device__ __forceinline__ void decode_rec(const Rec& rc, unsigned long long hp, LAS unsigned* S, int lane, unsigned (&v)[16]) {
  const unsigned base = (unsigned)hp & 0xFFu, tt = (unsigned)(hp >> 16) & 0xFFFFFFFu;
  const bool modeB = ((hp >> 44) & 1u) != 0u;
  const unsigned n0 = (unsigned)(hp >> 45) & 7u;
  const u32x4 P0 = rc.P0, P1 = rc.P1, sv = rc.sv, sw = rc.P1; const u32x2 F = rc.F;
  unsigned tab1 = 0, tab2 = 0;
#pragma unroll
  for (int i = 0; i < 4; ++i) tab1 |= sym_of(OUT, base, (tt >> (4 * i)) & 15u) << (8 * i);
#pragma unroll
  for (int i = 0; i < 3; ++i) tab2 |= sym_of(OUT, base, (tt >> (16 + 4 * i)) & 15u) << (8 * i);
  const unsigned B4 = base * 0x01010101u, T1 = ((tt >> 4) & 15u) * 0x01010101u, N0 = n0 * 0x01010101u;
  *(LAS u32x4*)(S + lane * 4) = sv;
  if (OUT && lane < 32) *(LAS u32x4*)(S + 256 + lane * 4) = sw;
  unsigned c1[8], e2[8], e0[8], n2 = 0, c0 = 0;
#pragma unroll
  for (int g = 0; g < 8; ++g) {
    c1[g] = (((g < 4) ? F.x : F.y) >> (2 * (g & 3))) & 0x03030303u;
    e2[g] = lut4(0x01000000u, c1[g]); n2 += pc(e2[g]);
    if (OUT) { e0[g] = modeB ? lut4(0x00000001u, c1[g]) : 0u; c0 += pc(e0[g]); } else e0[g] = 0u;
  }
  const unsigned x1 = c0 | (n2 << 16), s1 = dpp_shr_add(x1), t1 = rdl(s1, 63), p1 = s1 - x1;
  const unsigned C0 = t1 & 0xFFFFu, N2 = t1 >> 16;
  const unsigned aM1 = OUT ? 4u * (2048u - C0) : 0u, a2a = aM1 + C0, aM2 = a2a + N2;
  const unsigned oN = 4u * (32u * (unsigned)lane - (p1 & 0xFFFFu));
  unsigned hn[8], m2[8], e2b[8], cm2 = 0, n2b = 0;
  {
    unsigned w2a = fetch32(S, a2a + (p1 >> 16)), wM1 = 0; unsigned long long wN = 0, wN2 = 0;
    if (OUT) {
      wM1 = fetch32(S, aM1 + (p1 & 0xFFFFu)); wN = fetch64(S, oN);
      wN2 = fetch64(S, oN + 4u * (16u - pc(e0[0]) - pc(e0[1]) - pc(e0[2]) - pc(e0[3])));
    }
#pragma unroll
    for (int g = 0; g < 8; ++g) {
      e2b[g] = unpack_bits<1>(w2a, e2[g]); w2a >>= pc(e2[g]); n2b += pc(e2b[g]);
      if (OUT) {
        if (g == 4) wN = wN2;
        const unsigned eN = e0[g] ^ 0x01010101u;
        hn[g] = unpack_bits<4>((unsigned)wN, eN); wN >>= 4u * pc(eN);
        m2[g] = unpack_bits<1>(wM1, e0[g]); wM1 >>= pc(e0[g]); cm2 += pc(m2[g]);
      }
    }
  }
  const unsigned x2 = cm2 | (n2b << 16), s2 = dpp_shr_add(x2), t2 = rdl(s2, 63), p2 = s2 - x2;
  const unsigned CM2 = t2 & 0xFFFFu, N2B = t2 >> 16, a2b = aM2 + CM2, a3 = a2b + 2u * N2B;
  unsigned wM2 = OUT ? fetch32(S, aM2 + (p2 & 0xFFFFu)) : 0u;
  unsigned long long w2b = fetch64(S, a2b + 2u * (p2 >> 16));
  unsigned H[8], L[8], e3[8], n3 = 0;
#pragma unroll
  for (int g = 0; g < 8; ++g) {
    const unsigned c2 = unpack_bits<2>((unsigned)w2b, e2b[g]); w2b >>= 2u * pc(e2b[g]);
    e3[g] = lut4(0x01000000u, c2); n3 += pc(e3[g]);
    H[g] = bfi(bmask(e2b[g]), lut4(tab2, c2), lut4(tab1, c1[g]));
    if (OUT) {
      const unsigned b2 = unpack_bits<1>(wM2, m2[g]); wM2 >>= pc(m2[g]);
      const unsigned nib = hn[g] | ((N0 - m2[g] - b2) & bmask(e0[g]));
      const unsigned p4 = (g >> 1) == 0 ? P0.x : (g >> 1) == 1 ? P0.y : (g >> 1) == 2 ? P0.z : P0.w;
      L[g] = ((p4 >> (4 * (g & 1))) & 0x0F0F0F0Fu) | (nib << 4);
    } else {
      L[g] = g == 0 ? P0.x : g == 1 ? P0.y : g == 2 ? P0.z : g == 3 ? P0.w : g == 4 ? P1.x : g == 5 ? P1.y : g == 6 ? P1.z : P1.w;
    }
  }
  const unsigned s3 = dpp_shr_add(n3), N3 = rdl(s3, 63);
  const unsigned o3 = a3 + 4u * (s3 - n3);
  unsigned long long w3 = fetch64(S, o3);
  const unsigned long long w3b = fetch64(S, o3 + 4u * (pc(e3[0]) + pc(e3[1]) + pc(e3[2]) + pc(e3[3])));
  unsigned e4[8], any4 = 0;
#pragma unroll
  for (int g = 0; g < 8; ++g) {
    if (g == 4) w3 = w3b;
    unsigned R = unpack_bits<4>((unsigned)w3, e3[g]); w3 >>= 4u * pc(e3[g]);
    e4[g] = e3[g] & ~nz4(R ^ T1);
    R &= ~bmask(e4[g]);
    const unsigned H3 = OUT ? ((B4 - (R & 0x07070707u)) | ((R & 0x08080808u) << 4)) : (B4 - R);
    H[g] = bfi(bmask(e3[g]), H3, H[g]);
    any4 |= e4[g];
  }
  if (__any(any4 != 0u)) {
    unsigned n4 = 0;
#pragma unroll
    for (int g = 0; g < 8; ++g) n4 += pc(e4[g]);
    unsigned o4 = a3 + 4u * N3 + 8u * (dpp_shr_add(n4) - n4);
#pragma unroll
    for (int g = 0; g < 8; ++g) { H[g] = bfi(bmask(e4[g]), unpack_bits<8>(fetch32(S, o4), e4[g]), H[g]); o4 += 8u * pc(e4[g]); }
  }
#pragma unroll
  for (int g = 0; g < 8; ++g) {
    const unsigned Hs = OUT ? H[g] : bfi(0x80808080u, L[g], H[g] >> 1), Ls = OUT ? L[g] : bfi(0x80808080u, H[g] << 7, L[g]);
    v[2 * g] = vperm(Hs, Ls, 0x05010400u);
    v[2 * g + 1] = vperm(Hs, Ls, 0x07030602u);
  }
}

struct MapX { unsigned base; __device__ __forceinline__ unsigned operator()(int w, int j, int l) const { return base + (unsigned)(w * 4096 + j * 1024 + l * 16); } };
struct MapW { unsigned base; __device__ __forceinline__ unsigned operator()(int w, int j, int l) const { return base + (unsigned)((j >> 1) * 16384 + ((w >> 1) * 2 + (j & 1)) * 2048 + (w & 1) * 1024 + l * 16); } };
struct MapO { unsigned base; __device__ __forceinline__ unsigned operator()(int w, int j, int l) const { return base + (unsigned)((64 * (w >> 2) + 16 * (j >> 1) + (l & 15)) * (ND * 2) + (64 * (w & 3) + 32 * (j & 1) + 8 * (l >> 4)) * 2); } };
__device__ __forceinline__ unsigned out_task_base(int rank, int task) {
  const int tile = task >> 2, q = task & 3, pm = tile >> 3, pn = tile & 7;
  return (unsigned)(((rank >> 2) * 4096 + pm * 256 + (q >> 1) * 128 + (q & 1) * 32) * (ND * 2) + ((rank & 3) * 2048 + pn * 256) * 2);
}

__device__ __forceinline__ unsigned long long mk_granule(unsigned long long payload, unsigned call) { return payload | ((unsigned long long)(call & 0xFFFFu) << 48); }
__device__ __forceinline__ unsigned long long uni64(unsigned long long g) { return ((unsigned long long)rfl((unsigned)(g >> 32)) << 32) | rfl((unsigned)g); }
__device__ __forceinline__ unsigned long long ld_granule(const char* ws, unsigned off) {
  return __hip_atomic_load((const gu64*)(ws + off), __ATOMIC_RELAXED, __HIP_MEMORY_SCOPE_SYSTEM);
}
__device__ __forceinline__ bool tag_ok(unsigned long long g, unsigned call) { return (unsigned)(g >> 48) == (call & 0xFFFFu); }
__device__ __forceinline__ bool poll_granule(const Ctx& cx, const char* ws, unsigned off, unsigned long long& g) {
  g = uni64(g);
  if (!tag_ok(g, cx.call)) {
    const unsigned long long t0 = rt();
    do {
      if (cx.give_up(t0)) return false;
      g = uni64(ld_granule(ws, off));
    } while (!tag_ok(g, cx.call));
  }
  asm volatile("" ::: "memory");
  return true;
}

template <class Map>
__device__ __forceinline__ void nan_chunk(rsrc_t fin, const Map& map, int wid, int lane) {
#pragma unroll
  for (int j = 0; j < 4; ++j) st128s((u32x4){0x7FC07FC0u, 0x7FC07FC0u, 0x7FC07FC0u, 0x7FC07FC0u}, fin, map(wid, j, lane));
}

struct InTask { int kt, isw, idx; };
__device__ __forceinline__ InTask in_task(int i) {
  InTask t;
  if (i < 768) { t.kt = i / 48; const int u = i - t.kt * 48, u3 = u / 3, r3 = u - u3 * 3;
    if (r3 == 0) { t.isw = 0; t.idx = (u3 & 1) * 16 + (u3 >> 1); } else { t.isw = 1; const int c = u3 * 2 + r3 - 1; t.idx = (c & 3) * 8 + (c >> 2); } }
  else { const int u = i - 768; t.kt = (u >> 4) & 15; t.isw = 0; t.idx = (u & 1) * 16 + 8 + ((u & 15) >> 1); }
  return t;
}
__device__ __forceinline__ int otask_of(int isw, int idx, int kt) { return isw * 512 + idx * 16 + kt; }
__device__ __forceinline__ void in_load(const Params& p, const InTask& t, u32x4 (&r)[4], int wid, int lane) {
  const int rr = lane >> 2, cp = (lane & 3) ^ ((rr >> 3) << 1);
  if (!t.isw) {
    const rsrc_t rx = mkr(p.x, IN_BYTES);
#pragma unroll
    for (int j = 0; j < 4; ++j) {
      const unsigned off = (unsigned)((256 * t.idx + 32 * wid + 16 * (j >> 1) + rr) * (KS * 2) + (64 * t.kt + 32 * (j & 1) + 8 * cp) * 2);
      r[j] = __builtin_amdgcn_raw_buffer_load_b128(rx, (int)off, 0, 0);
    }
  } else {
    const rsrc_t rw = mkr(p.w, IN_BYTES);
    const unsigned off0 = (unsigned)((64 * t.kt + 32 * (wid & 1) + 8 * cp) * (ND * 2) + (256 * t.idx + 64 * (wid >> 1) + 4 * rr) * 2);
#pragma unroll
    for (int d = 0; d < 4; ++d) {
      const u32x2 a = __builtin_amdgcn_raw_buffer_load_b64(rw, (int)(off0 + (unsigned)(2 * d) * (ND * 2)), 0, 0);
      const u32x2 b = __builtin_amdgcn_raw_buffer_load_b64(rw, (int)(off0 + (unsigned)(2 * d + 1) * (ND * 2)), 0, 0);
      r[d] = __builtin_shufflevector(a, b, 0, 1, 2, 3);
    }
  }
}
__device__ __forceinline__ void in_prep(int isw, const u32x4 (&r)[4], unsigned (&v)[16]) {
  if (isw) {
#pragma unroll
    for (int d = 0; d < 4; ++d) {
      v[0 + d] = __builtin_amdgcn_perm(r[d].z, r[d].x, 0x05040100u);
      v[4 + d] = __builtin_amdgcn_perm(r[d].z, r[d].x, 0x07060302u);
      v[8 + d] = __builtin_amdgcn_perm(r[d].w, r[d].y, 0x05040100u);
      v[12 + d] = __builtin_amdgcn_perm(r[d].w, r[d].y, 0x07060302u);
    }
  } else {
#pragma unroll
    for (int j = 0; j < 4; ++j) { v[4 * j] = r[j].x; v[4 * j + 1] = r[j].y; v[4 * j + 2] = r[j].z; v[4 * j + 3] = r[j].w; }
  }
}

__device__ __forceinline__ void role_encode(const Params& p, const Ctx& cx, LAS unsigned* S, int e, int ne) {
  const int tid = threadIdx.x, wid = (int)rfl(tid >> 6), lane = tid & 63;
  char* ws = p.ws[p.rank];
  const rsrc_t lw = mkr(ws, WS_END);
  u32x4 A[4];
  in_load(p, in_task(e), A, wid, lane);
  DRAIN_ALL();
  for (int i = e; i < NIT; i += ne) {
    const InTask t = in_task(i);
    unsigned v[16];
    in_prep(t.isw, A, v);
    if (i + ne < NIT) in_load(p, in_task(i + ne), A, wid, lane);
    u32x4 P0, P1; u32x2 F;
    const unsigned long long hp = encode_chunk<false>(v, S, lane, P0, P1, F);
    const int ot = otask_of(t.isw, t.idx, t.kt);
    if (!hp_raw(hp)) {
      store_chunk<false>(lw, WS_OBI + (unsigned)ot * PKT + (unsigned)wid * 3072u, S, hp_len(hp), lane, P0, P1, F);
    } else {
      const bool mine = t.isw ? ((t.idx >> 3) == (p.rank & 3)) : ((t.idx >> 4) == (p.rank >> 2));
      const int T = t.kt * 8 + p.rank;
      const unsigned ib = WS_IMG + (unsigned)(t.isw ? 2048 + (t.idx & 7) * 128 + T : (t.idx & 15) * 128 + T) * IMG;
#pragma unroll
      for (int j = 0; j < 4; ++j) {
        const u32x4 q = (u32x4){v[4 * j], v[4 * j + 1], v[4 * j + 2], v[4 * j + 3]};
        st128s(q, lw, WS_OBR + (unsigned)ot * IMG + (unsigned)(wid * 4096 + j * 1024 + lane * 16));
        if (mine) st128s(q, lw, t.isw ? MapW{ib}(wid, j, lane) : MapX{ib}(wid, j, lane));
      }
    }
    DRAIN_ALL();
    if (lane == 0) __hip_atomic_store((gu64*)(ws + WS_HOBI + (unsigned)ot * 64u) + wid, mk_granule(hp, cx.call), __ATOMIC_RELAXED, __HIP_MEMORY_SCOPE_SYSTEM);
  }
}

__device__ __forceinline__ bool eo_ready(const Ctx& cx, const gu32* f, bool block) {
  if (rfl(__hip_atomic_load(f, __ATOMIC_RELAXED, __HIP_MEMORY_SCOPE_AGENT)) >= cx.call) return true;
  if (!block) return false;
  const unsigned long long t0 = rt();
  while (rfl(__hip_atomic_load(f, __ATOMIC_RELAXED, __HIP_MEMORY_SCOPE_AGENT)) < cx.call) { if (cx.give_up(t0)) return false; }
  return true;
}
__device__ __forceinline__ void role_encode_out(const Params& p, const Ctx& cx, LAS unsigned* S, int e, int ne) {
  const int tid = threadIdx.x, wid = (int)rfl(tid >> 6), lane = tid & 63;
  char* ws = p.ws[p.rank];
  const rsrc_t lw = mkr(ws, WS_END), ro = mkr(p.out[p.rank], OUT_BYTES);
  const gu32* rf = (const gu32*)(ws + WS_RFLG) + wid * 4;
  auto load = [&](int task, u32x4 (&A)[4]) { const MapO map{out_task_base(p.rank, task)};
#pragma unroll
    for (int j = 0; j < 4; ++j) A[j] = ld128s(ro, map(wid, j, lane)); };
  u32x4 A[4];
  if (!eo_ready(cx, rf + e * 32, true)) return;
  load(e, A);
  DRAIN_ALL();
  for (int i = e; i < NOT; i += ne) {
    unsigned v[16];
#pragma unroll
    for (int j = 0; j < 4; ++j) { v[4 * j] = A[j].x; v[4 * j + 1] = A[j].y; v[4 * j + 2] = A[j].z; v[4 * j + 3] = A[j].w; }
    const int nx = i + ne;
    bool have = false;
    if (nx < NOT) { have = eo_ready(cx, rf + nx * 32, false); if (have) load(nx, A); }
    u32x4 P0, P1; u32x2 F;
    const unsigned long long hp = encode_chunk<true>(v, S, lane, P0, P1, F);
    if (!hp_raw(hp)) store_chunk<true>(lw, WS_OBO + (unsigned)i * PKT + (unsigned)wid * 3072u, S, hp_len(hp), lane, P0, P1, F);
    DRAIN_ALL();
    if (lane == 0) __hip_atomic_store((gu64*)(ws + WS_HOBO + (unsigned)i * 64u) + wid, mk_granule(hp, cx.call), __ATOMIC_RELAXED, __HIP_MEMORY_SCOPE_SYSTEM);
    if (nx < NOT && !have) {
      if (!eo_ready(cx, rf + nx * 32, true)) return;
      load(nx, A);
      DRAIN_ALL();
    }
  }
}

struct PItem { unsigned hsrc, dsrc, hdst, ddst, rs, rd; int kind; };
__device__ __forceinline__ PItem p_item(int rank, int d, int i) {
  PItem r;
  if (i < 384) {
    int kt, isw, k;
    if (i < 256) { kt = i >> 4; isw = i & 1; k = (i & 15) >> 1; } else { kt = (i - 256) >> 3; isw = 0; k = 8 + ((i - 256) & 7); }
    const int idx = isw ? (d & 3) * 8 + k : (d >> 2) * 16 + k;
    const int ot = otask_of(isw, idx, kt), T = kt * 8 + rank, img = isw ? 2048 + k * 128 + T : k * 128 + T;
    r.hsrc = WS_HOBI + (unsigned)ot * 64u; r.dsrc = WS_OBI + (unsigned)ot * PKT;
    r.hdst = WS_HIN + (unsigned)img * 64u; r.ddst = WS_PIN + (unsigned)img * PKT;
    r.rs = WS_OBR + (unsigned)ot * IMG; r.rd = WS_IMG + (unsigned)img * IMG; r.kind = isw;
  } else {
    const int task = (i - 384) & (NOT - 1);
    r.hsrc = WS_HOBO + (unsigned)task * 64u; r.dsrc = WS_OBO + (unsigned)task * PKT;
    r.hdst = WS_HOUT + (unsigned)(rank * NOT + task) * 64u; r.ddst = WS_POUT + (unsigned)(rank * NOT + task) * PKT;
    r.rs = r.rd = out_task_base(rank, task); r.kind = 2;
  }
  return r;
}

#define P_LDG(ws, off) ld_granule(ws, off)

__device__ __forceinline__ void p_publish(const Ctx& cx, const char* ws, char* wd, unsigned hsrc, unsigned hdst, unsigned long long hl, int lane) {
  if (hsrc == 0xFFFFFFFFu) hl = mk_granule(0x8000ull, cx.call);
  else {
    const gu64* src = (const gu64*)(ws + hsrc) + (lane & 7);
    while (!__all(tag_ok(hl, cx.call))) hl = __hip_atomic_load(src, __ATOMIC_RELAXED, __HIP_MEMORY_SCOPE_SYSTEM);
  }
  if (lane < 8) __hip_atomic_store((gu64*)(wd + hdst) + lane, hl, __ATOMIC_RELAXED, __HIP_MEMORY_SCOPE_SYSTEM);
}

__device__ __forceinline__ void role_push(const Params& p, Ctx& cx, unsigned callraw, int link, int v0, int nv) {
  constexpr int NC = NPI * 8;
  const int lane = threadIdx.x & 63;
  int d = p.rank + 1 + link; if (d >= NRANK) d -= NRANK;
  if ((unsigned)d >= (unsigned)NRANK) return;
  char* ws = p.ws[p.rank]; char* wd = p.ws[d];
  const rsrc_t lw = mkr(ws, WS_END), dw = mkr(wd, WS_END);
  gu32* tkc = (gu32*)(ws + WS_TK) + link * 32;
  gu32* pcnt = (gu32*)(ws + WS_PCNT) + link * NPI;
  const unsigned lo = (unsigned)lane * 16u;
  int prev = v0; unsigned prev_hsrc = 0, prev_hdst;
  {
    const int i0 = v0 >> 3, w = v0 & 7;
    InTask t; t.kt = 0; t.isw = i0 & 1; t.idx = t.isw ? (d & 3) * 8 + (i0 >> 1) : (d >> 2) * 16 + (i0 >> 1);
    u32x4 A[4];
    in_load(p, t, A, w, lane);
    unsigned dummy = 0;
    DRAIN_PIN2(callraw, dummy);
    cx.call = rfl(callraw) + 1u;
    unsigned v[16];
    in_prep(t.isw, A, v);
    const PItem f = p_item(p.rank, d, i0);
#pragma unroll
    for (int j = 0; j < 4; ++j) st128s((u32x4){v[4 * j], v[4 * j + 1], v[4 * j + 2], v[4 * j + 3]}, dw, t.isw ? MapW{f.rd}(w, j, lane) : MapX{f.rd}(w, j, lane));
    prev_hdst = f.hdst;
  }
  STAMP(p, 3);
  const unsigned tkbase = (cx.call - 1u) * (unsigned)(NC - nv) - 4u * (unsigned)nv;
  int it = v0 + nv, nx = v0 + 2 * nv, n2 = v0 + 3 * nv;
  PItem cur = p_item(p.rank, d, it >> 3);
  unsigned long long gc = P_LDG(ws, cur.hsrc + (it & 7) * 8), gn;
  DRAIN_ALL();
  gc = uni64(gc);
  if (!tag_ok(gc, cx.call)) {
    unsigned pc = atomic_add_lane0_nowait(pcnt + (prev >> 3)), dummy = 0;
    DRAIN_PIN2(pc, dummy);
    if (rfl(pc) == 8u * cx.call - 1u) p_publish(cx, ws, wd, 0xFFFFFFFFu, prev_hdst, 0ull, lane);
    prev = -1;
    if (!poll_granule(cx, ws, cur.hsrc + (it & 7) * 8, gc)) return;
  }
  auto load3 = [&](const PItem& t, int u, unsigned long long g, u32x4& a, u32x4& b, u32x4& c) {
    const unsigned o = t.dsrc + (unsigned)(u & 7) * 3072u + lo, len = hp_len(g);
    a = ld128s(lw, o); b = ld128s(lw, 1024u + lo < len ? o + 1024 : 0xFFFFFF00u); c = ld128s(lw, 2048u + lo < len ? o + 2048 : 0xFFFFFF00u);
  };
  u32x4 c0, c1, c2;
  load3(cur, it, gc, c0, c1, c2);
  gn = P_LDG(ws, p_item(p.rank, d, nx >> 3).hsrc + (nx & 7) * 8);
  if (prev >= 0) prev_hsrc = 0xFFFFFFFFu;
  DRAIN_ALL();
  while (it < NC) {
    const int w = it & 7;
    if (!hp_raw(gc)) {
      const unsigned o = cur.ddst + (unsigned)w * 3072u + lo, len = hp_len(gc);
      st128s(c0, dw, o);
      if (1024u + lo < len) st128s(c1, dw, o + 1024);
      if (2048u + lo < len) st128s(c2, dw, o + 2048);
    } else if (cur.kind == 2) {
      const rsrc_t so = mkr(p.out[p.rank], OUT_BYTES), dd = mkr(p.out[d], OUT_BYTES); const MapO map{cur.rs};
#pragma unroll
      for (int j = 0; j < 4; ++j) st128s(ld128s(so, map(w, j, lane)), dd, map(w, j, lane));
    } else {
#pragma unroll
      for (int j = 0; j < 4; ++j) { const u32x4 q = ld128s(lw, cur.rs + (unsigned)(w * 4096 + j * 1024) + lo);
        st128s(q, dw, cur.kind ? MapW{cur.rd}(w, j, lane) : MapX{cur.rd}(w, j, lane)); }
    }
    PItem nxt = cur; u32x4 n0 = c0, n1 = c1, n2v = c2; unsigned long long g2 = 0;
    bool have = true;
    if (nx < NC) {
      nxt = p_item(p.rank, d, nx >> 3);
      gn = uni64(gn);
      have = tag_ok(gn, cx.call);
      if (have) load3(nxt, nx, gn, n0, n1, n2v);
      if (n2 < NC) g2 = P_LDG(ws, p_item(p.rank, d, n2 >> 3).hsrc + (n2 & 7) * 8);
    }
    unsigned tk = atomic_add_lane0_nowait(tkc);
    unsigned pc = 0; unsigned long long hl = 0;
    if (prev >= 0) {
      pc = atomic_add_lane0_nowait(pcnt + (prev >> 3));
      if (prev_hsrc != 0xFFFFFFFFu) hl = __hip_atomic_load((const gu64*)(ws + prev_hsrc) + (lane & 7), __ATOMIC_RELAXED, __HIP_MEMORY_SCOPE_SYSTEM);
    }
    DRAIN_PIN2(tk, pc);
    if (prev >= 0 && rfl(pc) == 8u * cx.call - 1u) p_publish(cx, ws, wd, prev_hsrc, prev_hdst, hl, lane);
    const int n3 = (int)(rfl(tk) - tkbase);
    if (!have) {
      unsigned pc2 = atomic_add_lane0_nowait(pcnt + (it >> 3)), dummy = 0;
      hl = __hip_atomic_load((const gu64*)(ws + cur.hsrc) + (lane & 7), __ATOMIC_RELAXED, __HIP_MEMORY_SCOPE_SYSTEM);
      DRAIN_PIN2(pc2, dummy);
      if (rfl(pc2) == 8u * cx.call - 1u) p_publish(cx, ws, wd, cur.hsrc, cur.hdst, hl, lane);
      prev = -1;
      if (!poll_granule(cx, ws, nxt.hsrc + (nx & 7) * 8, gn)) return;
      load3(nxt, nx, gn, n0, n1, n2v);
      DRAIN_ALL();
    } else {
      prev = it; prev_hsrc = cur.hsrc; prev_hdst = cur.hdst;
    }
    cur = nxt; gc = gn; gn = g2; it = nx; nx = n2; n2 = n3;
    c0 = n0; c1 = n1; c2 = n2v;
  }
  if (prev >= 0) {
    unsigned pc = atomic_add_lane0_nowait(pcnt + (prev >> 3)), dummy = 0;
    unsigned long long hl = 0;
    if (prev_hsrc != 0xFFFFFFFFu) hl = __hip_atomic_load((const gu64*)(ws + prev_hsrc) + (lane & 7), __ATOMIC_RELAXED, __HIP_MEMORY_SCOPE_SYSTEM);
    DRAIN_PIN2(pc, dummy);
    if (rfl(pc) == 8u * cx.call - 1u) p_publish(cx, ws, wd, prev_hsrc, prev_hdst, hl, lane);
  }
  STAMP(p, 2);
}

struct DTask { unsigned hl, pk, dst; int kind; int sig; };
__device__ __forceinline__ unsigned d_map(const DTask& t, int w, int j, int l) {
  return t.kind == 0 ? MapX{t.dst}(w, j, l) : t.kind == 1 ? MapW{t.dst}(w, j, l) : MapO{t.dst}(w, j, l);
}
template <bool OUT> __device__ __forceinline__ DTask d_task(int rank, int i) {
  DTask t;
  if (!OUT) {
    int k, T; int isw = 0;
    if (i < 2048) { T = i >> 4; isw = i & 1; k = (i & 15) >> 1; } else { const int u = (i - 2048) & 1023; T = u >> 3; k = 8 + (u & 7); }
    const int img = isw ? 2048 + k * 128 + T : k * 128 + T;
    t.hl = WS_HIN + (unsigned)img * 64u; t.pk = WS_PIN + (unsigned)img * PKT;
    if ((T & 7) == rank) {
      const int ot = otask_of(isw, isw ? (rank & 3) * 8 + k : (rank >> 2) * 16 + k, T >> 3);
      t.hl = WS_HOBI + (unsigned)ot * 64u; t.pk = WS_OBI + (unsigned)ot * PKT;
    }
    t.dst = WS_IMG + (unsigned)img * IMG; t.kind = isw; t.sig = img;
  } else {
    const int task = (i / (NRANK - 1)) & (NOT - 1), k = i % (NRANK - 1);
    int s = rank + 1 + k; if (s >= NRANK) s -= NRANK;
    const unsigned tidx = (unsigned)(s * NOT + task);
    t.hl = WS_HOUT + tidx * 64u; t.pk = WS_POUT + tidx * PKT; t.dst = out_task_base(s, task); t.kind = 2; t.sig = 0;
  }
  return t;
}
#define WAIT_VM4() do { asm volatile("s_waitcnt vmcnt(4)" ::: "memory"); __builtin_amdgcn_s_waitcnt(0x0F74); } while (0)

template <bool OUT>
__device__ __forceinline__ void role_decode(const Params& p, const Ctx& cx, LAS unsigned* S, int dw, int nw) {
  constexpr int TOTAL = OUT ? NOT * (NRANK - 1) : NIMG;
  const int tid = threadIdx.x, wid = (int)rfl(tid >> 6), lane = tid & 63;
  char* ws = p.ws[p.rank];
  const rsrc_t lw = mkr(ws, WS_END), fin = OUT ? mkr(p.out[p.rank], OUT_BYTES) : lw;
  gu32* cnt = (gu32*)(ws + WS_CNT);
  int i = dw;
  if (i >= TOTAL) return;
  DTask cur = d_task<OUT>(p.rank, i);
  unsigned long long gc = ld_granule(ws, cur.hl + wid * 8), gn = 0;
  bool ok = poll_granule(cx, ws, cur.hl + wid * 8, gc);
  Rec rc = load_rec<OUT>(lw, cur.pk + (unsigned)wid * 3072u, gc, lane);
  if (i + nw < TOTAL) gn = ld_granule(ws, d_task<OUT>(p.rank, i + nw).hl + wid * 8);
  DRAIN_ALL();
  int prev = -1;
  while (ok) {
    const int nx = i + nw;
    DTask nxt = cur; Rec rn = rc; unsigned long long g2 = 0; bool have = true;
    if (nx < TOTAL) {
      nxt = d_task<OUT>(p.rank, nx);
      gn = uni64(gn);
      have = tag_ok(gn, cx.call);
      if (have) rn = load_rec<OUT>(lw, nxt.pk + (unsigned)wid * 3072u, gn, lane);
      if (nx + nw < TOTAL) g2 = ld_granule(ws, d_task<OUT>(p.rank, nx + nw).hl + wid * 8);
    }
    if (!hp_raw(gc)) {
      unsigned v[16];
      decode_rec<OUT>(rc, gc, S, lane, v);
#pragma unroll
      for (int j = 0; j < 4; ++j)
        __builtin_amdgcn_raw_buffer_store_b128((u32x4){v[4 * j], v[4 * j + 1], v[4 * j + 2], v[4 * j + 3]}, fin, (int)d_map(cur, wid, j, lane), 0, OUT ? CFG_AUX_DOUT : CFG_AUX_DIN);
      WAIT_VM4();
    } else {
      DRAIN_ALL();
    }
    if (!OUT && prev >= 0 && lane == 0) __hip_atomic_fetch_add(cnt + prev * 32, 1u, __ATOMIC_RELAXED, __HIP_MEMORY_SCOPE_AGENT);
    prev = cur.sig;
    if (nx >= TOTAL) break;
    if (!have) {
      if (!OUT) { VM0(); if (lane == 0) __hip_atomic_fetch_add(cnt + prev * 32, 1u, __ATOMIC_RELAXED, __HIP_MEMORY_SCOPE_AGENT); prev = -1; }
      ok = poll_granule(cx, ws, nxt.hl + wid * 8, gn);
      rn = load_rec<OUT>(lw, nxt.pk + (unsigned)wid * 3072u, gn, lane);
      DRAIN_ALL();
    }
    cur = nxt; rc = rn; gc = gn; gn = g2; i = nx;
  }
  if (!OUT) {
    VM0();
    if (ok && prev >= 0 && lane == 0) __hip_atomic_fetch_add(cnt + prev * 32, 1u, __ATOMIC_RELAXED, __HIP_MEMORY_SCOPE_AGENT);
  } else if (!ok) {
    for (; i < TOTAL; i += nw) {
      const DTask t = d_task<OUT>(p.rank, i);
#pragma unroll
      for (int j = 0; j < 4; ++j) st128s((u32x4){0x7FC07FC0u, 0x7FC07FC0u, 0x7FC07FC0u, 0x7FC07FC0u}, fin, d_map(t, wid, j, lane));
    }
  }
}

__host__ __device__ __forceinline__ int lds_byte(int r, int c) { const int st = (r >> 4) * 2 + (c >> 5), rr = r & 15, cc = c & 31, ob = rr * 64 + cc * 2; return st * 1024 + (ob ^ (((ob >> 9) & 1) << 5)); }
__device__ __forceinline__ unsigned cvt_pk_bf16(float lo, float hi) { unsigned r; asm volatile("v_cvt_pk_bf16_f32 %0, %1, %2" : "=v"(r) : "v"(lo), "v"(hi)); return r; }
__device__ __forceinline__ float gelu1(float y) {
  const float k1 = -2.0f * 0.7978845608028654f * 1.4426950408889634f, k3 = k1 * 0.044715f;
  const float t = y * (k3 * (y * y) + k1);
  return y * __builtin_amdgcn_rcpf(1.0f + __builtin_amdgcn_exp2f(t));
}

__device__ __forceinline__ void gemm_unit(const Params& p, const Ctx& cx, LAS unsigned char* lds, int pm, int pn, int h, int ph) {
  constexpr int NTH = NT / 2;
  const int tid = threadIdx.x, wid = (int)rfl(tid >> 6), lane = tid & 63, wr = wid >> 2, wc = wid & 3, fr = lane & 15, fq = lane >> 4;
  char* ws = p.ws[p.rank];
  const char* cA = ws + WS_IMG + (size_t)(pm * 128 + 8 * h) * IMG;
  const char* cB = ws + WS_IMG + (size_t)(2048 + pn * 128 + 8 * h) * IMG;
  const gu32* fl = (const gu32*)(ws + WS_CNT) + (size_t)(((lane & 2) ? 2048 + pn * 128 : pm * 128) + 8 * h + (lane & 1)) * 32;
  const unsigned need = 8u * cx.call;
  const unsigned voff = (unsigned)tid * 16u;
  const unsigned ldsw = (unsigned)wid * 1024u;
  const int aoff = lds_byte(wr * 64 + fr, fq * 8), boff = lds_byte(wc * 32 + fr, fq * 8);
  bool dead = false;
#define KT(t) ((((t) >> 3) << 4) + ((t) & 7))
  auto wait_tiles = [&](int t) {
    if (wid == 0 && !dead) {
      const gu32* f = fl + KT(t) * 32;
      if (!__all(__hip_atomic_load(f, __ATOMIC_RELAXED, __HIP_MEMORY_SCOPE_AGENT) >= need)) {
        const unsigned long long t0 = rt();
        while (!__all(__hip_atomic_load(f, __ATOMIC_RELAXED, __HIP_MEMORY_SCOPE_AGENT) >= need)) { if (cx.give_up(t0)) { dead = true; break; } }
      }
    }
  };
#define SA(b, h) (((b) * 2 + (h)) * HTB)
#define SB(b, h) ((4 + (b) * 2 + (h)) * HTB)
#define STAGE(bufoff, gbase) do { _Pragma("unroll") for (int _i = 0; _i < 2; ++_i) \
    __builtin_amdgcn_global_load_lds((const GAS unsigned*)((const char*)(gbase) + voff + _i * 8192), (LAS unsigned*)(lds + (bufoff) + ldsw + _i * 8192), 16, 0, AUX_SYS); } while (0)
#define LDA(dst, b, h) do { _Pragma("unroll") for (int m = 0; m < 4; ++m) _Pragma("unroll") for (int k = 0; k < 2; ++k) dst[m][k] = *(const LAS bf16x8*)(lds + SA(b, h) + aoff + m * 2048 + k * 1024); } while (0)
#define LDB(dst, b, h) do { _Pragma("unroll") for (int n = 0; n < 2; ++n) _Pragma("unroll") for (int k = 0; k < 2; ++k) dst[n][k] = *(const LAS bf16x8*)(lds + SB(b, h) + boff + n * 2048 + k * 1024); } while (0)
#define MMA(ai, bj, At, Bt) do { __builtin_amdgcn_s_setprio(1); _Pragma("unroll") for (int m = 0; m < 4; ++m) _Pragma("unroll") for (int n = 0; n < 2; ++n) _Pragma("unroll") for (int k = 0; k < 2; ++k) \
    acc[ai][bj][m][n] = __builtin_amdgcn_mfma_f32_16x16x32_bf16(Bt[n][k], At[m][k], acc[ai][bj][m][n], 0, 0, 0); __builtin_amdgcn_s_setprio(0); } while (0)
#define WAIT_V(n) asm volatile("s_waitcnt vmcnt(" #n ")" ::: "memory")
#define WAIT_L(n) asm volatile("s_waitcnt lgkmcnt(" #n ")" ::: "memory")
#define BAR __builtin_amdgcn_s_barrier()
#define SCHED __builtin_amdgcn_sched_barrier(0)
  f32x4 acc[2][2][4][2];
#pragma unroll
  for (int a = 0; a < 2; ++a)
#pragma unroll
    for (int b = 0; b < 2; ++b)
#pragma unroll
      for (int m = 0; m < 4; ++m)
#pragma unroll
        for (int n = 0; n < 2; ++n) acc[a][b][m][n] = (f32x4){0.f, 0.f, 0.f, 0.f};
  bf16x8 At[4][2], B0[2][2], B1[2][2];
  wait_tiles(0); wait_tiles(2);
  BARRIER();
  STAMP(p, 1 + 4 * ph);
  STAGE(SB(0, 0), cB); STAGE(SA(0, 0), cA); STAGE(SB(0, 1), cB + HTB); STAGE(SA(0, 1), cA + HTB);
  if (wr == 1) BAR;
  WAIT_V(4); BAR;
  STAGE(SB(1, 0), cB + IMG); STAGE(SA(1, 0), cA + IMG); STAGE(SB(1, 1), cB + IMG + HTB);
  WAIT_V(6); BAR;
  for (int t = 0; t < NTH; t += 2) {
    const bool last = (t == NTH - 2);
    const char* a1 = cA + (size_t)(KT(t) + 1) * IMG;
    const size_t o2 = last ? 0 : (size_t)KT(t + 2) * IMG;
    const char* a2 = cA + o2; const char* b2 = cB + o2;
    const char* a3 = a2 + IMG; const char* b3 = b2 + IMG;
    LAS unsigned* fslot = (LAS unsigned*)(lds + LDS_FLAG + ((t >> 1) & 1) * 16);
    const bool chk = (t + 4 < NTH);
    if (chk && wid == 0 && lane < 4) __builtin_amdgcn_global_load_lds(fl + KT(t + 4) * 32, fslot, 4, 0, AUX_SYS);
    LDB(B0, 0, 0); SCHED; LDA(At, 0, 0); STAGE(SA(1, 1), a1 + HTB);
    WAIT_L(8); BAR; WAIT_L(0); MMA(0, 0, At, B0); BAR; SCHED;
    LDB(B1, 0, 1); STAGE(SB(0, 0), b2);
    BAR; WAIT_L(0); MMA(0, 1, At, B1); BAR;
    LDA(At, 0, 1); STAGE(SA(0, 0), a2);
    BAR; WAIT_L(0); MMA(1, 0, At, B0); BAR; SCHED;
    STAGE(SB(0, 1), b2 + HTB);
    WAIT_V(6); BAR; MMA(1, 1, At, B1); BAR;
    LDB(B0, 1, 0); SCHED; LDA(At, 1, 0); STAGE(SA(0, 1), a2 + HTB);
    WAIT_L(8); BAR; WAIT_L(0); MMA(0, 0, At, B0); BAR; SCHED;
    LDB(B1, 1, 1); STAGE(SB(1, 0), b3);
    BAR; WAIT_L(0); MMA(0, 1, At, B1); BAR;
    LDA(At, 1, 1); STAGE(SA(1, 0), a3);
    BAR; WAIT_L(0); MMA(1, 0, At, B0); BAR; SCHED;
    const u32x4 f = *(const LAS u32x4*)fslot;
    STAGE(SB(1, 1), b3 + HTB);
    WAIT_V(6); BAR; MMA(1, 1, At, B1); BAR;
    if (chk) {
      const unsigned fm = rfl(min(min(f.x, f.y), min(f.z, f.w)));
      if (fm < need) { wait_tiles(t + 4); BARRIER(); }
    }
  }
  if (wr == 0) BAR;
  STAMP(p, 2 + 4 * ph);
  const int tile = pm * 8 + pn;
  {
    int tz = threadIdx.x; asm volatile("" : "+v"(tz));
    const int ln = tz & 63;
    const rsrc_t ro = mkr(p.out[p.rank], OUT_BYTES), lw = mkr(ws, WS_END);
    {
      const unsigned po = WS_PART + (unsigned)(tile * 2 + h) * 131072u + (unsigned)(wid * 16384 + ln * 16);
      const unsigned pi = WS_PART + (unsigned)(tile * 2 + 1 - h) * 131072u + (unsigned)(wid * 16384 + ln * 16);
#pragma unroll
      for (int i = 0; i < 16; ++i) {
        const f32x4 q = h ? acc[0][i >> 3][(i >> 1) & 3][i & 1] : acc[1][i >> 3][(i >> 1) & 3][i & 1];
        st128s(__builtin_bit_cast(u32x4, q), lw, po + i * 1024);
      }
      VM0();
      gu32* fo = (gu32*)(ws + WS_PFLG) + ((tile * 2 + h) * 8 + wid) * 32;
      const gu32* fi = (const gu32*)(ws + WS_PFLG) + ((tile * 2 + 1 - h) * 8 + wid) * 32;
      if (!dead && !cx.timed_out() && ln == 0) __hip_atomic_store(fo, cx.call, __ATOMIC_RELAXED, __HIP_MEMORY_SCOPE_AGENT);
      if (rfl(__hip_atomic_load(fi, __ATOMIC_RELAXED, __HIP_MEMORY_SCOPE_AGENT)) < cx.call) {
        const unsigned long long t0 = rt();
        while (rfl(__hip_atomic_load(fi, __ATOMIC_RELAXED, __HIP_MEMORY_SCOPE_AGENT)) < cx.call) { if (cx.give_up(t0)) break; }
      }
      asm volatile("" ::: "memory");
#pragma unroll
      for (int i = 0; i < 16; ++i) {
        const f32x4 q = __builtin_bit_cast(f32x4, ld128s(lw, pi + i * 1024));
        if (h) acc[1][i >> 3][(i >> 1) & 3][i & 1] += q; else acc[0][i >> 3][(i >> 1) & 3][i & 1] += q;
      }
    }
    STAMP(p, 12 + 4 * ph);
    const bool bad = cx.timed_out();
#pragma unroll
    for (int qq = 0; qq < 2; ++qq) {
      unsigned u[16];
#pragma unroll
      for (int mm = 0; mm < 2; ++mm) {
        const int m = qq * 2 + mm;
        unsigned d[8];
#pragma unroll
        for (int c = 0; c < 8; ++c) {
          const float y0 = h ? acc[1][c & 1][m][0][c >> 1] : acc[0][c & 1][m][0][c >> 1], y1 = h ? acc[1][c & 1][m][1][c >> 1] : acc[0][c & 1][m][1][c >> 1];
          d[c] = cvt_pk_bf16(gelu1(y0), gelu1(y1)); if (bad) d[c] = 0x7FC07FC0u; }
#pragma unroll
        for (int c = 0; c < 4; ++c) {
          auto r = __builtin_amdgcn_permlane16_swap(d[c], d[c + 4], false, false); d[c] = r[0]; d[c + 4] = r[1];
          auto s2 = __builtin_amdgcn_permlane32_swap(d[c], d[c + 4], false, false); d[c] = s2[0]; d[c + 4] = s2[1];
        }
#pragma unroll
        for (int c = 0; c < 8; ++c) u[8 * mm + c] = d[c];
      }
      const MapO map{out_task_base(p.rank, tile * 4 + h * 2 + qq)};
#pragma unroll
      for (int j = 0; j < 4; ++j) st128s((u32x4){u[4 * j], u[4 * j + 1], u[4 * j + 2], u[4 * j + 3]}, ro, map(wid, j, ln));
    }
    VM0();
    if (!bad && ln < 2)
      __hip_atomic_store((gu32*)(ws + WS_RFLG) + ((tile * 4 + h * 2 + ln) * 8 + wid) * 4, cx.call, __ATOMIC_RELAXED, __HIP_MEMORY_SCOPE_AGENT);
  }
  STAMP(p, 3 + 4 * ph);
#undef KT
#undef SA
#undef SB
#undef STAGE
#undef LDA
#undef LDB
#undef MMA
#undef WAIT_V
#undef WAIT_L
#undef BAR
#undef SCHED
}

__global__ __launch_bounds__(NTHR, 2) void fused_kernel(Params p) {
  extern __shared__ __attribute__((aligned(16))) unsigned char shm[];
  LAS unsigned char* lds = (LAS unsigned char*)shm;
  const int b = blockIdx.x, xcd = b & 7, slot = b >> 3;
  char* ws = p.ws[p.rank];
  const int wave = (int)rfl(threadIdx.x >> 6);
  unsigned callraw = atomic_add_lane0_nowait((gu32*)(ws + WS_CALL) + (b * 8 + wave) * 4);
  Ctx cx; cx.tmo = (gu32*)(ws + WS_TMO); cx.call = 0;
  LAS unsigned* S = (LAS unsigned*)(lds + LDS_SCR + wave * SCR_BYTES);
  STAMP(p, 0);
  if (b >= N_P) { unsigned dummy = 0; DRAIN_PIN2(callraw, dummy); cx.call = rfl(callraw) + 1u; }
  if (b < N_P) {
    { const int gw = b * 8 + wave; role_push(p, cx, callraw, gw % 7, gw / 7, PJ * 8); }
  } else if (b < N_P + N_EO) {
    role_encode_out(p, cx, S, b - N_P, N_EO);
    STAMP(p, 2);
  } else if (slot < 16) {
    const int dw = b - N_P - N_EO;
    role_decode<false>(p, cx, S, dw, N_D);
    STAMP(p, 1);
    role_decode<true>(p, cx, S, dw, N_D + 128);
    STAMP(p, 2);
  } else {
    const int g = slot - 16, h = g & 1, s = g >> 1;
    const int pm = (xcd >> 1) * 2 + (s & 1), pn = (xcd & 1) * 4 + (s >> 1);
    role_encode(p, cx, S, b - 128, 128);
    STAMP(p, 9);
#pragma unroll 1
    for (int ph = 0; ph < 2; ++ph) gemm_unit(p, cx, lds, pm + 8 * ph, pn, h, ph);
    role_decode<true>(p, cx, S, N_D + (b - 128), N_D + 128);
    STAMP(p, 10);
  }
}

}

extern "C" void kernel_launch(int rank, int nranks, ncclComm_t comm, hipStream_t stream,
                              void* const* d_in, const int* in_sizes, int n_in,
                              void* d_out, int out_size,
                              const int* in_dtypes, int out_dtype,
                              void* const* d_out_all, void* const* d_ws_all, size_t ws_bytes) {
  using namespace fk;
  (void)comm; (void)in_dtypes; (void)out_dtype;
  if (nranks != NRANK || n_in < 2 || rank < 0 || rank >= NRANK || ws_bytes < (size_t)WS_END ||
      in_sizes[0] != MD * KS || in_sizes[1] != KS * ND || out_size != MD * ND) {
    fprintf(stderr, "kernel_launch: unexpected problem shape\n");
    return;
  }
  static bool attr_set[64] = {};
  int dev = 0;
  if (hipGetDevice(&dev) != hipSuccess) return;
  if (dev >= 0 && dev < 64 && !attr_set[dev]) {
    hipError_t e = hipFuncSetAttribute((const void*)fused_kernel, hipFuncAttributeMaxDynamicSharedMemorySize, LDS_BYTES);
    if (e != hipSuccess) { fprintf(stderr, "HIP error %s\n", hipGetErrorString(e)); return; }
    attr_set[dev] = true;
  }
  Params p;
  p.x = (const char*)d_in[0]; p.w = (const char*)d_in[1];
  for (int i = 0; i < NRANK; ++i) { p.ws[i] = (char*)d_ws_all[i]; p.out[i] = (char*)d_out_all[i]; }
  p.out[rank] = (char*)d_out;
  p.rank = rank; p.pad = 0;
  hipLaunchKernelGGL(fused_kernel, dim3(GRID), dim3(NTHR), LDS_BYTES, stream, p);
  hipError_t e = hipGetLastError();
  if (e != hipSuccess) fprintf(stderr, "HIP launch error %s\n", hipGetErrorString(e));
}
```

```cpp
#include <hip/hip_runtime.h>
#include <rccl/rccl.h>
#include <cstdio>

namespace fk {

#define LAS __attribute__((address_space(3)))
#define GAS __attribute__((address_space(1)))
typedef unsigned short bf16_t;
typedef short bf16x8 __attribute__((ext_vector_type(8)));
typedef float f32x4 __attribute__((ext_vector_type(4)));
typedef unsigned u32x4 __attribute__((ext_vector_type(4)));
typedef unsigned u32x2 __attribute__((ext_vector_type(2)));
typedef unsigned short u16x2 __attribute__((ext_vector_type(2)));
typedef GAS unsigned gu32;
typedef GAS unsigned long long gu64;
typedef __amdgpu_buffer_rsrc_t rsrc_t;

constexpr int NRANK = 8;
constexpr int MD = 8192, ND = 8192, KD = 8192, KS = KD / NRANK;
constexpr int NT = KD / 64;
constexpr unsigned OUT_BYTES = (unsigned)MD * ND * 2u;
constexpr unsigned IN_BYTES = (unsigned)MD * KS * 2u;

constexpr int GRID = 256, NTHR = 512;
#define CFG_SLEEP 2
#define CFG_PJ 5
#define CFG_AUX_DIN 17
#define CFG_AUX_DOUT 17
#define CFG_SEQIN 1
#define CFG_GMAP 1
#define CFG_RAWTAIL 0
#define CFG_AUX_G 17
#define CFG_RAWHEAD 2
#define CFG_HEADSEQ 1
#define CFG_PMAP 1
constexpr int PJ = CFG_PJ;
#define CFG_NEO 28
constexpr int N_EO = CFG_NEO;
constexpr int N_P = 7 * PJ, N_D = 128 - N_P - N_EO;
constexpr int AUX_SYS = 17;

constexpr int NIMG = 3072;
constexpr unsigned IMG = 32768, PKT = 24576;
constexpr int NIT = 1024;
constexpr int NOT = 512;
constexpr int NPI = 384 + NOT;
constexpr int HCAP = 13;

constexpr unsigned WS_CALL = 0;
constexpr unsigned WS_TMO  = WS_CALL + 256 * 128;
constexpr unsigned WS_TK   = WS_TMO + 1024;
constexpr unsigned WS_CNT  = WS_TMO + 4096;
constexpr unsigned WS_PCNT = WS_CNT + NIMG * 128;
constexpr unsigned WS_HIN  = WS_PCNT + 32768;
constexpr unsigned WS_HOUT = WS_HIN + NIMG * 128;
constexpr unsigned WS_HOBI = WS_HOUT + NRANK * NOT * 128;
constexpr unsigned WS_HOBO = WS_HOBI + NIT * 128;
constexpr unsigned WS_PFLG = WS_HOBO + NOT * 128;
constexpr unsigned WS_RFLG = WS_PFLG + 2048 * 128;
constexpr unsigned WS_DBG  = WS_RFLG + NOT * 8 * 16;
constexpr unsigned WS_PIN  = 4u << 20;
constexpr unsigned WS_POUT = WS_PIN + NIMG * PKT;
constexpr unsigned WS_OBI  = WS_POUT + NRANK * NOT * PKT;
constexpr unsigned WS_OBR  = WS_OBI + NIT * PKT;
constexpr unsigned WS_OBO  = WS_OBR + NIT * IMG;
constexpr unsigned WS_PART = WS_OBO + NOT * PKT;
constexpr unsigned WS_IMG  = WS_PART + 256 * 131072;
constexpr unsigned WS_END  = WS_IMG + NIMG * IMG;
static_assert(WS_DBG + 256 * 64 * 8 <= WS_PIN, "control region overflow");
static_assert((size_t)WS_IMG + (size_t)NIMG * IMG < (1ull << 31), "workspace offsets must fit 31 bits");

constexpr unsigned long long DEADLINE = 40000000ull;

constexpr int HTB = 16384, STAGE_BYTES = 8 * HTB;
constexpr int LDS_FLAG = STAGE_BYTES;
constexpr int LDS_CALL = STAGE_BYTES + 64;
constexpr int LDS_TK = STAGE_BYTES + 96;
constexpr int LDS_DEAD = STAGE_BYTES + 80;
constexpr int LDS_GHDR = STAGE_BYTES + 128;
constexpr int LDS_HDR = 0;
constexpr int LDS_SCR = STAGE_BYTES + 512;
constexpr int LDS_BYTES = STAGE_BYTES + 512 + 8 * 1664;

struct Params {
  const char* x; const char* w;
  char* ws[NRANK]; char* out[NRANK];
  int rank; int pad;
};

__device__ __forceinline__ rsrc_t mkr(const void* p, unsigned bytes) { return __builtin_amdgcn_make_buffer_rsrc((void*)p, 0, (int)bytes, 0x00020000); }
__device__ __forceinline__ u32x4 ld128s(rsrc_t r, unsigned off) { return __builtin_amdgcn_raw_buffer_load_b128(r, (int)off, 0, AUX_SYS); }
__device__ __forceinline__ void st128s(u32x4 v, rsrc_t r, unsigned off) { __builtin_amdgcn_raw_buffer_store_b128(v, r, (int)off, 0, AUX_SYS); }
__device__ __forceinline__ unsigned long long rt() { return __builtin_amdgcn_s_memrealtime(); }
__device__ __forceinline__ unsigned rfl(unsigned v) { return (unsigned)__builtin_amdgcn_readfirstlane((int)v); }
__device__ __forceinline__ unsigned rdl(unsigned v, int l) { return (unsigned)__builtin_amdgcn_readlane((int)v, l); }
#define VM0() asm volatile("s_waitcnt vmcnt(0)" ::: "memory")
#define DRAIN_ALL() do { asm volatile("s_waitcnt vmcnt(0) lgkmcnt(0)" ::: "memory"); __builtin_amdgcn_s_waitcnt(0x0070); } while (0)
#define DRAIN_PIN2(a, b) do { asm volatile("s_waitcnt vmcnt(0) lgkmcnt(0)" : "+v"(a), "+v"(b) :: "memory"); __builtin_amdgcn_s_waitcnt(0x0070); } while (0)
#define BARRIER() do { asm volatile("" ::: "memory"); __builtin_amdgcn_s_barrier(); asm volatile("" ::: "memory"); } while (0)

#define STAMP(p, idx) do { } while (0)

__device__ __forceinline__ unsigned atomic_add_lane0_nowait(gu32* ptr) {
  unsigned r; unsigned long long sv; const unsigned one = 1u;
  asm volatile("s_mov_b64 %1, exec\n\ts_mov_b64 exec, 1\n\tglobal_atomic_add %0, %2, %3, off sc0\n\ts_mov_b64 exec, %1"
               : "=&v"(r), "=&s"(sv) : "v"(ptr), "v"(one) : "memory");
  return r;
}

struct Ctx {
  gu32* tmo; unsigned call;
  __device__ __forceinline__ bool timed_out() const { return rfl(__hip_atomic_load(tmo, __ATOMIC_RELAXED, __HIP_MEMORY_SCOPE_AGENT)) == call; }
  __device__ __forceinline__ bool give_up(unsigned long long t0) const {
    if (timed_out() || rt() - t0 > DEADLINE) { __hip_atomic_store(tmo, call, __ATOMIC_RELAXED, __HIP_MEMORY_SCOPE_AGENT); return true; }
    __builtin_amdgcn_s_sleep(CFG_SLEEP);
    return false;
  }
};

__device__ __forceinline__ unsigned dpp_shr_add(unsigned x) {
  x += (unsigned)__builtin_amdgcn_update_dpp(0, (int)x, 0x111, 0xf, 0xf, false);
  x += (unsigned)__builtin_amdgcn_update_dpp(0, (int)x, 0x112, 0xf, 0xf, false);
  x += (unsigned)__builtin_amdgcn_update_dpp(0, (int)x, 0x114, 0xf, 0xf, false);
  x += (unsigned)__builtin_amdgcn_update_dpp(0, (int)x, 0x118, 0xf, 0xf, false);
  x += (unsigned)__builtin_amdgcn_update_dpp(0, (int)x, 0x142, 0xa, 0xf, false);
  x += (unsigned)__builtin_amdgcn_update_dpp(0, (int)x, 0x143, 0xc, 0xf, false);
  return x;
}
__device__ __forceinline__ unsigned row_max(unsigned x) {
  x = max(x, (unsigned)__builtin_amdgcn_update_dpp(0, (int)x, 0x111, 0xf, 0xf, false));
  x = max(x, (unsigned)__builtin_amdgcn_update_dpp(0, (int)x, 0x112, 0xf, 0xf, false));
  x = max(x, (unsigned)__builtin_amdgcn_update_dpp(0, (int)x, 0x114, 0xf, 0xf, false));
  x = max(x, (unsigned)__builtin_amdgcn_update_dpp(0, (int)x, 0x118, 0xf, 0xf, false));
  return x;
}
__device__ __forceinline__ unsigned wave_max_u32(unsigned x) {
  x = row_max(x);
  x = max(x, (unsigned)__builtin_amdgcn_update_dpp(0, (int)x, 0x142, 0xa, 0xf, false));
  x = max(x, (unsigned)__builtin_amdgcn_update_dpp(0, (int)x, 0x143, 0xc, 0xf, false));
  return rdl(x, 63);
}
__device__ __forceinline__ unsigned vperm(unsigned a, unsigned b, unsigned sel) { return __builtin_amdgcn_perm(a, b, sel); }
__device__ __forceinline__ unsigned lut4(unsigned tab, unsigned sel) { return __builtin_amdgcn_perm(0u, tab, sel); }
__device__ __forceinline__ unsigned bmask(unsigned e) { return __builtin_amdgcn_perm(0u, 0u, e | 0x0C0C0C0Cu); }
__device__ __forceinline__ unsigned bfi(unsigned m, unsigned a, unsigned b) { return (m & a) | (~m & b); }
__device__ __forceinline__ unsigned mul24(unsigned a, unsigned k) { unsigned r; asm("v_mul_u32_u24 %0, %1, %2" : "=v"(r) : "s"(k), "v"(a)); return r; }
__device__ __forceinline__ unsigned rank4(unsigned e) { return mul24(e, 0x010101u) << 8; }
__device__ __forceinline__ unsigned pc(unsigned e) { return (unsigned)__builtin_popcount(e); }
__device__ __forceinline__ unsigned nz4(unsigned x) { return ((x + 0x0F0F0F0Fu) >> 4) & 0x01010101u; }

constexpr int SCR_BYTES = 1664;

template <int W> __device__ __forceinline__ unsigned pack_bits(unsigned c, unsigned e) {
  const unsigned pre = rank4(e), em = bmask(e);
  if (W == 1) return __builtin_amdgcn_udot4(c, lut4(0x08040201u, pre) & em, 0u, false);
  if (W == 2) return __builtin_amdgcn_udot4(c, lut4(0x40100401u, pre) & em, 0u, false);
  return __builtin_amdgcn_udot4(c, lut4(0x00001001u, pre) & em, 0u, false) | (__builtin_amdgcn_udot4(c, lut4(0x10010000u, pre) & em, 0u, false) << 8);
}
template <int W> __device__ __forceinline__ unsigned unpack_bits(unsigned w, unsigned e) {
  unsigned s;
  if (W == 1) s = mul24(w & 0xFu, 0x00204081u) & 0x01010101u;
  else if (W == 2) { w &= 0xFFu; const unsigned t = (w | (w << 12)) & 0x000F000Fu; s = (t | (t << 6)) & 0x03030303u; }
  else if (W == 4) { w &= 0xFFFFu; const unsigned t = (w | (w << 8)) & 0x00FF00FFu; s = (t | (t << 4)) & 0x0F0F0F0Fu; }
  else s = w;
  return lut4(s, rank4(e)) & bmask(e);
}
__device__ __forceinline__ void lds_or(LAS unsigned* q, unsigned x) { if (x != 0u) __hip_atomic_fetch_or(q, x, __ATOMIC_RELAXED, __HIP_MEMORY_SCOPE_WAVEFRONT); }
__device__ __forceinline__ void flush32(LAS unsigned* S, unsigned off, unsigned acc) {
  const unsigned long long x = (unsigned long long)acc << (off & 31u);
  LAS unsigned* q = S + (off >> 5);
  lds_or(q, (unsigned)x); lds_or(q + 1, (unsigned)(x >> 32));
}
__device__ __forceinline__ void flush64(LAS unsigned* S, unsigned off, unsigned long long acc) {
  const unsigned sh = off & 31u;
  const unsigned long long x = acc << sh;
  LAS unsigned* q = S + (off >> 5);
  lds_or(q, (unsigned)x); lds_or(q + 1, (unsigned)(x >> 32)); lds_or(q + 2, ((unsigned)(acc >> 32) >> 1) >> (31u - sh));
}
__device__ __forceinline__ unsigned fetch32(const LAS unsigned* S, unsigned off) {
  const LAS unsigned* q = S + (off >> 5);
  return __builtin_amdgcn_alignbit(q[1], q[0], off & 31u);
}
__device__ __forceinline__ unsigned long long fetch64(const LAS unsigned* S, unsigned off) {
  const LAS unsigned* q = S + (off >> 5);
  const unsigned a = q[0], b = q[1], c = q[2];
  return ((unsigned long long)__builtin_amdgcn_alignbit(c, b, off & 31u) << 32) | __builtin_amdgcn_alignbit(b, a, off & 31u);
}
__device__ __forceinline__ unsigned wave_sum_lane63(unsigned x) { return dpp_shr_add(x); }
__device__ __forceinline__ unsigned sym_of(bool out, unsigned base, unsigned t) { return out ? ((base - (t & 7u)) | ((t & 8u) << 4)) & 0xFFu : (base - t) & 0xFFu; }

template <bool OUT>
__device__ __forceinline__ unsigned long long encode_chunk(const unsigned (&v)[16], LAS unsigned* S, int lane, u32x4& P0, u32x4& P1, u32x2& F) {
  u16x2 mx = __builtin_bit_cast(u16x2, v[0] & 0x7FFF7FFFu);
#pragma unroll
  for (int i = 1; i < 16; ++i) mx = __builtin_elementwise_max(mx, __builtin_bit_cast(u16x2, v[i] & 0x7FFF7FFFu));
  const unsigned base = wave_max_u32(max((unsigned)mx.x, (unsigned)mx.y)) >> (OUT ? 8 : 7);
  const unsigned B4 = base * 0x01010101u;
  *(LAS u32x4*)(S + lane * 4) = (u32x4){0u, 0u, 0u, 0u};
  if (lane < (OUT ? 34 : 1)) *(LAS u32x4*)(S + 256 + lane * 4) = (u32x4){0u, 0u, 0u, 0u};
  unsigned H[8], L[8], R[8], oor[8], anyo = 0;
#pragma unroll
  for (int g = 0; g < 8; ++g) {
    const unsigned Hs = vperm(v[2 * g + 1], v[2 * g], 0x07050301u), Ls = vperm(v[2 * g + 1], v[2 * g], 0x06040200u);
    if (OUT) { H[g] = Hs; L[g] = Ls; const unsigned D = B4 - (Hs & 0x7F7F7F7Fu); R[g] = (D & 0x07070707u) | ((Hs >> 4) & 0x08080808u); oor[g] = D & 0x78787878u; }
    else { H[g] = bfi(0x01010101u, Ls >> 7, Hs << 1); L[g] = bfi(0x80808080u, Hs, Ls); const unsigned D = B4 - H[g]; R[g] = D & 0x0F0F0F0Fu; oor[g] = D & 0xF0F0F0F0u; }
    anyo |= oor[g];
  }
  unsigned key;
  {
    unsigned long long hc = 0;
#pragma unroll
    for (int b = 0; b < 8; ++b) hc += 1ull << (((R[b >> 2] >> (8 * (b & 3))) & 15u) * 4u);
    const unsigned lo = (unsigned)hc, hi = (unsigned)(hc >> 32);
    unsigned q[8];
#pragma unroll
    for (int k = 0; k < 8; ++k) q[k] = rdl(dpp_shr_add(((lo >> (4 * k)) & 15u) | (((hi >> (4 * k)) & 15u) << 16)), 63);
    const int k7 = lane & 7;
    const unsigned qa = (k7 & 1) ? ((k7 & 2) ? q[3] : q[1]) : ((k7 & 2) ? q[2] : q[0]), qb = (k7 & 1) ? ((k7 & 2) ? q[7] : q[5]) : ((k7 & 2) ? q[6] : q[4]);
    const unsigned cnt = (((k7 & 4) ? qb : qa) >> ((lane & 8) * 2)) & 0xFFFFu;
    key = (lane < 16) ? (((cnt + 1u) << 4) | (unsigned)lane) : 0u;
  }
  unsigned tt = 0, cl = 0x1Fu;
#pragma unroll
  for (int i = 0; i < 7; ++i) {
    const unsigned bias = (OUT && i == 0 && lane >= 8 && lane < 16) ? (32u << 4) : 0u;
    const unsigned m = rdl(row_max(key + bias), 15);
    tt |= (m & 15u) << (4 * i);
    if (key + bias == m) { key = 0u; cl = (i < 3) ? (unsigned)i : (i == 3) ? 3u : (7u | ((unsigned)(i - 4) << 3)); }
  }
  cl <<= 8 * (lane & 3);
  cl |= (unsigned)__builtin_amdgcn_update_dpp(0, (int)cl, 0xB1, 0xf, 0xf, false);
  cl |= (unsigned)__builtin_amdgcn_update_dpp(0, (int)cl, 0x4E, 0xf, 0xf, false);
  const unsigned tb0 = rdl(cl, 0), tb1 = rdl(cl, 4), tb2 = rdl(cl, 8), tb3 = rdl(cl, 12);
  unsigned cls[8];
#pragma unroll
  for (int g = 0; g < 8; ++g) {
    const unsigned r7 = R[g] & 0x07070707u;
    cls[g] = bfi(bmask((R[g] >> 3) & 0x01010101u), vperm(tb3, tb2, r7), vperm(tb1, tb0, r7));
  }
  const bool has_oor = __any(anyo != 0u);
  unsigned e4[8];
#pragma unroll
  for (int g = 0; g < 8; ++g) e4[g] = 0u;
  if (has_oor) {
    const unsigned T1 = ((tt >> 4) & 15u) * 0x01010101u;
#pragma unroll
    for (int g = 0; g < 8; ++g) {
      e4[g] = nz4(OUT ? (oor[g] >> 3) : (oor[g] >> 4));
      const unsigned m = bmask(e4[g]);
      cls[g] |= m & 0x1F1F1F1Fu; R[g] = bfi(m, T1, R[g]);
    }
  }
  bool modeB = false; unsigned n0 = 0;
  if (OUT) {
    unsigned z[8], mxn = 0;
#pragma unroll
    for (int g = 0; g < 8; ++g) z[g] = lut4(0x000000FFu, cls[g] & 0x03030303u);
#pragma unroll
    for (int g = 0; g < 2; ++g) {
      const unsigned hn = (L[g] >> 4) & z[g] & 0x0F0F0F0Fu;
      mxn = max(mxn, max(max(hn & 0xFFu, (hn >> 8) & 0xFFu), max((hn >> 16) & 0xFFu, hn >> 24)));
    }
    n0 = wave_max_u32(mxn);
    unsigned viol[8], nv = 0;
    const unsigned N4 = (n0 | 0x80u) * 0x01010101u;
#pragma unroll
    for (int g = 0; g < 8; ++g) {
      const unsigned dn = (N4 - ((L[g] >> 4) & 0x0F0F0F0Fu)) & 0x7F7F7F7Fu;
      viol[g] = z[g] & bmask(((dn + 0x7D7D7D7Du) >> 7) & 0x01010101u);
      nv += pc(viol[g] & 0x01010101u);
    }
    modeB = n0 < 8u && rdl(dpp_shr_add(nv), 63) < 24u;
    if (modeB) {
#pragma unroll
      for (int g = 0; g < 8; ++g) cls[g] |= viol[g] & 0x1F1F1F1Fu;
    }
  }
  unsigned e2[8], e2b[8], e3[8], e0[8], m2[8], n2 = 0, n2b = 0, n3 = 0, n4 = 0, c0 = 0, cm2 = 0;
  const unsigned N0 = n0 * 0x01010101u;
#pragma unroll
  for (int g = 0; g < 8; ++g) {
    const unsigned c1 = cls[g] & 0x03030303u;
    e2[g] = lut4(0x01000000u, c1); e2b[g] = (cls[g] >> 2) & 0x01010101u; e3[g] = lut4(0x01000000u, (cls[g] >> 3) & 0x03030303u);
    n2 += pc(e2[g]); n2b += pc(e2b[g]); n3 += pc(e3[g]); n4 += pc(e4[g]);
    if (OUT) {
      e0[g] = modeB ? lut4(0x00000001u, c1) : 0u;
      m2[g] = e0[g] & nz4(((L[g] >> 4) & 0x0F0F0F0Fu) ^ N0);
      c0 += pc(e0[g]); cm2 += pc(m2[g]);
    } else { e0[g] = 0u; m2[g] = 0u; }
  }
  const unsigned x1 = c0 | (n2 << 16), x2 = cm2 | (n2b << 16), x3 = n3 | (n4 << 16);
  const unsigned s1 = dpp_shr_add(x1), s2 = dpp_shr_add(x2), s3 = dpp_shr_add(x3);
  const unsigned t1 = rdl(s1, 63), t2 = rdl(s2, 63), t3 = rdl(s3, 63);
  const unsigned C0 = t1 & 0xFFFFu, N2 = t1 >> 16, CM2 = t2 & 0xFFFFu, N2B = t2 >> 16, N3 = t3 & 0xFFFFu, N4c = t3 >> 16;
  const unsigned bSN = OUT ? 4u * (2048u - C0) : 0u;
  const unsigned aM1 = bSN, a2a = aM1 + C0, aM2 = a2a + N2, a2b = aM2 + CM2, a3 = a2b + 2u * N2B, a4 = a3 + 4u * N3, bits = a4 + 8u * N4c;
  const bool raw = bits > (OUT ? 12288u : 4096u);
  const unsigned len = ((OUT ? 12288u : 20480u) + bits + 255u) >> 8;
  if (!raw) {
    const unsigned p1 = s1 - x1, p2 = s2 - x2, p3 = s3 - x3;
    const unsigned oN = 4u * (32u * (unsigned)lane - (p1 & 0xFFFFu)), oM1 = aM1 + (p1 & 0xFFFFu), o2a = a2a + (p1 >> 16), oM2 = aM2 + (p2 & 0xFFFFu),
                   o2b = a2b + 2u * (p2 >> 16), o3 = a3 + 4u * (p3 & 0xFFFFu);
    unsigned o4 = a4 + 8u * (p3 >> 16);
    unsigned wM1 = 0, wM2 = 0, w2a = 0, kM1 = 0, kM2 = 0, k2a = 0, k2b = 0, kN = 0, k3 = 0, kNa = 0, k3a = 0;
    unsigned long long w2b = 0, wN = 0, w3 = 0;
#pragma unroll
    for (int g = 0; g < 8; ++g) {
      if (g == 4) {
        if (OUT) { flush64(S, oN, wN); wN = 0; kNa = kN; kN = 0; }
        flush64(S, o3, w3); w3 = 0; k3a = k3; k3 = 0;
      }
      if (OUT) {
        const unsigned hn = (L[g] >> 4) & 0x0F0F0F0Fu, eN = e0[g] ^ 0x01010101u;
        wN |= (unsigned long long)pack_bits<4>(hn, eN) << kN; kN += 4u * pc(eN);
        wM1 |= pack_bits<1>(m2[g], e0[g]) << kM1; kM1 += pc(e0[g]);
        wM2 |= pack_bits<1>(nz4(((N0 | 0x10101010u) - 0x01010101u - hn) & 0x0F0F0F0Fu), m2[g]) << kM2; kM2 += pc(m2[g]);
      }
      w2a |= pack_bits<1>(e2b[g], e2[g]) << k2a; k2a += pc(e2[g]);
      w2b |= (unsigned long long)pack_bits<2>((cls[g] >> 3) & 0x03030303u, e2b[g]) << k2b; k2b += 2u * pc(e2b[g]);
      w3 |= (unsigned long long)pack_bits<4>(R[g], e3[g]) << k3; k3 += 4u * pc(e3[g]);
    }
    if (OUT) { flush64(S, oN + kNa, wN); flush32(S, oM1, wM1); flush32(S, oM2, wM2); }
    flush32(S, o2a, w2a); flush64(S, o2b, w2b); flush64(S, o3 + k3a, w3);
    if (has_oor) {
#pragma unroll
      for (int g = 0; g < 8; ++g)
#pragma unroll
        for (int b = 0; b < 4; ++b)
          if ((e4[g] >> (8 * b)) & 1u) {
            const unsigned long long x = (unsigned long long)((H[g] >> (8 * b)) & 0xFFu) << (o4 & 31u);
            __hip_atomic_fetch_or(S + (o4 >> 5), (unsigned)x, __ATOMIC_RELAXED, __HIP_MEMORY_SCOPE_WAVEFRONT);
            __hip_atomic_fetch_or(S + (o4 >> 5) + 1, (unsigned)(x >> 32), __ATOMIC_RELAXED, __HIP_MEMORY_SCOPE_WAVEFRONT);
            o4 += 8u;
          }
    }
  }
  unsigned Fw[2];
#pragma unroll
  for (int i = 0; i < 2; ++i)
    Fw[i] = (cls[4 * i] & 0x03030303u) | ((cls[4 * i + 1] & 0x03030303u) << 2) | ((cls[4 * i + 2] & 0x03030303u) << 4) | ((cls[4 * i + 3] & 0x03030303u) << 6);
  F = (u32x2){Fw[0], Fw[1]};
  if (OUT) {
    unsigned p4[4];
#pragma unroll
    for (int i = 0; i < 4; ++i) p4[i] = (L[2 * i] & 0x0F0F0F0Fu) | ((L[2 * i + 1] & 0x0F0F0F0Fu) << 4);
    P0 = (u32x4){p4[0], p4[1], p4[2], p4[3]}; P1 = P0;
  } else {
    P0 = (u32x4){L[0], L[1], L[2], L[3]}; P1 = (u32x4){L[4], L[5], L[6], L[7]};
  }
  return (unsigned long long)(base | (raw ? 0x8000u : (len << 8))) | ((unsigned long long)tt << 16) | (modeB ? (1ull << 44) | ((unsigned long long)n0 << 45) : 0ull);
}
__device__ __forceinline__ bool hp_raw(unsigned long long hp) { return ((unsigned)hp & 0x8000u) != 0u; }
__device__ __forceinline__ unsigned hp_len(unsigned long long hp) { return (((unsigned)hp >> 8) & 127u) * 32u; }

template <bool OUT>
__device__ __forceinline__ void store_chunk(rsrc_t r, unsigned o, const LAS unsigned* S, unsigned len, int lane, u32x4 P0, u32x4 P1, u32x2 F) {
  const unsigned l16 = (unsigned)lane * 16u;
  st128s(P0, r, o + l16);
  const u32x4 sv = *(const LAS u32x4*)(S + lane * 4);
  if (OUT) {
    const u32x4 sw = *(const LAS u32x4*)(S + 256 + (lane & 31) * 4);
    __builtin_amdgcn_raw_buffer_store_b64(F, r, (int)(o + 1024 + lane * 8), 0, AUX_SYS);
    if (1536u + l16 < len) st128s(sv, r, o + 1536 + l16);
    if (lane < 32 && 2560u + l16 < len) st128s(sw, r, o + 2560 + l16);
  } else {
    st128s(P1, r, o + 1024 + l16);
    __builtin_amdgcn_raw_buffer_store_b64(F, r, (int)(o + 2048 + lane * 8), 0, AUX_SYS);
    if (lane < 32 && 2560u + l16 < len) st128s(sv, r, o + 2560 + l16);
  }
}

struct Rec { u32x4 P0, P1, sv; u32x2 F; };
template <bool OUT>
__device__ __forceinline__ Rec load_rec(rsrc_t r, unsigned o, unsigned long long hp, int lane) {
  const unsigned len = hp_len(hp), l16 = (unsigned)lane * 16u;
  Rec c;
  c.P0 = ld128s(r, o + l16);
  if (OUT) {
    c.F = __builtin_amdgcn_raw_buffer_load_b64(r, (int)(o + 1024 + lane * 8), 0, AUX_SYS);
    c.sv = ld128s(r, (1536u + l16 < len) ? o + 1536 + l16 : 0xFFFFFF00u);
    c.P1 = ld128s(r, (lane < 32 && 2560u + l16 < len) ? o + 2560 + l16 : 0xFFFFFF00u);
  } else {
    c.P1 = ld128s(r, o + 1024 + l16);
    c.F = __builtin_amdgcn_raw_buffer_load_b64(r, (int)(o + 2048 + lane * 8), 0, AUX_SYS);
    c.sv = ld128s(r, (lane < 32 && 2560u + l16 < len) ? o + 2560 + l16 : 0xFFFFFF00u);
  }
  return c;
}
template <bool OUT>
__device__ __forceinline__ void decode_rec(const Rec& rc, unsigned long long hp, LAS unsigned* S, int lane, unsigned (&v)[16]) {
  const unsigned base = (unsigned)hp & 0xFFu, tt = (unsigned)(hp >> 16) & 0xFFFFFFFu;
  const bool modeB = ((hp >> 44) & 1u) != 0u;
  const unsigned n0 = (unsigned)(hp >> 45) & 7u;
  const u32x4 P0 = rc.P0, P1 = rc.P1, sv = rc.sv, sw = rc.P1; const u32x2 F = rc.F;
  unsigned tab1 = 0, tab2 = 0;
#pragma unroll
  for (int i = 0; i < 4; ++i) tab1 |= sym_of(OUT, base, (tt >> (4 * i)) & 15u) << (8 * i);
#pragma unroll
  for (int i = 0; i < 3; ++i) tab2 |= sym_of(OUT, base, (tt >> (16 + 4 * i)) & 15u) << (8 * i);
  const unsigned B4 = base * 0x01010101u, T1 = ((tt >> 4) & 15u) * 0x01010101u, N0 = n0 * 0x01010101u;
  *(LAS u32x4*)(S + lane * 4) = sv;
  if (OUT && lane < 32) *(LAS u32x4*)(S + 256 + lane * 4) = sw;
  unsigned c1[8], e2[8], e0[8], n2 = 0, c0 = 0;
#pragma unroll
  for (int g = 0; g < 8; ++g) {
    c1[g] = (((g < 4) ? F.x : F.y) >> (2 * (g & 3))) & 0x03030303u;
    e2[g] = lut4(0x01000000u, c1[g]); n2 += pc(e2[g]);
    if (OUT) { e0[g] = modeB ? lut4(0x00000001u, c1[g]) : 0u; c0 += pc(e0[g]); } else e0[g] = 0u;
  }
  const unsigned x1 = c0 | (n2 << 16), s1 = dpp_shr_add(x1), t1 = rdl(s1, 63), p1 = s1 - x1;
  const unsigned C0 = t1 & 0xFFFFu, N2 = t1 >> 16;
  const unsigned aM1 = OUT ? 4u * (2048u - C0) : 0u, a2a = aM1 + C0, aM2 = a2a + N2;
  const unsigned oN = 4u * (32u * (unsigned)lane - (p1 & 0xFFFFu));
  unsigned hn[8], m2[8], e2b[8], cm2 = 0, n2b = 0;
  {
    unsigned w2a = fetch32(S, a2a + (p1 >> 16)), wM1 = 0; unsigned long long wN = 0, wN2 = 0;
    if (OUT) {
      wM1 = fetch32(S, aM1 + (p1 & 0xFFFFu)); wN = fetch64(S, oN);
      wN2 = fetch64(S, oN + 4u * (16u - pc(e0[0]) - pc(e0[1]) - pc(e0[2]) - pc(e0[3])));
    }
#pragma unroll
    for (int g = 0; g < 8; ++g) {
      e2b[g] = unpack_bits<1>(w2a, e2[g]); w2a >>= pc(e2[g]); n2b += pc(e2b[g]);
      if (OUT) {
        if (g == 4) wN = wN2;
        const unsigned eN = e0[g] ^ 0x01010101u;
        hn[g] = unpack_bits<4>((unsigned)wN, eN); wN >>= 4u * pc(eN);
        m2[g] = unpack_bits<1>(wM1, e0[g]); wM1 >>= pc(e0[g]); cm2 += pc(m2[g]);
      }
    }
  }
  const unsigned x2 = cm2 | (n2b << 16), s2 = dpp_shr_add(x2), t2 = rdl(s2, 63), p2 = s2 - x2;
  const unsigned CM2 = t2 & 0xFFFFu, N2B = t2 >> 16, a2b = aM2 + CM2, a3 = a2b + 2u * N2B;
  unsigned wM2 = OUT ? fetch32(S, aM2 + (p2 & 0xFFFFu)) : 0u;
  unsigned long long w2b = fetch64(S, a2b + 2u * (p2 >> 16));
  unsigned H[8], L[8], e3[8], n3 = 0;
#pragma unroll
  for (int g = 0; g < 8; ++g) {
    const unsigned c2 = unpack_bits<2>((unsigned)w2b, e2b[g]); w2b >>= 2u * pc(e2b[g]);
    e3[g] = lut4(0x01000000u, c2); n3 += pc(e3[g]);
    H[g] = bfi(bmask(e2b[g]), lut4(tab2, c2), lut4(tab1, c1[g]));
    if (OUT) {
      const unsigned b2 = unpack_bits<1>(wM2, m2[g]); wM2 >>= pc(m2[g]);
      const unsigned nib = hn[g] | ((N0 - m2[g] - b2) & bmask(e0[g]));
      const unsigned p4 = (g >> 1) == 0 ? P0.x : (g >> 1) == 1 ? P0.y : (g >> 1) == 2 ? P0.z : P0.w;
      L[g] = ((p4 >> (4 * (g & 1))) & 0x0F0F0F0Fu) | (nib << 4);
    } else {
      L[g] = g == 0 ? P0.x : g == 1 ? P0.y : g == 2 ? P0.z : g == 3 ? P0.w : g == 4 ? P1.x : g == 5 ? P1.y : g == 6 ? P1.z : P1.w;
    }
  }
  const unsigned s3 = dpp_shr_add(n3), N3 = rdl(s3, 63);
  const unsigned o3 = a3 + 4u * (s3 - n3);
  unsigned long long w3 = fetch64(S, o3);
  const unsigned long long w3b = fetch64(S, o3 + 4u * (pc(e3[0]) + pc(e3[1]) + pc(e3[2]) + pc(e3[3])));
  unsigned e4[8], any4 = 0;
#pragma unroll
  for (int g = 0; g < 8; ++g) {
    if (g == 4) w3 = w3b;
    unsigned R = unpack_bits<4>((unsigned)w3, e3[g]); w3 >>= 4u * pc(e3[g]);
    e4[g] = e3[g] & ~nz4(R ^ T1);
    R &= ~bmask(e4[g]);
    const unsigned H3 = OUT ? ((B4 - (R & 0x07070707u)) | ((R & 0x08080808u) << 4)) : (B4 - R);
    H[g] = bfi(bmask(e3[g]), H3, H[g]);
    any4 |= e4[g];
  }
  if (__any(any4 != 0u)) {
    unsigned n4 = 0;
#pragma unroll
    for (int g = 0; g < 8; ++g) n4 += pc(e4[g]);
    unsigned o4 = a3 + 4u * N3 + 8u * (dpp_shr_add(n4) - n4);
#pragma unroll
    for (int g = 0; g < 8; ++g) { H[g] = bfi(bmask(e4[g]), unpack_bits<8>(fetch32(S, o4), e4[g]), H[g]); o4 += 8u * pc(e4[g]); }
  }
#pragma unroll
  for (int g = 0; g < 8; ++g) {
    const unsigned Hs = OUT ? H[g] : bfi(0x80808080u, L[g], H[g] >> 1), Ls = OUT ? L[g] : bfi(0x80808080u, H[g] << 7, L[g]);
    v[2 * g] = vperm(Hs, Ls, 0x05010400u);
    v[2 * g + 1] = vperm(Hs, Ls, 0x07030602u);
  }
}

struct MapX { unsigned base; __device__ __forceinline__ unsigned operator()(int w, int j, int l) const { return base + (unsigned)(w * 4096 + j * 1024 + l * 16); } };
struct MapW { unsigned base; __device__ __forceinline__ unsigned operator()(int w, int j, int l) const { return base + (unsigned)((j >> 1) * 16384 + ((w >> 1) * 2 + (j & 1)) * 2048 + (w & 1) * 1024 + l * 16); } };
struct MapO { unsigned base; __device__ __forceinline__ unsigned operator()(int w, int j, int l) const { return base + (unsigned)((64 * (w >> 2) + 16 * (j >> 1) + (l & 15)) * (ND * 2) + (64 * (w & 3) + 32 * (j & 1) + 8 * (l >> 4)) * 2); } };
__device__ __forceinline__ unsigned out_task_base(int rank, int task) {
  const int tile = task >> 2, q = task & 3, pm = tile >> 3, pn = tile & 7;
  return (unsigned)(((rank >> 2) * 4096 + pm * 256 + (q >> 1) * 128 + (q & 1) * 32) * (ND * 2) + ((rank & 3) * 2048 + pn * 256) * 2);
}

__device__ __forceinline__ unsigned long long mk_granule(unsigned long long payload, unsigned call) { return payload | ((unsigned long long)(call & 0xFFFFu) << 48); }
__device__ __forceinline__ unsigned long long uni64(unsigned long long g) { return ((unsigned long long)rfl((unsigned)(g >> 32)) << 32) | rfl((unsigned)g); }
__device__ __forceinline__ unsigned long long ld_granule(const char* ws, unsigned off) {
  return __hip_atomic_load((const gu64*)(ws + off), __ATOMIC_RELAXED, __HIP_MEMORY_SCOPE_SYSTEM);
}
__device__ __forceinline__ bool tag_ok(unsigned long long g, unsigned call) { return (unsigned)(g >> 48) == (call & 0xFFFFu); }
__device__ __forceinline__ bool poll_granule(const Ctx& cx, const char* ws, unsigned off, unsigned long long& g) {
  g = uni64(g);
  if (!tag_ok(g, cx.call)) {
    const unsigned long long t0 = rt();
    do {
      if (cx.give_up(t0)) return false;
      g = uni64(ld_granule(ws, off));
    } while (!tag_ok(g, cx.call));
  }
  asm volatile("" ::: "memory");
  return true;
}

template <class Map>
__device__ __forceinline__ void nan_chunk(rsrc_t fin, const Map& map, int wid, int lane) {
#pragma unroll
  for (int j = 0; j < 4; ++j) st128s((u32x4){0x7FC07FC0u, 0x7FC07FC0u, 0x7FC07FC0u, 0x7FC07FC0u}, fin, map(wid, j, lane));
}

struct InTask { int kt, isw, idx; };
__device__ __forceinline__ InTask in_task(int i) {
  InTask t;
  if (i < 768) { t.kt = i / 48; const int u = i - t.kt * 48, u3 = u / 3, r3 = u - u3 * 3;
    if (r3 == 0) { t.isw = 0; t.idx = (u3 & 1) * 16 + (u3 >> 1); } else { t.isw = 1; const int c = u3 * 2 + r3 - 1; t.idx = (c & 3) * 8 + (c >> 2); } }
  else { const int u = i - 768; t.kt = (u >> 4) & 15; t.isw = 0; t.idx = (u & 1) * 16 + 8 + ((u & 15) >> 1); }
  return t;
}
__device__ __forceinline__ int otask_of(int isw, int idx, int kt) { return isw * 512 + idx * 16 + kt; }
__device__ __forceinline__ void in_load(const Params& p, const InTask& t, u32x4 (&r)[4], int wid, int lane) {
  const int rr = lane >> 2, cp = (lane & 3) ^ ((rr >> 3) << 1);
  if (!t.isw) {
    const rsrc_t rx = mkr(p.x, IN_BYTES);
#pragma unroll
    for (int j = 0; j < 4; ++j) {
      const unsigned off = (unsigned)((256 * t.idx + 32 * wid + 16 * (j >> 1) + rr) * (KS * 2) + (64 * t.kt + 32 * (j & 1) + 8 * cp) * 2);
      r[j] = __builtin_amdgcn_raw_buffer_load_b128(rx, (int)off, 0, 0);
    }
  } else {
    const rsrc_t rw = mkr(p.w, IN_BYTES);
    const unsigned off0 = (unsigned)((64 * t.kt + 32 * (wid & 1) + 8 * cp) * (ND * 2) + (256 * t.idx + 64 * (wid >> 1) + 4 * rr) * 2);
#pragma unroll
    for (int d = 0; d < 4; ++d) {
      const u32x2 a = __builtin_amdgcn_raw_buffer_load_b64(rw, (int)(off0 + (unsigned)(2 * d) * (ND * 2)), 0, 0);
      const u32x2 b = __builtin_amdgcn_raw_buffer_load_b64(rw, (int)(off0 + (unsigned)(2 * d + 1) * (ND * 2)), 0, 0);
      r[d] = __builtin_shufflevector(a, b, 0, 1, 2, 3);
    }
  }
}
__device__ __forceinline__ void in_prep(int isw, const u32x4 (&r)[4], unsigned (&v)[16]) {
  if (isw) {
#pragma unroll
    for (int d = 0; d < 4; ++d) {
      v[0 + d] = __builtin_amdgcn_perm(r[d].z, r[d].x, 0x05040100u);
      v[4 + d] = __builtin_amdgcn_perm(r[d].z, r[d].x, 0x07060302u);
      v[8 + d] = __builtin_amdgcn_perm(r[d].w, r[d].y, 0x05040100u);
      v[12 + d] = __builtin_amdgcn_perm(r[d].w, r[d].y, 0x07060302u);
    }
  } else {
#pragma unroll
    for (int j = 0; j < 4; ++j) { v[4 * j] = r[j].x; v[4 * j + 1] = r[j].y; v[4 * j + 2] = r[j].z; v[4 * j + 3] = r[j].w; }
  }
}

__device__ __forceinline__ void role_encode(const Params& p, const Ctx& cx, LAS unsigned* S, int e, int ne) {
  const int tid = threadIdx.x, wid = (int)rfl(tid >> 6), lane = tid & 63;
  char* ws = p.ws[p.rank];
  const rsrc_t lw = mkr(ws, WS_END);
  u32x4 A[4];
  in_load(p, in_task(e), A, wid, lane);
  DRAIN_ALL();
  for (int i = e; i < NIT; i += ne) {
    const InTask t = in_task(i);
    unsigned v[16];
    in_prep(t.isw, A, v);
    if (i + ne < NIT) in_load(p, in_task(i + ne), A, wid, lane);
    u32x4 P0, P1; u32x2 F;
    const unsigned long long hp = encode_chunk<false>(v, S, lane, P0, P1, F);
    const int ot = otask_of(t.isw, t.idx, t.kt);
    const bool mine = t.isw ? ((t.idx >> 3) == (p.rank & 3)) : ((t.idx >> 4) == (p.rank >> 2));
    const int img = t.isw ? 2048 + (t.idx & 7) * 128 + t.kt * 8 + p.rank : (t.idx & 15) * 128 + t.kt * 8 + p.rank;
    if (!hp_raw(hp)) {
      store_chunk<false>(lw, WS_OBI + (unsigned)ot * PKT + (unsigned)wid * 3072u, S, hp_len(hp), lane, P0, P1, F);
    } else {
#pragma unroll
      for (int j = 0; j < 4; ++j) st128s((u32x4){v[4 * j], v[4 * j + 1], v[4 * j + 2], v[4 * j + 3]}, lw, WS_OBR + (unsigned)ot * IMG + (unsigned)(wid * 4096 + j * 1024 + lane * 16));
    }
    if (mine) {
      const unsigned ib = WS_IMG + (unsigned)img * IMG;
#pragma unroll
      for (int j = 0; j < 4; ++j) st128s((u32x4){v[4 * j], v[4 * j + 1], v[4 * j + 2], v[4 * j + 3]}, lw, t.isw ? MapW{ib}(wid, j, lane) : MapX{ib}(wid, j, lane));
    }
    DRAIN_ALL();
    if (lane == 0) {
      __hip_atomic_store((gu64*)(ws + WS_HOBI + (unsigned)ot * 64u) + wid, mk_granule(hp, cx.call), __ATOMIC_RELAXED, __HIP_MEMORY_SCOPE_SYSTEM);
      if (mine) __hip_atomic_fetch_add((gu32*)(ws + WS_CNT) + img * 32, 1u, __ATOMIC_RELAXED, __HIP_MEMORY_SCOPE_AGENT);
    }
  }
}

__device__ __forceinline__ bool eo_ready(const Ctx& cx, const gu32* f, bool block) {
  if (rfl(__hip_atomic_load(f, __ATOMIC_RELAXED, __HIP_MEMORY_SCOPE_AGENT)) >= cx.call) return true;
  if (!block) return false;
  const unsigned long long t0 = rt();
  while (rfl(__hip_atomic_load(f, __ATOMIC_RELAXED, __HIP_MEMORY_SCOPE_AGENT)) < cx.call) { if (cx.give_up(t0)) return false; }
  return true;
}
__device__ __forceinline__ void role_encode_out(const Params& p, const Ctx& cx, LAS unsigned* S, int e, int ne) {
  const int tid = threadIdx.x, wid = (int)rfl(tid >> 6), lane = tid & 63;
  char* ws = p.ws[p.rank];
  const rsrc_t lw = mkr(ws, WS_END), ro = mkr(p.out[p.rank], OUT_BYTES);
  const gu32* rf = (const gu32*)(ws + WS_RFLG) + wid * 4;
  auto load = [&](int task, u32x4 (&A)[4]) { const MapO map{out_task_base(p.rank, task)};
#pragma unroll
    for (int j = 0; j < 4; ++j) A[j] = ld128s(ro, map(wid, j, lane)); };
  u32x4 A[4];
  if (!eo_ready(cx, rf + e * 32, true)) return;
  load(e, A);
  DRAIN_ALL();
  for (int i = e; i < NOT; i += ne) {
    unsigned v[16];
#pragma unroll
    for (int j = 0; j < 4; ++j) { v[4 * j] = A[j].x; v[4 * j + 1] = A[j].y; v[4 * j + 2] = A[j].z; v[4 * j + 3] = A[j].w; }
    const int nx = i + ne;
    bool have = false;
    if (nx < NOT) { have = eo_ready(cx, rf + nx * 32, false); if (have) load(nx, A); }
    u32x4 P0, P1; u32x2 F;
    const unsigned long long hp = encode_chunk<true>(v, S, lane, P0, P1, F);
    if (!hp_raw(hp)) store_chunk<true>(lw, WS_OBO + (unsigned)i * PKT + (unsigned)wid * 3072u, S, hp_len(hp), lane, P0, P1, F);
    DRAIN_ALL();
    if (lane == 0) __hip_atomic_store((gu64*)(ws + WS_HOBO + (unsigned)i * 64u) + wid, mk_granule(hp, cx.call), __ATOMIC_RELAXED, __HIP_MEMORY_SCOPE_SYSTEM);
    if (nx < NOT && !have) {
      if (!eo_ready(cx, rf + nx * 32, true)) return;
      load(nx, A);
      DRAIN_ALL();
    }
  }
}

struct PItem { unsigned hsrc, dsrc, hdst, ddst, rs, rd; int kind; };
__device__ __forceinline__ PItem p_item(int rank, int d, int i) {
  PItem r;
  if (i < 384) {
    int kt, isw, k;
    if (i < 256) { kt = i >> 4; isw = i & 1; k = (i & 15) >> 1; } else { kt = (i - 256) >> 3; isw = 0; k = 8 + ((i - 256) & 7); }
    const int idx = isw ? (d & 3) * 8 + k : (d >> 2) * 16 + k;
    const int ot = otask_of(isw, idx, kt), T = kt * 8 + rank, img = isw ? 2048 + k * 128 + T : k * 128 + T;
    const unsigned slot = CFG_SEQIN ? (unsigned)(rank * 384 + i) : (unsigned)img;
    r.hsrc = WS_HOBI + (unsigned)ot * 64u; r.dsrc = WS_OBI + (unsigned)ot * PKT;
    r.hdst = WS_HIN + slot * 64u; r.ddst = WS_PIN + slot * PKT;
    r.rs = WS_OBR + (unsigned)ot * IMG; r.rd = WS_IMG + (unsigned)img * IMG; r.kind = isw;
  } else {
    const int task = (i - 384) & (NOT - 1);
    r.hsrc = WS_HOBO + (unsigned)task * 64u; r.dsrc = WS_OBO + (unsigned)task * PKT;
    r.hdst = WS_HOUT + (unsigned)(rank * NOT + task) * 64u; r.ddst = WS_POUT + (unsigned)(rank * NOT + task) * PKT;
    r.rs = r.rd = out_task_base(rank, task); r.kind = 2;
  }
  return r;
}

#define P_LDG(ws, off) ld_granule(ws, off)

__device__ __forceinline__ void p_publish(const Ctx& cx, const char* ws, char* wd, unsigned hsrc, unsigned hdst, unsigned long long hl, int lane) {
  if (hsrc == 0xFFFFFFFFu) hl = mk_granule(0x8000ull, cx.call);
  else {
    const gu64* src = (const gu64*)(ws + hsrc) + (lane & 7);
    while (!__all(tag_ok(hl, cx.call))) hl = __hip_atomic_load(src, __ATOMIC_RELAXED, __HIP_MEMORY_SCOPE_SYSTEM);
  }
  if (lane < 8) __hip_atomic_store((gu64*)(wd + hdst) + lane, hl, __ATOMIC_RELAXED, __HIP_MEMORY_SCOPE_SYSTEM);
}

__device__ __forceinline__ void role_push(const Params& p, Ctx& cx, unsigned callraw, int link, int v0, int nv) {
  constexpr int NC = NPI * 8;
  const int lane = threadIdx.x & 63;
  int d = p.rank + 1 + link; if (d >= NRANK) d -= NRANK;
  if ((unsigned)d >= (unsigned)NRANK) return;
  char* ws = p.ws[p.rank]; char* wd = p.ws[d];
  const rsrc_t lw = mkr(ws, WS_END), dw = mkr(wd, WS_END);
  gu32* tkc = (gu32*)(ws + WS_TK) + link * 32;
  gu32* pcnt = (gu32*)(ws + WS_PCNT) + link * NPI;
  const unsigned lo = (unsigned)lane * 16u;
  constexpr int R0 = CFG_RAWHEAD;
  static_assert(R0 * PJ <= 16, "raw head chunks must be K-tile 0 items");
  int prev; unsigned prev_hsrc = 0, prev_hdst;
  {
    auto head_task = [&](int u) { const int i0 = u >> 3; InTask t; t.kt = 0; t.isw = i0 & 1; t.idx = t.isw ? (d & 3) * 8 + (i0 >> 1) : (d >> 2) * 16 + (i0 >> 1); return t; };
    u32x4 A[R0][4];
#pragma unroll
    for (int r = 0; r < R0; ++r) in_load(p, head_task(v0 + r * nv), A[r], v0 & 7, lane);
    unsigned dummy = 0;
    DRAIN_PIN2(callraw, dummy);
    cx.call = rfl(callraw) + 1u;
#pragma unroll
    for (int r = 0; r < R0; ++r) {
      const int u = v0 + r * nv, w = u & 7;
      const InTask t = head_task(u);
      unsigned v[16];
      in_prep(t.isw, A[r], v);
      const PItem f = p_item(p.rank, d, u >> 3);
      unsigned pc = 0;
      if (CFG_HEADSEQ && r > 0) { VM0(); pc = atomic_add_lane0_nowait(pcnt + (prev >> 3)); }
#pragma unroll
      for (int j = 0; j < 4; ++j) st128s((u32x4){v[4 * j], v[4 * j + 1], v[4 * j + 2], v[4 * j + 3]}, dw, t.isw ? MapW{f.rd}(w, j, lane) : MapX{f.rd}(w, j, lane));
      if (CFG_HEADSEQ && r > 0) {
        asm volatile("s_waitcnt vmcnt(4)" : "+v"(pc) :: "memory");
        if (rfl(pc) == 8u * cx.call - 1u) p_publish(cx, ws, wd, 0xFFFFFFFFu, prev_hdst, 0ull, lane);
      }
      if (r == 0) STAMP(p, 3);
      prev = u; prev_hdst = f.hdst;
    }
    if (!CFG_HEADSEQ && R0 > 1) {
      VM0();
#pragma unroll
      for (int r = 0; r + 1 < R0; ++r) {
        const int u = v0 + r * nv;
        unsigned pc = atomic_add_lane0_nowait(pcnt + (u >> 3)), dummy2 = 0;
        DRAIN_PIN2(pc, dummy2);
        if (rfl(pc) == 8u * cx.call - 1u) p_publish(cx, ws, wd, 0xFFFFFFFFu, p_item(p.rank, d, u >> 3).hdst, 0ull, lane);
      }
    }
  }
  const unsigned tkbase = (cx.call - 1u) * (unsigned)(NC - R0 * nv) - (unsigned)(R0 + 3) * (unsigned)nv;
  int it = v0 + R0 * nv, nx = v0 + (R0 + 1) * nv, n2 = v0 + (R0 + 2) * nv;
  PItem cur = p_item(p.rank, d, it >> 3);
  unsigned long long gc = P_LDG(ws, cur.hsrc + (it & 7) * 8), gn;
  DRAIN_ALL();
  gc = uni64(gc);
  if (!tag_ok(gc, cx.call)) {
    unsigned pc = atomic_add_lane0_nowait(pcnt + (prev >> 3)), dummy = 0;
    DRAIN_PIN2(pc, dummy);
    if (rfl(pc) == 8u * cx.call - 1u) p_publish(cx, ws, wd, 0xFFFFFFFFu, prev_hdst, 0ull, lane);
    prev = -1;
    if (!poll_granule(cx, ws, cur.hsrc + (it & 7) * 8, gc)) return;
  }
  const rsrc_t so = mkr(p.out[p.rank], OUT_BYTES), dd = mkr(p.out[d], OUT_BYTES);
  auto load3 = [&](const PItem& t, int u, unsigned long long g, u32x4& a, u32x4& b, u32x4& c, u32x4& e) {
    if (CFG_RAWTAIL > 0 && u >= NC - 8 * CFG_RAWTAIL) {
      const MapO map{t.rs};
      a = ld128s(so, map(u & 7, 0, lane)); b = ld128s(so, map(u & 7, 1, lane)); c = ld128s(so, map(u & 7, 2, lane)); e = ld128s(so, map(u & 7, 3, lane));
    } else {
      const unsigned o = t.dsrc + (unsigned)(u & 7) * 3072u + lo, len = hp_len(g);
      a = ld128s(lw, o); b = ld128s(lw, 1024u + lo < len ? o + 1024 : 0xFFFFFF00u); c = ld128s(lw, 2048u + lo < len ? o + 2048 : 0xFFFFFF00u);
    }
  };
  u32x4 c0, c1, c2, c3 = (u32x4){0u, 0u, 0u, 0u};
  load3(cur, it, gc, c0, c1, c2, c3);
  gn = P_LDG(ws, p_item(p.rank, d, nx >> 3).hsrc + (nx & 7) * 8);
  if (prev >= 0) prev_hsrc = 0xFFFFFFFFu;
  DRAIN_ALL();
  while (it < NC) {
    const int w = it & 7;
    unsigned tk = atomic_add_lane0_nowait(tkc);
    unsigned pc = 0; unsigned long long hl = 0;
    if (prev >= 0) {
      pc = atomic_add_lane0_nowait(pcnt + (prev >> 3));
      if (prev_hsrc != 0xFFFFFFFFu) hl = __hip_atomic_load((const gu64*)(ws + prev_hsrc) + (lane & 7), __ATOMIC_RELAXED, __HIP_MEMORY_SCOPE_SYSTEM);
    }
    const bool rawtail = CFG_RAWTAIL > 0 && it >= NC - 8 * CFG_RAWTAIL;
    if (rawtail) {
      cur.hsrc = 0xFFFFFFFFu;
      const MapO map{cur.rs};
      st128s(c0, dd, map(w, 0, lane)); st128s(c1, dd, map(w, 1, lane)); st128s(c2, dd, map(w, 2, lane)); st128s(c3, dd, map(w, 3, lane));
    } else if (!hp_raw(gc)) {
      const unsigned o = cur.ddst + (unsigned)w * 3072u + lo, len = hp_len(gc);
      st128s(c0, dw, o);
      if (1024u + lo < len) st128s(c1, dw, o + 1024);
      if (2048u + lo < len) st128s(c2, dw, o + 2048);
    } else if (cur.kind == 2) {
      const MapO map{cur.rs};
      u32x4 q[4];
#pragma unroll
      for (int j = 0; j < 4; ++j) q[j] = ld128s(so, map(w, j, lane));
#pragma unroll
      for (int j = 0; j < 4; ++j) st128s(q[j], dd, map(w, j, lane));
    } else {
      u32x4 q[4];
#pragma unroll
      for (int j = 0; j < 4; ++j) q[j] = ld128s(lw, cur.rs + (unsigned)(w * 4096 + j * 1024) + lo);
#pragma unroll
      for (int j = 0; j < 4; ++j) st128s(q[j], dw, cur.kind ? MapW{cur.rd}(w, j, lane) : MapX{cur.rd}(w, j, lane));
    }
    PItem nxt = cur; u32x4 n0 = c0, n1 = c1, n2v = c2, n3v = c3; unsigned long long g2 = 0;
    bool have = true;
    if (nx < NC) {
      nxt = p_item(p.rank, d, nx >> 3);
      gn = uni64(gn);
      have = tag_ok(gn, cx.call);
      if (have) load3(nxt, nx, gn, n0, n1, n2v, n3v);
      if (n2 < NC) g2 = P_LDG(ws, p_item(p.rank, d, n2 >> 3).hsrc + (n2 & 7) * 8);
    }
    DRAIN_PIN2(tk, pc);
    if (prev >= 0 && rfl(pc) == 8u * cx.call - 1u) p_publish(cx, ws, wd, prev_hsrc, prev_hdst, hl, lane);
    const int n3 = (int)(rfl(tk) - tkbase);
    if (!have) {
      unsigned pc2 = atomic_add_lane0_nowait(pcnt + (it >> 3)), dummy = 0;
      if (cur.hsrc != 0xFFFFFFFFu) hl = __hip_atomic_load((const gu64*)(ws + cur.hsrc) + (lane & 7), __ATOMIC_RELAXED, __HIP_MEMORY_SCOPE_SYSTEM);
      DRAIN_PIN2(pc2, dummy);
      if (rfl(pc2) == 8u * cx.call - 1u) p_publish(cx, ws, wd, cur.hsrc, cur.hdst, hl, lane);
      prev = -1;
      if (!poll_granule(cx, ws, nxt.hsrc + (nx & 7) * 8, gn)) return;
      load3(nxt, nx, gn, n0, n1, n2v, n3v);
      DRAIN_ALL();
    } else {
      prev = it; prev_hsrc = cur.hsrc; prev_hdst = cur.hdst;
    }
    cur = nxt; gc = gn; gn = g2; it = nx; nx = n2; n2 = n3;
    c0 = n0; c1 = n1; c2 = n2v; c3 = n3v;
  }
  if (prev >= 0) {
    unsigned pc = atomic_add_lane0_nowait(pcnt + (prev >> 3)), dummy = 0;
    unsigned long long hl = 0;
    if (prev_hsrc != 0xFFFFFFFFu) hl = __hip_atomic_load((const gu64*)(ws + prev_hsrc) + (lane & 7), __ATOMIC_RELAXED, __HIP_MEMORY_SCOPE_SYSTEM);
    DRAIN_PIN2(pc, dummy);
    if (rfl(pc) == 8u * cx.call - 1u) p_publish(cx, ws, wd, prev_hsrc, prev_hdst, hl, lane);
  }
  STAMP(p, 2);
}

struct DTask { unsigned hl, pk, dst; int kind; int sig; };
__device__ __forceinline__ unsigned d_map(const DTask& t, int w, int j, int l) {
  return t.kind == 0 ? MapX{t.dst}(w, j, l) : t.kind == 1 ? MapW{t.dst}(w, j, l) : MapO{t.dst}(w, j, l);
}
template <bool OUT> __device__ __forceinline__ DTask d_task(int rank, int i) {
  DTask t;
  if (!OUT) {
    int kt, sl, k, isw;
    if (i < 7 * 256) { kt = i / 112; const int u = i - kt * 112; sl = u >> 4; isw = u & 1; k = (u & 15) >> 1; }
    else { const int u = i - 7 * 256; kt = (u / 56) & 15; const int u2 = u - (u / 56) * 56; sl = u2 >> 3; isw = 0; k = 8 + (u2 & 7); }
    int src = rank + 1 + sl; if (src >= NRANK) src -= NRANK;
    const int T = kt * 8 + src, img = isw ? 2048 + k * 128 + T : k * 128 + T;
    const unsigned slot = CFG_SEQIN ? (unsigned)(src * 384 + (k < 8 ? kt * 16 + k * 2 + isw : 256 + kt * 8 + (k - 8))) : (unsigned)img;
    t.hl = WS_HIN + slot * 64u; t.pk = WS_PIN + slot * PKT;
    t.dst = WS_IMG + (unsigned)img * IMG; t.kind = isw; t.sig = img;
  } else {
    const int task = (i / (NRANK - 1)) & (NOT - 1), k = i % (NRANK - 1);
    int s = rank + 1 + k; if (s >= NRANK) s -= NRANK;
    const unsigned tidx = (unsigned)(s * NOT + task);
    t.hl = WS_HOUT + tidx * 64u; t.pk = WS_POUT + tidx * PKT; t.dst = out_task_base(s, task); t.kind = 2; t.sig = 0;
  }
  return t;
}
#define WAIT_VM8() do { asm volatile("s_waitcnt vmcnt(8)" ::: "memory"); __builtin_amdgcn_s_waitcnt(0x0F78); } while (0)

template <bool OUT>
__device__ __forceinline__ void role_decode(const Params& p, const Ctx& cx, LAS unsigned* S, int dw, int nw) {
  constexpr int TOTAL = OUT ? NOT * (NRANK - 1) : 7 * 384;
  const int tid = threadIdx.x, wid = (int)rfl(tid >> 6), lane = tid & 63;
  char* ws = p.ws[p.rank];
  const rsrc_t lw = mkr(ws, WS_END), fin = OUT ? mkr(p.out[p.rank], OUT_BYTES) : lw;
  gu32* cnt = (gu32*)(ws + WS_CNT);
  const unsigned co = (unsigned)wid * 3072u, ho = (unsigned)wid * 8u;
  int i = dw;
  if (i >= TOTAL) return;
  DTask cur = d_task<OUT>(p.rank, i), t1 = cur;
  unsigned long long gc = ld_granule(ws, cur.hl + ho), g1 = 0, g2 = 0;
  bool ok = poll_granule(cx, ws, cur.hl + ho, gc);
  Rec rc = load_rec<OUT>(lw, cur.pk + co, gc, lane), r1 = rc;
  bool have1 = false;
  if (i + nw < TOTAL) { t1 = d_task<OUT>(p.rank, i + nw); g1 = ld_granule(ws, t1.hl + ho); }
  if (i + 2 * nw < TOTAL) g2 = ld_granule(ws, d_task<OUT>(p.rank, i + 2 * nw).hl + ho);
  DRAIN_ALL();
  if (i + nw < TOTAL) { g1 = uni64(g1); have1 = tag_ok(g1, cx.call); if (have1) r1 = load_rec<OUT>(lw, t1.pk + co, g1, lane); }
  DRAIN_ALL();
  int prev = -1;
  while (ok) {
    const int n1 = i + nw, n2 = i + 2 * nw, n3 = i + 3 * nw;
    DTask t2 = t1; Rec r2 = r1; unsigned long long g3 = 0; bool have2 = false;
    if (n2 < TOTAL) {
      t2 = d_task<OUT>(p.rank, n2);
      g2 = uni64(g2);
      have2 = tag_ok(g2, cx.call);
      if (have2) r2 = load_rec<OUT>(lw, t2.pk + co, g2, lane);
      if (n3 < TOTAL) g3 = ld_granule(ws, d_task<OUT>(p.rank, n3).hl + ho);
    }
    if (!hp_raw(gc)) {
      unsigned v[16];
      decode_rec<OUT>(rc, gc, S, lane, v);
#pragma unroll
      for (int j = 0; j < 4; ++j)
        __builtin_amdgcn_raw_buffer_store_b128((u32x4){v[4 * j], v[4 * j + 1], v[4 * j + 2], v[4 * j + 3]}, fin, (int)d_map(cur, wid, j, lane), 0, OUT ? CFG_AUX_DOUT : CFG_AUX_DIN);
      if (have2) WAIT_VM8(); else DRAIN_ALL();
    } else {
      DRAIN_ALL();
    }
    if (!OUT && prev >= 0 && lane == 0) __hip_atomic_fetch_add(cnt + prev * 32, 1u, __ATOMIC_RELAXED, __HIP_MEMORY_SCOPE_AGENT);
    prev = cur.sig;
    if (n1 >= TOTAL) break;
    if (!have1) {
      if (!OUT) { VM0(); if (lane == 0) __hip_atomic_fetch_add(cnt + prev * 32, 1u, __ATOMIC_RELAXED, __HIP_MEMORY_SCOPE_AGENT); prev = -1; }
      ok = poll_granule(cx, ws, t1.hl + ho, g1);
      r1 = load_rec<OUT>(lw, t1.pk + co, g1, lane);
      DRAIN_ALL();
    }
    cur = t1; rc = r1; gc = g1; t1 = t2; r1 = r2; g1 = g2; have1 = have2; g2 = g3; i = n1;
  }
  if (!OUT) {
    VM0();
    if (ok && prev >= 0 && lane == 0) __hip_atomic_fetch_add(cnt + prev * 32, 1u, __ATOMIC_RELAXED, __HIP_MEMORY_SCOPE_AGENT);
  } else if (!ok) {
    for (; i < TOTAL; i += nw) {
      const DTask t = d_task<OUT>(p.rank, i);
#pragma unroll
      for (int j = 0; j < 4; ++j) st128s((u32x4){0x7FC07FC0u, 0x7FC07FC0u, 0x7FC07FC0u, 0x7FC07FC0u}, fin, d_map(t, wid, j, lane));
    }
  }
}

__host__ __device__ __forceinline__ int lds_byte(int r, int c) { const int st = (r >> 4) * 2 + (c >> 5), rr = r & 15, cc = c & 31, ob = rr * 64 + cc * 2; return st * 1024 + (ob ^ (((ob >> 9) & 1) << 5)); }
__device__ __forceinline__ unsigned cvt_pk_bf16(float lo, float hi) { unsigned r; asm volatile("v_cvt_pk_bf16_f32 %0, %1, %2" : "=v"(r) : "v"(lo), "v"(hi)); return r; }
__device__ __forceinline__ float gelu1(float y) {
  const float k1 = -2.0f * 0.7978845608028654f * 1.4426950408889634f, k3 = k1 * 0.044715f;
  const float t = y * (k3 * (y * y) + k1);
  return y * __builtin_amdgcn_rcpf(1.0f + __builtin_amdgcn_exp2f(t));
}

__device__ __forceinline__ void gemm_unit(const Params& p, const Ctx& cx, LAS unsigned char* lds, int pm, int pn, int h, int ph) {
  constexpr int NTH = NT / 2;
  const int tid = threadIdx.x, wid = (int)rfl(tid >> 6), lane = tid & 63, wr = wid >> 2, wc = wid & 3, fr = lane & 15, fq = lane >> 4;
  char* ws = p.ws[p.rank];
  const char* cA = ws + WS_IMG + (size_t)(pm * 128 + 8 * h) * IMG;
  const char* cB = ws + WS_IMG + (size_t)(2048 + pn * 128 + 8 * h) * IMG;
  const gu32* fl = (const gu32*)(ws + WS_CNT) + (size_t)(((lane & 2) ? 2048 + pn * 128 : pm * 128) + 8 * h + (lane & 1)) * 32;
  const unsigned need = 8u * cx.call;
  const unsigned voff = (unsigned)tid * 16u;
  const unsigned ldsw = (unsigned)wid * 1024u;
  const int aoff = lds_byte(wr * 64 + fr, fq * 8), boff = lds_byte(wc * 32 + fr, fq * 8);
  bool dead = false;
#define KT(t) ((((t) >> 3) << 4) + ((t) & 7))
  auto wait_tiles = [&](int t) {
    if (wid == 0 && !dead) {
      const gu32* f = fl + KT(t) * 32;
      if (!__all(__hip_atomic_load(f, __ATOMIC_RELAXED, __HIP_MEMORY_SCOPE_AGENT) >= need)) {
        const unsigned long long t0 = rt();
        while (!__all(__hip_atomic_load(f, __ATOMIC_RELAXED, __HIP_MEMORY_SCOPE_AGENT) >= need)) { if (cx.give_up(t0)) { dead = true; break; } }
      }
    }
  };
#define SA(b, h) (((b) * 2 + (h)) * HTB)
#define SB(b, h) ((4 + (b) * 2 + (h)) * HTB)
#define STAGE(bufoff, gbase) do { _Pragma("unroll") for (int _i = 0; _i < 2; ++_i) \
    __builtin_amdgcn_global_load_lds((const GAS unsigned*)((const char*)(gbase) + voff + _i * 8192), (LAS unsigned*)(lds + (bufoff) + ldsw + _i * 8192), 16, 0, CFG_AUX_G); } while (0)
#define LDA(dst, b, h) do { _Pragma("unroll") for (int m = 0; m < 4; ++m) _Pragma("unroll") for (int k = 0; k < 2; ++k) dst[m][k] = *(const LAS bf16x8*)(lds + SA(b, h) + aoff + m * 2048 + k * 1024); } while (0)
#define LDB(dst, b, h) do { _Pragma("unroll") for (int n = 0; n < 2; ++n) _Pragma("unroll") for (int k = 0; k < 2; ++k) dst[n][k] = *(const LAS bf16x8*)(lds + SB(b, h) + boff + n * 2048 + k * 1024); } while (0)
#define MMA(ai, bj, At, Bt) do { __builtin_amdgcn_s_setprio(1); _Pragma("unroll") for (int m = 0; m < 4; ++m) _Pragma("unroll") for (int n = 0; n < 2; ++n) _Pragma("unroll") for (int k = 0; k < 2; ++k) \
    acc[ai][bj][m][n] = __builtin_amdgcn_mfma_f32_16x16x32_bf16(Bt[n][k], At[m][k], acc[ai][bj][m][n], 0, 0, 0); __builtin_amdgcn_s_setprio(0); } while (0)
#define WAIT_V(n) asm volatile("s_waitcnt vmcnt(" #n ")" ::: "memory")
#define WAIT_L(n) asm volatile("s_waitcnt lgkmcnt(" #n ")" ::: "memory")
#define BAR __builtin_amdgcn_s_barrier()
#define SCHED __builtin_amdgcn_sched_barrier(0)
  f32x4 acc[2][2][4][2];
#pragma unroll
  for (int a = 0; a < 2; ++a)
#pragma unroll
    for (int b = 0; b < 2; ++b)
#pragma unroll
      for (int m = 0; m < 4; ++m)
#pragma unroll
        for (int n = 0; n < 2; ++n) acc[a][b][m][n] = (f32x4){0.f, 0.f, 0.f, 0.f};
  bf16x8 At[4][2], B0[2][2], B1[2][2];
  wait_tiles(0); wait_tiles(2);
  BARRIER();
  STAMP(p, 1 + 4 * ph);
  STAGE(SB(0, 0), cB); STAGE(SA(0, 0), cA); STAGE(SB(0, 1), cB + HTB); STAGE(SA(0, 1), cA + HTB);
  if (wr == 1) BAR;
  WAIT_V(4); BAR;
  STAGE(SB(1, 0), cB + IMG); STAGE(SA(1, 0), cA + IMG); STAGE(SB(1, 1), cB + IMG + HTB);
  WAIT_V(6); BAR;
  for (int t = 0; t < NTH; t += 2) {
    const bool last = (t == NTH - 2);
    const char* a1 = cA + (size_t)(KT(t) + 1) * IMG;
    const size_t o2 = last ? 0 : (size_t)KT(t + 2) * IMG;
    const char* a2 = cA + o2; const char* b2 = cB + o2;
    const char* a3 = a2 + IMG; const char* b3 = b2 + IMG;
    LAS unsigned* fslot = (LAS unsigned*)(lds + LDS_FLAG + ((t >> 1) & 1) * 16);
    const bool chk = (t + 4 < NTH);
    if (chk && wid == 0 && lane < 4) __builtin_amdgcn_global_load_lds(fl + KT(t + 4) * 32, fslot, 4, 0, AUX_SYS);
    LDB(B0, 0, 0); SCHED; LDA(At, 0, 0); STAGE(SA(1, 1), a1 + HTB);
    WAIT_L(8); BAR; WAIT_L(0); MMA(0, 0, At, B0); BAR; SCHED;
    LDB(B1, 0, 1); STAGE(SB(0, 0), b2);
    BAR; WAIT_L(0); MMA(0, 1, At, B1); BAR;
    LDA(At, 0, 1); STAGE(SA(0, 0), a2);
    BAR; WAIT_L(0); MMA(1, 0, At, B0); BAR; SCHED;
    STAGE(SB(0, 1), b2 + HTB);
    WAIT_V(6); BAR; MMA(1, 1, At, B1); BAR;
    LDB(B0, 1, 0); SCHED; LDA(At, 1, 0); STAGE(SA(0, 1), a2 + HTB);
    WAIT_L(8); BAR; WAIT_L(0); MMA(0, 0, At, B0); BAR; SCHED;
    LDB(B1, 1, 1); STAGE(SB(1, 0), b3);
    BAR; WAIT_L(0); MMA(0, 1, At, B1); BAR;
    LDA(At, 1, 1); STAGE(SA(1, 0), a3);
    BAR; WAIT_L(0); MMA(1, 0, At, B0); BAR; SCHED;
    const u32x4 f = *(const LAS u32x4*)fslot;
    STAGE(SB(1, 1), b3 + HTB);
    WAIT_V(6); BAR; MMA(1, 1, At, B1); BAR;
    if (chk) {
      const unsigned fm = rfl(min(min(f.x, f.y), min(f.z, f.w)));
      if (fm < need) { wait_tiles(t + 4); BARRIER(); }
    }
  }
  if (wr == 0) BAR;
  STAMP(p, 2 + 4 * ph);
  const int tile = pm * 8 + pn;
  {
    int tz = threadIdx.x; asm volatile("" : "+v"(tz));
    const int ln = tz & 63;
    const rsrc_t ro = mkr(p.out[p.rank], OUT_BYTES), lw = mkr(ws, WS_END);
    {
      const unsigned po = WS_PART + (unsigned)(tile * 2 + h) * 131072u + (unsigned)(wid * 16384 + ln * 16);
      const unsigned pi = WS_PART + (unsigned)(tile * 2 + 1 - h) * 131072u + (unsigned)(wid * 16384 + ln * 16);
#pragma unroll
      for (int i = 0; i < 16; ++i) {
        const f32x4 q = h ? acc[0][i >> 3][(i >> 1) & 3][i & 1] : acc[1][i >> 3][(i >> 1) & 3][i & 1];
        st128s(__builtin_bit_cast(u32x4, q), lw, po + i * 1024);
      }
      VM0();
      gu32* fo = (gu32*)(ws + WS_PFLG) + ((tile * 2 + h) * 8 + wid) * 32;
      const gu32* fi = (const gu32*)(ws + WS_PFLG) + ((tile * 2 + 1 - h) * 8 + wid) * 32;
      if (!dead && !cx.timed_out() && ln == 0) __hip_atomic_store(fo, cx.call, __ATOMIC_RELAXED, __HIP_MEMORY_SCOPE_AGENT);
      if (rfl(__hip_atomic_load(fi, __ATOMIC_RELAXED, __HIP_MEMORY_SCOPE_AGENT)) < cx.call) {
        const unsigned long long t0 = rt();
        while (rfl(__hip_atomic_load(fi, __ATOMIC_RELAXED, __HIP_MEMORY_SCOPE_AGENT)) < cx.call) { if (cx.give_up(t0)) break; }
      }
      asm volatile("" ::: "memory");
#pragma unroll
      for (int i = 0; i < 16; ++i) {
        const f32x4 q = __builtin_bit_cast(f32x4, ld128s(lw, pi + i * 1024));
        if (h) acc[1][i >> 3][(i >> 1) & 3][i & 1] += q; else acc[0][i >> 3][(i >> 1) & 3][i & 1] += q;
      }
    }
    STAMP(p, 12 + 4 * ph);
    const bool bad = cx.timed_out();
#pragma unroll
    for (int qq = 0; qq < 2; ++qq) {
      unsigned u[16];
#pragma unroll
      for (int mm = 0; mm < 2; ++mm) {
        const int m = qq * 2 + mm;
        unsigned d[8];
#pragma unroll
        for (int c = 0; c < 8; ++c) {
          const float y0 = h ? acc[1][c & 1][m][0][c >> 1] : acc[0][c & 1][m][0][c >> 1], y1 = h ? acc[1][c & 1][m][1][c >> 1] : acc[0][c & 1][m][1][c >> 1];
          d[c] = cvt_pk_bf16(gelu1(y0), gelu1(y1)); if (bad) d[c] = 0x7FC07FC0u; }
#pragma unroll
        for (int c = 0; c < 4; ++c) {
          auto r = __builtin_amdgcn_permlane16_swap(d[c], d[c + 4], false, false); d[c] = r[0]; d[c + 4] = r[1];
          auto s2 = __builtin_amdgcn_permlane32_swap(d[c], d[c + 4], false, false); d[c] = s2[0]; d[c + 4] = s2[1];
        }
#pragma unroll
        for (int c = 0; c < 8; ++c) u[8 * mm + c] = d[c];
      }
      const MapO map{out_task_base(p.rank, tile * 4 + h * 2 + qq)};
#pragma unroll
      for (int j = 0; j < 4; ++j) st128s((u32x4){u[4 * j], u[4 * j + 1], u[4 * j + 2], u[4 * j + 3]}, ro, map(wid, j, ln));
    }
    VM0();
    if (!bad && ln < 2)
      __hip_atomic_store((gu32*)(ws + WS_RFLG) + ((tile * 4 + h * 2 + ln) * 8 + wid) * 4, cx.call, __ATOMIC_RELAXED, __HIP_MEMORY_SCOPE_AGENT);
  }
  STAMP(p, 3 + 4 * ph);
#undef KT
#undef SA
#undef SB
#undef STAGE
#undef LDA
#undef LDB
#undef MMA
#undef WAIT_V
#undef WAIT_L
#undef BAR
#undef SCHED
}

__global__ __launch_bounds__(NTHR, 2) void fused_kernel(Params p) {
  extern __shared__ __attribute__((aligned(16))) unsigned char shm[];
  LAS unsigned char* lds = (LAS unsigned char*)shm;
  const int b = blockIdx.x, xcd = b & 7, slot = b >> 3;
  char* ws = p.ws[p.rank];
  const int wave = (int)rfl(threadIdx.x >> 6);
  unsigned callraw = atomic_add_lane0_nowait((gu32*)(ws + WS_CALL) + (b * 8 + wave) * 4);
  Ctx cx; cx.tmo = (gu32*)(ws + WS_TMO); cx.call = 0;
  LAS unsigned* S = (LAS unsigned*)(lds + LDS_SCR + wave * SCR_BYTES);
  STAMP(p, 0);
  if (b >= N_P) { unsigned dummy = 0; DRAIN_PIN2(callraw, dummy); cx.call = rfl(callraw) + 1u; }
  if (b < N_P) {
    { const int gw = b * 8 + wave; role_push(p, cx, callraw, gw % 7, gw / 7, PJ * 8); }
  } else if (b < N_P + N_EO) {
    role_encode_out(p, cx, S, b - N_P, N_EO);
    STAMP(p, 2);
  } else if (slot < 16) {
    const int dw = b - N_P - N_EO;
    role_decode<false>(p, cx, S, dw, N_D);
    STAMP(p, 1);
    role_decode<true>(p, cx, S, dw, N_D + 128);
    STAMP(p, 2);
  } else {
    const int g = slot - 16, h = xcd & 1, q = xcd >> 1;
    const int pm = (q >> 1) * 4 + (g & 3), pn = (q & 1) * 4 + (g >> 2);
    role_encode(p, cx, S, b - 128, 128);
    STAMP(p, 9);
#pragma unroll 1
    for (int ph = 0; ph < 2; ++ph) gemm_unit(p, cx, lds, pm + 8 * ph, pn, h, ph);
    role_decode<true>(p, cx, S, N_D + (b - 128), N_D + 128);
    STAMP(p, 10);
  }
}

}

extern "C" void kernel_launch(int rank, int nranks, ncclComm_t comm, hipStream_t stream,
                              void* const* d_in, const int* in_sizes, int n_in,
                              void* d_out, int out_size,
                              const int* in_dtypes, int out_dtype,
                              void* const* d_out_all, void* const* d_ws_all, size_t ws_bytes) {
  using namespace fk;
  (void)comm; (void)in_dtypes; (void)out_dtype;
  if (nranks != NRANK || n_in < 2 || rank < 0 || rank >= NRANK || ws_bytes < (size_t)WS_END ||
      in_sizes[0] != MD * KS || in_sizes[1] != KS * ND || out_size != MD * ND) {
    fprintf(stderr, "kernel_launch: unexpected problem shape\n");
    return;
  }
  static bool attr_set[64] = {};
  int dev = 0;
  if (hipGetDevice(&dev) != hipSuccess) return;
  if (dev >= 0 && dev < 64 && !attr_set[dev]) {
    hipError_t e = hipFuncSetAttribute((const void*)fused_kernel, hipFuncAttributeMaxDynamicSharedMemorySize, LDS_BYTES);
    if (e != hipSuccess) { fprintf(stderr, "HIP error %s\n", hipGetErrorString(e)); return; }
    attr_set[dev] = true;
  }
  Params p;
  p.x = (const char*)d_in[0]; p.w = (const char*)d_in[1];
  for (int i = 0; i < NRANK; ++i) { p.ws[i] = (char*)d_ws_all[i]; p.out[i] = (char*)d_out_all[i]; }
  p.out[rank] = (char*)d_out;
  p.rank = rank; p.pad = 0;
  hipLaunchKernelGGL(fused_kernel, dim3(GRID), dim3(NTHR), LDS_BYTES, stream, p);
  hipError_t e = hipGetLastError();
  if (e != hipSuccess) fprintf(stderr, "HIP launch error %s\n", hipGetErrorString(e));
}
```

```cpp
#include <hip/hip_runtime.h>
#include <rccl/rccl.h>
#include <cstdio>

namespace fk {

#define LAS __attribute__((address_space(3)))
#define GAS __attribute__((address_space(1)))
typedef unsigned short bf16_t;
typedef short bf16x8 __attribute__((ext_vector_type(8)));
typedef float f32x4 __attribute__((ext_vector_type(4)));
typedef unsigned u32x4 __attribute__((ext_vector_type(4)));
typedef unsigned u32x2 __attribute__((ext_vector_type(2)));
typedef unsigned short u16x2 __attribute__((ext_vector_type(2)));
typedef GAS unsigned gu32;
typedef GAS unsigned long long gu64;
typedef __amdgpu_buffer_rsrc_t rsrc_t;

constexpr int NRANK = 8;
constexpr int MD = 8192, ND = 8192, KD = 8192, KS = KD / NRANK;
constexpr int NT = KD / 64;
constexpr unsigned OUT_BYTES = (unsigned)MD * ND * 2u;
constexpr unsigned IN_BYTES = (unsigned)MD * KS * 2u;

constexpr int GRID = 256, NTHR = 512;
#define CFG_SLEEP 2
#define CFG_PJ 5
#define CFG_AUX_DIN 17
#define CFG_AUX_DOUT 17
#define CFG_SEQIN 1
#define CFG_GMAP 1
#define CFG_RAWTAIL 0
#define CFG_AUX_G 17
#define CFG_RAWHEAD 2
#define CFG_HEADSEQ 1
#define CFG_BSLEEP 0
#define CFG_ASLEEP 0
#define CFG_PMAP 1
constexpr int PJ = CFG_PJ;
#define CFG_NEO 24
constexpr int N_EO = CFG_NEO;
constexpr int N_P = 7 * PJ, N_D = 128 - N_P - N_EO;
constexpr int AUX_SYS = 17;

constexpr int NIMG = 3072;
constexpr unsigned IMG = 32768, PKT = 24576;
constexpr int NIT = 1024;
constexpr int NOT = 512;
constexpr int NPI = 384 + NOT;
constexpr int HCAP = 13;

constexpr unsigned WS_CALL = 0;
constexpr unsigned WS_TMO  = WS_CALL + 256 * 128;
constexpr unsigned WS_TK   = WS_TMO + 1024;
constexpr unsigned WS_CNT  = WS_TMO + 4096;
constexpr unsigned WS_PCNT = WS_CNT + NIMG * 128;
constexpr unsigned WS_HIN  = WS_PCNT + 32768;
constexpr unsigned WS_HOUT = WS_HIN + NIMG * 128;
constexpr unsigned WS_HOBI = WS_HOUT + NRANK * NOT * 128;
constexpr unsigned WS_HOBO = WS_HOBI + NIT * 128;
constexpr unsigned WS_PFLG = WS_HOBO + NOT * 128;
constexpr unsigned WS_RFLG = WS_PFLG + 2048 * 128;
constexpr unsigned WS_DBG  = WS_RFLG + NOT * 8 * 16;
constexpr unsigned WS_PIN  = 4u << 20;
constexpr unsigned WS_POUT = WS_PIN + NIMG * PKT;
constexpr unsigned WS_OBI  = WS_POUT + NRANK * NOT * PKT;
constexpr unsigned WS_OBR  = WS_OBI + NIT * PKT;
constexpr unsigned WS_OBO  = WS_OBR + NIT * IMG;
constexpr unsigned WS_PART = WS_OBO + NOT * PKT;
constexpr unsigned WS_IMG  = WS_PART + 256 * 131072;
constexpr unsigned WS_END  = WS_IMG + NIMG * IMG;
static_assert(WS_DBG + 256 * 64 * 8 <= WS_PIN, "control region overflow");
static_assert((size_t)WS_IMG + (size_t)NIMG * IMG < (1ull << 31), "workspace offsets must fit 31 bits");

constexpr unsigned long long DEADLINE = 40000000ull;

constexpr int HTB = 16384, STAGE_BYTES = 8 * HTB;
constexpr int LDS_FLAG = STAGE_BYTES;
constexpr int LDS_CALL = STAGE_BYTES + 64;
constexpr int LDS_TK = STAGE_BYTES + 96;
constexpr int LDS_DEAD = STAGE_BYTES + 80;
constexpr int LDS_GHDR = STAGE_BYTES + 128;
constexpr int LDS_HDR = 0;
constexpr int LDS_SCR = STAGE_BYTES + 512;
constexpr int LDS_BYTES = STAGE_BYTES + 512 + 8 * 1664;

struct Params {
  const char* x; const char* w;
  char* ws[NRANK]; char* out[NRANK];
  int rank; int pad;
};

__device__ __forceinline__ rsrc_t mkr(const void* p, unsigned bytes) { return __builtin_amdgcn_make_buffer_rsrc((void*)p, 0, (int)bytes, 0x00020000); }
__device__ __forceinline__ u32x4 ld128s(rsrc_t r, unsigned off) { return __builtin_amdgcn_raw_buffer_load_b128(r, (int)off, 0, AUX_SYS); }
__device__ __forceinline__ void st128s(u32x4 v, rsrc_t r, unsigned off) { __builtin_amdgcn_raw_buffer_store_b128(v, r, (int)off, 0, AUX_SYS); }
__device__ __forceinline__ unsigned long long rt() { return __builtin_amdgcn_s_memrealtime(); }
__device__ __forceinline__ unsigned rfl(unsigned v) { return (unsigned)__builtin_amdgcn_readfirstlane((int)v); }
__device__ __forceinline__ unsigned rdl(unsigned v, int l) { return (unsigned)__builtin_amdgcn_readlane((int)v, l); }
#define VM0() asm volatile("s_waitcnt vmcnt(0)" ::: "memory")
#define DRAIN_ALL() do { asm volatile("s_waitcnt vmcnt(0) lgkmcnt(0)" ::: "memory"); __builtin_amdgcn_s_waitcnt(0x0070); } while (0)
#define DRAIN_PIN2(a, b) do { asm volatile("s_waitcnt vmcnt(0) lgkmcnt(0)" : "+v"(a), "+v"(b) :: "memory"); __builtin_amdgcn_s_waitcnt(0x0070); } while (0)
#define BARRIER() do { asm volatile("" ::: "memory"); __builtin_amdgcn_s_barrier(); asm volatile("" ::: "memory"); } while (0)

#define STAMP(p, idx) do { } while (0)
#define STAMPL(p, idx) do { } while (0)

__device__ __forceinline__ unsigned atomic_add_lane0_nowait(gu32* ptr) {
  unsigned r; unsigned long long sv; const unsigned one = 1u;
  asm volatile("s_mov_b64 %1, exec\n\ts_mov_b64 exec, 1\n\tglobal_atomic_add %0, %2, %3, off sc0\n\ts_mov_b64 exec, %1"
               : "=&v"(r), "=&s"(sv) : "v"(ptr), "v"(one) : "memory");
  return r;
}

struct Ctx {
  gu32* tmo; unsigned call;
  __device__ __forceinline__ bool timed_out() const { return rfl(__hip_atomic_load(tmo, __ATOMIC_RELAXED, __HIP_MEMORY_SCOPE_AGENT)) == call; }
  __device__ __forceinline__ bool give_up(unsigned long long t0) const {
    if (timed_out() || rt() - t0 > DEADLINE) { __hip_atomic_store(tmo, call, __ATOMIC_RELAXED, __HIP_MEMORY_SCOPE_AGENT); return true; }
    __builtin_amdgcn_s_sleep(CFG_SLEEP);
    return false;
  }
};

__device__ __forceinline__ unsigned dpp_shr_add(unsigned x) {
  x += (unsigned)__builtin_amdgcn_update_dpp(0, (int)x, 0x111, 0xf, 0xf, false);
  x += (unsigned)__builtin_amdgcn_update_dpp(0, (int)x, 0x112, 0xf, 0xf, false);
  x += (unsigned)__builtin_amdgcn_update_dpp(0, (int)x, 0x114, 0xf, 0xf, false);
  x += (unsigned)__builtin_amdgcn_update_dpp(0, (int)x, 0x118, 0xf, 0xf, false);
  x += (unsigned)__builtin_amdgcn_update_dpp(0, (int)x, 0x142, 0xa, 0xf, false);
  x += (unsigned)__builtin_amdgcn_update_dpp(0, (int)x, 0x143, 0xc, 0xf, false);
  return x;
}
__device__ __forceinline__ unsigned row_max(unsigned x) {
  x = max(x, (unsigned)__builtin_amdgcn_update_dpp(0, (int)x, 0x111, 0xf, 0xf, false));
  x = max(x, (unsigned)__builtin_amdgcn_update_dpp(0, (int)x, 0x112, 0xf, 0xf, false));
  x = max(x, (unsigned)__builtin_amdgcn_update_dpp(0, (int)x, 0x114, 0xf, 0xf, false));
  x = max(x, (unsigned)__builtin_amdgcn_update_dpp(0, (int)x, 0x118, 0xf, 0xf, false));
  return x;
}
__device__ __forceinline__ unsigned wave_max_u32(unsigned x) {
  x = row_max(x);
  x = max(x, (unsigned)__builtin_amdgcn_update_dpp(0, (int)x, 0x142, 0xa, 0xf, false));
  x = max(x, (unsigned)__builtin_amdgcn_update_dpp(0, (int)x, 0x143, 0xc, 0xf, false));
  return rdl(x, 63);
}
__device__ __forceinline__ unsigned vperm(unsigned a, unsigned b, unsigned sel) { return __builtin_amdgcn_perm(a, b, sel); }
__device__ __forceinline__ unsigned lut4(unsigned tab, unsigned sel) { return __builtin_amdgcn_perm(0u, tab, sel); }
__device__ __forceinline__ unsigned bmask(unsigned e) { return __builtin_amdgcn_perm(0u, 0u, e | 0x0C0C0C0Cu); }
__device__ __forceinline__ unsigned bfi(unsigned m, unsigned a, unsigned b) { return (m & a) | (~m & b); }
__device__ __forceinline__ unsigned mul24(unsigned a, unsigned k) { unsigned r; asm("v_mul_u32_u24 %0, %1, %2" : "=v"(r) : "s"(k), "v"(a)); return r; }
__device__ __forceinline__ unsigned rank4(unsigned e) { return mul24(e, 0x010101u) << 8; }
__device__ __forceinline__ unsigned pc(unsigned e) { return (unsigned)__builtin_popcount(e); }
__device__ __forceinline__ unsigned nz4(unsigned x) { return ((x + 0x0F0F0F0Fu) >> 4) & 0x01010101u; }

constexpr int SCR_BYTES = 1664;

template <int W> __device__ __forceinline__ unsigned pack_bits(unsigned c, unsigned e) {
  const unsigned pre = rank4(e), em = bmask(e);
  if (W == 1) return __builtin_amdgcn_udot4(c, lut4(0x08040201u, pre) & em, 0u, false);
  if (W == 2) return __builtin_amdgcn_udot4(c, lut4(0x40100401u, pre) & em, 0u, false);
  return __builtin_amdgcn_udot4(c, lut4(0x00001001u, pre) & em, 0u, false) | (__builtin_amdgcn_udot4(c, lut4(0x10010000u, pre) & em, 0u, false) << 8);
}
template <int W> __device__ __forceinline__ unsigned unpack_bits(unsigned w, unsigned e) {
  unsigned s;
  if (W == 1) s = mul24(w & 0xFu, 0x00204081u) & 0x01010101u;
  else if (W == 2) { w &= 0xFFu; const unsigned t = (w | (w << 12)) & 0x000F000Fu; s = (t | (t << 6)) & 0x03030303u; }
  else if (W == 4) { w &= 0xFFFFu; const unsigned t = (w | (w << 8)) & 0x00FF00FFu; s = (t | (t << 4)) & 0x0F0F0F0Fu; }
  else s = w;
  return lut4(s, rank4(e)) & bmask(e);
}
__device__ __forceinline__ void lds_or(LAS unsigned* q, unsigned x) { if (x != 0u) __hip_atomic_fetch_or(q, x, __ATOMIC_RELAXED, __HIP_MEMORY_SCOPE_WAVEFRONT); }
__device__ __forceinline__ void flush32(LAS unsigned* S, unsigned off, unsigned acc) {
  const unsigned long long x = (unsigned long long)acc << (off & 31u);
  LAS unsigned* q = S + (off >> 5);
  lds_or(q, (unsigned)x); lds_or(q + 1, (unsigned)(x >> 32));
}
__device__ __forceinline__ void flush64(LAS unsigned* S, unsigned off, unsigned long long acc) {
  const unsigned sh = off & 31u;
  const unsigned long long x = acc << sh;
  LAS unsigned* q = S + (off >> 5);
  lds_or(q, (unsigned)x); lds_or(q + 1, (unsigned)(x >> 32)); lds_or(q + 2, ((unsigned)(acc >> 32) >> 1) >> (31u - sh));
}
__device__ __forceinline__ unsigned fetch32(const LAS unsigned* S, unsigned off) {
  const LAS unsigned* q = S + (off >> 5);
  return __builtin_amdgcn_alignbit(q[1], q[0], off & 31u);
}
__device__ __forceinline__ unsigned long long fetch64(const LAS unsigned* S, unsigned off) {
  const LAS unsigned* q = S + (off >> 5);
  const unsigned a = q[0], b = q[1], c = q[2];
  return ((unsigned long long)__builtin_amdgcn_alignbit(c, b, off & 31u) << 32) | __builtin_amdgcn_alignbit(b, a, off & 31u);
}
__device__ __forceinline__ unsigned wave_sum_lane63(unsigned x) { return dpp_shr_add(x); }
__device__ __forceinline__ unsigned sym_of(bool out, unsigned base, unsigned t) { return out ? ((base - (t & 7u)) | ((t & 8u) << 4)) & 0xFFu : (base - t) & 0xFFu; }

template <bool OUT>
__device__ __forceinline__ unsigned long long encode_chunk(const unsigned (&v)[16], LAS unsigned* S, int lane, u32x4& P0, u32x4& P1, u32x2& F) {
  u16x2 mx = __builtin_bit_cast(u16x2, v[0] & 0x7FFF7FFFu);
#pragma unroll
  for (int i = 1; i < 16; ++i) mx = __builtin_elementwise_max(mx, __builtin_bit_cast(u16x2, v[i] & 0x7FFF7FFFu));
  const unsigned base = wave_max_u32(max((unsigned)mx.x, (unsigned)mx.y)) >> (OUT ? 8 : 7);
  const unsigned B4 = base * 0x01010101u;
  *(LAS u32x4*)(S + lane * 4) = (u32x4){0u, 0u, 0u, 0u};
  if (lane < (OUT ? 34 : 1)) *(LAS u32x4*)(S + 256 + lane * 4) = (u32x4){0u, 0u, 0u, 0u};
  unsigned H[8], L[8], R[8], oor[8], anyo = 0;
#pragma unroll
  for (int g = 0; g < 8; ++g) {
    const unsigned Hs = vperm(v[2 * g + 1], v[2 * g], 0x07050301u), Ls = vperm(v[2 * g + 1], v[2 * g], 0x06040200u);
    if (OUT) { H[g] = Hs; L[g] = Ls; const unsigned D = B4 - (Hs & 0x7F7F7F7Fu); R[g] = (D & 0x07070707u) | ((Hs >> 4) & 0x08080808u); oor[g] = D & 0x78787878u; }
    else { H[g] = bfi(0x01010101u, Ls >> 7, Hs << 1); L[g] = bfi(0x80808080u, Hs, Ls); const unsigned D = B4 - H[g]; R[g] = D & 0x0F0F0F0Fu; oor[g] = D & 0xF0F0F0F0u; }
    anyo |= oor[g];
  }
  unsigned key;
  {
    unsigned long long hc = 0;
#pragma unroll
    for (int b = 0; b < 8; ++b) hc += 1ull << (((R[b >> 2] >> (8 * (b & 3))) & 15u) * 4u);
    const unsigned lo = (unsigned)hc, hi = (unsigned)(hc >> 32);
    unsigned q[8];
#pragma unroll
    for (int k = 0; k < 8; ++k) q[k] = rdl(dpp_shr_add(((lo >> (4 * k)) & 15u) | (((hi >> (4 * k)) & 15u) << 16)), 63);
    const int k7 = lane & 7;
    const unsigned qa = (k7 & 1) ? ((k7 & 2) ? q[3] : q[1]) : ((k7 & 2) ? q[2] : q[0]), qb = (k7 & 1) ? ((k7 & 2) ? q[7] : q[5]) : ((k7 & 2) ? q[6] : q[4]);
    const unsigned cnt = (((k7 & 4) ? qb : qa) >> ((lane & 8) * 2)) & 0xFFFFu;
    key = (lane < 16) ? (((cnt + 1u) << 4) | (unsigned)lane) : 0u;
  }
  unsigned tt = 0, cl = 0x1Fu;
#pragma unroll
  for (int i = 0; i < 7; ++i) {
    const unsigned bias = (OUT && i == 0 && lane >= 8 && lane < 16) ? (32u << 4) : 0u;
    const unsigned m = rdl(row_max(key + bias), 15);
    tt |= (m & 15u) << (4 * i);
    if (key + bias == m) { key = 0u; cl = (i < 3) ? (unsigned)i : (i == 3) ? 3u : (7u | ((unsigned)(i - 4) << 3)); }
  }
  cl <<= 8 * (lane & 3);
  cl |= (unsigned)__builtin_amdgcn_update_dpp(0, (int)cl, 0xB1, 0xf, 0xf, false);
  cl |= (unsigned)__builtin_amdgcn_update_dpp(0, (int)cl, 0x4E, 0xf, 0xf, false);
  const unsigned tb0 = rdl(cl, 0), tb1 = rdl(cl, 4), tb2 = rdl(cl, 8), tb3 = rdl(cl, 12);
  unsigned cls[8];
#pragma unroll
  for (int g = 0; g < 8; ++g) {
    const unsigned r7 = R[g] & 0x07070707u;
    cls[g] = bfi(bmask((R[g] >> 3) & 0x01010101u), vperm(tb3, tb2, r7), vperm(tb1, tb0, r7));
  }
  const bool has_oor = __any(anyo != 0u);
  unsigned e4[8];
#pragma unroll
  for (int g = 0; g < 8; ++g) e4[g] = 0u;
  if (has_oor) {
    const unsigned T1 = ((tt >> 4) & 15u) * 0x01010101u;
#pragma unroll
    for (int g = 0; g < 8; ++g) {
      e4[g] = nz4(OUT ? (oor[g] >> 3) : (oor[g] >> 4));
      const unsigned m = bmask(e4[g]);
      cls[g] |= m & 0x1F1F1F1Fu; R[g] = bfi(m, T1, R[g]);
    }
  }
  bool modeB = false; unsigned n0 = 0;
  if (OUT) {
    unsigned z[8], mxn = 0;
#pragma unroll
    for (int g = 0; g < 8; ++g) z[g] = lut4(0x000000FFu, cls[g] & 0x03030303u);
#pragma unroll
    for (int g = 0; g < 2; ++g) {
      const unsigned hn = (L[g] >> 4) & z[g] & 0x0F0F0F0Fu;
      mxn = max(mxn, max(max(hn & 0xFFu, (hn >> 8) & 0xFFu), max((hn >> 16) & 0xFFu, hn >> 24)));
    }
    n0 = wave_max_u32(mxn);
    unsigned viol[8], nv = 0;
    const unsigned N4 = (n0 | 0x80u) * 0x01010101u;
#pragma unroll
    for (int g = 0; g < 8; ++g) {
      const unsigned dn = (N4 - ((L[g] >> 4) & 0x0F0F0F0Fu)) & 0x7F7F7F7Fu;
      viol[g] = z[g] & bmask(((dn + 0x7D7D7D7Du) >> 7) & 0x01010101u);
      nv += pc(viol[g] & 0x01010101u);
    }
    modeB = n0 < 8u && rdl(dpp_shr_add(nv), 63) < 24u;
    if (modeB) {
#pragma unroll
      for (int g = 0; g < 8; ++g) cls[g] |= viol[g] & 0x1F1F1F1Fu;
    }
  }
  unsigned e2[8], e2b[8], e3[8], e0[8], m2[8], n2 = 0, n2b = 0, n3 = 0, n4 = 0, c0 = 0, cm2 = 0;
  const unsigned N0 = n0 * 0x01010101u;
#pragma unroll
  for (int g = 0; g < 8; ++g) {
    const unsigned c1 = cls[g] & 0x03030303u;
    e2[g] = lut4(0x01000000u, c1); e2b[g] = (cls[g] >> 2) & 0x01010101u; e3[g] = lut4(0x01000000u, (cls[g] >> 3) & 0x03030303u);
    n2 += pc(e2[g]); n2b += pc(e2b[g]); n3 += pc(e3[g]); n4 += pc(e4[g]);
    if (OUT) {
      e0[g] = modeB ? lut4(0x00000001u, c1) : 0u;
      m2[g] = e0[g] & nz4(((L[g] >> 4) & 0x0F0F0F0Fu) ^ N0);
      c0 += pc(e0[g]); cm2 += pc(m2[g]);
    } else { e0[g] = 0u; m2[g] = 0u; }
  }
  const unsigned x1 = c0 | (n2 << 16), x2 = cm2 | (n2b << 16), x3 = n3 | (n4 << 16);
  const unsigned s1 = dpp_shr_add(x1), s2 = dpp_shr_add(x2), s3 = dpp_shr_add(x3);
  const unsigned t1 = rdl(s1, 63), t2 = rdl(s2, 63), t3 = rdl(s3, 63);
  const unsigned C0 = t1 & 0xFFFFu, N2 = t1 >> 16, CM2 = t2 & 0xFFFFu, N2B = t2 >> 16, N3 = t3 & 0xFFFFu, N4c = t3 >> 16;
  const unsigned bSN = OUT ? 4u * (2048u - C0) : 0u;
  const unsigned aM1 = bSN, a2a = aM1 + C0, aM2 = a2a + N2, a2b = aM2 + CM2, a3 = a2b + 2u * N2B, a4 = a3 + 4u * N3, bits = a4 + 8u * N4c;
  const bool raw = bits > (OUT ? 12288u : 4096u);
  const unsigned len = ((OUT ? 12288u : 20480u) + bits + 255u) >> 8;
  if (!raw) {
    const unsigned p1 = s1 - x1, p2 = s2 - x2, p3 = s3 - x3;
    const unsigned oN = 4u * (32u * (unsigned)lane - (p1 & 0xFFFFu)), oM1 = aM1 + (p1 & 0xFFFFu), o2a = a2a + (p1 >> 16), oM2 = aM2 + (p2 & 0xFFFFu),
                   o2b = a2b + 2u * (p2 >> 16), o3 = a3 + 4u * (p3 & 0xFFFFu);
    unsigned o4 = a4 + 8u * (p3 >> 16);
    unsigned wM1 = 0, wM2 = 0, w2a = 0, kM1 = 0, kM2 = 0, k2a = 0, k2b = 0, kN = 0, k3 = 0, kNa = 0, k3a = 0;
    unsigned long long w2b = 0, wN = 0, w3 = 0;
#pragma unroll
    for (int g = 0; g < 8; ++g) {
      if (g == 4) {
        if (OUT) { flush64(S, oN, wN); wN = 0; kNa = kN; kN = 0; }
        flush64(S, o3, w3); w3 = 0; k3a = k3; k3 = 0;
      }
      if (OUT) {
        const unsigned hn = (L[g] >> 4) & 0x0F0F0F0Fu, eN = e0[g] ^ 0x01010101u;
        wN |= (unsigned long long)pack_bits<4>(hn, eN) << kN; kN += 4u * pc(eN);
        wM1 |= pack_bits<1>(m2[g], e0[g]) << kM1; kM1 += pc(e0[g]);
        wM2 |= pack_bits<1>(nz4(((N0 | 0x10101010u) - 0x01010101u - hn) & 0x0F0F0F0Fu), m2[g]) << kM2; kM2 += pc(m2[g]);
      }
      w2a |= pack_bits<1>(e2b[g], e2[g]) << k2a; k2a += pc(e2[g]);
      w2b |= (unsigned long long)pack_bits<2>((cls[g] >> 3) & 0x03030303u, e2b[g]) << k2b; k2b += 2u * pc(e2b[g]);
      w3 |= (unsigned long long)pack_bits<4>(R[g], e3[g]) << k3; k3 += 4u * pc(e3[g]);
    }
    if (OUT) { flush64(S, oN + kNa, wN); flush32(S, oM1, wM1); flush32(S, oM2, wM2); }
    flush32(S, o2a, w2a); flush64(S, o2b, w2b); flush64(S, o3 + k3a, w3);
    if (has_oor) {
#pragma unroll
      for (int g = 0; g < 8; ++g)
#pragma unroll
        for (int b = 0; b < 4; ++b)
          if ((e4[g] >> (8 * b)) & 1u) {
            const unsigned long long x = (unsigned long long)((H[g] >> (8 * b)) & 0xFFu) << (o4 & 31u);
            __hip_atomic_fetch_or(S + (o4 >> 5), (unsigned)x, __ATOMIC_RELAXED, __HIP_MEMORY_SCOPE_WAVEFRONT);
            __hip_atomic_fetch_or(S + (o4 >> 5) + 1, (unsigned)(x >> 32), __ATOMIC_RELAXED, __HIP_MEMORY_SCOPE_WAVEFRONT);
            o4 += 8u;
          }
    }
  }
  unsigned Fw[2];
#pragma unroll
  for (int i = 0; i < 2; ++i)
    Fw[i] = (cls[4 * i] & 0x03030303u) | ((cls[4 * i + 1] & 0x03030303u) << 2) | ((cls[4 * i + 2] & 0x03030303u) << 4) | ((cls[4 * i + 3] & 0x03030303u) << 6);
  F = (u32x2){Fw[0], Fw[1]};
  if (OUT) {
    unsigned p4[4];
#pragma unroll
    for (int i = 0; i < 4; ++i) p4[i] = (L[2 * i] & 0x0F0F0F0Fu) | ((L[2 * i + 1] & 0x0F0F0F0Fu) << 4);
    P0 = (u32x4){p4[0], p4[1], p4[2], p4[3]}; P1 = P0;
  } else {
    P0 = (u32x4){L[0], L[1], L[2], L[3]}; P1 = (u32x4){L[4], L[5], L[6], L[7]};
  }
  return (unsigned long long)(base | (raw ? 0x8000u : (len << 8))) | ((unsigned long long)tt << 16) | (modeB ? (1ull << 44) | ((unsigned long long)n0 << 45) : 0ull);
}
__device__ __forceinline__ bool hp_raw(unsigned long long hp) { return ((unsigned)hp & 0x8000u) != 0u; }
__device__ __forceinline__ unsigned hp_len(unsigned long long hp) { return (((unsigned)hp >> 8) & 127u) * 32u; }

template <bool OUT>
__device__ __forceinline__ void store_chunk(rsrc_t r, unsigned o, const LAS unsigned* S, unsigned len, int lane, u32x4 P0, u32x4 P1, u32x2 F) {
  const unsigned l16 = (unsigned)lane * 16u;
  st128s(P0, r, o + l16);
  const u32x4 sv = *(const LAS u32x4*)(S + lane * 4);
  if (OUT) {
    const u32x4 sw = *(const LAS u32x4*)(S + 256 + (lane & 31) * 4);
    __builtin_amdgcn_raw_buffer_store_b64(F, r, (int)(o + 1024 + lane * 8), 0, AUX_SYS);
    if (1536u + l16 < len) st128s(sv, r, o + 1536 + l16);
    if (lane < 32 && 2560u + l16 < len) st128s(sw, r, o + 2560 + l16);
  } else {
    st128s(P1, r, o + 1024 + l16);
    __builtin_amdgcn_raw_buffer_store_b64(F, r, (int)(o + 2048 + lane * 8), 0, AUX_SYS);
    if (lane < 32 && 2560u + l16 < len) st128s(sv, r, o + 2560 + l16);
  }
}

struct Rec { u32x4 P0, P1, sv; u32x2 F; };
template <bool OUT>
__device__ __forceinline__ Rec load_rec(rsrc_t r, unsigned o, unsigned long long hp, int lane) {
  const unsigned len = hp_len(hp), l16 = (unsigned)lane * 16u;
  Rec c;
  c.P0 = ld128s(r, o + l16);
  if (OUT) {
    c.F = __builtin_amdgcn_raw_buffer_load_b64(r, (int)(o + 1024 + lane * 8), 0, AUX_SYS);
    c.sv = ld128s(r, (1536u + l16 < len) ? o + 1536 + l16 : 0xFFFFFF00u);
    c.P1 = ld128s(r, (lane < 32 && 2560u + l16 < len) ? o + 2560 + l16 : 0xFFFFFF00u);
  } else {
    c.P1 = ld128s(r, o + 1024 + l16);
    c.F = __builtin_amdgcn_raw_buffer_load_b64(r, (int)(o + 2048 + lane * 8), 0, AUX_SYS);
    c.sv = ld128s(r, (lane < 32 && 2560u + l16 < len) ? o + 2560 + l16 : 0xFFFFFF00u);
  }
  return c;
}
template <bool OUT>
__device__ __forceinline__ void decode_rec(const Rec& rc, unsigned long long hp, LAS unsigned* S, int lane, unsigned (&v)[16]) {
  const unsigned base = (unsigned)hp & 0xFFu, tt = (unsigned)(hp >> 16) & 0xFFFFFFFu;
  const bool modeB = ((hp >> 44) & 1u) != 0u;
  const unsigned n0 = (unsigned)(hp >> 45) & 7u;
  const u32x4 P0 = rc.P0, P1 = rc.P1, sv = rc.sv, sw = rc.P1; const u32x2 F = rc.F;
  unsigned tab1 = 0, tab2 = 0;
#pragma unroll
  for (int i = 0; i < 4; ++i) tab1 |= sym_of(OUT, base, (tt >> (4 * i)) & 15u) << (8 * i);
#pragma unroll
  for (int i = 0; i < 3; ++i) tab2 |= sym_of(OUT, base, (tt >> (16 + 4 * i)) & 15u) << (8 * i);
  const unsigned B4 = base * 0x01010101u, T1 = ((tt >> 4) & 15u) * 0x01010101u, N0 = n0 * 0x01010101u;
  *(LAS u32x4*)(S + lane * 4) = sv;
  if (OUT && lane < 32) *(LAS u32x4*)(S + 256 + lane * 4) = sw;
  unsigned c1[8], e2[8], e0[8], n2 = 0, c0 = 0;
#pragma unroll
  for (int g = 0; g < 8; ++g) {
    c1[g] = (((g < 4) ? F.x : F.y) >> (2 * (g & 3))) & 0x03030303u;
    e2[g] = lut4(0x01000000u, c1[g]); n2 += pc(e2[g]);
    if (OUT) { e0[g] = modeB ? lut4(0x00000001u, c1[g]) : 0u; c0 += pc(e0[g]); } else e0[g] = 0u;
  }
  const unsigned x1 = c0 | (n2 << 16), s1 = dpp_shr_add(x1), t1 = rdl(s1, 63), p1 = s1 - x1;
  const unsigned C0 = t1 & 0xFFFFu, N2 = t1 >> 16;
  const unsigned aM1 = OUT ? 4u * (2048u - C0) : 0u, a2a = aM1 + C0, aM2 = a2a + N2;
  const unsigned oN = 4u * (32u * (unsigned)lane - (p1 & 0xFFFFu));
  unsigned hn[8], m2[8], e2b[8], cm2 = 0, n2b = 0;
  {
    unsigned w2a = fetch32(S, a2a + (p1 >> 16)), wM1 = 0; unsigned long long wN = 0, wN2 = 0;
    if (OUT) {
      wM1 = fetch32(S, aM1 + (p1 & 0xFFFFu)); wN = fetch64(S, oN);
      wN2 = fetch64(S, oN + 4u * (16u - pc(e0[0]) - pc(e0[1]) - pc(e0[2]) - pc(e0[3])));
    }
#pragma unroll
    for (int g = 0; g < 8; ++g) {
      e2b[g] = unpack_bits<1>(w2a, e2[g]); w2a >>= pc(e2[g]); n2b += pc(e2b[g]);
      if (OUT) {
        if (g == 4) wN = wN2;
        const unsigned eN = e0[g] ^ 0x01010101u;
        hn[g] = unpack_bits<4>((unsigned)wN, eN); wN >>= 4u * pc(eN);
        m2[g] = unpack_bits<1>(wM1, e0[g]); wM1 >>= pc(e0[g]); cm2 += pc(m2[g]);
      }
    }
  }
  const unsigned x2 = cm2 | (n2b << 16), s2 = dpp_shr_add(x2), t2 = rdl(s2, 63), p2 = s2 - x2;
  const unsigned CM2 = t2 & 0xFFFFu, N2B = t2 >> 16, a2b = aM2 + CM2, a3 = a2b + 2u * N2B;
  unsigned wM2 = OUT ? fetch32(S, aM2 + (p2 & 0xFFFFu)) : 0u;
  unsigned long long w2b = fetch64(S, a2b + 2u * (p2 >> 16));
  unsigned H[8], L[8], e3[8], n3 = 0;
#pragma unroll
  for (int g = 0; g < 8; ++g) {
    const unsigned c2 = unpack_bits<2>((unsigned)w2b, e2b[g]); w2b >>= 2u * pc(e2b[g]);
    e3[g] = lut4(0x01000000u, c2); n3 += pc(e3[g]);
    H[g] = bfi(bmask(e2b[g]), lut4(tab2, c2), lut4(tab1, c1[g]));
    if (OUT) {
      const unsigned b2 = unpack_bits<1>(wM2, m2[g]); wM2 >>= pc(m2[g]);
      const unsigned nib = hn[g] | ((N0 - m2[g] - b2) & bmask(e0[g]));
      const unsigned p4 = (g >> 1) == 0 ? P0.x : (g >> 1) == 1 ? P0.y : (g >> 1) == 2 ? P0.z : P0.w;
      L[g] = ((p4 >> (4 * (g & 1))) & 0x0F0F0F0Fu) | (nib << 4);
    } else {
      L[g] = g == 0 ? P0.x : g == 1 ? P0.y : g == 2 ? P0.z : g == 3 ? P0.w : g == 4 ? P1.x : g == 5 ? P1.y : g == 6 ? P1.z : P1.w;
    }
  }
  const unsigned s3 = dpp_shr_add(n3), N3 = rdl(s3, 63);
  const unsigned o3 = a3 + 4u * (s3 - n3);
  unsigned long long w3 = fetch64(S, o3);
  const unsigned long long w3b = fetch64(S, o3 + 4u * (pc(e3[0]) + pc(e3[1]) + pc(e3[2]) + pc(e3[3])));
  unsigned e4[8], any4 = 0;
#pragma unroll
  for (int g = 0; g < 8; ++g) {
    if (g == 4) w3 = w3b;
    unsigned R = unpack_bits<4>((unsigned)w3, e3[g]); w3 >>= 4u * pc(e3[g]);
    e4[g] = e3[g] & ~nz4(R ^ T1);
    R &= ~bmask(e4[g]);
    const unsigned H3 = OUT ? ((B4 - (R & 0x07070707u)) | ((R & 0x08080808u) << 4)) : (B4 - R);
    H[g] = bfi(bmask(e3[g]), H3, H[g]);
    any4 |= e4[g];
  }
  if (__any(any4 != 0u)) {
    unsigned n4 = 0;
#pragma unroll
    for (int g = 0; g < 8; ++g) n4 += pc(e4[g]);
    unsigned o4 = a3 + 4u * N3 + 8u * (dpp_shr_add(n4) - n4);
#pragma unroll
    for (int g = 0; g < 8; ++g) { H[g] = bfi(bmask(e4[g]), unpack_bits<8>(fetch32(S, o4), e4[g]), H[g]); o4 += 8u * pc(e4[g]); }
  }
#pragma unroll
  for (int g = 0; g < 8; ++g) {
    const unsigned Hs = OUT ? H[g] : bfi(0x80808080u, L[g], H[g] >> 1), Ls = OUT ? L[g] : bfi(0x80808080u, H[g] << 7, L[g]);
    v[2 * g] = vperm(Hs, Ls, 0x05010400u);
    v[2 * g + 1] = vperm(Hs, Ls, 0x07030602u);
  }
}

struct MapX { unsigned base; __device__ __forceinline__ unsigned operator()(int w, int j, int l) const { return base + (unsigned)(w * 4096 + j * 1024 + l * 16); } };
struct MapW { unsigned base; __device__ __forceinline__ unsigned operator()(int w, int j, int l) const { return base + (unsigned)((j >> 1) * 16384 + ((w >> 1) * 2 + (j & 1)) * 2048 + (w & 1) * 1024 + l * 16); } };
struct MapO { unsigned base; __device__ __forceinline__ unsigned operator()(int w, int j, int l) const { return base + (unsigned)((64 * (w >> 2) + 16 * (j >> 1) + (l & 15)) * (ND * 2) + (64 * (w & 3) + 32 * (j & 1) + 8 * (l >> 4)) * 2); } };
__device__ __forceinline__ unsigned out_task_base(int rank, int task) {
  const int tile = task >> 2, q = task & 3, pm = tile >> 3, pn = tile & 7;
  return (unsigned)(((rank >> 2) * 4096 + pm * 256 + (q >> 1) * 128 + (q & 1) * 32) * (ND * 2) + ((rank & 3) * 2048 + pn * 256) * 2);
}

__device__ __forceinline__ unsigned long long mk_granule(unsigned long long payload, unsigned call) { return payload | ((unsigned long long)(call & 0xFFFFu) << 48); }
__device__ __forceinline__ unsigned long long uni64(unsigned long long g) { return ((unsigned long long)rfl((unsigned)(g >> 32)) << 32) | rfl((unsigned)g); }
__device__ __forceinline__ unsigned long long ld_granule(const char* ws, unsigned off) {
  return __hip_atomic_load((const gu64*)(ws + off), __ATOMIC_RELAXED, __HIP_MEMORY_SCOPE_SYSTEM);
}
__device__ __forceinline__ bool tag_ok(unsigned long long g, unsigned call) { return (unsigned)(g >> 48) == (call & 0xFFFFu); }
__device__ __forceinline__ bool poll_granule(const Ctx& cx, const char* ws, unsigned off, unsigned long long& g) {
  g = uni64(g);
  if (!tag_ok(g, cx.call)) {
    const unsigned long long t0 = rt();
    do {
      if (cx.give_up(t0)) return false;
      g = uni64(ld_granule(ws, off));
    } while (!tag_ok(g, cx.call));
  }
  asm volatile("" ::: "memory");
  return true;
}

template <class Map>
__device__ __forceinline__ void nan_chunk(rsrc_t fin, const Map& map, int wid, int lane) {
#pragma unroll
  for (int j = 0; j < 4; ++j) st128s((u32x4){0x7FC07FC0u, 0x7FC07FC0u, 0x7FC07FC0u, 0x7FC07FC0u}, fin, map(wid, j, lane));
}

struct InTask { int kt, isw, idx; };
__device__ __forceinline__ InTask in_task(int i) {
  InTask t;
  if (i < 768) { t.kt = i / 48; const int u = i - t.kt * 48, u3 = u / 3, r3 = u - u3 * 3;
    if (r3 == 0) { t.isw = 0; t.idx = (u3 & 1) * 16 + (u3 >> 1); } else { t.isw = 1; const int c = u3 * 2 + r3 - 1; t.idx = (c & 3) * 8 + (c >> 2); } }
  else { const int u = i - 768; t.kt = (u >> 4) & 15; t.isw = 0; t.idx = (u & 1) * 16 + 8 + ((u & 15) >> 1); }
  return t;
}
__device__ __forceinline__ int otask_of(int isw, int idx, int kt) { return isw * 512 + idx * 16 + kt; }
__device__ __forceinline__ void in_load(const Params& p, const InTask& t, u32x4 (&r)[4], int wid, int lane) {
  const int rr = lane >> 2, cp = (lane & 3) ^ ((rr >> 3) << 1);
  if (!t.isw) {
    const rsrc_t rx = mkr(p.x, IN_BYTES);
#pragma unroll
    for (int j = 0; j < 4; ++j) {
      const unsigned off = (unsigned)((256 * t.idx + 32 * wid + 16 * (j >> 1) + rr) * (KS * 2) + (64 * t.kt + 32 * (j & 1) + 8 * cp) * 2);
      r[j] = __builtin_amdgcn_raw_buffer_load_b128(rx, (int)off, 0, 0);
    }
  } else {
    const rsrc_t rw = mkr(p.w, IN_BYTES);
    const unsigned off0 = (unsigned)((64 * t.kt + 32 * (wid & 1) + 8 * cp) * (ND * 2) + (256 * t.idx + 64 * (wid >> 1) + 4 * rr) * 2);
#pragma unroll
    for (int d = 0; d < 4; ++d) {
      const u32x2 a = __builtin_amdgcn_raw_buffer_load_b64(rw, (int)(off0 + (unsigned)(2 * d) * (ND * 2)), 0, 0);
      const u32x2 b = __builtin_amdgcn_raw_buffer_load_b64(rw, (int)(off0 + (unsigned)(2 * d + 1) * (ND * 2)), 0, 0);
      r[d] = __builtin_shufflevector(a, b, 0, 1, 2, 3);
    }
  }
}
__device__ __forceinline__ void in_prep(int isw, const u32x4 (&r)[4], unsigned (&v)[16]) {
  if (isw) {
#pragma unroll
    for (int d = 0; d < 4; ++d) {
      v[0 + d] = __builtin_amdgcn_perm(r[d].z, r[d].x, 0x05040100u);
      v[4 + d] = __builtin_amdgcn_perm(r[d].z, r[d].x, 0x07060302u);
      v[8 + d] = __builtin_amdgcn_perm(r[d].w, r[d].y, 0x05040100u);
      v[12 + d] = __builtin_amdgcn_perm(r[d].w, r[d].y, 0x07060302u);
    }
  } else {
#pragma unroll
    for (int j = 0; j < 4; ++j) { v[4 * j] = r[j].x; v[4 * j + 1] = r[j].y; v[4 * j + 2] = r[j].z; v[4 * j + 3] = r[j].w; }
  }
}

__device__ __forceinline__ void role_encode(const Params& p, const Ctx& cx, LAS unsigned* S, int e, int ne) {
  const int tid = threadIdx.x, wid = (int)rfl(tid >> 6), lane = tid & 63;
  char* ws = p.ws[p.rank];
  const rsrc_t lw = mkr(ws, WS_END);
  u32x4 A[4];
  in_load(p, in_task(e), A, wid, lane);
  DRAIN_ALL();
  for (int i = e; i < NIT; i += ne) {
    const InTask t = in_task(i);
    unsigned v[16];
    in_prep(t.isw, A, v);
    if (i + ne < NIT) in_load(p, in_task(i + ne), A, wid, lane);
    u32x4 P0, P1; u32x2 F;
    const unsigned long long hp = encode_chunk<false>(v, S, lane, P0, P1, F);
    const int ot = otask_of(t.isw, t.idx, t.kt);
    const bool mine = t.isw ? ((t.idx >> 3) == (p.rank & 3)) : ((t.idx >> 4) == (p.rank >> 2));
    const int img = t.isw ? 2048 + (t.idx & 7) * 128 + t.kt * 8 + p.rank : (t.idx & 15) * 128 + t.kt * 8 + p.rank;
    if (!hp_raw(hp)) {
      store_chunk<false>(lw, WS_OBI + (unsigned)ot * PKT + (unsigned)wid * 3072u, S, hp_len(hp), lane, P0, P1, F);
    } else {
#pragma unroll
      for (int j = 0; j < 4; ++j) st128s((u32x4){v[4 * j], v[4 * j + 1], v[4 * j + 2], v[4 * j + 3]}, lw, WS_OBR + (unsigned)ot * IMG + (unsigned)(wid * 4096 + j * 1024 + lane * 16));
    }
    if (mine) {
      const unsigned ib = WS_IMG + (unsigned)img * IMG;
#pragma unroll
      for (int j = 0; j < 4; ++j) st128s((u32x4){v[4 * j], v[4 * j + 1], v[4 * j + 2], v[4 * j + 3]}, lw, t.isw ? MapW{ib}(wid, j, lane) : MapX{ib}(wid, j, lane));
    }
    DRAIN_ALL();
    if (lane == 0) {
      __hip_atomic_store((gu64*)(ws + WS_HOBI + (unsigned)ot * 64u) + wid, mk_granule(hp, cx.call), __ATOMIC_RELAXED, __HIP_MEMORY_SCOPE_SYSTEM);
      if (mine) __hip_atomic_fetch_add((gu32*)(ws + WS_CNT) + img * 32, 1u, __ATOMIC_RELAXED, __HIP_MEMORY_SCOPE_AGENT);
    }
  }
}

__device__ __forceinline__ bool eo_ready(const Ctx& cx, const gu32* f, bool block) {
  if (rfl(__hip_atomic_load(f, __ATOMIC_RELAXED, __HIP_MEMORY_SCOPE_AGENT)) >= cx.call) return true;
  if (!block) return false;
  const unsigned long long t0 = rt();
  while (rfl(__hip_atomic_load(f, __ATOMIC_RELAXED, __HIP_MEMORY_SCOPE_AGENT)) < cx.call) { if (cx.give_up(t0)) return false; }
  return true;
}
__device__ __forceinline__ void role_encode_out(const Params& p, const Ctx& cx, LAS unsigned* S, int e, int ne) {
  const int tid = threadIdx.x, wid = (int)rfl(tid >> 6), lane = tid & 63;
  char* ws = p.ws[p.rank];
  const rsrc_t lw = mkr(ws, WS_END), ro = mkr(p.out[p.rank], OUT_BYTES);
  const gu32* rf = (const gu32*)(ws + WS_RFLG) + wid * 4;
  auto load = [&](int task, u32x4 (&A)[4]) { const MapO map{out_task_base(p.rank, task)};
#pragma unroll
    for (int j = 0; j < 4; ++j) A[j] = ld128s(ro, map(wid, j, lane)); };
  u32x4 A[4];
  if (!eo_ready(cx, rf + e * 32, true)) return;
  load(e, A);
  DRAIN_ALL();
  for (int i = e; i < NOT; i += ne) {
    unsigned v[16];
#pragma unroll
    for (int j = 0; j < 4; ++j) { v[4 * j] = A[j].x; v[4 * j + 1] = A[j].y; v[4 * j + 2] = A[j].z; v[4 * j + 3] = A[j].w; }
    const int nx = i + ne;
    bool have = false;
    if (nx < NOT) { have = eo_ready(cx, rf + nx * 32, false); if (have) load(nx, A); }
    u32x4 P0, P1; u32x2 F;
    const unsigned long long hp = encode_chunk<true>(v, S, lane, P0, P1, F);
    if (!hp_raw(hp)) store_chunk<true>(lw, WS_OBO + (unsigned)i * PKT + (unsigned)wid * 3072u, S, hp_len(hp), lane, P0, P1, F);
    DRAIN_ALL();
    if (lane == 0) __hip_atomic_store((gu64*)(ws + WS_HOBO + (unsigned)i * 64u) + wid, mk_granule(hp, cx.call), __ATOMIC_RELAXED, __HIP_MEMORY_SCOPE_SYSTEM);
    if (nx < NOT && !have) {
      if (!eo_ready(cx, rf + nx * 32, true)) return;
      load(nx, A);
      DRAIN_ALL();
    }
  }
}

struct PItem { unsigned hsrc, dsrc, hdst, ddst, rs, rd; int kind; };
__device__ __forceinline__ PItem p_item(int rank, int d, int i) {
  PItem r;
  if (i < 384) {
    int kt, isw, k;
    if (i < 256) { kt = i >> 4; isw = i & 1; k = (i & 15) >> 1; } else { kt = (i - 256) >> 3; isw = 0; k = 8 + ((i - 256) & 7); }
    const int idx = isw ? (d & 3) * 8 + k : (d >> 2) * 16 + k;
    const int ot = otask_of(isw, idx, kt), T = kt * 8 + rank, img = isw ? 2048 + k * 128 + T : k * 128 + T;
    const unsigned slot = CFG_SEQIN ? (unsigned)(rank * 384 + i) : (unsigned)img;
    r.hsrc = WS_HOBI + (unsigned)ot * 64u; r.dsrc = WS_OBI + (unsigned)ot * PKT;
    r.hdst = WS_HIN + slot * 64u; r.ddst = WS_PIN + slot * PKT;
    r.rs = WS_OBR + (unsigned)ot * IMG; r.rd = WS_IMG + (unsigned)img * IMG; r.kind = isw;
  } else {
    const int task = (i - 384) & (NOT - 1);
    r.hsrc = WS_HOBO + (unsigned)task * 64u; r.dsrc = WS_OBO + (unsigned)task * PKT;
    r.hdst = WS_HOUT + (unsigned)(rank * NOT + task) * 64u; r.ddst = WS_POUT + (unsigned)(rank * NOT + task) * PKT;
    r.rs = r.rd = out_task_base(rank, task); r.kind = 2;
  }
  return r;
}

#define P_LDG(ws, off) ld_granule(ws, off)

__device__ __forceinline__ void p_publish(const Ctx& cx, const char* ws, char* wd, unsigned hsrc, unsigned hdst, unsigned long long hl, int lane) {
  if (hsrc == 0xFFFFFFFFu) hl = mk_granule(0x8000ull, cx.call);
  else {
    const gu64* src = (const gu64*)(ws + hsrc) + (lane & 7);
    while (!__all(tag_ok(hl, cx.call))) hl = __hip_atomic_load(src, __ATOMIC_RELAXED, __HIP_MEMORY_SCOPE_SYSTEM);
  }
  if (lane < 8) __hip_atomic_store((gu64*)(wd + hdst) + lane, hl, __ATOMIC_RELAXED, __HIP_MEMORY_SCOPE_SYSTEM);
}

__device__ __forceinline__ void role_push(const Params& p, Ctx& cx, unsigned callraw, int link, int v0, int nv) {
  constexpr int NC = NPI * 8;
  const int lane = threadIdx.x & 63;
  int d = p.rank + 1 + link; if (d >= NRANK) d -= NRANK;
  if ((unsigned)d >= (unsigned)NRANK) return;
  char* ws = p.ws[p.rank]; char* wd = p.ws[d];
  const rsrc_t lw = mkr(ws, WS_END), dw = mkr(wd, WS_END);
  gu32* tkc = (gu32*)(ws + WS_TK) + link * 32;
  gu32* pcnt = (gu32*)(ws + WS_PCNT) + link * NPI;
  const unsigned lo = (unsigned)lane * 16u;
  constexpr int R0 = CFG_RAWHEAD;
  static_assert(R0 * PJ <= 16, "raw head chunks must be K-tile 0 items");
  int prev; unsigned prev_hsrc = 0, prev_hdst;
  {
    auto head_task = [&](int u) { const int i0 = u >> 3; InTask t; t.kt = 0; t.isw = i0 & 1; t.idx = t.isw ? (d & 3) * 8 + (i0 >> 1) : (d >> 2) * 16 + (i0 >> 1); return t; };
    u32x4 A[R0][4];
#pragma unroll
    for (int r = 0; r < R0; ++r) in_load(p, head_task(v0 + r * nv), A[r], v0 & 7, lane);
    unsigned dummy = 0;
    DRAIN_PIN2(callraw, dummy);
    cx.call = rfl(callraw) + 1u;
#pragma unroll
    for (int r = 0; r < R0; ++r) {
      const int u = v0 + r * nv, w = u & 7;
      const InTask t = head_task(u);
      unsigned v[16];
      in_prep(t.isw, A[r], v);
      const PItem f = p_item(p.rank, d, u >> 3);
      unsigned pc = 0;
      if (CFG_HEADSEQ && r > 0) { VM0(); pc = atomic_add_lane0_nowait(pcnt + (prev >> 3)); }
#pragma unroll
      for (int j = 0; j < 4; ++j) st128s((u32x4){v[4 * j], v[4 * j + 1], v[4 * j + 2], v[4 * j + 3]}, dw, t.isw ? MapW{f.rd}(w, j, lane) : MapX{f.rd}(w, j, lane));
      if (CFG_HEADSEQ && r > 0) {
        asm volatile("s_waitcnt vmcnt(4)" : "+v"(pc) :: "memory");
        if (rfl(pc) == 8u * cx.call - 1u) p_publish(cx, ws, wd, 0xFFFFFFFFu, prev_hdst, 0ull, lane);
      }
      if (r == 0) STAMP(p, 3);
      prev = u; prev_hdst = f.hdst;
    }
    if (!CFG_HEADSEQ && R0 > 1) {
      VM0();
#pragma unroll
      for (int r = 0; r + 1 < R0; ++r) {
        const int u = v0 + r * nv;
        unsigned pc = atomic_add_lane0_nowait(pcnt + (u >> 3)), dummy2 = 0;
        DRAIN_PIN2(pc, dummy2);
        if (rfl(pc) == 8u * cx.call - 1u) p_publish(cx, ws, wd, 0xFFFFFFFFu, p_item(p.rank, d, u >> 3).hdst, 0ull, lane);
      }
    }
  }
  const unsigned tkbase = (cx.call - 1u) * (unsigned)(NC - R0 * nv) - (unsigned)(R0 + 3) * (unsigned)nv;
  int it = v0 + R0 * nv, nx = v0 + (R0 + 1) * nv, n2 = v0 + (R0 + 2) * nv;
  PItem cur = p_item(p.rank, d, it >> 3);
  unsigned long long gc = P_LDG(ws, cur.hsrc + (it & 7) * 8), gn;
  DRAIN_ALL();
  gc = uni64(gc);
  if (!tag_ok(gc, cx.call)) {
    unsigned pc = atomic_add_lane0_nowait(pcnt + (prev >> 3)), dummy = 0;
    DRAIN_PIN2(pc, dummy);
    if (rfl(pc) == 8u * cx.call - 1u) p_publish(cx, ws, wd, 0xFFFFFFFFu, prev_hdst, 0ull, lane);
    prev = -1;
    if (!poll_granule(cx, ws, cur.hsrc + (it & 7) * 8, gc)) return;
  }
  const rsrc_t so = mkr(p.out[p.rank], OUT_BYTES), dd = mkr(p.out[d], OUT_BYTES);
  auto load3 = [&](const PItem& t, int u, unsigned long long g, u32x4& a, u32x4& b, u32x4& c, u32x4& e) {
    if (CFG_RAWTAIL > 0 && u >= NC - 8 * CFG_RAWTAIL) {
      const MapO map{t.rs};
      a = ld128s(so, map(u & 7, 0, lane)); b = ld128s(so, map(u & 7, 1, lane)); c = ld128s(so, map(u & 7, 2, lane)); e = ld128s(so, map(u & 7, 3, lane));
    } else {
      const unsigned o = t.dsrc + (unsigned)(u & 7) * 3072u + lo, len = hp_len(g);
      a = ld128s(lw, o); b = ld128s(lw, 1024u + lo < len ? o + 1024 : 0xFFFFFF00u); c = ld128s(lw, 2048u + lo < len ? o + 2048 : 0xFFFFFF00u);
    }
  };
  u32x4 c0, c1, c2, c3 = (u32x4){0u, 0u, 0u, 0u};
  load3(cur, it, gc, c0, c1, c2, c3);
  gn = P_LDG(ws, p_item(p.rank, d, nx >> 3).hsrc + (nx & 7) * 8);
  if (prev >= 0) prev_hsrc = 0xFFFFFFFFu;
  DRAIN_ALL();
  while (it < NC) {
    const int w = it & 7;
    unsigned tk = atomic_add_lane0_nowait(tkc);
    unsigned pc = 0; unsigned long long hl = 0;
    if (prev >= 0) {
      pc = atomic_add_lane0_nowait(pcnt + (prev >> 3));
      if (prev_hsrc != 0xFFFFFFFFu) hl = __hip_atomic_load((const gu64*)(ws + prev_hsrc) + (lane & 7), __ATOMIC_RELAXED, __HIP_MEMORY_SCOPE_SYSTEM);
    }
    const bool rawtail = CFG_RAWTAIL > 0 && it >= NC - 8 * CFG_RAWTAIL;
    if (rawtail) {
      cur.hsrc = 0xFFFFFFFFu;
      const MapO map{cur.rs};
      st128s(c0, dd, map(w, 0, lane)); st128s(c1, dd, map(w, 1, lane)); st128s(c2, dd, map(w, 2, lane)); st128s(c3, dd, map(w, 3, lane));
    } else if (!hp_raw(gc)) {
      const unsigned o = cur.ddst + (unsigned)w * 3072u + lo, len = hp_len(gc);
      st128s(c0, dw, o);
      if (1024u + lo < len) st128s(c1, dw, o + 1024);
      if (2048u + lo < len) st128s(c2, dw, o + 2048);
    } else if (cur.kind == 2) {
      const MapO map{cur.rs};
      u32x4 q[4];
#pragma unroll
      for (int j = 0; j < 4; ++j) q[j] = ld128s(so, map(w, j, lane));
#pragma unroll
      for (int j = 0; j < 4; ++j) st128s(q[j], dd, map(w, j, lane));
    } else {
      u32x4 q[4];
#pragma unroll
      for (int j = 0; j < 4; ++j) q[j] = ld128s(lw, cur.rs + (unsigned)(w * 4096 + j * 1024) + lo);
#pragma unroll
      for (int j = 0; j < 4; ++j) st128s(q[j], dw, cur.kind ? MapW{cur.rd}(w, j, lane) : MapX{cur.rd}(w, j, lane));
    }
    PItem nxt = cur; u32x4 n0 = c0, n1 = c1, n2v = c2, n3v = c3; unsigned long long g2 = 0;
    bool have = true;
    if (nx < NC) {
      nxt = p_item(p.rank, d, nx >> 3);
      gn = uni64(gn);
      have = tag_ok(gn, cx.call);
      if (have) load3(nxt, nx, gn, n0, n1, n2v, n3v);
      if (n2 < NC) g2 = P_LDG(ws, p_item(p.rank, d, n2 >> 3).hsrc + (n2 & 7) * 8);
    }
    DRAIN_PIN2(tk, pc);
    if (prev >= 0 && rfl(pc) == 8u * cx.call - 1u) p_publish(cx, ws, wd, prev_hsrc, prev_hdst, hl, lane);
    const int n3 = (int)(rfl(tk) - tkbase);
    if (!have) {
      unsigned pc2 = atomic_add_lane0_nowait(pcnt + (it >> 3)), dummy = 0;
      if (cur.hsrc != 0xFFFFFFFFu) hl = __hip_atomic_load((const gu64*)(ws + cur.hsrc) + (lane & 7), __ATOMIC_RELAXED, __HIP_MEMORY_SCOPE_SYSTEM);
      DRAIN_PIN2(pc2, dummy);
      if (rfl(pc2) == 8u * cx.call - 1u) p_publish(cx, ws, wd, cur.hsrc, cur.hdst, hl, lane);
      prev = -1;
      if (!poll_granule(cx, ws, nxt.hsrc + (nx & 7) * 8, gn)) return;
      load3(nxt, nx, gn, n0, n1, n2v, n3v);
      DRAIN_ALL();
    } else {
      prev = it; prev_hsrc = cur.hsrc; prev_hdst = cur.hdst;
    }
    cur = nxt; gc = gn; gn = g2; it = nx; nx = n2; n2 = n3;
    c0 = n0; c1 = n1; c2 = n2v; c3 = n3v;
  }
  if (prev >= 0) {
    unsigned pc = atomic_add_lane0_nowait(pcnt + (prev >> 3)), dummy = 0;
    unsigned long long hl = 0;
    if (prev_hsrc != 0xFFFFFFFFu) hl = __hip_atomic_load((const gu64*)(ws + prev_hsrc) + (lane & 7), __ATOMIC_RELAXED, __HIP_MEMORY_SCOPE_SYSTEM);
    DRAIN_PIN2(pc, dummy);
    if (rfl(pc) == 8u * cx.call - 1u) p_publish(cx, ws, wd, prev_hsrc, prev_hdst, hl, lane);
  }
  STAMPL(p, 2);
}

struct DTask { unsigned hl, pk, dst; int kind; int sig; };
__device__ __forceinline__ unsigned d_map(const DTask& t, int w, int j, int l) {
  return t.kind == 0 ? MapX{t.dst}(w, j, l) : t.kind == 1 ? MapW{t.dst}(w, j, l) : MapO{t.dst}(w, j, l);
}
template <bool OUT> __device__ __forceinline__ DTask d_task(int rank, int i) {
  DTask t;
  if (!OUT) {
    int kt, sl, k, isw;
    if (i < 7 * 256) { kt = i / 112; const int u = i - kt * 112; sl = u >> 4; isw = u & 1; k = (u & 15) >> 1; }
    else { const int u = i - 7 * 256; kt = (u / 56) & 15; const int u2 = u - (u / 56) * 56; sl = u2 >> 3; isw = 0; k = 8 + (u2 & 7); }
    int src = rank + 1 + sl; if (src >= NRANK) src -= NRANK;
    const int T = kt * 8 + src, img = isw ? 2048 + k * 128 + T : k * 128 + T;
    const unsigned slot = CFG_SEQIN ? (unsigned)(src * 384 + (k < 8 ? kt * 16 + k * 2 + isw : 256 + kt * 8 + (k - 8))) : (unsigned)img;
    t.hl = WS_HIN + slot * 64u; t.pk = WS_PIN + slot * PKT;
    t.dst = WS_IMG + (unsigned)img * IMG; t.kind = isw; t.sig = img;
  } else {
    const int task = (i / (NRANK - 1)) & (NOT - 1), k = i % (NRANK - 1);
    int s = rank + 1 + k; if (s >= NRANK) s -= NRANK;
    const unsigned tidx = (unsigned)(s * NOT + task);
    t.hl = WS_HOUT + tidx * 64u; t.pk = WS_POUT + tidx * PKT; t.dst = out_task_base(s, task); t.kind = 2; t.sig = 0;
  }
  return t;
}
#define WAIT_VM8() do { asm volatile("s_waitcnt vmcnt(8)" ::: "memory"); __builtin_amdgcn_s_waitcnt(0x0F78); } while (0)

template <bool OUT>
__device__ __forceinline__ void role_decode(const Params& p, const Ctx& cx, LAS unsigned* S, int dw, int nw) {
  constexpr int TOTAL = OUT ? NOT * (NRANK - 1) : 7 * 384;
  const int tid = threadIdx.x, wid = (int)rfl(tid >> 6), lane = tid & 63;
  char* ws = p.ws[p.rank];
  const rsrc_t lw = mkr(ws, WS_END), fin = OUT ? mkr(p.out[p.rank], OUT_BYTES) : lw;
  gu32* cnt = (gu32*)(ws + WS_CNT);
  const unsigned co = (unsigned)wid * 3072u, ho = (unsigned)wid * 8u;
  int i = dw;
  if (i >= TOTAL) return;
  DTask cur = d_task<OUT>(p.rank, i), t1 = cur;
  unsigned long long gc = ld_granule(ws, cur.hl + ho), g1 = 0, g2 = 0;
  bool ok = poll_granule(cx, ws, cur.hl + ho, gc);
  Rec rc = load_rec<OUT>(lw, cur.pk + co, gc, lane), r1 = rc;
  bool have1 = false;
  if (i + nw < TOTAL) { t1 = d_task<OUT>(p.rank, i + nw); g1 = ld_granule(ws, t1.hl + ho); }
  if (i + 2 * nw < TOTAL) g2 = ld_granule(ws, d_task<OUT>(p.rank, i + 2 * nw).hl + ho);
  DRAIN_ALL();
  if (i + nw < TOTAL) { g1 = uni64(g1); have1 = tag_ok(g1, cx.call); if (have1) r1 = load_rec<OUT>(lw, t1.pk + co, g1, lane); }
  DRAIN_ALL();
  int prev = -1;
  while (ok) {
    const int n1 = i + nw, n2 = i + 2 * nw, n3 = i + 3 * nw;
    DTask t2 = t1; Rec r2 = r1; unsigned long long g3 = 0; bool have2 = false;
    if (n2 < TOTAL) {
      t2 = d_task<OUT>(p.rank, n2);
      g2 = uni64(g2);
      have2 = tag_ok(g2, cx.call);
      if (have2) r2 = load_rec<OUT>(lw, t2.pk + co, g2, lane);
      if (n3 < TOTAL) g3 = ld_granule(ws, d_task<OUT>(p.rank, n3).hl + ho);
    }
    if (!hp_raw(gc)) {
      unsigned v[16];
      decode_rec<OUT>(rc, gc, S, lane, v);
#pragma unroll
      for (int j = 0; j < 4; ++j)
        __builtin_amdgcn_raw_buffer_store_b128((u32x4){v[4 * j], v[4 * j + 1], v[4 * j + 2], v[4 * j + 3]}, fin, (int)d_map(cur, wid, j, lane), 0, OUT ? CFG_AUX_DOUT : CFG_AUX_DIN);
      if (have2) WAIT_VM8(); else DRAIN_ALL();
    } else {
      DRAIN_ALL();
    }
    if (!OUT && prev >= 0 && lane == 0) __hip_atomic_fetch_add(cnt + prev * 32, 1u, __ATOMIC_RELAXED, __HIP_MEMORY_SCOPE_AGENT);
    prev = cur.sig;
    if (n1 >= TOTAL) break;
    if (!have1) {
      if (!OUT) { VM0(); if (lane == 0) __hip_atomic_fetch_add(cnt + prev * 32, 1u, __ATOMIC_RELAXED, __HIP_MEMORY_SCOPE_AGENT); prev = -1; }
      ok = poll_granule(cx, ws, t1.hl + ho, g1);
      r1 = load_rec<OUT>(lw, t1.pk + co, g1, lane);
      DRAIN_ALL();
    }
    cur = t1; rc = r1; gc = g1; t1 = t2; r1 = r2; g1 = g2; have1 = have2; g2 = g3; i = n1;
  }
  if (!OUT) {
    VM0();
    if (ok && prev >= 0 && lane == 0) __hip_atomic_fetch_add(cnt + prev * 32, 1u, __ATOMIC_RELAXED, __HIP_MEMORY_SCOPE_AGENT);
  } else if (!ok) {
    for (; i < TOTAL; i += nw) {
      const DTask t = d_task<OUT>(p.rank, i);
#pragma unroll
      for (int j = 0; j < 4; ++j) st128s((u32x4){0x7FC07FC0u, 0x7FC07FC0u, 0x7FC07FC0u, 0x7FC07FC0u}, fin, d_map(t, wid, j, lane));
    }
  }
}

__host__ __device__ __forceinline__ int lds_byte(int r, int c) { const int st = (r >> 4) * 2 + (c >> 5), rr = r & 15, cc = c & 31, ob = rr * 64 + cc * 2; return st * 1024 + (ob ^ (((ob >> 9) & 1) << 5)); }
__device__ __forceinline__ unsigned cvt_pk_bf16(float lo, float hi) { unsigned r; asm volatile("v_cvt_pk_bf16_f32 %0, %1, %2" : "=v"(r) : "v"(lo), "v"(hi)); return r; }
__device__ __forceinline__ float gelu1(float y) {
  const float k1 = -2.0f * 0.7978845608028654f * 1.4426950408889634f, k3 = k1 * 0.044715f;
  const float t = y * (k3 * (y * y) + k1);
  return y * __builtin_amdgcn_rcpf(1.0f + __builtin_amdgcn_exp2f(t));
}

__device__ __forceinline__ void gemm_unit(const Params& p, const Ctx& cx, LAS unsigned char* lds, int pm, int pn, int h, int ph) {
  constexpr int NTH = NT / 2;
  const int tid = threadIdx.x, wid = (int)rfl(tid >> 6), lane = tid & 63, wr = wid >> 2, wc = wid & 3, fr = lane & 15, fq = lane >> 4;
  char* ws = p.ws[p.rank];
  const char* cA = ws + WS_IMG + (size_t)(pm * 128 + 8 * h) * IMG;
  const char* cB = ws + WS_IMG + (size_t)(2048 + pn * 128 + 8 * h) * IMG;
  const gu32* fl = (const gu32*)(ws + WS_CNT) + (size_t)(((lane & 2) ? 2048 + pn * 128 : pm * 128) + 8 * h + (lane & 1)) * 32;
  const unsigned need = 8u * cx.call;
  const unsigned voff = (unsigned)tid * 16u;
  const unsigned ldsw = (unsigned)wid * 1024u;
  const int aoff = lds_byte(wr * 64 + fr, fq * 8), boff = lds_byte(wc * 32 + fr, fq * 8);
  bool dead = false;
#define KT(t) ((((t) >> 3) << 4) + ((t) & 7))
  auto wait_tiles = [&](int t) {
    if (wid == 0 && !dead) {
      const gu32* f = fl + KT(t) * 32;
      if (!__all(__hip_atomic_load(f, __ATOMIC_RELAXED, __HIP_MEMORY_SCOPE_AGENT) >= need)) {
        const unsigned long long t0 = rt();
        while (!__all(__hip_atomic_load(f, __ATOMIC_RELAXED, __HIP_MEMORY_SCOPE_AGENT) >= need)) { if (cx.give_up(t0)) { dead = true; break; } }
      }
    }
  };
#define SA(b, h) (((b) * 2 + (h)) * HTB)
#define SB(b, h) ((4 + (b) * 2 + (h)) * HTB)
#define STAGE(bufoff, gbase) do { _Pragma("unroll") for (int _i = 0; _i < 2; ++_i) \
    __builtin_amdgcn_global_load_lds((const GAS unsigned*)((const char*)(gbase) + voff + _i * 8192), (LAS unsigned*)(lds + (bufoff) + ldsw + _i * 8192), 16, 0, CFG_AUX_G); } while (0)
#define LDA(dst, b, h) do { _Pragma("unroll") for (int m = 0; m < 4; ++m) _Pragma("unroll") for (int k = 0; k < 2; ++k) dst[m][k] = *(const LAS bf16x8*)(lds + SA(b, h) + aoff + m * 2048 + k * 1024); } while (0)
#define LDB(dst, b, h) do { _Pragma("unroll") for (int n = 0; n < 2; ++n) _Pragma("unroll") for (int k = 0; k < 2; ++k) dst[n][k] = *(const LAS bf16x8*)(lds + SB(b, h) + boff + n * 2048 + k * 1024); } while (0)
#define MMA(ai, bj, At, Bt) do { __builtin_amdgcn_s_setprio(1); _Pragma("unroll") for (int m = 0; m < 4; ++m) _Pragma("unroll") for (int n = 0; n < 2; ++n) _Pragma("unroll") for (int k = 0; k < 2; ++k) \
    acc[ai][bj][m][n] = __builtin_amdgcn_mfma_f32_16x16x32_bf16(Bt[n][k], At[m][k], acc[ai][bj][m][n], 0, 0, 0); __builtin_amdgcn_s_setprio(0); } while (0)
#define WAIT_V(n) asm volatile("s_waitcnt vmcnt(" #n ")" ::: "memory")
#define WAIT_L(n) asm volatile("s_waitcnt lgkmcnt(" #n ")" ::: "memory")
#define BAR __builtin_amdgcn_s_barrier()
#define SCHED __builtin_amdgcn_sched_barrier(0)
  f32x4 acc[2][2][4][2];
#pragma unroll
  for (int a = 0; a < 2; ++a)
#pragma unroll
    for (int b = 0; b < 2; ++b)
#pragma unroll
      for (int m = 0; m < 4; ++m)
#pragma unroll
        for (int n = 0; n < 2; ++n) acc[a][b][m][n] = (f32x4){0.f, 0.f, 0.f, 0.f};
  bf16x8 At[4][2], B0[2][2], B1[2][2];
  wait_tiles(0); wait_tiles(2);
  BARRIER();
  STAMP(p, 1 + 4 * ph);
  STAGE(SB(0, 0), cB); STAGE(SA(0, 0), cA); STAGE(SB(0, 1), cB + HTB); STAGE(SA(0, 1), cA + HTB);
  if (wr == 1) BAR;
  WAIT_V(4); BAR;
  STAGE(SB(1, 0), cB + IMG); STAGE(SA(1, 0), cA + IMG); STAGE(SB(1, 1), cB + IMG + HTB);
  WAIT_V(6); BAR;
  for (int t = 0; t < NTH; t += 2) {
    const bool last = (t == NTH - 2);
    const char* a1 = cA + (size_t)(KT(t) + 1) * IMG;
    const size_t o2 = last ? 0 : (size_t)KT(t + 2) * IMG;
    const char* a2 = cA + o2; const char* b2 = cB + o2;
    const char* a3 = a2 + IMG; const char* b3 = b2 + IMG;
    LAS unsigned* fslot = (LAS unsigned*)(lds + LDS_FLAG + ((t >> 1) & 1) * 16);
    const bool chk = (t + 4 < NTH);
    if (chk && wid == 0 && lane < 4) __builtin_amdgcn_global_load_lds(fl + KT(t + 4) * 32, fslot, 4, 0, AUX_SYS);
    LDB(B0, 0, 0); SCHED; LDA(At, 0, 0); STAGE(SA(1, 1), a1 + HTB);
    WAIT_L(8); BAR; WAIT_L(0); MMA(0, 0, At, B0); BAR; SCHED;
    LDB(B1, 0, 1); STAGE(SB(0, 0), b2);
    BAR; WAIT_L(0); MMA(0, 1, At, B1); BAR;
    LDA(At, 0, 1); STAGE(SA(0, 0), a2);
    BAR; WAIT_L(0); MMA(1, 0, At, B0); BAR; SCHED;
    STAGE(SB(0, 1), b2 + HTB);
    WAIT_V(6); BAR; MMA(1, 1, At, B1); BAR;
    LDB(B0, 1, 0); SCHED; LDA(At, 1, 0); STAGE(SA(0, 1), a2 + HTB);
    WAIT_L(8); BAR; WAIT_L(0); MMA(0, 0, At, B0); BAR; SCHED;
    LDB(B1, 1, 1); STAGE(SB(1, 0), b3);
    BAR; WAIT_L(0); MMA(0, 1, At, B1); BAR;
    LDA(At, 1, 1); STAGE(SA(1, 0), a3);
    BAR; WAIT_L(0); MMA(1, 0, At, B0); BAR; SCHED;
    const u32x4 f = *(const LAS u32x4*)fslot;
    STAGE(SB(1, 1), b3 + HTB);
    WAIT_V(6); BAR; MMA(1, 1, At, B1); BAR;
    if (chk) {
      const unsigned fm = rfl(min(min(f.x, f.y), min(f.z, f.w)));
      if (fm < need) { wait_tiles(t + 4); BARRIER(); }
    }
    if (CFG_BSLEEP > 0 && ph == 1) __builtin_amdgcn_s_sleep(CFG_BSLEEP);
    if (CFG_ASLEEP > 0 && ph == 0) __builtin_amdgcn_s_sleep(CFG_ASLEEP);
  }
  if (wr == 0) BAR;
  STAMP(p, 2 + 4 * ph);
  const int tile = pm * 8 + pn;
  {
    int tz = threadIdx.x; asm volatile("" : "+v"(tz));
    const int ln = tz & 63;
    const rsrc_t ro = mkr(p.out[p.rank], OUT_BYTES), lw = mkr(ws, WS_END);
    {
      const unsigned po = WS_PART + (unsigned)(tile * 2 + h) * 131072u + (unsigned)(wid * 16384 + ln * 16);
      const unsigned pi = WS_PART + (unsigned)(tile * 2 + 1 - h) * 131072u + (unsigned)(wid * 16384 + ln * 16);
#pragma unroll
      for (int i = 0; i < 16; ++i) {
        const f32x4 q = h ? acc[0][i >> 3][(i >> 1) & 3][i & 1] : acc[1][i >> 3][(i >> 1) & 3][i & 1];
        st128s(__builtin_bit_cast(u32x4, q), lw, po + i * 1024);
      }
      VM0();
      gu32* fo = (gu32*)(ws + WS_PFLG) + ((tile * 2 + h) * 8 + wid) * 32;
      const gu32* fi = (const gu32*)(ws + WS_PFLG) + ((tile * 2 + 1 - h) * 8 + wid) * 32;
      if (!dead && !cx.timed_out() && ln == 0) __hip_atomic_store(fo, cx.call, __ATOMIC_RELAXED, __HIP_MEMORY_SCOPE_AGENT);
      if (rfl(__hip_atomic_load(fi, __ATOMIC_RELAXED, __HIP_MEMORY_SCOPE_AGENT)) < cx.call) {
        const unsigned long long t0 = rt();
        while (rfl(__hip_atomic_load(fi, __ATOMIC_RELAXED, __HIP_MEMORY_SCOPE_AGENT)) < cx.call) { if (cx.give_up(t0)) break; }
      }
      asm volatile("" ::: "memory");
#pragma unroll
      for (int i = 0; i < 16; ++i) {
        const f32x4 q = __builtin_bit_cast(f32x4, ld128s(lw, pi + i * 1024));
        if (h) acc[1][i >> 3][(i >> 1) & 3][i & 1] += q; else acc[0][i >> 3][(i >> 1) & 3][i & 1] += q;
      }
    }
    STAMP(p, 12 + 4 * ph);
    const bool bad = cx.timed_out();
#pragma unroll
    for (int qq = 0; qq < 2; ++qq) {
      unsigned u[16];
#pragma unroll
      for (int mm = 0; mm < 2; ++mm) {
        const int m = qq * 2 + mm;
        unsigned d[8];
#pragma unroll
        for (int c = 0; c < 8; ++c) {
          const float y0 = h ? acc[1][c & 1][m][0][c >> 1] : acc[0][c & 1][m][0][c >> 1], y1 = h ? acc[1][c & 1][m][1][c >> 1] : acc[0][c & 1][m][1][c >> 1];
          d[c] = cvt_pk_bf16(gelu1(y0), gelu1(y1)); if (bad) d[c] = 0x7FC07FC0u; }
#pragma unroll
        for (int c = 0; c < 4; ++c) {
          auto r = __builtin_amdgcn_permlane16_swap(d[c], d[c + 4], false, false); d[c] = r[0]; d[c + 4] = r[1];
          auto s2 = __builtin_amdgcn_permlane32_swap(d[c], d[c + 4], false, false); d[c] = s2[0]; d[c + 4] = s2[1];
        }
#pragma unroll
        for (int c = 0; c < 8; ++c) u[8 * mm + c] = d[c];
      }
      const MapO map{out_task_base(p.rank, tile * 4 + h * 2 + qq)};
#pragma unroll
      for (int j = 0; j < 4; ++j) st128s((u32x4){u[4 * j], u[4 * j + 1], u[4 * j + 2], u[4 * j + 3]}, ro, map(wid, j, ln));
    }
    VM0();
    if (!bad && ln < 2)
      __hip_atomic_store((gu32*)(ws + WS_RFLG) + ((tile * 4 + h * 2 + ln) * 8 + wid) * 4, cx.call, __ATOMIC_RELAXED, __HIP_MEMORY_SCOPE_AGENT);
  }
  STAMP(p, 3 + 4 * ph);
#undef KT
#undef SA
#undef SB
#undef STAGE
#undef LDA
#undef LDB
#undef MMA
#undef WAIT_V
#undef WAIT_L
#undef BAR
#undef SCHED
}

__global__ __launch_bounds__(NTHR, 2) void fused_kernel(Params p) {
  extern __shared__ __attribute__((aligned(16))) unsigned char shm[];
  LAS unsigned char* lds = (LAS unsigned char*)shm;
  const int b = blockIdx.x, xcd = b & 7, slot = b >> 3;
  char* ws = p.ws[p.rank];
  const int wave = (int)rfl(threadIdx.x >> 6);
  unsigned callraw = atomic_add_lane0_nowait((gu32*)(ws + WS_CALL) + (b * 8 + wave) * 4);
  Ctx cx; cx.tmo = (gu32*)(ws + WS_TMO); cx.call = 0;
  LAS unsigned* S = (LAS unsigned*)(lds + LDS_SCR + wave * SCR_BYTES);
  if (b >= N_P) { unsigned dummy = 0; DRAIN_PIN2(callraw, dummy); cx.call = rfl(callraw) + 1u; }
  if (b < N_P) {
    { const int gw = b * 8 + wave; role_push(p, cx, callraw, gw % 7, gw / 7, PJ * 8); }
  } else if (b < N_P + N_EO) {
    role_encode_out(p, cx, S, b - N_P, N_EO);
    STAMP(p, 2);
  } else if (slot < 16) {
    const int dw = b - N_P - N_EO;
    role_decode<false>(p, cx, S, dw, N_D);
    STAMP(p, 1);
    role_decode<true>(p, cx, S, dw, N_D + 128);
    STAMP(p, 2);
  } else {
    const int g = slot - 16, h = xcd & 1, q = xcd >> 1;
    const int pm = (q >> 1) * 4 + (g & 3), pn = (q & 1) * 4 + (g >> 2);
    role_encode(p, cx, S, b - 128, 128);
    STAMP(p, 9);
#pragma unroll 1
    for (int ph = 0; ph < 2; ++ph) gemm_unit(p, cx, lds, pm + 8 * ph, pn, h, ph);
    role_decode<true>(p, cx, S, N_D + (b - 128), N_D + 128);
    STAMP(p, 10);
  }
}

}

extern "C" void kernel_launch(int rank, int nranks, ncclComm_t comm, hipStream_t stream,
                              void* const* d_in, const int* in_sizes, int n_in,
                              void* d_out, int out_size,
                              const int* in_dtypes, int out_dtype,
                              void* const* d_out_all, void* const* d_ws_all, size_t ws_bytes) {
  using namespace fk;
  (void)comm; (void)in_dtypes; (void)out_dtype;
  if (nranks != NRANK || n_in < 2 || rank < 0 || rank >= NRANK || ws_bytes < (size_t)WS_END ||
      in_sizes[0] != MD * KS || in_sizes[1] != KS * ND || out_size != MD * ND) {
    fprintf(stderr, "kernel_launch: unexpected problem shape\n");
    return;
  }
  static bool attr_set[64] = {};
  int dev = 0;
  if (hipGetDevice(&dev) != hipSuccess) return;
  if (dev >= 0 && dev < 64 && !attr_set[dev]) {
    hipError_t e = hipFuncSetAttribute((const void*)fused_kernel, hipFuncAttributeMaxDynamicSharedMemorySize, LDS_BYTES);
    if (e != hipSuccess) { fprintf(stderr, "HIP error %s\n", hipGetErrorString(e)); return; }
    attr_set[dev] = true;
  }
  Params p;
  p.x = (const char*)d_in[0]; p.w = (const char*)d_in[1];
  for (int i = 0; i < NRANK; ++i) { p.ws[i] = (char*)d_ws_all[i]; p.out[i] = (char*)d_out_all[i]; }
  p.out[rank] = (char*)d_out;
  p.rank = rank; p.pad = 0;
  hipLaunchKernelGGL(fused_kernel, dim3(GRID), dim3(NTHR), LDS_BYTES, stream, p);
  hipError_t e = hipGetLastError();
  if (e != hipSuccess) fprintf(stderr, "HIP launch error %s\n", hipGetErrorString(e));
}
```

```cpp
#include <hip/hip_runtime.h>
#include <rccl/rccl.h>
#include <cstdio>

namespace fk {

#define LAS __attribute__((address_space(3)))
#define GAS __attribute__((address_space(1)))
typedef unsigned short bf16_t;
typedef short bf16x8 __attribute__((ext_vector_type(8)));
typedef float f32x4 __attribute__((ext_vector_type(4)));
typedef unsigned u32x4 __attribute__((ext_vector_type(4)));
typedef unsigned u32x2 __attribute__((ext_vector_type(2)));
typedef unsigned short u16x2 __attribute__((ext_vector_type(2)));
typedef GAS unsigned gu32;
typedef GAS unsigned long long gu64;
typedef __amdgpu_buffer_rsrc_t rsrc_t;

constexpr int NRANK = 8;
constexpr int MD = 8192, ND = 8192, KD = 8192, KS = KD / NRANK;
constexpr int NT = KD / 64;
constexpr unsigned OUT_BYTES = (unsigned)MD * ND * 2u;
constexpr unsigned IN_BYTES = (unsigned)MD * KS * 2u;

constexpr int GRID = 256, NTHR = 512;
#define CFG_SLEEP 2
#define CFG_PJ 4
#define CFG_AUX_DIN 17
#define CFG_AUX_DOUT 17
#define CFG_SEQIN 1
#define CFG_GMAP 1
#define CFG_RAWTAIL 0
#define CFG_AUX_G 17
#define CFG_RAWHEAD 3
#define CFG_HEADSEQ 1
#define CFG_BSLEEP 0
#define CFG_ASLEEP 0
#define CFG_PMAP 1
constexpr int PJ = CFG_PJ;
#define CFG_NEO 24
constexpr int N_EO = CFG_NEO;
constexpr int N_P = 7 * PJ, N_D = 128 - N_P - N_EO;
constexpr int AUX_SYS = 17;

constexpr int NIMG = 3072;
constexpr unsigned IMG = 32768, PKT = 24576;
constexpr int NIT = 1024;
constexpr int NOT = 512;
constexpr int NPI = 384 + NOT;
constexpr int HCAP = 13;

constexpr unsigned WS_CALL = 0;
constexpr unsigned WS_TMO  = WS_CALL + 256 * 128;
constexpr unsigned WS_TK   = WS_TMO + 1024;
constexpr unsigned WS_CNT  = WS_TMO + 4096;
constexpr unsigned WS_PCNT = WS_CNT + NIMG * 128;
constexpr unsigned WS_HIN  = WS_PCNT + 32768;
constexpr unsigned WS_HOUT = WS_HIN + NIMG * 128;
constexpr unsigned WS_HOBI = WS_HOUT + NRANK * NOT * 128;
constexpr unsigned WS_HOBO = WS_HOBI + NIT * 128;
constexpr unsigned WS_PFLG = WS_HOBO + NOT * 128;
constexpr unsigned WS_RFLG = WS_PFLG + 2048 * 128;
constexpr unsigned WS_DBG  = WS_RFLG + NOT * 8 * 16;
constexpr unsigned WS_PIN  = 4u << 20;
constexpr unsigned WS_POUT = WS_PIN + NIMG * PKT;
constexpr unsigned WS_OBI  = WS_POUT + NRANK * NOT * PKT;
constexpr unsigned WS_OBR  = WS_OBI + NIT * PKT;
constexpr unsigned WS_OBO  = WS_OBR + NIT * IMG;
constexpr unsigned WS_PART = WS_OBO + NOT * PKT;
constexpr unsigned WS_IMG  = WS_PART + 256 * 131072;
constexpr unsigned WS_END  = WS_IMG + NIMG * IMG;
static_assert(WS_DBG + 256 * 64 * 8 <= WS_PIN, "control region overflow");
static_assert((size_t)WS_IMG + (size_t)NIMG * IMG < (1ull << 31), "workspace offsets must fit 31 bits");

constexpr unsigned long long DEADLINE = 40000000ull;

constexpr int HTB = 16384, STAGE_BYTES = 8 * HTB;
constexpr int LDS_FLAG = STAGE_BYTES;
constexpr int LDS_CALL = STAGE_BYTES + 64;
constexpr int LDS_TK = STAGE_BYTES + 96;
constexpr int LDS_DEAD = STAGE_BYTES + 80;
constexpr int LDS_GHDR = STAGE_BYTES + 128;
constexpr int LDS_HDR = 0;
constexpr int LDS_SCR = STAGE_BYTES + 512;
constexpr int LDS_BYTES = STAGE_BYTES + 512 + 8 * 1664;

struct Params {
  const char* x; const char* w;
  char* ws[NRANK]; char* out[NRANK];
  int rank; int pad;
};

__device__ __forceinline__ rsrc_t mkr(const void* p, unsigned bytes) { return __builtin_amdgcn_make_buffer_rsrc((void*)p, 0, (int)bytes, 0x00020000); }
__device__ __forceinline__ u32x4 ld128s(rsrc_t r, unsigned off) { return __builtin_amdgcn_raw_buffer_load_b128(r, (int)off, 0, AUX_SYS); }
__device__ __forceinline__ void st128s(u32x4 v, rsrc_t r, unsigned off) { __builtin_amdgcn_raw_buffer_store_b128(v, r, (int)off, 0, AUX_SYS); }
__device__ __forceinline__ unsigned long long rt() { return __builtin_amdgcn_s_memrealtime(); }
__device__ __forceinline__ unsigned rfl(unsigned v) { return (unsigned)__builtin_amdgcn_readfirstlane((int)v); }
__device__ __forceinline__ unsigned rdl(unsigned v, int l) { return (unsigned)__builtin_amdgcn_readlane((int)v, l); }
#define VM0() do { asm volatile("s_waitcnt vmcnt(0)" ::: "memory"); __builtin_amdgcn_s_waitcnt(0x0F70); } while (0)
#define DRAIN_ALL() do { asm volatile("s_waitcnt vmcnt(0) lgkmcnt(0)" ::: "memory"); __builtin_amdgcn_s_waitcnt(0x0070); } while (0)
#define DRAIN_PIN2(a, b) do { asm volatile("s_waitcnt vmcnt(0) lgkmcnt(0)" : "+v"(a), "+v"(b) :: "memory"); __builtin_amdgcn_s_waitcnt(0x0070); } while (0)
#define BARRIER() do { asm volatile("" ::: "memory"); __builtin_amdgcn_s_barrier(); asm volatile("" ::: "memory"); } while (0)

#define STAMP(p, idx) do { } while (0)
#define STAMPL(p, idx) do { } while (0)

__device__ __forceinline__ unsigned atomic_add_lane0_nowait(gu32* ptr) {
  unsigned r; unsigned long long sv; const unsigned one = 1u;
  asm volatile("s_mov_b64 %1, exec\n\ts_mov_b64 exec, 1\n\tglobal_atomic_add %0, %2, %3, off sc0\n\ts_mov_b64 exec, %1"
               : "=&v"(r), "=&s"(sv) : "v"(ptr), "v"(one) : "memory");
  return r;
}

struct Ctx {
  gu32* tmo; unsigned call;
  __device__ __forceinline__ bool timed_out() const { return rfl(__hip_atomic_load(tmo, __ATOMIC_RELAXED, __HIP_MEMORY_SCOPE_AGENT)) == call; }
  __device__ __forceinline__ bool give_up(unsigned long long t0) const {
    if (timed_out() || rt() - t0 > DEADLINE) { __hip_atomic_store(tmo, call, __ATOMIC_RELAXED, __HIP_MEMORY_SCOPE_AGENT); return true; }
    __builtin_amdgcn_s_sleep(CFG_SLEEP);
    return false;
  }
};

__device__ __forceinline__ unsigned dpp_shr_add(unsigned x) {
  x += (unsigned)__builtin_amdgcn_update_dpp(0, (int)x, 0x111, 0xf, 0xf, false);
  x += (unsigned)__builtin_amdgcn_update_dpp(0, (int)x, 0x112, 0xf, 0xf, false);
  x += (unsigned)__builtin_amdgcn_update_dpp(0, (int)x, 0x114, 0xf, 0xf, false);
  x += (unsigned)__builtin_amdgcn_update_dpp(0, (int)x, 0x118, 0xf, 0xf, false);
  x += (unsigned)__builtin_amdgcn_update_dpp(0, (int)x, 0x142, 0xa, 0xf, false);
  x += (unsigned)__builtin_amdgcn_update_dpp(0, (int)x, 0x143, 0xc, 0xf, false);
  return x;
}
__device__ __forceinline__ unsigned row_max(unsigned x) {
  x = max(x, (unsigned)__builtin_amdgcn_update_dpp(0, (int)x, 0x111, 0xf, 0xf, false));
  x = max(x, (unsigned)__builtin_amdgcn_update_dpp(0, (int)x, 0x112, 0xf, 0xf, false));
  x = max(x, (unsigned)__builtin_amdgcn_update_dpp(0, (int)x, 0x114, 0xf, 0xf, false));
  x = max(x, (unsigned)__builtin_amdgcn_update_dpp(0, (int)x, 0x118, 0xf, 0xf, false));
  return x;
}
__device__ __forceinline__ unsigned wave_max_u32(unsigned x) {
  x = row_max(x);
  x = max(x, (unsigned)__builtin_amdgcn_update_dpp(0, (int)x, 0x142, 0xa, 0xf, false));
  x = max(x, (unsigned)__builtin_amdgcn_update_dpp(0, (int)x, 0x143, 0xc, 0xf, false));
  return rdl(x, 63);
}
__device__ __forceinline__ unsigned vperm(unsigned a, unsigned b, unsigned sel) { return __builtin_amdgcn_perm(a, b, sel); }
__device__ __forceinline__ unsigned lut4(unsigned tab, unsigned sel) { return __builtin_amdgcn_perm(0u, tab, sel); }
__device__ __forceinline__ unsigned bmask(unsigned e) { return __builtin_amdgcn_perm(0u, 0u, e | 0x0C0C0C0Cu); }
__device__ __forceinline__ unsigned bfi(unsigned m, unsigned a, unsigned b) { return (m & a) | (~m & b); }
__device__ __forceinline__ unsigned mul24(unsigned a, unsigned k) { unsigned r; asm("v_mul_u32_u24 %0, %1, %2" : "=v"(r) : "s"(k), "v"(a)); return r; }
__device__ __forceinline__ unsigned rank4(unsigned e) { return mul24(e, 0x010101u) << 8; }
__device__ __forceinline__ unsigned pc(unsigned e) { return (unsigned)__builtin_popcount(e); }
__device__ __forceinline__ unsigned nz4(unsigned x) { return ((x + 0x0F0F0F0Fu) >> 4) & 0x01010101u; }

constexpr int SCR_BYTES = 1664;

template <int W> __device__ __forceinline__ unsigned pack_bits(unsigned c, unsigned e) {
  const unsigned pre = rank4(e), em = bmask(e);
  if (W == 1) return __builtin_amdgcn_udot4(c, lut4(0x08040201u, pre) & em, 0u, false);
  if (W == 2) return __builtin_amdgcn_udot4(c, lut4(0x40100401u, pre) & em, 0u, false);
  return __builtin_amdgcn_udot4(c, lut4(0x00001001u, pre) & em, 0u, false) | (__builtin_amdgcn_udot4(c, lut4(0x10010000u, pre) & em, 0u, false) << 8);
}
template <int W> __device__ __forceinline__ unsigned unpack_bits(unsigned w, unsigned e) {
  unsigned s;
  if (W == 1) s = mul24(w & 0xFu, 0x00204081u) & 0x01010101u;
  else if (W == 2) { w &= 0xFFu; const unsigned t = (w | (w << 12)) & 0x000F000Fu; s = (t | (t << 6)) & 0x03030303u; }
  else if (W == 4) { w &= 0xFFFFu; const unsigned t = (w | (w << 8)) & 0x00FF00FFu; s = (t | (t << 4)) & 0x0F0F0F0Fu; }
  else s = w;
  return lut4(s, rank4(e)) & bmask(e);
}
__device__ __forceinline__ void lds_or(LAS unsigned* q, unsigned x) { if (x != 0u) __hip_atomic_fetch_or(q, x, __ATOMIC_RELAXED, __HIP_MEMORY_SCOPE_WAVEFRONT); }
__device__ __forceinline__ void flush32(LAS unsigned* S, unsigned off, unsigned acc) {
  const unsigned long long x = (unsigned long long)acc << (off & 31u);
  LAS unsigned* q = S + (off >> 5);
  lds_or(q, (unsigned)x); lds_or(q + 1, (unsigned)(x >> 32));
}
__device__ __forceinline__ void flush64(LAS unsigned* S, unsigned off, unsigned long long acc) {
  const unsigned sh = off & 31u;
  const unsigned long long x = acc << sh;
  LAS unsigned* q = S + (off >> 5);
  lds_or(q, (unsigned)x); lds_or(q + 1, (unsigned)(x >> 32)); lds_or(q + 2, ((unsigned)(acc >> 32) >> 1) >> (31u - sh));
}
__device__ __forceinline__ unsigned fetch32(const LAS unsigned* S, unsigned off) {
  const LAS unsigned* q = S + (off >> 5);
  return __builtin_amdgcn_alignbit(q[1], q[0], off & 31u);
}
__device__ __forceinline__ unsigned long long fetch64(const LAS unsigned* S, unsigned off) {
  const LAS unsigned* q = S + (off >> 5);
  const unsigned a = q[0], b = q[1], c = q[2];
  return ((unsigned long long)__builtin_amdgcn_alignbit(c, b, off & 31u) << 32) | __builtin_amdgcn_alignbit(b, a, off & 31u);
}
__device__ __forceinline__ unsigned wave_sum_lane63(unsigned x) { return dpp_shr_add(x); }
__device__ __forceinline__ unsigned sym_of(bool out, unsigned base, unsigned t) { return out ? ((base - (t & 7u)) | ((t & 8u) << 4)) & 0xFFu : (base - t) & 0xFFu; }

template <bool OUT>
__device__ __forceinline__ unsigned long long encode_chunk(const unsigned (&v)[16], LAS unsigned* S, int lane, u32x4& P0, u32x4& P1, u32x2& F) {
  u16x2 mx = __builtin_bit_cast(u16x2, v[0] & 0x7FFF7FFFu);
#pragma unroll
  for (int i = 1; i < 16; ++i) mx = __builtin_elementwise_max(mx, __builtin_bit_cast(u16x2, v[i] & 0x7FFF7FFFu));
  const unsigned base = wave_max_u32(max((unsigned)mx.x, (unsigned)mx.y)) >> (OUT ? 8 : 7);
  const unsigned B4 = base * 0x01010101u;
  *(LAS u32x4*)(S + lane * 4) = (u32x4){0u, 0u, 0u, 0u};
  if (lane < (OUT ? 34 : 1)) *(LAS u32x4*)(S + 256 + lane * 4) = (u32x4){0u, 0u, 0u, 0u};
  unsigned H[8], L[8], R[8], oor[8], anyo = 0;
#pragma unroll
  for (int g = 0; g < 8; ++g) {
    const unsigned Hs = vperm(v[2 * g + 1], v[2 * g], 0x07050301u), Ls = vperm(v[2 * g + 1], v[2 * g], 0x06040200u);
    if (OUT) { H[g] = Hs; L[g] = Ls; const unsigned D = B4 - (Hs & 0x7F7F7F7Fu); R[g] = (D & 0x07070707u) | ((Hs >> 4) & 0x08080808u); oor[g] = D & 0x78787878u; }
    else { H[g] = bfi(0x01010101u, Ls >> 7, Hs << 1); L[g] = bfi(0x80808080u, Hs, Ls); const unsigned D = B4 - H[g]; R[g] = D & 0x0F0F0F0Fu; oor[g] = D & 0xF0F0F0F0u; }
    anyo |= oor[g];
  }
  unsigned key;
  {
    unsigned long long hc = 0;
#pragma unroll
    for (int b = 0; b < 8; ++b) hc += 1ull << (((R[b >> 2] >> (8 * (b & 3))) & 15u) * 4u);
    const unsigned lo = (unsigned)hc, hi = (unsigned)(hc >> 32);
    unsigned q[8];
#pragma unroll
    for (int k = 0; k < 8; ++k) q[k] = rdl(dpp_shr_add(((lo >> (4 * k)) & 15u) | (((hi >> (4 * k)) & 15u) << 16)), 63);
    const int k7 = lane & 7;
    const unsigned qa = (k7 & 1) ? ((k7 & 2) ? q[3] : q[1]) : ((k7 & 2) ? q[2] : q[0]), qb = (k7 & 1) ? ((k7 & 2) ? q[7] : q[5]) : ((k7 & 2) ? q[6] : q[4]);
    const unsigned cnt = (((k7 & 4) ? qb : qa) >> ((lane & 8) * 2)) & 0xFFFFu;
    key = (lane < 16) ? (((cnt + 1u) << 4) | (unsigned)lane) : 0u;
  }
  unsigned tt = 0, cl = 0x1Fu;
#pragma unroll
  for (int i = 0; i < 7; ++i) {
    const unsigned bias = (OUT && i == 0 && lane >= 8 && lane < 16) ? (32u << 4) : 0u;
    const unsigned m = rdl(row_max(key + bias), 15);
    tt |= (m & 15u) << (4 * i);
    if (key + bias == m) { key = 0u; cl = (i < 3) ? (unsigned)i : (i == 3) ? 3u : (7u | ((unsigned)(i - 4) << 3)); }
  }
  cl <<= 8 * (lane & 3);
  cl |= (unsigned)__builtin_amdgcn_update_dpp(0, (int)cl, 0xB1, 0xf, 0xf, false);
  cl |= (unsigned)__builtin_amdgcn_update_dpp(0, (int)cl, 0x4E, 0xf, 0xf, false);
  const unsigned tb0 = rdl(cl, 0), tb1 = rdl(cl, 4), tb2 = rdl(cl, 8), tb3 = rdl(cl, 12);
  unsigned cls[8];
#pragma unroll
  for (int g = 0; g < 8; ++g) {
    const unsigned r7 = R[g] & 0x07070707u;
    cls[g] = bfi(bmask((R[g] >> 3) & 0x01010101u), vperm(tb3, tb2, r7), vperm(tb1, tb0, r7));
  }
  const bool has_oor = __any(anyo != 0u);
  unsigned e4[8];
#pragma unroll
  for (int g = 0; g < 8; ++g) e4[g] = 0u;
  if (has_oor) {
    const unsigned T1 = ((tt >> 4) & 15u) * 0x01010101u;
#pragma unroll
    for (int g = 0; g < 8; ++g) {
      e4[g] = nz4(OUT ? (oor[g] >> 3) : (oor[g] >> 4));
      const unsigned m = bmask(e4[g]);
      cls[g] |= m & 0x1F1F1F1Fu; R[g] = bfi(m, T1, R[g]);
    }
  }
  bool modeB = false; unsigned n0 = 0;
  if (OUT) {
    unsigned z[8], mxn = 0;
#pragma unroll
    for (int g = 0; g < 8; ++g) z[g] = lut4(0x000000FFu, cls[g] & 0x03030303u);
#pragma unroll
    for (int g = 0; g < 2; ++g) {
      const unsigned hn = (L[g] >> 4) & z[g] & 0x0F0F0F0Fu;
      mxn = max(mxn, max(max(hn & 0xFFu, (hn >> 8) & 0xFFu), max((hn >> 16) & 0xFFu, hn >> 24)));
    }
    n0 = wave_max_u32(mxn);
    unsigned viol[8], nv = 0;
    const unsigned N4 = (n0 | 0x80u) * 0x01010101u;
#pragma unroll
    for (int g = 0; g < 8; ++g) {
      const unsigned dn = (N4 - ((L[g] >> 4) & 0x0F0F0F0Fu)) & 0x7F7F7F7Fu;
      viol[g] = z[g] & bmask(((dn + 0x7D7D7D7Du) >> 7) & 0x01010101u);
      nv += pc(viol[g] & 0x01010101u);
    }
    modeB = n0 < 8u && rdl(dpp_shr_add(nv), 63) < 24u;
    if (modeB) {
#pragma unroll
      for (int g = 0; g < 8; ++g) cls[g] |= viol[g] & 0x1F1F1F1Fu;
    }
  }
  unsigned e2[8], e2b[8], e3[8], e0[8], m2[8], n2 = 0, n2b = 0, n3 = 0, n4 = 0, c0 = 0, cm2 = 0;
  const unsigned N0 = n0 * 0x01010101u;
#pragma unroll
  for (int g = 0; g < 8; ++g) {
    const unsigned c1 = cls[g] & 0x03030303u;
    e2[g] = lut4(0x01000000u, c1); e2b[g] = (cls[g] >> 2) & 0x01010101u; e3[g] = lut4(0x01000000u, (cls[g] >> 3) & 0x03030303u);
    n2 += pc(e2[g]); n2b += pc(e2b[g]); n3 += pc(e3[g]); n4 += pc(e4[g]);
    if (OUT) {
      e0[g] = modeB ? lut4(0x00000001u, c1) : 0u;
      m2[g] = e0[g] & nz4(((L[g] >> 4) & 0x0F0F0F0Fu) ^ N0);
      c0 += pc(e0[g]); cm2 += pc(m2[g]);
    } else { e0[g] = 0u; m2[g] = 0u; }
  }
  const unsigned x1 = c0 | (n2 << 16), x2 = cm2 | (n2b << 16), x3 = n3 | (n4 << 16);
  const unsigned s1 = dpp_shr_add(x1), s2 = dpp_shr_add(x2), s3 = dpp_shr_add(x3);
  const unsigned t1 = rdl(s1, 63), t2 = rdl(s2, 63), t3 = rdl(s3, 63);
  const unsigned C0 = t1 & 0xFFFFu, N2 = t1 >> 16, CM2 = t2 & 0xFFFFu, N2B = t2 >> 16, N3 = t3 & 0xFFFFu, N4c = t3 >> 16;
  const unsigned bSN = OUT ? 4u * (2048u - C0) : 0u;
  const unsigned aM1 = bSN, a2a = aM1 + C0, aM2 = a2a + N2, a2b = aM2 + CM2, a3 = a2b + 2u * N2B, a4 = a3 + 4u * N3, bits = a4 + 8u * N4c;
  const bool raw = bits > (OUT ? 12288u : 4096u);
  const unsigned len = ((OUT ? 12288u : 20480u) + bits + 255u) >> 8;
  if (!raw) {
    const unsigned p1 = s1 - x1, p2 = s2 - x2, p3 = s3 - x3;
    const unsigned oN = 4u * (32u * (unsigned)lane - (p1 & 0xFFFFu)), oM1 = aM1 + (p1 & 0xFFFFu), o2a = a2a + (p1 >> 16), oM2 = aM2 + (p2 & 0xFFFFu),
                   o2b = a2b + 2u * (p2 >> 16), o3 = a3 + 4u * (p3 & 0xFFFFu);
    unsigned o4 = a4 + 8u * (p3 >> 16);
    unsigned wM1 = 0, wM2 = 0, w2a = 0, kM1 = 0, kM2 = 0, k2a = 0, k2b = 0, kN = 0, k3 = 0, kNa = 0, k3a = 0;
    unsigned long long w2b = 0, wN = 0, w3 = 0;
#pragma unroll
    for (int g = 0; g < 8; ++g) {
      if (g == 4) {
        if (OUT) { flush64(S, oN, wN); wN = 0; kNa = kN; kN = 0; }
        flush64(S, o3, w3); w3 = 0; k3a = k3; k3 = 0;
      }
      if (OUT) {
        const unsigned hn = (L[g] >> 4) & 0x0F0F0F0Fu, eN = e0[g] ^ 0x01010101u;
        wN |= (unsigned long long)pack_bits<4>(hn, eN) << kN; kN += 4u * pc(eN);
        wM1 |= pack_bits<1>(m2[g], e0[g]) << kM1; kM1 += pc(e0[g]);
        wM2 |= pack_bits<1>(nz4(((N0 | 0x10101010u) - 0x01010101u - hn) & 0x0F0F0F0Fu), m2[g]) << kM2; kM2 += pc(m2[g]);
      }
      w2a |= pack_bits<1>(e2b[g], e2[g]) << k2a; k2a += pc(e2[g]);
      w2b |= (unsigned long long)pack_bits<2>((cls[g] >> 3) & 0x03030303u, e2b[g]) << k2b; k2b += 2u * pc(e2b[g]);
      w3 |= (unsigned long long)pack_bits<4>(R[g], e3[g]) << k3; k3 += 4u * pc(e3[g]);
    }
    if (OUT) { flush64(S, oN + kNa, wN); flush32(S, oM1, wM1); flush32(S, oM2, wM2); }
    flush32(S, o2a, w2a); flush64(S, o2b, w2b); flush64(S, o3 + k3a, w3);
    if (has_oor) {
#pragma unroll
      for (int g = 0; g < 8; ++g)
#pragma unroll
        for (int b = 0; b < 4; ++b)
          if ((e4[g] >> (8 * b)) & 1u) {
            const unsigned long long x = (unsigned long long)((H[g] >> (8 * b)) & 0xFFu) << (o4 & 31u);
            __hip_atomic_fetch_or(S + (o4 >> 5), (unsigned)x, __ATOMIC_RELAXED, __HIP_MEMORY_SCOPE_WAVEFRONT);
            __hip_atomic_fetch_or(S + (o4 >> 5) + 1, (unsigned)(x >> 32), __ATOMIC_RELAXED, __HIP_MEMORY_SCOPE_WAVEFRONT);
            o4 += 8u;
          }
    }
  }
  unsigned Fw[2];
#pragma unroll
  for (int i = 0; i < 2; ++i)
    Fw[i] = (cls[4 * i] & 0x03030303u) | ((cls[4 * i + 1] & 0x03030303u) << 2) | ((cls[4 * i + 2] & 0x03030303u) << 4) | ((cls[4 * i + 3] & 0x03030303u) << 6);
  F = (u32x2){Fw[0], Fw[1]};
  if (OUT) {
    unsigned p4[4];
#pragma unroll
    for (int i = 0; i < 4; ++i) p4[i] = (L[2 * i] & 0x0F0F0F0Fu) | ((L[2 * i + 1] & 0x0F0F0F0Fu) << 4);
    P0 = (u32x4){p4[0], p4[1], p4[2], p4[3]}; P1 = P0;
  } else {
    P0 = (u32x4){L[0], L[1], L[2], L[3]}; P1 = (u32x4){L[4], L[5], L[6], L[7]};
  }
  return (unsigned long long)(base | (raw ? 0x8000u : (len << 8))) | ((unsigned long long)tt << 16) | (modeB ? (1ull << 44) | ((unsigned long long)n0 << 45) : 0ull);
}
__device__ __forceinline__ bool hp_raw(unsigned long long hp) { return ((unsigned)hp & 0x8000u) != 0u; }
__device__ __forceinline__ unsigned hp_len(unsigned long long hp) { return (((unsigned)hp >> 8) & 127u) * 32u; }

template <bool OUT>
__device__ __forceinline__ void store_chunk(rsrc_t r, unsigned o, const LAS unsigned* S, unsigned len, int lane, u32x4 P0, u32x4 P1, u32x2 F) {
  const unsigned l16 = (unsigned)lane * 16u;
  st128s(P0, r, o + l16);
  const u32x4 sv = *(const LAS u32x4*)(S + lane * 4);
  if (OUT) {
    const u32x4 sw = *(const LAS u32x4*)(S + 256 + (lane & 31) * 4);
    __builtin_amdgcn_raw_buffer_store_b64(F, r, (int)(o + 1024 + lane * 8), 0, AUX_SYS);
    if (1536u + l16 < len) st128s(sv, r, o + 1536 + l16);
    if (lane < 32 && 2560u + l16 < len) st128s(sw, r, o + 2560 + l16);
  } else {
    st128s(P1, r, o + 1024 + l16);
    __builtin_amdgcn_raw_buffer_store_b64(F, r, (int)(o + 2048 + lane * 8), 0, AUX_SYS);
    if (lane < 32 && 2560u + l16 < len) st128s(sv, r, o + 2560 + l16);
  }
}

struct Rec { u32x4 P0, P1, sv; u32x2 F; };
template <bool OUT>
__device__ __forceinline__ Rec load_rec(rsrc_t r, unsigned o, unsigned long long hp, int lane) {
  const unsigned len = hp_len(hp), l16 = (unsigned)lane * 16u;
  Rec c;
  c.P0 = ld128s(r, o + l16);
  if (OUT) {
    c.F = __builtin_amdgcn_raw_buffer_load_b64(r, (int)(o + 1024 + lane * 8), 0, AUX_SYS);
    c.sv = ld128s(r, (1536u + l16 < len) ? o + 1536 + l16 : 0xFFFFFF00u);
    c.P1 = ld128s(r, (lane < 32 && 2560u + l16 < len) ? o + 2560 + l16 : 0xFFFFFF00u);
  } else {
    c.P1 = ld128s(r, o + 1024 + l16);
    c.F = __builtin_amdgcn_raw_buffer_load_b64(r, (int)(o + 2048 + lane * 8), 0, AUX_SYS);
    c.sv = ld128s(r, (lane < 32 && 2560u + l16 < len) ? o + 2560 + l16 : 0xFFFFFF00u);
  }
  return c;
}
template <bool OUT>
__device__ __forceinline__ void decode_rec(const Rec& rc, unsigned long long hp, LAS unsigned* S, int lane, unsigned (&v)[16]) {
  const unsigned base = (unsigned)hp & 0xFFu, tt = (unsigned)(hp >> 16) & 0xFFFFFFFu;
  const bool modeB = ((hp >> 44) & 1u) != 0u;
  const unsigned n0 = (unsigned)(hp >> 45) & 7u;
  const u32x4 P0 = rc.P0, P1 = rc.P1, sv = rc.sv, sw = rc.P1; const u32x2 F = rc.F;
  unsigned tab1 = 0, tab2 = 0;
#pragma unroll
  for (int i = 0; i < 4; ++i) tab1 |= sym_of(OUT, base, (tt >> (4 * i)) & 15u) << (8 * i);
#pragma unroll
  for (int i = 0; i < 3; ++i) tab2 |= sym_of(OUT, base, (tt >> (16 + 4 * i)) & 15u) << (8 * i);
  const unsigned B4 = base * 0x01010101u, T1 = ((tt >> 4) & 15u) * 0x01010101u, N0 = n0 * 0x01010101u;
  *(LAS u32x4*)(S + lane * 4) = sv;
  if (OUT && lane < 32) *(LAS u32x4*)(S + 256 + lane * 4) = sw;
  unsigned c1[8], e2[8], e0[8], n2 = 0, c0 = 0;
#pragma unroll
  for (int g = 0; g < 8; ++g) {
    c1[g] = (((g < 4) ? F.x : F.y) >> (2 * (g & 3))) & 0x03030303u;
    e2[g] = lut4(0x01000000u, c1[g]); n2 += pc(e2[g]);
    if (OUT) { e0[g] = modeB ? lut4(0x00000001u, c1[g]) : 0u; c0 += pc(e0[g]); } else e0[g] = 0u;
  }
  const unsigned x1 = c0 | (n2 << 16), s1 = dpp_shr_add(x1), t1 = rdl(s1, 63), p1 = s1 - x1;
  const unsigned C0 = t1 & 0xFFFFu, N2 = t1 >> 16;
  const unsigned aM1 = OUT ? 4u * (2048u - C0) : 0u, a2a = aM1 + C0, aM2 = a2a + N2;
  const unsigned oN = 4u * (32u * (unsigned)lane - (p1 & 0xFFFFu));
  unsigned hn[8], m2[8], e2b[8], cm2 = 0, n2b = 0;
  {
    unsigned w2a = fetch32(S, a2a + (p1 >> 16)), wM1 = 0; unsigned long long wN = 0, wN2 = 0;
    if (OUT) {
      wM1 = fetch32(S, aM1 + (p1 & 0xFFFFu)); wN = fetch64(S, oN);
      wN2 = fetch64(S, oN + 4u * (16u - pc(e0[0]) - pc(e0[1]) - pc(e0[2]) - pc(e0[3])));
    }
#pragma unroll
    for (int g = 0; g < 8; ++g) {
      e2b[g] = unpack_bits<1>(w2a, e2[g]); w2a >>= pc(e2[g]); n2b += pc(e2b[g]);
      if (OUT) {
        if (g == 4) wN = wN2;
        const unsigned eN = e0[g] ^ 0x01010101u;
        hn[g] = unpack_bits<4>((unsigned)wN, eN); wN >>= 4u * pc(eN);
        m2[g] = unpack_bits<1>(wM1, e0[g]); wM1 >>= pc(e0[g]); cm2 += pc(m2[g]);
      }
    }
  }
  const unsigned x2 = cm2 | (n2b << 16), s2 = dpp_shr_add(x2), t2 = rdl(s2, 63), p2 = s2 - x2;
  const unsigned CM2 = t2 & 0xFFFFu, N2B = t2 >> 16, a2b = aM2 + CM2, a3 = a2b + 2u * N2B;
  unsigned wM2 = OUT ? fetch32(S, aM2 + (p2 & 0xFFFFu)) : 0u;
  unsigned long long w2b = fetch64(S, a2b + 2u * (p2 >> 16));
  unsigned H[8], L[8], e3[8], n3 = 0;
#pragma unroll
  for (int g = 0; g < 8; ++g) {
    const unsigned c2 = unpack_bits<2>((unsigned)w2b, e2b[g]); w2b >>= 2u * pc(e2b[g]);
    e3[g] = lut4(0x01000000u, c2); n3 += pc(e3[g]);
    H[g] = bfi(bmask(e2b[g]), lut4(tab2, c2), lut4(tab1, c1[g]));
    if (OUT) {
      const unsigned b2 = unpack_bits<1>(wM2, m2[g]); wM2 >>= pc(m2[g]);
      const unsigned nib = hn[g] | ((N0 - m2[g] - b2) & bmask(e0[g]));
      const unsigned p4 = (g >> 1) == 0 ? P0.x : (g >> 1) == 1 ? P0.y : (g >> 1) == 2 ? P0.z : P0.w;
      L[g] = ((p4 >> (4 * (g & 1))) & 0x0F0F0F0Fu) | (nib << 4);
    } else {
      L[g] = g == 0 ? P0.x : g == 1 ? P0.y : g == 2 ? P0.z : g == 3 ? P0.w : g == 4 ? P1.x : g == 5 ? P1.y : g == 6 ? P1.z : P1.w;
    }
  }
  const unsigned s3 = dpp_shr_add(n3), N3 = rdl(s3, 63);
  const unsigned o3 = a3 + 4u * (s3 - n3);
  unsigned long long w3 = fetch64(S, o3);
  const unsigned long long w3b = fetch64(S, o3 + 4u * (pc(e3[0]) + pc(e3[1]) + pc(e3[2]) + pc(e3[3])));
  unsigned e4[8], any4 = 0;
#pragma unroll
  for (int g = 0; g < 8; ++g) {
    if (g == 4) w3 = w3b;
    unsigned R = unpack_bits<4>((unsigned)w3, e3[g]); w3 >>= 4u * pc(e3[g]);
    e4[g] = e3[g] & ~nz4(R ^ T1);
    R &= ~bmask(e4[g]);
    const unsigned H3 = OUT ? ((B4 - (R & 0x07070707u)) | ((R & 0x08080808u) << 4)) : (B4 - R);
    H[g] = bfi(bmask(e3[g]), H3, H[g]);
    any4 |= e4[g];
  }
  if (__any(any4 != 0u)) {
    unsigned n4 = 0;
#pragma unroll
    for (int g = 0; g < 8; ++g) n4 += pc(e4[g]);
    unsigned o4 = a3 + 4u * N3 + 8u * (dpp_shr_add(n4) - n4);
#pragma unroll
    for (int g = 0; g < 8; ++g) { H[g] = bfi(bmask(e4[g]), unpack_bits<8>(fetch32(S, o4), e4[g]), H[g]); o4 += 8u * pc(e4[g]); }
  }
#pragma unroll
  for (int g = 0; g < 8; ++g) {
    const unsigned Hs = OUT ? H[g] : bfi(0x80808080u, L[g], H[g] >> 1), Ls = OUT ? L[g] : bfi(0x80808080u, H[g] << 7, L[g]);
    v[2 * g] = vperm(Hs, Ls, 0x05010400u);
    v[2 * g + 1] = vperm(Hs, Ls, 0x07030602u);
  }
}

struct MapX { unsigned base; __device__ __forceinline__ unsigned operator()(int w, int j, int l) const { return base + (unsigned)(w * 4096 + j * 1024 + l * 16); } };
struct MapW { unsigned base; __device__ __forceinline__ unsigned operator()(int w, int j, int l) const { return base + (unsigned)((j >> 1) * 16384 + ((w >> 1) * 2 + (j & 1)) * 2048 + (w & 1) * 1024 + l * 16); } };
struct MapO { unsigned base; __device__ __forceinline__ unsigned operator()(int w, int j, int l) const { return base + (unsigned)((64 * (w >> 2) + 16 * (j >> 1) + (l & 15)) * (ND * 2) + (64 * (w & 3) + 32 * (j & 1) + 8 * (l >> 4)) * 2); } };
__device__ __forceinline__ unsigned out_task_base(int rank, int task) {
  const int tile = task >> 2, q = task & 3, pm = tile >> 3, pn = tile & 7;
  return (unsigned)(((rank >> 2) * 4096 + pm * 256 + (q >> 1) * 128 + (q & 1) * 32) * (ND * 2) + ((rank & 3) * 2048 + pn * 256) * 2);
}

__device__ __forceinline__ unsigned long long mk_granule(unsigned long long payload, unsigned call) { return payload | ((unsigned long long)(call & 0xFFFFu) << 48); }
__device__ __forceinline__ unsigned long long uni64(unsigned long long g) { return ((unsigned long long)rfl((unsigned)(g >> 32)) << 32) | rfl((unsigned)g); }
__device__ __forceinline__ unsigned long long ld_granule(const char* ws, unsigned off) {
  return __hip_atomic_load((const gu64*)(ws + off), __ATOMIC_RELAXED, __HIP_MEMORY_SCOPE_SYSTEM);
}
__device__ __forceinline__ bool tag_ok(unsigned long long g, unsigned call) { return (unsigned)(g >> 48) == (call & 0xFFFFu); }
__device__ __forceinline__ bool poll_granule(const Ctx& cx, const char* ws, unsigned off, unsigned long long& g) {
  g = uni64(g);
  if (!tag_ok(g, cx.call)) {
    const unsigned long long t0 = rt();
    do {
      if (cx.give_up(t0)) return false;
      g = uni64(ld_granule(ws, off));
    } while (!tag_ok(g, cx.call));
  }
  asm volatile("" ::: "memory");
  return true;
}

template <class Map>
__device__ __forceinline__ void nan_chunk(rsrc_t fin, const Map& map, int wid, int lane) {
#pragma unroll
  for (int j = 0; j < 4; ++j) st128s((u32x4){0x7FC07FC0u, 0x7FC07FC0u, 0x7FC07FC0u, 0x7FC07FC0u}, fin, map(wid, j, lane));
}

struct InTask { int kt, isw, idx; };
__device__ __forceinline__ InTask in_task(int i) {
  InTask t;
  if (i < 768) { t.kt = i / 48; const int u = i - t.kt * 48, u3 = u / 3, r3 = u - u3 * 3;
    if (r3 == 0) { t.isw = 0; t.idx = (u3 & 1) * 16 + (u3 >> 1); } else { t.isw = 1; const int c = u3 * 2 + r3 - 1; t.idx = (c & 3) * 8 + (c >> 2); } }
  else { const int u = i - 768; t.kt = (u >> 4) & 15; t.isw = 0; t.idx = (u & 1) * 16 + 8 + ((u & 15) >> 1); }
  return t;
}
__device__ __forceinline__ int otask_of(int isw, int idx, int kt) { return isw * 512 + idx * 16 + kt; }
__device__ __forceinline__ void in_load(const Params& p, const InTask& t, u32x4 (&r)[4], int wid, int lane) {
  const int rr = lane >> 2, cp = (lane & 3) ^ ((rr >> 3) << 1);
  if (!t.isw) {
    const rsrc_t rx = mkr(p.x, IN_BYTES);
#pragma unroll
    for (int j = 0; j < 4; ++j) {
      const unsigned off = (unsigned)((256 * t.idx + 32 * wid + 16 * (j >> 1) + rr) * (KS * 2) + (64 * t.kt + 32 * (j & 1) + 8 * cp) * 2);
      r[j] = __builtin_amdgcn_raw_buffer_load_b128(rx, (int)off, 0, 0);
    }
  } else {
    const rsrc_t rw = mkr(p.w, IN_BYTES);
    const unsigned off0 = (unsigned)((64 * t.kt + 32 * (wid & 1) + 8 * cp) * (ND * 2) + (256 * t.idx + 64 * (wid >> 1) + 4 * rr) * 2);
#pragma unroll
    for (int d = 0; d < 4; ++d) {
      const u32x2 a = __builtin_amdgcn_raw_buffer_load_b64(rw, (int)(off0 + (unsigned)(2 * d) * (ND * 2)), 0, 0);
      const u32x2 b = __builtin_amdgcn_raw_buffer_load_b64(rw, (int)(off0 + (unsigned)(2 * d + 1) * (ND * 2)), 0, 0);
      r[d] = __builtin_shufflevector(a, b, 0, 1, 2, 3);
    }
  }
}
__device__ __forceinline__ void in_prep(int isw, const u32x4 (&r)[4], unsigned (&v)[16]) {
  if (isw) {
#pragma unroll
    for (int d = 0; d < 4; ++d) {
      v[0 + d] = __builtin_amdgcn_perm(r[d].z, r[d].x, 0x05040100u);
      v[4 + d] = __builtin_amdgcn_perm(r[d].z, r[d].x, 0x07060302u);
      v[8 + d] = __builtin_amdgcn_perm(r[d].w, r[d].y, 0x05040100u);
      v[12 + d] = __builtin_amdgcn_perm(r[d].w, r[d].y, 0x07060302u);
    }
  } else {
#pragma unroll
    for (int j = 0; j < 4; ++j) { v[4 * j] = r[j].x; v[4 * j + 1] = r[j].y; v[4 * j + 2] = r[j].z; v[4 * j + 3] = r[j].w; }
  }
}

__device__ __forceinline__ void role_encode(const Params& p, const Ctx& cx, LAS unsigned* S, int e, int ne) {
  const int tid = threadIdx.x, wid = (int)rfl(tid >> 6), lane = tid & 63;
  char* ws = p.ws[p.rank];
  const rsrc_t lw = mkr(ws, WS_END);
  u32x4 A[4];
  in_load(p, in_task(e), A, wid, lane);
  DRAIN_ALL();
  for (int i = e; i < NIT; i += ne) {
    const InTask t = in_task(i);
    unsigned v[16];
    in_prep(t.isw, A, v);
    if (i + ne < NIT) in_load(p, in_task(i + ne), A, wid, lane);
    u32x4 P0, P1; u32x2 F;
    const unsigned long long hp = encode_chunk<false>(v, S, lane, P0, P1, F);
    const int ot = otask_of(t.isw, t.idx, t.kt);
    const bool mine = t.isw ? ((t.idx >> 3) == (p.rank & 3)) : ((t.idx >> 4) == (p.rank >> 2));
    const int img = t.isw ? 2048 + (t.idx & 7) * 128 + t.kt * 8 + p.rank : (t.idx & 15) * 128 + t.kt * 8 + p.rank;
    if (!hp_raw(hp)) {
      store_chunk<false>(lw, WS_OBI + (unsigned)ot * PKT + (unsigned)wid * 3072u, S, hp_len(hp), lane, P0, P1, F);
    } else {
#pragma unroll
      for (int j = 0; j < 4; ++j) st128s((u32x4){v[4 * j], v[4 * j + 1], v[4 * j + 2], v[4 * j + 3]}, lw, WS_OBR + (unsigned)ot * IMG + (unsigned)(wid * 4096 + j * 1024 + lane * 16));
    }
    if (mine) {
      const unsigned ib = WS_IMG + (unsigned)img * IMG;
#pragma unroll
      for (int j = 0; j < 4; ++j) st128s((u32x4){v[4 * j], v[4 * j + 1], v[4 * j + 2], v[4 * j + 3]}, lw, t.isw ? MapW{ib}(wid, j, lane) : MapX{ib}(wid, j, lane));
    }
    DRAIN_ALL();
    if (lane == 0) {
      __hip_atomic_store((gu64*)(ws + WS_HOBI + (unsigned)ot * 64u) + wid, mk_granule(hp, cx.call), __ATOMIC_RELAXED, __HIP_MEMORY_SCOPE_SYSTEM);
      if (mine) __hip_atomic_fetch_add((gu32*)(ws + WS_CNT) + img * 32, 1u, __ATOMIC_RELAXED, __HIP_MEMORY_SCOPE_AGENT);
    }
  }
}

__device__ __forceinline__ bool eo_ready(const Ctx& cx, const gu32* f, bool block) {
  if (rfl(__hip_atomic_load(f, __ATOMIC_RELAXED, __HIP_MEMORY_SCOPE_AGENT)) >= cx.call) return true;
  if (!block) return false;
  const unsigned long long t0 = rt();
  while (rfl(__hip_atomic_load(f, __ATOMIC_RELAXED, __HIP_MEMORY_SCOPE_AGENT)) < cx.call) { if (cx.give_up(t0)) return false; }
  return true;
}
__device__ __forceinline__ void role_encode_out(const Params& p, const Ctx& cx, LAS unsigned* S, int e, int ne) {
  const int tid = threadIdx.x, wid = (int)rfl(tid >> 6), lane = tid & 63;
  char* ws = p.ws[p.rank];
  const rsrc_t lw = mkr(ws, WS_END), ro = mkr(p.out[p.rank], OUT_BYTES);
  const gu32* rf = (const gu32*)(ws + WS_RFLG) + wid * 4;
  auto load = [&](int task, u32x4 (&A)[4]) { const MapO map{out_task_base(p.rank, task)};
#pragma unroll
    for (int j = 0; j < 4; ++j) A[j] = ld128s(ro, map(wid, j, lane)); };
  u32x4 A[4];
  if (!eo_ready(cx, rf + e * 32, true)) return;
  load(e, A);
  DRAIN_ALL();
  for (int i = e; i < NOT; i += ne) {
    unsigned v[16];
#pragma unroll
    for (int j = 0; j < 4; ++j) { v[4 * j] = A[j].x; v[4 * j + 1] = A[j].y; v[4 * j + 2] = A[j].z; v[4 * j + 3] = A[j].w; }
    const int nx = i + ne;
    bool have = false;
    if (nx < NOT) { have = eo_ready(cx, rf + nx * 32, false); if (have) load(nx, A); }
    u32x4 P0, P1; u32x2 F;
    const unsigned long long hp = encode_chunk<true>(v, S, lane, P0, P1, F);
    if (!hp_raw(hp)) store_chunk<true>(lw, WS_OBO + (unsigned)i * PKT + (unsigned)wid * 3072u, S, hp_len(hp), lane, P0, P1, F);
    DRAIN_ALL();
    if (lane == 0) __hip_atomic_store((gu64*)(ws + WS_HOBO + (unsigned)i * 64u) + wid, mk_granule(hp, cx.call), __ATOMIC_RELAXED, __HIP_MEMORY_SCOPE_SYSTEM);
    if (nx < NOT && !have) {
      if (!eo_ready(cx, rf + nx * 32, true)) return;
      load(nx, A);
      DRAIN_ALL();
    }
  }
}

struct PItem { unsigned hsrc, dsrc, hdst, ddst, rs, rd; int kind; };
__device__ __forceinline__ PItem p_item(int rank, int d, int i) {
  PItem r;
  if (i < 384) {
    int kt, isw, k;
    if (i < 256) { kt = i >> 4; isw = i & 1; k = (i & 15) >> 1; } else { kt = (i - 256) >> 3; isw = 0; k = 8 + ((i - 256) & 7); }
    const int idx = isw ? (d & 3) * 8 + k : (d >> 2) * 16 + k;
    const int ot = otask_of(isw, idx, kt), T = kt * 8 + rank, img = isw ? 2048 + k * 128 + T : k * 128 + T;
    const unsigned slot = CFG_SEQIN ? (unsigned)(rank * 384 + i) : (unsigned)img;
    r.hsrc = WS_HOBI + (unsigned)ot * 64u; r.dsrc = WS_OBI + (unsigned)ot * PKT;
    r.hdst = WS_HIN + slot * 64u; r.ddst = WS_PIN + slot * PKT;
    r.rs = WS_OBR + (unsigned)ot * IMG; r.rd = WS_IMG + (unsigned)img * IMG; r.kind = isw;
  } else {
    const int task = (i - 384) & (NOT - 1);
    r.hsrc = WS_HOBO + (unsigned)task * 64u; r.dsrc = WS_OBO + (unsigned)task * PKT;
    r.hdst = WS_HOUT + (unsigned)(rank * NOT + task) * 64u; r.ddst = WS_POUT + (unsigned)(rank * NOT + task) * PKT;
    r.rs = r.rd = out_task_base(rank, task); r.kind = 2;
  }
  return r;
}

#define P_LDG(ws, off) ld_granule(ws, off)

__device__ __forceinline__ void p_publish(const Ctx& cx, const char* ws, char* wd, unsigned hsrc, unsigned hdst, unsigned long long hl, int lane) {
  if (hsrc == 0xFFFFFFFFu) hl = mk_granule(0x8000ull, cx.call);
  else {
    const char* src = ws + hsrc + (lane & 7) * 8;
    while (!__all(tag_ok(hl, cx.call))) {
      u32x2 r; asm volatile("global_load_dwordx2 %0, %1, off sc0 sc1\n\ts_waitcnt vmcnt(0)" : "=&v"(r) : "v"(src) : "memory");
      hl = ((unsigned long long)r.y << 32) | r.x;
    }
  }
  if (lane < 8) {
    const u32x2 v = (u32x2){(unsigned)hl, (unsigned)(hl >> 32)};
    asm volatile("global_store_dwordx2 %0, %1, off sc0 sc1\n\ts_nop 1" :: "v"(wd + hdst + lane * 8), "v"(v) : "memory");
  }
}

__device__ __forceinline__ void st_peer(u32x4 v, char* ptr) { asm volatile("global_store_dwordx4 %0, %1, off sc0 sc1\n\ts_nop 1" :: "v"(ptr), "v"(v) : "memory"); }
__device__ __forceinline__ void glds16(const char* src, unsigned dst) {
  unsigned keep; asm volatile("s_mov_b32 %0, m0\n\ts_mov_b32 m0, %2\n\ts_nop 0\n\tglobal_load_lds_dwordx4 %1, off sc0 sc1\n\ts_mov_b32 m0, %0" : "=&s"(keep) : "v"(src), "s"(dst) : "memory");
}
__device__ __forceinline__ void glds4(const char* src, unsigned dst) {
  unsigned keep; asm volatile("s_mov_b32 %0, m0\n\ts_mov_b32 m0, %2\n\ts_nop 0\n\tglobal_load_lds_dword %1, off sc0 sc1\n\ts_mov_b32 m0, %0" : "=&s"(keep) : "v"(src), "s"(dst) : "memory");
}
__device__ __forceinline__ u32x2 ld64_nowait(const char* src) {
  u32x2 r; asm volatile("global_load_dwordx2 %0, %1, off sc0 sc1" : "=&v"(r) : "v"(src) : "memory");
  return r;
}
__device__ __forceinline__ unsigned long long u64_of(u32x2 v) { return ((unsigned long long)v.y << 32) | v.x; }
__device__ __forceinline__ void role_push(const Params& p, Ctx& cx, unsigned callraw, unsigned lds_ring, int link, int v0, int nv) {
  constexpr int NC = NPI * 8;
  const int lane = threadIdx.x & 63;
  int d = p.rank + 1 + link; if (d >= NRANK) d -= NRANK;
  if ((unsigned)d >= (unsigned)NRANK) return;
  char* ws = p.ws[p.rank]; char* wd = p.ws[d];
  const rsrc_t lw = mkr(ws, WS_END), dw = mkr(wd, WS_END);
  gu32* tkc = (gu32*)(ws + WS_TK) + link * 32;
  gu32* pcnt = (gu32*)(ws + WS_PCNT) + link * NPI;
  const unsigned lo = (unsigned)lane * 16u;
  constexpr int R0 = CFG_RAWHEAD;
  static_assert(R0 * PJ <= 16, "raw head chunks must be K-tile 0 items");
  int prev; unsigned prev_hsrc = 0, prev_hdst;
  {
    auto head_task = [&](int u) { const int i0 = u >> 3; InTask t; t.kt = 0; t.isw = i0 & 1; t.idx = t.isw ? (d & 3) * 8 + (i0 >> 1) : (d >> 2) * 16 + (i0 >> 1); return t; };
    u32x4 A[R0][4];
#pragma unroll
    for (int r = 0; r < R0; ++r) in_load(p, head_task(v0 + r * nv), A[r], v0 & 7, lane);
    unsigned dummy = 0;
    DRAIN_PIN2(callraw, dummy);
    cx.call = rfl(callraw) + 1u;
#pragma unroll
    for (int r = 0; r < R0; ++r) {
      const int u = v0 + r * nv, w = u & 7;
      const InTask t = head_task(u);
      unsigned v[16];
      in_prep(t.isw, A[r], v);
      const PItem f = p_item(p.rank, d, u >> 3);
      unsigned pc = 0;
      if (CFG_HEADSEQ && r > 0) { VM0(); pc = atomic_add_lane0_nowait(pcnt + (prev >> 3)); }
#pragma unroll
      for (int j = 0; j < 4; ++j) st128s((u32x4){v[4 * j], v[4 * j + 1], v[4 * j + 2], v[4 * j + 3]}, dw, t.isw ? MapW{f.rd}(w, j, lane) : MapX{f.rd}(w, j, lane));
      if (CFG_HEADSEQ && r > 0) {
        asm volatile("s_waitcnt vmcnt(4)" : "+v"(pc) :: "memory");
        if (rfl(pc) == 8u * cx.call - 1u) p_publish(cx, ws, wd, 0xFFFFFFFFu, prev_hdst, 0ull, lane);
      }
      if (r == 0) STAMP(p, 3);
      prev = u; prev_hdst = f.hdst;
    }
    if (!CFG_HEADSEQ && R0 > 1) {
      VM0();
#pragma unroll
      for (int r = 0; r + 1 < R0; ++r) {
        const int u = v0 + r * nv;
        unsigned pc = atomic_add_lane0_nowait(pcnt + (u >> 3)), dummy2 = 0;
        DRAIN_PIN2(pc, dummy2);
        if (rfl(pc) == 8u * cx.call - 1u) p_publish(cx, ws, wd, 0xFFFFFFFFu, p_item(p.rank, d, u >> 3).hdst, 0ull, lane);
      }
    }
  }
  const unsigned tkbase = (cx.call - 1u) * (unsigned)(NC - R0 * nv) - (unsigned)(R0 + 4) * (unsigned)nv;
  gu32* dummy_cnt = (gu32*)(ws + WS_TK) + 8 * 32 + (v0 & 7);
  const unsigned ring = lds_ring, gsl = lds_ring + 3u * 3072u;
  auto hoff = [&](int u) { return p_item(p.rank, d, u >> 3).hsrc + (unsigned)(u & 7) * 8u; };
  auto tail = [&](int u) { return CFG_RAWTAIL > 0 && u >= NC - 8 * CFG_RAWTAIL; };
  auto dma_rec = [&](int u, unsigned long long g, bool on, unsigned sl) {
    const unsigned base = on ? p_item(p.rank, d, u >> 3).dsrc + (unsigned)(u & 7) * 3072u : WS_OBI, len = on ? hp_len(g) : 0u;
    const char* q = ws + base;
#pragma unroll
    for (int k = 0; k < 3; ++k) glds16(q + ((unsigned)k * 1024u + lo < len ? (unsigned)k * 1024u + lo : lo), ring + sl * 3072u + (unsigned)k * 1024u);
  };
  auto dma_granule = [&](int u, unsigned sl) { if (lane < 2) glds4(ws + (u < NC ? hoff(u) : WS_HOBI) + lane * 4, gsl + sl * 8u); };
  auto rd_granule = [&](unsigned sl) { const LAS unsigned* q = (const LAS unsigned*)(size_t)(gsl + sl * 8u); return ((unsigned long long)rfl(q[1]) << 32) | rfl(q[0]); };
  int c = v0 + R0 * nv, c1 = c + nv, c2 = c1 + nv, c3 = c2 + nv;
  unsigned long long gc = P_LDG(ws, hoff(c)), g1 = P_LDG(ws, hoff(c1));
  dma_granule(c2, 0u);
  DRAIN_ALL();
  gc = uni64(gc); g1 = uni64(g1);
  if (!tag_ok(gc, cx.call)) {
    unsigned pc = atomic_add_lane0_nowait(pcnt + (prev >> 3)), dummy = 0;
    DRAIN_PIN2(pc, dummy);
    if (rfl(pc) == 8u * cx.call - 1u) p_publish(cx, ws, wd, 0xFFFFFFFFu, prev_hdst, 0ull, lane);
    prev = -1;
    if (!poll_granule(cx, ws, hoff(c), gc)) return;
  }
  if (prev >= 0) prev_hsrc = 0xFFFFFFFFu;
  unsigned sc = 0, gp = 0;
  bool have1 = tag_ok(g1, cx.call);
  dma_rec(c, gc, !hp_raw(gc) && !tail(c), 0u);
  dma_rec(c1, g1, have1 && !hp_raw(g1) && !tail(c1), 1u);
  DRAIN_ALL();
  while (c < NC) {
    const int w = c & 7;
    const PItem cur = p_item(p.rank, d, c >> 3);
    unsigned tk = atomic_add_lane0_nowait(tkc);
    unsigned pc = atomic_add_lane0_nowait(prev >= 0 ? pcnt + (prev >> 3) : dummy_cnt);
    u32x2 hl = ld64_nowait(ws + ((prev >= 0 && prev_hsrc != 0xFFFFFFFFu) ? prev_hsrc : WS_HOBI) + (lane & 7) * 8);
    dma_granule(c3, gp ^ 1u);
    const bool rawc = hp_raw(gc) || tail(c);
    if (!rawc) {
      const LAS u32x4* q = (const LAS u32x4*)(size_t)(ring + sc * 3072u + lo);
      const u32x4 d0 = q[0], d1 = q[64], d2 = q[128];
      char* o = wd + (size_t)(cur.ddst + (unsigned)w * 3072u + lo); const unsigned len = hp_len(gc);
      st_peer(d0, o);
      if (1024u + lo < len) st_peer(d1, o + 1024);
      if (2048u + lo < len) st_peer(d2, o + 2048);
    } else {
      u32x4 q[4];
      if (cur.kind == 2) {
        const rsrc_t so = mkr(p.out[p.rank], OUT_BYTES), dd = mkr(p.out[d], OUT_BYTES); const MapO map{cur.rs};
#pragma unroll
        for (int j = 0; j < 4; ++j) q[j] = ld128s(so, map(w, j, lane));
#pragma unroll
        for (int j = 0; j < 4; ++j) st128s(q[j], dd, map(w, j, lane));
      } else {
#pragma unroll
        for (int j = 0; j < 4; ++j) q[j] = ld128s(lw, cur.rs + (unsigned)(w * 4096 + j * 1024) + lo);
#pragma unroll
        for (int j = 0; j < 4; ++j) st128s(q[j], dw, cur.kind ? MapW{cur.rd}(w, j, lane) : MapX{cur.rd}(w, j, lane));
      }
      VM0();
    }
    const unsigned long long g2 = rd_granule(gp);
    const bool have2 = c2 < NC && tag_ok(g2, cx.call);
    const unsigned s2 = sc >= 1u ? sc - 1u : 2u;
    dma_rec(c2 < NC ? c2 : 0, g2, have2 && !hp_raw(g2) && !tail(c2), s2);
    asm volatile("s_waitcnt vmcnt(3)" : "+v"(tk), "+v"(pc), "+v"(hl) :: "memory");
    if (prev >= 0 && rfl(pc) == 8u * cx.call - 1u) p_publish(cx, ws, wd, prev_hsrc, prev_hdst, u64_of(hl), lane);
    const int c4 = (int)(rfl(tk) - tkbase);
    prev = c; prev_hsrc = tail(c) ? 0xFFFFFFFFu : cur.hsrc; prev_hdst = cur.hdst;
    const unsigned s1 = sc == 2u ? 0u : sc + 1u;
    if (c1 < NC && !have1) {
      VM0();
      unsigned pc2 = atomic_add_lane0_nowait(pcnt + (c >> 3));
      u32x2 hl2 = ld64_nowait(ws + (prev_hsrc != 0xFFFFFFFFu ? prev_hsrc : WS_HOBI) + (lane & 7) * 8);
      asm volatile("s_waitcnt vmcnt(0)" : "+v"(pc2), "+v"(hl2) :: "memory");
      if (rfl(pc2) == 8u * cx.call - 1u) p_publish(cx, ws, wd, prev_hsrc, prev_hdst, u64_of(hl2), lane);
      prev = -1;
      g1 = 0;
      if (!poll_granule(cx, ws, hoff(c1), g1)) return;
      dma_rec(c1, g1, !hp_raw(g1) && !tail(c1), s1);
      VM0();
    }
    c = c1; c1 = c2; c2 = c3; c3 = c4;
    gc = g1; g1 = g2; have1 = have2; sc = s1; gp ^= 1u;
  }
  if (prev >= 0) {
    VM0();
    unsigned pc = atomic_add_lane0_nowait(pcnt + (prev >> 3));
    u32x2 hl2 = ld64_nowait(ws + (prev_hsrc != 0xFFFFFFFFu ? prev_hsrc : WS_HOBI) + (lane & 7) * 8);
    asm volatile("s_waitcnt vmcnt(0)" : "+v"(pc), "+v"(hl2) :: "memory");
    if (rfl(pc) == 8u * cx.call - 1u) p_publish(cx, ws, wd, prev_hsrc, prev_hdst, u64_of(hl2), lane);
  }
  STAMPL(p, 2);
}

struct DTask { unsigned hl, pk, dst; int kind; int sig; };
__device__ __forceinline__ unsigned d_map(const DTask& t, int w, int j, int l) {
  return t.kind == 0 ? MapX{t.dst}(w, j, l) : t.kind == 1 ? MapW{t.dst}(w, j, l) : MapO{t.dst}(w, j, l);
}
template <bool OUT> __device__ __forceinline__ DTask d_task(int rank, int i) {
  DTask t;
  if (!OUT) {
    int kt, sl, k, isw;
    if (i < 7 * 256) { kt = i / 112; const int u = i - kt * 112; sl = u >> 4; isw = u & 1; k = (u & 15) >> 1; }
    else { const int u = i - 7 * 256; kt = (u / 56) & 15; const int u2 = u - (u / 56) * 56; sl = u2 >> 3; isw = 0; k = 8 + (u2 & 7); }
    int src = rank + 1 + sl; if (src >= NRANK) src -= NRANK;
    const int T = kt * 8 + src, img = isw ? 2048 + k * 128 + T : k * 128 + T;
    const unsigned slot = CFG_SEQIN ? (unsigned)(src * 384 + (k < 8 ? kt * 16 + k * 2 + isw : 256 + kt * 8 + (k - 8))) : (unsigned)img;
    t.hl = WS_HIN + slot * 64u; t.pk = WS_PIN + slot * PKT;
    t.dst = WS_IMG + (unsigned)img * IMG; t.kind = isw; t.sig = img;
  } else {
    const int task = (i / (NRANK - 1)) & (NOT - 1), k = i % (NRANK - 1);
    int s = rank + 1 + k; if (s >= NRANK) s -= NRANK;
    const unsigned tidx = (unsigned)(s * NOT + task);
    t.hl = WS_HOUT + tidx * 64u; t.pk = WS_POUT + tidx * PKT; t.dst = out_task_base(s, task); t.kind = 2; t.sig = 0;
  }
  return t;
}
#define WAIT_VM8() do { asm volatile("s_waitcnt vmcnt(8)" ::: "memory"); __builtin_amdgcn_s_waitcnt(0x0F78); } while (0)

template <bool OUT>
__device__ __forceinline__ void role_decode(const Params& p, const Ctx& cx, LAS unsigned* S, int dw, int nw) {
  constexpr int TOTAL = OUT ? NOT * (NRANK - 1) : 7 * 384;
  const int tid = threadIdx.x, wid = (int)rfl(tid >> 6), lane = tid & 63;
  char* ws = p.ws[p.rank];
  const rsrc_t lw = mkr(ws, WS_END), fin = OUT ? mkr(p.out[p.rank], OUT_BYTES) : lw;
  gu32* cnt = (gu32*)(ws + WS_CNT);
  const unsigned co = (unsigned)wid * 3072u, ho = (unsigned)wid * 8u;
  int i = dw;
  if (i >= TOTAL) return;
  DTask cur = d_task<OUT>(p.rank, i), t1 = cur;
  unsigned long long gc = ld_granule(ws, cur.hl + ho), g1 = 0, g2 = 0;
  bool ok = poll_granule(cx, ws, cur.hl + ho, gc);
  Rec rc = load_rec<OUT>(lw, cur.pk + co, gc, lane), r1 = rc;
  bool have1 = false;
  if (i + nw < TOTAL) { t1 = d_task<OUT>(p.rank, i + nw); g1 = ld_granule(ws, t1.hl + ho); }
  if (i + 2 * nw < TOTAL) g2 = ld_granule(ws, d_task<OUT>(p.rank, i + 2 * nw).hl + ho);
  DRAIN_ALL();
  if (i + nw < TOTAL) { g1 = uni64(g1); have1 = tag_ok(g1, cx.call); if (have1) r1 = load_rec<OUT>(lw, t1.pk + co, g1, lane); }
  DRAIN_ALL();
  int prev = -1;
  while (ok) {
    const int n1 = i + nw, n2 = i + 2 * nw, n3 = i + 3 * nw;
    DTask t2 = t1; Rec r2 = r1; unsigned long long g3 = 0; bool have2 = false;
    if (n2 < TOTAL) {
      t2 = d_task<OUT>(p.rank, n2);
      g2 = uni64(g2);
      have2 = tag_ok(g2, cx.call);
      if (have2) r2 = load_rec<OUT>(lw, t2.pk + co, g2, lane);
      if (n3 < TOTAL) g3 = ld_granule(ws, d_task<OUT>(p.rank, n3).hl + ho);
    }
    if (!hp_raw(gc)) {
      unsigned v[16];
      decode_rec<OUT>(rc, gc, S, lane, v);
#pragma unroll
      for (int j = 0; j < 4; ++j)
        __builtin_amdgcn_raw_buffer_store_b128((u32x4){v[4 * j], v[4 * j + 1], v[4 * j + 2], v[4 * j + 3]}, fin, (int)d_map(cur, wid, j, lane), 0, OUT ? CFG_AUX_DOUT : CFG_AUX_DIN);
      if (have2) WAIT_VM8(); else DRAIN_ALL();
    } else {
      DRAIN_ALL();
    }
    if (!OUT && prev >= 0 && lane == 0) __hip_atomic_fetch_add(cnt + prev * 32, 1u, __ATOMIC_RELAXED, __HIP_MEMORY_SCOPE_AGENT);
    prev = cur.sig;
    if (n1 >= TOTAL) break;
    if (!have1) {
      if (!OUT) { VM0(); if (lane == 0) __hip_atomic_fetch_add(cnt + prev * 32, 1u, __ATOMIC_RELAXED, __HIP_MEMORY_SCOPE_AGENT); prev = -1; }
      ok = poll_granule(cx, ws, t1.hl + ho, g1);
      r1 = load_rec<OUT>(lw, t1.pk + co, g1, lane);
      DRAIN_ALL();
    }
    cur = t1; rc = r1; gc = g1; t1 = t2; r1 = r2; g1 = g2; have1 = have2; g2 = g3; i = n1;
  }
  if (!OUT) {
    VM0();
    if (ok && prev >= 0 && lane == 0) __hip_atomic_fetch_add(cnt + prev * 32, 1u, __ATOMIC_RELAXED, __HIP_MEMORY_SCOPE_AGENT);
  } else if (!ok) {
    for (; i < TOTAL; i += nw) {
      const DTask t = d_task<OUT>(p.rank, i);
#pragma unroll
      for (int j = 0; j < 4; ++j) st128s((u32x4){0x7FC07FC0u, 0x7FC07FC0u, 0x7FC07FC0u, 0x7FC07FC0u}, fin, d_map(t, wid, j, lane));
    }
  }
}

__host__ __device__ __forceinline__ int lds_byte(int r, int c) { const int st = (r >> 4) * 2 + (c >> 5), rr = r & 15, cc = c & 31, ob = rr * 64 + cc * 2; return st * 1024 + (ob ^ (((ob >> 9) & 1) << 5)); }
__device__ __forceinline__ unsigned cvt_pk_bf16(float lo, float hi) { unsigned r; asm volatile("v_cvt_pk_bf16_f32 %0, %1, %2" : "=v"(r) : "v"(lo), "v"(hi)); return r; }
__device__ __forceinline__ float gelu1(float y) {
  const float k1 = -2.0f * 0.7978845608028654f * 1.4426950408889634f, k3 = k1 * 0.044715f;
  const float t = y * (k3 * (y * y) + k1);
  return y * __builtin_amdgcn_rcpf(1.0f + __builtin_amdgcn_exp2f(t));
}

__device__ __forceinline__ void gemm_unit(const Params& p, const Ctx& cx, LAS unsigned char* lds, int pm, int pn, int h, int ph) {
  constexpr int NTH = NT / 2;
  const int tid = threadIdx.x, wid = (int)rfl(tid >> 6), lane = tid & 63, wr = wid >> 2, wc = wid & 3, fr = lane & 15, fq = lane >> 4;
  char* ws = p.ws[p.rank];
  const char* cA = ws + WS_IMG + (size_t)(pm * 128 + 8 * h) * IMG;
  const char* cB = ws + WS_IMG + (size_t)(2048 + pn * 128 + 8 * h) * IMG;
  const gu32* fl = (const gu32*)(ws + WS_CNT) + (size_t)(((lane & 2) ? 2048 + pn * 128 : pm * 128) + 8 * h + (lane & 1)) * 32;
  const unsigned need = 8u * cx.call;
  const unsigned voff = (unsigned)tid * 16u;
  const unsigned ldsw = (unsigned)wid * 1024u;
  const int aoff = lds_byte(wr * 64 + fr, fq * 8), boff = lds_byte(wc * 32 + fr, fq * 8);
  bool dead = false;
#define KT(t) ((((t) >> 3) << 4) + ((t) & 7))
  auto wait_tiles = [&](int t) {
    if (wid == 0 && !dead) {
      const gu32* f = fl + KT(t) * 32;
      if (!__all(__hip_atomic_load(f, __ATOMIC_RELAXED, __HIP_MEMORY_SCOPE_AGENT) >= need)) {
        const unsigned long long t0 = rt();
        while (!__all(__hip_atomic_load(f, __ATOMIC_RELAXED, __HIP_MEMORY_SCOPE_AGENT) >= need)) { if (cx.give_up(t0)) { dead = true; break; } }
      }
    }
  };
#define SA(b, h) (((b) * 2 + (h)) * HTB)
#define SB(b, h) ((4 + (b) * 2 + (h)) * HTB)
#define STAGE(bufoff, gbase) do { _Pragma("unroll") for (int _i = 0; _i < 2; ++_i) \
    __builtin_amdgcn_global_load_lds((const GAS unsigned*)((const char*)(gbase) + voff + _i * 8192), (LAS unsigned*)(lds + (bufoff) + ldsw + _i * 8192), 16, 0, CFG_AUX_G); } while (0)
#define LDA(dst, b, h) do { _Pragma("unroll") for (int m = 0; m < 4; ++m) _Pragma("unroll") for (int k = 0; k < 2; ++k) dst[m][k] = *(const LAS bf16x8*)(lds + SA(b, h) + aoff + m * 2048 + k * 1024); } while (0)
#define LDB(dst, b, h) do { _Pragma("unroll") for (int n = 0; n < 2; ++n) _Pragma("unroll") for (int k = 0; k < 2; ++k) dst[n][k] = *(const LAS bf16x8*)(lds + SB(b, h) + boff + n * 2048 + k * 1024); } while (0)
#define MMA(ai, bj, At, Bt) do { __builtin_amdgcn_s_setprio(1); _Pragma("unroll") for (int m = 0; m < 4; ++m) _Pragma("unroll") for (int n = 0; n < 2; ++n) _Pragma("unroll") for (int k = 0; k < 2; ++k) \
    acc[ai][bj][m][n] = __builtin_amdgcn_mfma_f32_16x16x32_bf16(Bt[n][k], At[m][k], acc[ai][bj][m][n], 0, 0, 0); __builtin_amdgcn_s_setprio(0); } while (0)
#define WAIT_V(n) asm volatile("s_waitcnt vmcnt(" #n ")" ::: "memory")
#define WAIT_L(n) asm volatile("s_waitcnt lgkmcnt(" #n ")" ::: "memory")
#define BAR __builtin_amdgcn_s_barrier()
#define SCHED __builtin_amdgcn_sched_barrier(0)
  f32x4 acc[2][2][4][2];
#pragma unroll
  for (int a = 0; a < 2; ++a)
#pragma unroll
    for (int b = 0; b < 2; ++b)
#pragma unroll
      for (int m = 0; m < 4; ++m)
#pragma unroll
        for (int n = 0; n < 2; ++n) acc[a][b][m][n] = (f32x4){0.f, 0.f, 0.f, 0.f};
  bf16x8 At[4][2], B0[2][2], B1[2][2];
  wait_tiles(0); wait_tiles(2);
  BARRIER();
  STAMP(p, 1 + 4 * ph);
  STAGE(SB(0, 0), cB); STAGE(SA(0, 0), cA); STAGE(SB(0, 1), cB + HTB); STAGE(SA(0, 1), cA + HTB);
  if (wr == 1) BAR;
  WAIT_V(4); BAR;
  STAGE(SB(1, 0), cB + IMG); STAGE(SA(1, 0), cA + IMG); STAGE(SB(1, 1), cB + IMG + HTB);
  WAIT_V(6); BAR;
  for (int t = 0; t < NTH; t += 2) {
    const bool last = (t == NTH - 2);
    const char* a1 = cA + (size_t)(KT(t) + 1) * IMG;
    const size_t o2 = last ? 0 : (size_t)KT(t + 2) * IMG;
    const char* a2 = cA + o2; const char* b2 = cB + o2;
    const char* a3 = a2 + IMG; const char* b3 = b2 + IMG;
    LAS unsigned* fslot = (LAS unsigned*)(lds + LDS_FLAG + ((t >> 1) & 1) * 16);
    const bool chk = (t + 4 < NTH);
    if (chk && wid == 0 && lane < 4) __builtin_amdgcn_global_load_lds(fl + KT(t + 4) * 32, fslot, 4, 0, AUX_SYS);
    LDB(B0, 0, 0); SCHED; LDA(At, 0, 0); STAGE(SA(1, 1), a1 + HTB);
    WAIT_L(8); BAR; WAIT_L(0); MMA(0, 0, At, B0); BAR; SCHED;
    LDB(B1, 0, 1); STAGE(SB(0, 0), b2);
    BAR; WAIT_L(0); MMA(0, 1, At, B1); BAR;
    LDA(At, 0, 1); STAGE(SA(0, 0), a2);
    BAR; WAIT_L(0); MMA(1, 0, At, B0); BAR; SCHED;
    STAGE(SB(0, 1), b2 + HTB);
    WAIT_V(6); BAR; MMA(1, 1, At, B1); BAR;
    LDB(B0, 1, 0); SCHED; LDA(At, 1, 0); STAGE(SA(0, 1), a2 + HTB);
    WAIT_L(8); BAR; WAIT_L(0); MMA(0, 0, At, B0); BAR; SCHED;
    LDB(B1, 1, 1); STAGE(SB(1, 0), b3);
    BAR; WAIT_L(0); MMA(0, 1, At, B1); BAR;
    LDA(At, 1, 1); STAGE(SA(1, 0), a3);
    BAR; WAIT_L(0); MMA(1, 0, At, B0); BAR; SCHED;
    const u32x4 f = *(const LAS u32x4*)fslot;
    STAGE(SB(1, 1), b3 + HTB);
    WAIT_V(6); BAR; MMA(1, 1, At, B1); BAR;
    if (chk) {
      const unsigned fm = rfl(min(min(f.x, f.y), min(f.z, f.w)));
      if (fm < need) { wait_tiles(t + 4); BARRIER(); }
    }
    if (CFG_BSLEEP > 0 && ph == 1) __builtin_amdgcn_s_sleep(CFG_BSLEEP);
    if (CFG_ASLEEP > 0 && ph == 0) __builtin_amdgcn_s_sleep(CFG_ASLEEP);
  }
  if (wr == 0) BAR;
  STAMP(p, 2 + 4 * ph);
  const int tile = pm * 8 + pn;
  {
    int tz = threadIdx.x; asm volatile("" : "+v"(tz));
    const int ln = tz & 63;
    const rsrc_t ro = mkr(p.out[p.rank], OUT_BYTES), lw = mkr(ws, WS_END);
    {
      const unsigned po = WS_PART + (unsigned)(tile * 2 + h) * 131072u + (unsigned)(wid * 16384 + ln * 16);
      const unsigned pi = WS_PART + (unsigned)(tile * 2 + 1 - h) * 131072u + (unsigned)(wid * 16384 + ln * 16);
#pragma unroll
      for (int i = 0; i < 16; ++i) {
        const f32x4 q = h ? acc[0][i >> 3][(i >> 1) & 3][i & 1] : acc[1][i >> 3][(i >> 1) & 3][i & 1];
        st128s(__builtin_bit_cast(u32x4, q), lw, po + i * 1024);
      }
      VM0();
      gu32* fo = (gu32*)(ws + WS_PFLG) + ((tile * 2 + h) * 8 + wid) * 32;
      const gu32* fi = (const gu32*)(ws + WS_PFLG) + ((tile * 2 + 1 - h) * 8 + wid) * 32;
      if (!dead && !cx.timed_out() && ln == 0) __hip_atomic_store(fo, cx.call, __ATOMIC_RELAXED, __HIP_MEMORY_SCOPE_AGENT);
      if (rfl(__hip_atomic_load(fi, __ATOMIC_RELAXED, __HIP_MEMORY_SCOPE_AGENT)) < cx.call) {
        const unsigned long long t0 = rt();
        while (rfl(__hip_atomic_load(fi, __ATOMIC_RELAXED, __HIP_MEMORY_SCOPE_AGENT)) < cx.call) { if (cx.give_up(t0)) break; }
      }
      asm volatile("" ::: "memory");
#pragma unroll
      for (int i = 0; i < 16; ++i) {
        const f32x4 q = __builtin_bit_cast(f32x4, ld128s(lw, pi + i * 1024));
        if (h) acc[1][i >> 3][(i >> 1) & 3][i & 1] += q; else acc[0][i >> 3][(i >> 1) & 3][i & 1] += q;
      }
    }
    STAMP(p, 12 + 4 * ph);
    const bool bad = cx.timed_out();
#pragma unroll
    for (int qq = 0; qq < 2; ++qq) {
      unsigned u[16];
#pragma unroll
      for (int mm = 0; mm < 2; ++mm) {
        const int m = qq * 2 + mm;
        unsigned d[8];
#pragma unroll
        for (int c = 0; c < 8; ++c) {
          const float y0 = h ? acc[1][c & 1][m][0][c >> 1] : acc[0][c & 1][m][0][c >> 1], y1 = h ? acc[1][c & 1][m][1][c >> 1] : acc[0][c & 1][m][1][c >> 1];
          d[c] = cvt_pk_bf16(gelu1(y0), gelu1(y1)); if (bad) d[c] = 0x7FC07FC0u; }
#pragma unroll
        for (int c = 0; c < 4; ++c) {
          auto r = __builtin_amdgcn_permlane16_swap(d[c], d[c + 4], false, false); d[c] = r[0]; d[c + 4] = r[1];
          auto s2 = __builtin_amdgcn_permlane32_swap(d[c], d[c + 4], false, false); d[c] = s2[0]; d[c + 4] = s2[1];
        }
#pragma unroll
        for (int c = 0; c < 8; ++c) u[8 * mm + c] = d[c];
      }
      const MapO map{out_task_base(p.rank, tile * 4 + h * 2 + qq)};
#pragma unroll
      for (int j = 0; j < 4; ++j) st128s((u32x4){u[4 * j], u[4 * j + 1], u[4 * j + 2], u[4 * j + 3]}, ro, map(wid, j, ln));
    }
    VM0();
    if (!bad && ln < 2)
      __hip_atomic_store((gu32*)(ws + WS_RFLG) + ((tile * 4 + h * 2 + ln) * 8 + wid) * 4, cx.call, __ATOMIC_RELAXED, __HIP_MEMORY_SCOPE_AGENT);
  }
  STAMP(p, 3 + 4 * ph);
#undef KT
#undef SA
#undef SB
#undef STAGE
#undef LDA
#undef LDB
#undef MMA
#undef WAIT_V
#undef WAIT_L
#undef BAR
#undef SCHED
}

__global__ __launch_bounds__(NTHR, 2) void fused_kernel(Params p) {
  extern __shared__ __attribute__((aligned(16))) unsigned char shm[];
  LAS unsigned char* lds = (LAS unsigned char*)shm;
  const int b = blockIdx.x, xcd = b & 7, slot = b >> 3;
  char* ws = p.ws[p.rank];
  const int wave = (int)rfl(threadIdx.x >> 6);
  unsigned callraw = atomic_add_lane0_nowait((gu32*)(ws + WS_CALL) + (b * 8 + wave) * 4);
  Ctx cx; cx.tmo = (gu32*)(ws + WS_TMO); cx.call = 0;
  LAS unsigned* S = (LAS unsigned*)(lds + LDS_SCR + wave * SCR_BYTES);
  if (b >= N_P) { unsigned dummy = 0; DRAIN_PIN2(callraw, dummy); cx.call = rfl(callraw) + 1u; }
  if (b < N_P) {
    { const int gw = b * 8 + wave; role_push(p, cx, callraw, (unsigned)(size_t)(lds + wave * 9728), gw % 7, gw / 7, PJ * 8); }
  } else if (b < N_P + N_EO) {
    role_encode_out(p, cx, S, b - N_P, N_EO);
    STAMP(p, 2);
  } else if (slot < 16) {
    const int dw = b - N_P - N_EO;
    role_decode<false>(p, cx, S, dw, N_D);
    STAMP(p, 1);
    role_decode<true>(p, cx, S, dw, N_D + 128);
    STAMP(p, 2);
  } else {
    const int g = slot - 16, h = xcd & 1, q = xcd >> 1;
    const int pm = (q >> 1) * 4 + (g & 3), pn = (q & 1) * 4 + (g >> 2);
    role_encode(p, cx, S, b - 128, 128);
    STAMP(p, 9);
#pragma unroll 1
    for (int ph = 0; ph < 2; ++ph) gemm_unit(p, cx, lds, pm + 8 * ph, pn, h, ph);
    role_decode<true>(p, cx, S, N_D + (b - 128), N_D + 128);
    STAMP(p, 10);
  }
}

}

extern "C" void kernel_launch(int rank, int nranks, ncclComm_t comm, hipStream_t stream,
                              void* const* d_in, const int* in_sizes, int n_in,
                              void* d_out, int out_size,
                              const int* in_dtypes, int out_dtype,
                              void* const* d_out_all, void* const* d_ws_all, size_t ws_bytes) {
  using namespace fk;
  (void)comm; (void)in_dtypes; (void)out_dtype;
  if (nranks != NRANK || n_in < 2 || rank < 0 || rank >= NRANK || ws_bytes < (size_t)WS_END ||
      in_sizes[0] != MD * KS || in_sizes[1] != KS * ND || out_size != MD * ND) {
    fprintf(stderr, "kernel_launch: unexpected problem shape\n");
    return;
  }
  static bool attr_set[64] = {};
  int dev = 0;
  if (hipGetDevice(&dev) != hipSuccess) return;
  if (dev >= 0 && dev < 64 && !attr_set[dev]) {
    hipError_t e = hipFuncSetAttribute((const void*)fused_kernel, hipFuncAttributeMaxDynamicSharedMemorySize, LDS_BYTES);
    if (e != hipSuccess) { fprintf(stderr, "HIP error %s\n", hipGetErrorString(e)); return; }
    attr_set[dev] = true;
  }
  Params p;
  p.x = (const char*)d_in[0]; p.w = (const char*)d_in[1];
  for (int i = 0; i < NRANK; ++i) { p.ws[i] = (char*)d_ws_all[i]; p.out[i] = (char*)d_out_all[i]; }
  p.out[rank] = (char*)d_out;
  p.rank = rank; p.pad = 0;
  hipLaunchKernelGGL(fused_kernel, dim3(GRID), dim3(NTHR), LDS_BYTES, stream, p);
  hipError_t e = hipGetLastError();
  if (e != hipSuccess) fprintf(stderr, "HIP launch error %s\n", hipGetErrorString(e));
}
```

```cpp
#include <hip/hip_runtime.h>
#include <rccl/rccl.h>
#include <cstdio>

namespace fk {

#define LAS __attribute__((address_space(3)))
#define GAS __attribute__((address_space(1)))
typedef unsigned short bf16_t;
typedef short bf16x8 __attribute__((ext_vector_type(8)));
typedef float f32x4 __attribute__((ext_vector_type(4)));
typedef unsigned u32x4 __attribute__((ext_vector_type(4)));
typedef unsigned u32x2 __attribute__((ext_vector_type(2)));
typedef unsigned short u16x2 __attribute__((ext_vector_type(2)));
typedef GAS unsigned gu32;
typedef GAS unsigned long long gu64;
typedef __amdgpu_buffer_rsrc_t rsrc_t;

constexpr int NRANK = 8;
constexpr int MD = 8192, ND = 8192, KD = 8192, KS = KD / NRANK;
constexpr int NT = KD / 64;
constexpr unsigned OUT_BYTES = (unsigned)MD * ND * 2u;
constexpr unsigned IN_BYTES = (unsigned)MD * KS * 2u;

constexpr int GRID = 256, NTHR = 512;
#define CFG_SLEEP 2
#define CFG_PJ 4
#define CFG_AUX_DIN 17
#define CFG_AUX_DOUT 17
#define CFG_SEQIN 1
#define CFG_GMAP 1
#define CFG_RAWTAIL 0
#define CFG_AUX_G 17
#define CFG_RAWHEAD 3
#define CFG_HEADSEQ 1
#define CFG_BSLEEP 0
#define CFG_ASLEEP 0
#define CFG_EK 8
#define CFG_PMAP 1
constexpr int PJ = CFG_PJ;
#define CFG_NEO 28
constexpr int N_EO = CFG_NEO;
constexpr int N_P = 7 * PJ, N_D = 128 - N_P - N_EO;
constexpr int AUX_SYS = 17;

constexpr int NIMG = 3072;
constexpr unsigned IMG = 32768, PKT = 24576;
constexpr int NIT = 1024;
constexpr int NOT = 512;
constexpr int NPI = 384 + NOT;
constexpr int HCAP = 13;

constexpr unsigned WS_CALL = 0;
constexpr unsigned WS_TMO  = WS_CALL + 256 * 128;
constexpr unsigned WS_TK   = WS_TMO + 1024;
constexpr unsigned WS_CNT  = WS_TMO + 4096;
constexpr unsigned WS_PCNT = WS_CNT + NIMG * 128;
constexpr unsigned WS_HIN  = WS_PCNT + 32768;
constexpr unsigned WS_HOUT = WS_HIN + NIMG * 128;
constexpr unsigned WS_HOBI = WS_HOUT + NRANK * NOT * 128;
constexpr unsigned WS_HOBO = WS_HOBI + NIT * 128;
constexpr unsigned WS_PFLG = WS_HOBO + NOT * 128;
constexpr unsigned WS_RFLG = WS_PFLG + 2048 * 128;
constexpr unsigned WS_DBG  = WS_RFLG + NOT * 8 * 16;
constexpr unsigned WS_PIN  = 4u << 20;
constexpr unsigned WS_POUT = WS_PIN + NIMG * PKT;
constexpr unsigned WS_OBI  = WS_POUT + NRANK * NOT * PKT;
constexpr unsigned WS_OBR  = WS_OBI + NIT * PKT;
constexpr unsigned WS_OBO  = WS_OBR + NIT * IMG;
constexpr unsigned WS_PART = WS_OBO + NOT * PKT;
constexpr unsigned WS_IMG  = WS_PART + 256 * 131072;
constexpr unsigned WS_END  = WS_IMG + NIMG * IMG;
static_assert(WS_DBG + 256 * 64 * 8 <= WS_PIN, "control region overflow");
static_assert((size_t)WS_IMG + (size_t)NIMG * IMG < (1ull << 31), "workspace offsets must fit 31 bits");

constexpr unsigned long long DEADLINE = 40000000ull;

constexpr int HTB = 16384, STAGE_BYTES = 8 * HTB;
constexpr int LDS_FLAG = STAGE_BYTES;
constexpr int LDS_CALL = STAGE_BYTES + 64;
constexpr int LDS_TK = STAGE_BYTES + 96;
constexpr int LDS_DEAD = STAGE_BYTES + 80;
constexpr int LDS_GHDR = STAGE_BYTES + 128;
constexpr int LDS_HDR = 0;
constexpr int LDS_SCR = STAGE_BYTES + 512;
constexpr int LDS_BYTES = STAGE_BYTES + 512 + 8 * 1664;

struct Params {
  const char* x; const char* w;
  char* ws[NRANK]; char* out[NRANK];
  int rank; int pad;
};

__device__ __forceinline__ rsrc_t mkr(const void* p, unsigned bytes) { return __builtin_amdgcn_make_buffer_rsrc((void*)p, 0, (int)bytes, 0x00020000); }
__device__ __forceinline__ u32x4 ld128s(rsrc_t r, unsigned off) { return __builtin_amdgcn_raw_buffer_load_b128(r, (int)off, 0, AUX_SYS); }
__device__ __forceinline__ void st128s(u32x4 v, rsrc_t r, unsigned off) { __builtin_amdgcn_raw_buffer_store_b128(v, r, (int)off, 0, AUX_SYS); }
__device__ __forceinline__ unsigned long long rt() { return __builtin_amdgcn_s_memrealtime(); }
__device__ __forceinline__ unsigned rfl(unsigned v) { return (unsigned)__builtin_amdgcn_readfirstlane((int)v); }
__device__ __forceinline__ unsigned rdl(unsigned v, int l) { return (unsigned)__builtin_amdgcn_readlane((int)v, l); }
#define VM0() do { asm volatile("s_waitcnt vmcnt(0)" ::: "memory"); __builtin_amdgcn_s_waitcnt(0x0F70); } while (0)
#define DRAIN_ALL() do { asm volatile("s_waitcnt vmcnt(0) lgkmcnt(0)" ::: "memory"); __builtin_amdgcn_s_waitcnt(0x0070); } while (0)
#define DRAIN_PIN2(a, b) do { asm volatile("s_waitcnt vmcnt(0) lgkmcnt(0)" : "+v"(a), "+v"(b) :: "memory"); __builtin_amdgcn_s_waitcnt(0x0070); } while (0)
#define BARRIER() do { asm volatile("" ::: "memory"); __builtin_amdgcn_s_barrier(); asm volatile("" ::: "memory"); } while (0)

#define STAMP(p, idx) do { } while (0)
#define STAMPL(p, idx) do { } while (0)

__device__ __forceinline__ unsigned atomic_add_lane0_nowait(gu32* ptr) {
  unsigned r; unsigned long long sv; const unsigned one = 1u;
  asm volatile("s_mov_b64 %1, exec\n\ts_mov_b64 exec, 1\n\tglobal_atomic_add %0, %2, %3, off sc0\n\ts_mov_b64 exec, %1"
               : "=&v"(r), "=&s"(sv) : "v"(ptr), "v"(one) : "memory");
  return r;
}

__device__ __forceinline__ int atomic_add_lane0_nowait_noret(gu32* ptr) {
  unsigned long long sv; const unsigned one = 1u;
  asm volatile("s_mov_b64 %0, exec\n\ts_mov_b64 exec, 1\n\tglobal_atomic_add %1, %2, off\n\ts_mov_b64 exec, %0" : "=&s"(sv) : "v"(ptr), "v"(one) : "memory");
  return 0;
}
struct Ctx {
  gu32* tmo; unsigned call;
  __device__ __forceinline__ bool timed_out() const { return rfl(__hip_atomic_load(tmo, __ATOMIC_RELAXED, __HIP_MEMORY_SCOPE_AGENT)) == call; }
  __device__ __forceinline__ bool give_up(unsigned long long t0) const {
    if (timed_out() || rt() - t0 > DEADLINE) { __hip_atomic_store(tmo, call, __ATOMIC_RELAXED, __HIP_MEMORY_SCOPE_AGENT); return true; }
    __builtin_amdgcn_s_sleep(CFG_SLEEP);
    return false;
  }
};

__device__ __forceinline__ unsigned dpp_shr_add(unsigned x) {
  x += (unsigned)__builtin_amdgcn_update_dpp(0, (int)x, 0x111, 0xf, 0xf, false);
  x += (unsigned)__builtin_amdgcn_update_dpp(0, (int)x, 0x112, 0xf, 0xf, false);
  x += (unsigned)__builtin_amdgcn_update_dpp(0, (int)x, 0x114, 0xf, 0xf, false);
  x += (unsigned)__builtin_amdgcn_update_dpp(0, (int)x, 0x118, 0xf, 0xf, false);
  x += (unsigned)__builtin_amdgcn_update_dpp(0, (int)x, 0x142, 0xa, 0xf, false);
  x += (unsigned)__builtin_amdgcn_update_dpp(0, (int)x, 0x143, 0xc, 0xf, false);
  return x;
}
__device__ __forceinline__ unsigned row_max(unsigned x) {
  x = max(x, (unsigned)__builtin_amdgcn_update_dpp(0, (int)x, 0x111, 0xf, 0xf, false));
  x = max(x, (unsigned)__builtin_amdgcn_update_dpp(0, (int)x, 0x112, 0xf, 0xf, false));
  x = max(x, (unsigned)__builtin_amdgcn_update_dpp(0, (int)x, 0x114, 0xf, 0xf, false));
  x = max(x, (unsigned)__builtin_amdgcn_update_dpp(0, (int)x, 0x118, 0xf, 0xf, false));
  return x;
}
__device__ __forceinline__ unsigned wave_max_u32(unsigned x) {
  x = row_max(x);
  x = max(x, (unsigned)__builtin_amdgcn_update_dpp(0, (int)x, 0x142, 0xa, 0xf, false));
  x = max(x, (unsigned)__builtin_amdgcn_update_dpp(0, (int)x, 0x143, 0xc, 0xf, false));
  return rdl(x, 63);
}
__device__ __forceinline__ unsigned vperm(unsigned a, unsigned b, unsigned sel) { return __builtin_amdgcn_perm(a, b, sel); }
__device__ __forceinline__ unsigned lut4(unsigned tab, unsigned sel) { return __builtin_amdgcn_perm(0u, tab, sel); }
__device__ __forceinline__ unsigned bmask(unsigned e) { return __builtin_amdgcn_perm(0u, 0u, e | 0x0C0C0C0Cu); }
__device__ __forceinline__ unsigned bfi(unsigned m, unsigned a, unsigned b) { return (m & a) | (~m & b); }
__device__ __forceinline__ unsigned mul24(unsigned a, unsigned k) { unsigned r; asm("v_mul_u32_u24 %0, %1, %2" : "=v"(r) : "s"(k), "v"(a)); return r; }
__device__ __forceinline__ unsigned rank4(unsigned e) { return mul24(e, 0x010101u) << 8; }
__device__ __forceinline__ unsigned pc(unsigned e) { return (unsigned)__builtin_popcount(e); }
__device__ __forceinline__ unsigned nz4(unsigned x) { return ((x + 0x0F0F0F0Fu) >> 4) & 0x01010101u; }

constexpr int SCR_BYTES = 1664;

template <int W> __device__ __forceinline__ unsigned pack_bits(unsigned c, unsigned e) {
  const unsigned pre = rank4(e), em = bmask(e);
  if (W == 1) return __builtin_amdgcn_udot4(c, lut4(0x08040201u, pre) & em, 0u, false);
  if (W == 2) return __builtin_amdgcn_udot4(c, lut4(0x40100401u, pre) & em, 0u, false);
  return __builtin_amdgcn_udot4(c, lut4(0x00001001u, pre) & em, 0u, false) | (__builtin_amdgcn_udot4(c, lut4(0x10010000u, pre) & em, 0u, false) << 8);
}
template <int W> __device__ __forceinline__ unsigned unpack_bits(unsigned w, unsigned e) {
  unsigned s;
  if (W == 1) s = mul24(w & 0xFu, 0x00204081u) & 0x01010101u;
  else if (W == 2) { w &= 0xFFu; const unsigned t = (w | (w << 12)) & 0x000F000Fu; s = (t | (t << 6)) & 0x03030303u; }
  else if (W == 4) { w &= 0xFFFFu; const unsigned t = (w | (w << 8)) & 0x00FF00FFu; s = (t | (t << 4)) & 0x0F0F0F0Fu; }
  else s = w;
  return lut4(s, rank4(e)) & bmask(e);
}
__device__ __forceinline__ void lds_or(LAS unsigned* q, unsigned x) { if (x != 0u) __hip_atomic_fetch_or(q, x, __ATOMIC_RELAXED, __HIP_MEMORY_SCOPE_WAVEFRONT); }
__device__ __forceinline__ void flush32(LAS unsigned* S, unsigned off, unsigned acc) {
  const unsigned long long x = (unsigned long long)acc << (off & 31u);
  LAS unsigned* q = S + (off >> 5);
  lds_or(q, (unsigned)x); lds_or(q + 1, (unsigned)(x >> 32));
}
__device__ __forceinline__ void flush64(LAS unsigned* S, unsigned off, unsigned long long acc) {
  const unsigned sh = off & 31u;
  const unsigned long long x = acc << sh;
  LAS unsigned* q = S + (off >> 5);
  lds_or(q, (unsigned)x); lds_or(q + 1, (unsigned)(x >> 32)); lds_or(q + 2, ((unsigned)(acc >> 32) >> 1) >> (31u - sh));
}
__device__ __forceinline__ unsigned fetch32(const LAS unsigned* S, unsigned off) {
  const LAS unsigned* q = S + (off >> 5);
  return __builtin_amdgcn_alignbit(q[1], q[0], off & 31u);
}
__device__ __forceinline__ unsigned long long fetch64(const LAS unsigned* S, unsigned off) {
  const LAS unsigned* q = S + (off >> 5);
  const unsigned a = q[0], b = q[1], c = q[2];
  return ((unsigned long long)__builtin_amdgcn_alignbit(c, b, off & 31u) << 32) | __builtin_amdgcn_alignbit(b, a, off & 31u);
}
__device__ __forceinline__ unsigned wave_sum_lane63(unsigned x) { return dpp_shr_add(x); }
__device__ __forceinline__ unsigned sym_of(bool out, unsigned base, unsigned t) { return out ? ((base - (t & 7u)) | ((t & 8u) << 4)) & 0xFFu : (base - t) & 0xFFu; }

template <bool OUT>
__device__ __forceinline__ unsigned long long encode_chunk(const unsigned (&v)[16], LAS unsigned* S, int lane, u32x4& P0, u32x4& P1, u32x2& F) {
  u16x2 mx = __builtin_bit_cast(u16x2, v[0] & 0x7FFF7FFFu);
#pragma unroll
  for (int i = 1; i < 16; ++i) mx = __builtin_elementwise_max(mx, __builtin_bit_cast(u16x2, v[i] & 0x7FFF7FFFu));
  const unsigned base = wave_max_u32(max((unsigned)mx.x, (unsigned)mx.y)) >> (OUT ? 8 : 7);
  const unsigned B4 = base * 0x01010101u;
  *(LAS u32x4*)(S + lane * 4) = (u32x4){0u, 0u, 0u, 0u};
  if (lane < (OUT ? 34 : 1)) *(LAS u32x4*)(S + 256 + lane * 4) = (u32x4){0u, 0u, 0u, 0u};
  unsigned H[8], L[8], R[8], oor[8], anyo = 0;
#pragma unroll
  for (int g = 0; g < 8; ++g) {
    const unsigned Hs = vperm(v[2 * g + 1], v[2 * g], 0x07050301u), Ls = vperm(v[2 * g + 1], v[2 * g], 0x06040200u);
    if (OUT) { H[g] = Hs; L[g] = Ls; const unsigned D = B4 - (Hs & 0x7F7F7F7Fu); R[g] = (D & 0x07070707u) | ((Hs >> 4) & 0x08080808u); oor[g] = D & 0x78787878u; }
    else { H[g] = bfi(0x01010101u, Ls >> 7, Hs << 1); L[g] = bfi(0x80808080u, Hs, Ls); const unsigned D = B4 - H[g]; R[g] = D & 0x0F0F0F0Fu; oor[g] = D & 0xF0F0F0F0u; }
    anyo |= oor[g];
  }
  unsigned key;
  {
    unsigned long long hc = 0;
#pragma unroll
    for (int b = 0; b < 8; ++b) hc += 1ull << (((R[b >> 2] >> (8 * (b & 3))) & 15u) * 4u);
    const unsigned lo = (unsigned)hc, hi = (unsigned)(hc >> 32);
    unsigned q[8];
#pragma unroll
    for (int k = 0; k < 8; ++k) q[k] = rdl(dpp_shr_add(((lo >> (4 * k)) & 15u) | (((hi >> (4 * k)) & 15u) << 16)), 63);
    const int k7 = lane & 7;
    const unsigned qa = (k7 & 1) ? ((k7 & 2) ? q[3] : q[1]) : ((k7 & 2) ? q[2] : q[0]), qb = (k7 & 1) ? ((k7 & 2) ? q[7] : q[5]) : ((k7 & 2) ? q[6] : q[4]);
    const unsigned cnt = (((k7 & 4) ? qb : qa) >> ((lane & 8) * 2)) & 0xFFFFu;
    key = (lane < 16) ? (((cnt + 1u) << 4) | (unsigned)lane) : 0u;
  }
  unsigned tt = 0, cl = 0x1Fu;
#pragma unroll
  for (int i = 0; i < 7; ++i) {
    const unsigned bias = (OUT && i == 0 && lane >= 8 && lane < 16) ? (32u << 4) : 0u;
    const unsigned m = rdl(row_max(key + bias), 15);
    tt |= (m & 15u) << (4 * i);
    if (key + bias == m) { key = 0u; cl = (i < 3) ? (unsigned)i : (i == 3) ? 3u : (7u | ((unsigned)(i - 4) << 3)); }
  }
  cl <<= 8 * (lane & 3);
  cl |= (unsigned)__builtin_amdgcn_update_dpp(0, (int)cl, 0xB1, 0xf, 0xf, false);
  cl |= (unsigned)__builtin_amdgcn_update_dpp(0, (int)cl, 0x4E, 0xf, 0xf, false);
  const unsigned tb0 = rdl(cl, 0), tb1 = rdl(cl, 4), tb2 = rdl(cl, 8), tb3 = rdl(cl, 12);
  unsigned cls[8];
#pragma unroll
  for (int g = 0; g < 8; ++g) {
    const unsigned r7 = R[g] & 0x07070707u;
    cls[g] = bfi(bmask((R[g] >> 3) & 0x01010101u), vperm(tb3, tb2, r7), vperm(tb1, tb0, r7));
  }
  const bool has_oor = __any(anyo != 0u);
  unsigned e4[8];
#pragma unroll
  for (int g = 0; g < 8; ++g) e4[g] = 0u;
  if (has_oor) {
    const unsigned T1 = ((tt >> 4) & 15u) * 0x01010101u;
#pragma unroll
    for (int g = 0; g < 8; ++g) {
      e4[g] = nz4(OUT ? (oor[g] >> 3) : (oor[g] >> 4));
      const unsigned m = bmask(e4[g]);
      cls[g] |= m & 0x1F1F1F1Fu; R[g] = bfi(m, T1, R[g]);
    }
  }
  bool modeB = false; unsigned n0 = 0;
  if (OUT) {
    unsigned z[8], mxn = 0;
#pragma unroll
    for (int g = 0; g < 8; ++g) z[g] = lut4(0x000000FFu, cls[g] & 0x03030303u);
#pragma unroll
    for (int g = 0; g < 2; ++g) {
      const unsigned hn = (L[g] >> 4) & z[g] & 0x0F0F0F0Fu;
      mxn = max(mxn, max(max(hn & 0xFFu, (hn >> 8) & 0xFFu), max((hn >> 16) & 0xFFu, hn >> 24)));
    }
    n0 = wave_max_u32(mxn);
    unsigned viol[8], nv = 0;
    const unsigned N4 = (n0 | 0x80u) * 0x01010101u;
#pragma unroll
    for (int g = 0; g < 8; ++g) {
      const unsigned dn = (N4 - ((L[g] >> 4) & 0x0F0F0F0Fu)) & 0x7F7F7F7Fu;
      viol[g] = z[g] & bmask(((dn + 0x7D7D7D7Du) >> 7) & 0x01010101u);
      nv += pc(viol[g] & 0x01010101u);
    }
    modeB = n0 < 8u && rdl(dpp_shr_add(nv), 63) < 24u;
    if (modeB) {
#pragma unroll
      for (int g = 0; g < 8; ++g) cls[g] |= viol[g] & 0x1F1F1F1Fu;
    }
  }
  unsigned e2[8], e2b[8], e3[8], e0[8], m2[8], n2 = 0, n2b = 0, n3 = 0, n4 = 0, c0 = 0, cm2 = 0;
  const unsigned N0 = n0 * 0x01010101u;
#pragma unroll
  for (int g = 0; g < 8; ++g) {
    const unsigned c1 = cls[g] & 0x03030303u;
    e2[g] = lut4(0x01000000u, c1); e2b[g] = (cls[g] >> 2) & 0x01010101u; e3[g] = lut4(0x01000000u, (cls[g] >> 3) & 0x03030303u);
    n2 += pc(e2[g]); n2b += pc(e2b[g]); n3 += pc(e3[g]); n4 += pc(e4[g]);
    if (OUT) {
      e0[g] = modeB ? lut4(0x00000001u, c1) : 0u;
      m2[g] = e0[g] & nz4(((L[g] >> 4) & 0x0F0F0F0Fu) ^ N0);
      c0 += pc(e0[g]); cm2 += pc(m2[g]);
    } else { e0[g] = 0u; m2[g] = 0u; }
  }
  const unsigned x1 = c0 | (n2 << 16), x2 = cm2 | (n2b << 16), x3 = n3 | (n4 << 16);
  const unsigned s1 = dpp_shr_add(x1), s2 = dpp_shr_add(x2), s3 = dpp_shr_add(x3);
  const unsigned t1 = rdl(s1, 63), t2 = rdl(s2, 63), t3 = rdl(s3, 63);
  const unsigned C0 = t1 & 0xFFFFu, N2 = t1 >> 16, CM2 = t2 & 0xFFFFu, N2B = t2 >> 16, N3 = t3 & 0xFFFFu, N4c = t3 >> 16;
  const unsigned bSN = OUT ? 4u * (2048u - C0) : 0u;
  const unsigned aM1 = bSN, a2a = aM1 + C0, aM2 = a2a + N2, a2b = aM2 + CM2, a3 = a2b + 2u * N2B, a4 = a3 + 4u * N3, bits = a4 + 8u * N4c;
  const bool raw = bits > (OUT ? 12288u : 4096u);
  const unsigned len = ((OUT ? 12288u : 20480u) + bits + 255u) >> 8;
  if (!raw) {
    const unsigned p1 = s1 - x1, p2 = s2 - x2, p3 = s3 - x3;
    const unsigned oN = 4u * (32u * (unsigned)lane - (p1 & 0xFFFFu)), oM1 = aM1 + (p1 & 0xFFFFu), o2a = a2a + (p1 >> 16), oM2 = aM2 + (p2 & 0xFFFFu),
                   o2b = a2b + 2u * (p2 >> 16), o3 = a3 + 4u * (p3 & 0xFFFFu);
    unsigned o4 = a4 + 8u * (p3 >> 16);
    unsigned wM1 = 0, wM2 = 0, w2a = 0, kM1 = 0, kM2 = 0, k2a = 0, k2b = 0, kN = 0, k3 = 0, kNa = 0, k3a = 0;
    unsigned long long w2b = 0, wN = 0, w3 = 0;
#pragma unroll
    for (int g = 0; g < 8; ++g) {
      if (g == 4) {
        if (OUT) { flush64(S, oN, wN); wN = 0; kNa = kN; kN = 0; }
        flush64(S, o3, w3); w3 = 0; k3a = k3; k3 = 0;
      }
      if (OUT) {
        const unsigned hn = (L[g] >> 4) & 0x0F0F0F0Fu, eN = e0[g] ^ 0x01010101u;
        wN |= (unsigned long long)pack_bits<4>(hn, eN) << kN; kN += 4u * pc(eN);
        wM1 |= pack_bits<1>(m2[g], e0[g]) << kM1; kM1 += pc(e0[g]);
        wM2 |= pack_bits<1>(nz4(((N0 | 0x10101010u) - 0x01010101u - hn) & 0x0F0F0F0Fu), m2[g]) << kM2; kM2 += pc(m2[g]);
      }
      w2a |= pack_bits<1>(e2b[g], e2[g]) << k2a; k2a += pc(e2[g]);
      w2b |= (unsigned long long)pack_bits<2>((cls[g] >> 3) & 0x03030303u, e2b[g]) << k2b; k2b += 2u * pc(e2b[g]);
      w3 |= (unsigned long long)pack_bits<4>(R[g], e3[g]) << k3; k3 += 4u * pc(e3[g]);
    }
    if (OUT) { flush64(S, oN + kNa, wN); flush32(S, oM1, wM1); flush32(S, oM2, wM2); }
    flush32(S, o2a, w2a); flush64(S, o2b, w2b); flush64(S, o3 + k3a, w3);
    if (has_oor) {
#pragma unroll
      for (int g = 0; g < 8; ++g)
#pragma unroll
        for (int b = 0; b < 4; ++b)
          if ((e4[g] >> (8 * b)) & 1u) {
            const unsigned long long x = (unsigned long long)((H[g] >> (8 * b)) & 0xFFu) << (o4 & 31u);
            __hip_atomic_fetch_or(S + (o4 >> 5), (unsigned)x, __ATOMIC_RELAXED, __HIP_MEMORY_SCOPE_WAVEFRONT);
            __hip_atomic_fetch_or(S + (o4 >> 5) + 1, (unsigned)(x >> 32), __ATOMIC_RELAXED, __HIP_MEMORY_SCOPE_WAVEFRONT);
            o4 += 8u;
          }
    }
  }
  unsigned Fw[2];
#pragma unroll
  for (int i = 0; i < 2; ++i)
    Fw[i] = (cls[4 * i] & 0x03030303u) | ((cls[4 * i + 1] & 0x03030303u) << 2) | ((cls[4 * i + 2] & 0x03030303u) << 4) | ((cls[4 * i + 3] & 0x03030303u) << 6);
  F = (u32x2){Fw[0], Fw[1]};
  if (OUT) {
    unsigned p4[4];
#pragma unroll
    for (int i = 0; i < 4; ++i) p4[i] = (L[2 * i] & 0x0F0F0F0Fu) | ((L[2 * i + 1] & 0x0F0F0F0Fu) << 4);
    P0 = (u32x4){p4[0], p4[1], p4[2], p4[3]}; P1 = P0;
  } else {
    P0 = (u32x4){L[0], L[1], L[2], L[3]}; P1 = (u32x4){L[4], L[5], L[6], L[7]};
  }
  return (unsigned long long)(base | (raw ? 0x8000u : (len << 8))) | ((unsigned long long)tt << 16) | (modeB ? (1ull << 44) | ((unsigned long long)n0 << 45) : 0ull);
}
__device__ __forceinline__ bool hp_raw(unsigned long long hp) { return ((unsigned)hp & 0x8000u) != 0u; }
__device__ __forceinline__ unsigned hp_len(unsigned long long hp) { return (((unsigned)hp >> 8) & 127u) * 32u; }

template <bool OUT>
__device__ __forceinline__ void store_chunk(rsrc_t r, unsigned o, const LAS unsigned* S, unsigned len, int lane, u32x4 P0, u32x4 P1, u32x2 F) {
  const unsigned l16 = (unsigned)lane * 16u;
  st128s(P0, r, o + l16);
  const u32x4 sv = *(const LAS u32x4*)(S + lane * 4);
  if (OUT) {
    const u32x4 sw = *(const LAS u32x4*)(S + 256 + (lane & 31) * 4);
    __builtin_amdgcn_raw_buffer_store_b64(F, r, (int)(o + 1024 + lane * 8), 0, AUX_SYS);
    if (1536u + l16 < len) st128s(sv, r, o + 1536 + l16);
    if (lane < 32 && 2560u + l16 < len) st128s(sw, r, o + 2560 + l16);
  } else {
    st128s(P1, r, o + 1024 + l16);
    __builtin_amdgcn_raw_buffer_store_b64(F, r, (int)(o + 2048 + lane * 8), 0, AUX_SYS);
    if (lane < 32 && 2560u + l16 < len) st128s(sv, r, o + 2560 + l16);
  }
}

template <bool OUT>
__device__ __forceinline__ void decode_rec(const LAS unsigned* R, unsigned long long hp, int lane, unsigned (&v)[16]) {
  const unsigned base = (unsigned)hp & 0xFFu, tt = (unsigned)(hp >> 16) & 0xFFFFFFFu;
  const bool modeB = ((hp >> 44) & 1u) != 0u;
  const unsigned n0 = (unsigned)(hp >> 45) & 7u;
  const u32x4 P0 = *(const LAS u32x4*)(R + lane * 4), P1 = OUT ? P0 : *(const LAS u32x4*)(R + 256 + lane * 4);
  const u32x2 F = *(const LAS u32x2*)(R + (OUT ? 256 : 512) + lane * 2);
  const LAS unsigned* S = R + (OUT ? 384 : 640);
  unsigned tab1 = 0, tab2 = 0;
#pragma unroll
  for (int i = 0; i < 4; ++i) tab1 |= sym_of(OUT, base, (tt >> (4 * i)) & 15u) << (8 * i);
#pragma unroll
  for (int i = 0; i < 3; ++i) tab2 |= sym_of(OUT, base, (tt >> (16 + 4 * i)) & 15u) << (8 * i);
  const unsigned B4 = base * 0x01010101u, T1 = ((tt >> 4) & 15u) * 0x01010101u, N0 = n0 * 0x01010101u;
  unsigned c1[8], e2[8], e0[8], n2 = 0, c0 = 0;
#pragma unroll
  for (int g = 0; g < 8; ++g) {
    c1[g] = (((g < 4) ? F.x : F.y) >> (2 * (g & 3))) & 0x03030303u;
    e2[g] = lut4(0x01000000u, c1[g]); n2 += pc(e2[g]);
    if (OUT) { e0[g] = modeB ? lut4(0x00000001u, c1[g]) : 0u; c0 += pc(e0[g]); } else e0[g] = 0u;
  }
  const unsigned x1 = c0 | (n2 << 16), s1 = dpp_shr_add(x1), t1 = rdl(s1, 63), p1 = s1 - x1;
  const unsigned C0 = t1 & 0xFFFFu, N2 = t1 >> 16;
  const unsigned aM1 = OUT ? 4u * (2048u - C0) : 0u, a2a = aM1 + C0, aM2 = a2a + N2;
  const unsigned oN = 4u * (32u * (unsigned)lane - (p1 & 0xFFFFu));
  unsigned hn[8], m2[8], e2b[8], cm2 = 0, n2b = 0;
  {
    unsigned w2a = fetch32(S, a2a + (p1 >> 16)), wM1 = 0; unsigned long long wN = 0, wN2 = 0;
    if (OUT) {
      wM1 = fetch32(S, aM1 + (p1 & 0xFFFFu)); wN = fetch64(S, oN);
      wN2 = fetch64(S, oN + 4u * (16u - pc(e0[0]) - pc(e0[1]) - pc(e0[2]) - pc(e0[3])));
    }
#pragma unroll
    for (int g = 0; g < 8; ++g) {
      e2b[g] = unpack_bits<1>(w2a, e2[g]); w2a >>= pc(e2[g]); n2b += pc(e2b[g]);
      if (OUT) {
        if (g == 4) wN = wN2;
        const unsigned eN = e0[g] ^ 0x01010101u;
        hn[g] = unpack_bits<4>((unsigned)wN, eN); wN >>= 4u * pc(eN);
        m2[g] = unpack_bits<1>(wM1, e0[g]); wM1 >>= pc(e0[g]); cm2 += pc(m2[g]);
      }
    }
  }
  const unsigned x2 = cm2 | (n2b << 16), s2 = dpp_shr_add(x2), t2 = rdl(s2, 63), p2 = s2 - x2;
  const unsigned CM2 = t2 & 0xFFFFu, N2B = t2 >> 16, a2b = aM2 + CM2, a3 = a2b + 2u * N2B;
  unsigned wM2 = OUT ? fetch32(S, aM2 + (p2 & 0xFFFFu)) : 0u;
  unsigned long long w2b = fetch64(S, a2b + 2u * (p2 >> 16));
  unsigned H[8], L[8], e3[8], n3 = 0;
#pragma unroll
  for (int g = 0; g < 8; ++g) {
    const unsigned c2 = unpack_bits<2>((unsigned)w2b, e2b[g]); w2b >>= 2u * pc(e2b[g]);
    e3[g] = lut4(0x01000000u, c2); n3 += pc(e3[g]);
    H[g] = bfi(bmask(e2b[g]), lut4(tab2, c2), lut4(tab1, c1[g]));
    if (OUT) {
      const unsigned b2 = unpack_bits<1>(wM2, m2[g]); wM2 >>= pc(m2[g]);
      const unsigned nib = hn[g] | ((N0 - m2[g] - b2) & bmask(e0[g]));
      const unsigned p4 = (g >> 1) == 0 ? P0.x : (g >> 1) == 1 ? P0.y : (g >> 1) == 2 ? P0.z : P0.w;
      L[g] = ((p4 >> (4 * (g & 1))) & 0x0F0F0F0Fu) | (nib << 4);
    } else {
      L[g] = g == 0 ? P0.x : g == 1 ? P0.y : g == 2 ? P0.z : g == 3 ? P0.w : g == 4 ? P1.x : g == 5 ? P1.y : g == 6 ? P1.z : P1.w;
    }
  }
  const unsigned s3 = dpp_shr_add(n3), N3 = rdl(s3, 63);
  const unsigned o3 = a3 + 4u * (s3 - n3);
  unsigned long long w3 = fetch64(S, o3);
  const unsigned long long w3b = fetch64(S, o3 + 4u * (pc(e3[0]) + pc(e3[1]) + pc(e3[2]) + pc(e3[3])));
  unsigned e4[8], any4 = 0;
#pragma unroll
  for (int g = 0; g < 8; ++g) {
    if (g == 4) w3 = w3b;
    unsigned R = unpack_bits<4>((unsigned)w3, e3[g]); w3 >>= 4u * pc(e3[g]);
    e4[g] = e3[g] & ~nz4(R ^ T1);
    R &= ~bmask(e4[g]);
    const unsigned H3 = OUT ? ((B4 - (R & 0x07070707u)) | ((R & 0x08080808u) << 4)) : (B4 - R);
    H[g] = bfi(bmask(e3[g]), H3, H[g]);
    any4 |= e4[g];
  }
  if (__any(any4 != 0u)) {
    unsigned n4 = 0;
#pragma unroll
    for (int g = 0; g < 8; ++g) n4 += pc(e4[g]);
    unsigned o4 = a3 + 4u * N3 + 8u * (dpp_shr_add(n4) - n4);
#pragma unroll
    for (int g = 0; g < 8; ++g) { H[g] = bfi(bmask(e4[g]), unpack_bits<8>(fetch32(S, o4), e4[g]), H[g]); o4 += 8u * pc(e4[g]); }
  }
#pragma unroll
  for (int g = 0; g < 8; ++g) {
    const unsigned Hs = OUT ? H[g] : bfi(0x80808080u, L[g], H[g] >> 1), Ls = OUT ? L[g] : bfi(0x80808080u, H[g] << 7, L[g]);
    v[2 * g] = vperm(Hs, Ls, 0x05010400u);
    v[2 * g + 1] = vperm(Hs, Ls, 0x07030602u);
  }
}

struct MapX { unsigned base; __device__ __forceinline__ unsigned operator()(int w, int j, int l) const { return base + (unsigned)(w * 4096 + j * 1024 + l * 16); } };
struct MapW { unsigned base; __device__ __forceinline__ unsigned operator()(int w, int j, int l) const { return base + (unsigned)((j >> 1) * 16384 + ((w >> 1) * 2 + (j & 1)) * 2048 + (w & 1) * 1024 + l * 16); } };
struct MapO { unsigned base; __device__ __forceinline__ unsigned operator()(int w, int j, int l) const { return base + (unsigned)((64 * (w >> 2) + 16 * (j >> 1) + (l & 15)) * (ND * 2) + (64 * (w & 3) + 32 * (j & 1) + 8 * (l >> 4)) * 2); } };
__device__ __forceinline__ unsigned out_task_base(int rank, int task) {
  const int tile = task >> 2, q = task & 3, pm = tile >> 3, pn = tile & 7;
  return (unsigned)(((rank >> 2) * 4096 + pm * 256 + (q >> 1) * 128 + (q & 1) * 32) * (ND * 2) + ((rank & 3) * 2048 + pn * 256) * 2);
}

__device__ __forceinline__ unsigned long long mk_granule(unsigned long long payload, unsigned call) { return payload | ((unsigned long long)(call & 0xFFFFu) << 48); }
__device__ __forceinline__ unsigned long long uni64(unsigned long long g) { return ((unsigned long long)rfl((unsigned)(g >> 32)) << 32) | rfl((unsigned)g); }
__device__ __forceinline__ unsigned long long ld_granule(const char* ws, unsigned off) {
  return __hip_atomic_load((const gu64*)(ws + off), __ATOMIC_RELAXED, __HIP_MEMORY_SCOPE_SYSTEM);
}
__device__ __forceinline__ bool tag_ok(unsigned long long g, unsigned call) { return (unsigned)(g >> 48) == (call & 0xFFFFu); }
__device__ __forceinline__ bool poll_granule(const Ctx& cx, const char* ws, unsigned off, unsigned long long& g) {
  g = uni64(g);
  if (!tag_ok(g, cx.call)) {
    const unsigned long long t0 = rt();
    do {
      if (cx.give_up(t0)) return false;
      g = uni64(ld_granule(ws, off));
    } while (!tag_ok(g, cx.call));
  }
  asm volatile("" ::: "memory");
  return true;
}

template <class Map>
__device__ __forceinline__ void nan_chunk(rsrc_t fin, const Map& map, int wid, int lane) {
#pragma unroll
  for (int j = 0; j < 4; ++j) st128s((u32x4){0x7FC07FC0u, 0x7FC07FC0u, 0x7FC07FC0u, 0x7FC07FC0u}, fin, map(wid, j, lane));
}

struct InTask { int kt, isw, idx; };
__device__ __forceinline__ InTask in_task(int i) {
  InTask t;
  if (i < 768) { t.kt = i / 48; const int u = i - t.kt * 48, u3 = u / 3, r3 = u - u3 * 3;
    if (r3 == 0) { t.isw = 0; t.idx = (u3 & 1) * 16 + (u3 >> 1); } else { t.isw = 1; const int c = u3 * 2 + r3 - 1; t.idx = (c & 3) * 8 + (c >> 2); } }
  else { const int u = i - 768; t.kt = (u >> 4) & 15; t.isw = 0; t.idx = (u & 1) * 16 + 8 + ((u & 15) >> 1); }
  return t;
}
__device__ __forceinline__ int otask_of(int isw, int idx, int kt) { return isw * 512 + idx * 16 + kt; }
__device__ __forceinline__ void in_load(const Params& p, const InTask& t, u32x4 (&r)[4], int wid, int lane) {
  const int rr = lane >> 2, cp = (lane & 3) ^ ((rr >> 3) << 1);
  if (!t.isw) {
    const rsrc_t rx = mkr(p.x, IN_BYTES);
#pragma unroll
    for (int j = 0; j < 4; ++j) {
      const unsigned off = (unsigned)((256 * t.idx + 32 * wid + 16 * (j >> 1) + rr) * (KS * 2) + (64 * t.kt + 32 * (j & 1) + 8 * cp) * 2);
      r[j] = __builtin_amdgcn_raw_buffer_load_b128(rx, (int)off, 0, 0);
    }
  } else {
    const rsrc_t rw = mkr(p.w, IN_BYTES);
    const unsigned off0 = (unsigned)((64 * t.kt + 32 * (wid & 1) + 8 * cp) * (ND * 2) + (256 * t.idx + 64 * (wid >> 1) + 4 * rr) * 2);
#pragma unroll
    for (int d = 0; d < 4; ++d) {
      const u32x2 a = __builtin_amdgcn_raw_buffer_load_b64(rw, (int)(off0 + (unsigned)(2 * d) * (ND * 2)), 0, 0);
      const u32x2 b = __builtin_amdgcn_raw_buffer_load_b64(rw, (int)(off0 + (unsigned)(2 * d + 1) * (ND * 2)), 0, 0);
      r[d] = __builtin_shufflevector(a, b, 0, 1, 2, 3);
    }
  }
}
__device__ __forceinline__ void in_prep(int isw, const u32x4 (&r)[4], unsigned (&v)[16]) {
  if (isw) {
#pragma unroll
    for (int d = 0; d < 4; ++d) {
      v[0 + d] = __builtin_amdgcn_perm(r[d].z, r[d].x, 0x05040100u);
      v[4 + d] = __builtin_amdgcn_perm(r[d].z, r[d].x, 0x07060302u);
      v[8 + d] = __builtin_amdgcn_perm(r[d].w, r[d].y, 0x05040100u);
      v[12 + d] = __builtin_amdgcn_perm(r[d].w, r[d].y, 0x07060302u);
    }
  } else {
#pragma unroll
    for (int j = 0; j < 4; ++j) { v[4 * j] = r[j].x; v[4 * j + 1] = r[j].y; v[4 * j + 2] = r[j].z; v[4 * j + 3] = r[j].w; }
  }
}

__device__ __forceinline__ void role_encode(const Params& p, const Ctx& cx, LAS unsigned* S, int e, int ne, int iend) {
  const int tid = threadIdx.x, wid = (int)rfl(tid >> 6), lane = tid & 63;
  char* ws = p.ws[p.rank];
  const rsrc_t lw = mkr(ws, WS_END);
  u32x4 A[4];
  if (e >= iend) return;
  in_load(p, in_task(e), A, wid, lane);
  DRAIN_ALL();
  for (int i = e; i < iend; i += ne) {
    const InTask t = in_task(i);
    unsigned v[16];
    in_prep(t.isw, A, v);
    if (i + ne < iend) in_load(p, in_task(i + ne), A, wid, lane);
    u32x4 P0, P1; u32x2 F;
    const unsigned long long hp = encode_chunk<false>(v, S, lane, P0, P1, F);
    const int ot = otask_of(t.isw, t.idx, t.kt);
    const bool mine = t.isw ? ((t.idx >> 3) == (p.rank & 3)) : ((t.idx >> 4) == (p.rank >> 2));
    const int img = t.isw ? 2048 + (t.idx & 7) * 128 + t.kt * 8 + p.rank : (t.idx & 15) * 128 + t.kt * 8 + p.rank;
    if (!hp_raw(hp)) {
      store_chunk<false>(lw, WS_OBI + (unsigned)ot * PKT + (unsigned)wid * 3072u, S, hp_len(hp), lane, P0, P1, F);
    } else {
#pragma unroll
      for (int j = 0; j < 4; ++j) st128s((u32x4){v[4 * j], v[4 * j + 1], v[4 * j + 2], v[4 * j + 3]}, lw, WS_OBR + (unsigned)ot * IMG + (unsigned)(wid * 4096 + j * 1024 + lane * 16));
    }
    if (mine) {
      const unsigned ib = WS_IMG + (unsigned)img * IMG;
#pragma unroll
      for (int j = 0; j < 4; ++j) st128s((u32x4){v[4 * j], v[4 * j + 1], v[4 * j + 2], v[4 * j + 3]}, lw, t.isw ? MapW{ib}(wid, j, lane) : MapX{ib}(wid, j, lane));
    }
    DRAIN_ALL();
    if (lane == 0) {
      __hip_atomic_store((gu64*)(ws + WS_HOBI + (unsigned)ot * 64u) + wid, mk_granule(hp, cx.call), __ATOMIC_RELAXED, __HIP_MEMORY_SCOPE_SYSTEM);
      if (mine) __hip_atomic_fetch_add((gu32*)(ws + WS_CNT) + img * 32, 1u, __ATOMIC_RELAXED, __HIP_MEMORY_SCOPE_AGENT);
    }
  }
}

__device__ __forceinline__ bool eo_ready(const Ctx& cx, const gu32* f, bool block) {
  if (rfl(__hip_atomic_load(f, __ATOMIC_RELAXED, __HIP_MEMORY_SCOPE_AGENT)) >= cx.call) return true;
  if (!block) return false;
  const unsigned long long t0 = rt();
  while (rfl(__hip_atomic_load(f, __ATOMIC_RELAXED, __HIP_MEMORY_SCOPE_AGENT)) < cx.call) { if (cx.give_up(t0)) return false; }
  return true;
}
__device__ __forceinline__ void role_encode_out(const Params& p, const Ctx& cx, LAS unsigned* S, int e, int ne) {
  const int tid = threadIdx.x, wid = (int)rfl(tid >> 6), lane = tid & 63;
  char* ws = p.ws[p.rank];
  const rsrc_t lw = mkr(ws, WS_END), ro = mkr(p.out[p.rank], OUT_BYTES);
  const gu32* rf = (const gu32*)(ws + WS_RFLG) + wid * 4;
  auto load = [&](int task, u32x4 (&A)[4]) { const MapO map{out_task_base(p.rank, task)};
#pragma unroll
    for (int j = 0; j < 4; ++j) A[j] = ld128s(ro, map(wid, j, lane)); };
  u32x4 A[4];
#pragma unroll
  for (int j = 0; j < 4; ++j) {
    unsigned t[4];
#pragma unroll
    for (int k = 0; k < 4; ++k) t[k] = ((0x3FBE3EBDu >> (((lane + 4 * j + k) & 3) * 8)) & 0xFFu) * 0x01000100u + ((unsigned)(lane * 29 + (4 * j + k) * 7) & 0x1Fu) * 0x00010003u;
    A[j] = (u32x4){t[0], t[1], t[2], t[3]};
  }
#pragma unroll 1
  for (int i = e - ne; i < NOT; i += ne) {
    unsigned v[16];
#pragma unroll
    for (int j = 0; j < 4; ++j) { v[4 * j] = A[j].x; v[4 * j + 1] = A[j].y; v[4 * j + 2] = A[j].z; v[4 * j + 3] = A[j].w; }
    const int nx = i + ne;
    bool have = false;
    if (nx < NOT) { have = eo_ready(cx, rf + nx * 32, false); if (have) load(nx, A); }
    u32x4 P0, P1; u32x2 F;
    const unsigned long long hp = encode_chunk<true>(v, S, lane, P0, P1, F);
    if (!hp_raw(hp)) store_chunk<true>(lw, i < 0 ? 0xFFFF0000u : WS_OBO + (unsigned)i * PKT + (unsigned)wid * 3072u, S, hp_len(hp), lane, P0, P1, F);
    DRAIN_ALL();
    if (i >= 0 && lane == 0) __hip_atomic_store((gu64*)(ws + WS_HOBO + (unsigned)i * 64u) + wid, mk_granule(hp, cx.call), __ATOMIC_RELAXED, __HIP_MEMORY_SCOPE_SYSTEM);
    if (nx < NOT && !have) {
      if (!eo_ready(cx, rf + nx * 32, true)) return;
      load(nx, A);
      DRAIN_ALL();
    }
  }
}

struct PItem { unsigned hsrc, dsrc, hdst, ddst, rs, rd; int kind; };
__device__ __forceinline__ PItem p_item(int rank, int d, int i) {
  PItem r;
  if (i < 384) {
    int kt, isw, k;
    if (i < 256) { kt = i >> 4; isw = i & 1; k = (i & 15) >> 1; } else { kt = (i - 256) >> 3; isw = 0; k = 8 + ((i - 256) & 7); }
    const int idx = isw ? (d & 3) * 8 + k : (d >> 2) * 16 + k;
    const int ot = otask_of(isw, idx, kt), T = kt * 8 + rank, img = isw ? 2048 + k * 128 + T : k * 128 + T;
    const unsigned slot = CFG_SEQIN ? (unsigned)(rank * 384 + i) : (unsigned)img;
    r.hsrc = WS_HOBI + (unsigned)ot * 64u; r.dsrc = WS_OBI + (unsigned)ot * PKT;
    r.hdst = WS_HIN + slot * 64u; r.ddst = WS_PIN + slot * PKT;
    r.rs = WS_OBR + (unsigned)ot * IMG; r.rd = WS_IMG + (unsigned)img * IMG; r.kind = isw;
  } else {
    const int task = (i - 384) & (NOT - 1);
    r.hsrc = WS_HOBO + (unsigned)task * 64u; r.dsrc = WS_OBO + (unsigned)task * PKT;
    r.hdst = WS_HOUT + (unsigned)(rank * NOT + task) * 64u; r.ddst = WS_POUT + (unsigned)(rank * NOT + task) * PKT;
    r.rs = r.rd = out_task_base(rank, task); r.kind = 2;
  }
  return r;
}

#define P_LDG(ws, off) ld_granule(ws, off)

__device__ __forceinline__ void p_publish(const Ctx& cx, const char* ws, char* wd, unsigned hsrc, unsigned hdst, unsigned long long hl, int lane) {
  if (hsrc == 0xFFFFFFFFu) hl = mk_granule(0x8000ull, cx.call);
  else {
    const char* src = ws + hsrc + (lane & 7) * 8;
    while (!__all(tag_ok(hl, cx.call))) {
      u32x2 r; asm volatile("global_load_dwordx2 %0, %1, off sc0 sc1\n\ts_waitcnt vmcnt(0)" : "=&v"(r) : "v"(src) : "memory");
      hl = ((unsigned long long)r.y << 32) | r.x;
    }
  }
  if (lane < 8) {
    const u32x2 v = (u32x2){(unsigned)hl, (unsigned)(hl >> 32)};
    asm volatile("global_store_dwordx2 %0, %1, off sc0 sc1\n\ts_nop 1" :: "v"(wd + hdst + lane * 8), "v"(v) : "memory");
  }
}

__device__ __forceinline__ void st_peer(u32x4 v, char* ptr) { asm volatile("global_store_dwordx4 %0, %1, off sc0 sc1\n\ts_nop 1" :: "v"(ptr), "v"(v) : "memory"); }
__device__ __forceinline__ void glds16(const char* src, unsigned dst) {
  unsigned keep; asm volatile("s_mov_b32 %0, m0\n\ts_mov_b32 m0, %2\n\ts_nop 0\n\tglobal_load_lds_dwordx4 %1, off sc0 sc1\n\ts_mov_b32 m0, %0" : "=&s"(keep) : "v"(src), "s"(rfl(dst)) : "memory");
}
__device__ __forceinline__ void glds4(const char* src, unsigned dst) {
  unsigned keep; asm volatile("s_mov_b32 %0, m0\n\ts_mov_b32 m0, %2\n\ts_nop 0\n\tglobal_load_lds_dword %1, off sc0 sc1\n\ts_mov_b32 m0, %0" : "=&s"(keep) : "v"(src), "s"(rfl(dst)) : "memory");
}
__device__ __forceinline__ u32x2 ld64_nowait(const char* src) {
  u32x2 r; asm volatile("global_load_dwordx2 %0, %1, off sc0 sc1" : "=&v"(r) : "v"(src) : "memory");
  return r;
}
__device__ __forceinline__ unsigned long long u64_of(u32x2 v) { return ((unsigned long long)v.y << 32) | v.x; }
__device__ __forceinline__ void role_push(const Params& p, Ctx& cx, unsigned callraw, unsigned lds_ring, int link, int v0, int nv) {
  constexpr int NC = NPI * 8;
  const int lane = threadIdx.x & 63;
  int d = p.rank + 1 + link; if (d >= NRANK) d -= NRANK;
  if ((unsigned)d >= (unsigned)NRANK) return;
  char* ws = p.ws[p.rank]; char* wd = p.ws[d];
  const rsrc_t lw = mkr(ws, WS_END), dw = mkr(wd, WS_END);
  gu32* tkc = (gu32*)(ws + WS_TK) + link * 32;
  gu32* pcnt = (gu32*)(ws + WS_PCNT) + link * NPI;
  const unsigned lo = (unsigned)lane * 16u;
  constexpr int R0 = CFG_RAWHEAD;
  static_assert(R0 * PJ <= 16, "raw head chunks must be K-tile 0 items");
  int prev; unsigned prev_hsrc = 0, prev_hdst;
  {
    auto head_task = [&](int u) { const int i0 = u >> 3; InTask t; t.kt = 0; t.isw = i0 & 1; t.idx = t.isw ? (d & 3) * 8 + (i0 >> 1) : (d >> 2) * 16 + (i0 >> 1); return t; };
    u32x4 A[R0][4];
#pragma unroll
    for (int r = 0; r < R0; ++r) in_load(p, head_task(v0 + r * nv), A[r], v0 & 7, lane);
    unsigned dummy = 0;
    DRAIN_PIN2(callraw, dummy);
    cx.call = rfl(callraw) + 1u;
#pragma unroll
    for (int r = 0; r < R0; ++r) {
      const int u = v0 + r * nv, w = u & 7;
      const InTask t = head_task(u);
      unsigned v[16];
      in_prep(t.isw, A[r], v);
      const PItem f = p_item(p.rank, d, u >> 3);
      unsigned pc = 0;
      if (CFG_HEADSEQ && r > 0) { VM0(); pc = atomic_add_lane0_nowait(pcnt + (prev >> 3)); }
#pragma unroll
      for (int j = 0; j < 4; ++j) st128s((u32x4){v[4 * j], v[4 * j + 1], v[4 * j + 2], v[4 * j + 3]}, dw, t.isw ? MapW{f.rd}(w, j, lane) : MapX{f.rd}(w, j, lane));
      if (CFG_HEADSEQ && r > 0) {
        asm volatile("s_waitcnt vmcnt(4)" : "+v"(pc) :: "memory");
        if (rfl(pc) == 8u * cx.call - 1u) p_publish(cx, ws, wd, 0xFFFFFFFFu, prev_hdst, 0ull, lane);
      }
      if (r == 0) STAMP(p, 3);
      prev = u; prev_hdst = f.hdst;
    }
    if (!CFG_HEADSEQ && R0 > 1) {
      VM0();
#pragma unroll
      for (int r = 0; r + 1 < R0; ++r) {
        const int u = v0 + r * nv;
        unsigned pc = atomic_add_lane0_nowait(pcnt + (u >> 3)), dummy2 = 0;
        DRAIN_PIN2(pc, dummy2);
        if (rfl(pc) == 8u * cx.call - 1u) p_publish(cx, ws, wd, 0xFFFFFFFFu, p_item(p.rank, d, u >> 3).hdst, 0ull, lane);
      }
    }
  }
  const unsigned tkbase = (cx.call - 1u) * (unsigned)(NC - R0 * nv) - (unsigned)(R0 + 4) * (unsigned)nv;
  gu32* dummy_cnt = (gu32*)(ws + WS_TK) + 8 * 32 + (v0 & 7);
  const unsigned ring = lds_ring, gsl = lds_ring + 3u * 3072u;
  auto hoff = [&](int u) { return p_item(p.rank, d, u >> 3).hsrc + (unsigned)(u & 7) * 8u; };
  auto tail = [&](int u) { return CFG_RAWTAIL > 0 && u >= NC - 8 * CFG_RAWTAIL; };
  auto dma_rec = [&](int u, unsigned long long g, bool on, unsigned sl) {
    const unsigned base = on ? p_item(p.rank, d, u >> 3).dsrc + (unsigned)(u & 7) * 3072u : WS_OBI, len = on ? hp_len(g) : 0u;
    const char* q = ws + base;
#pragma unroll
    for (int k = 0; k < 3; ++k) glds16(q + ((unsigned)k * 1024u + lo < len ? (unsigned)k * 1024u + lo : lo), ring + sl * 3072u + (unsigned)k * 1024u);
  };
  auto dma_granule = [&](int u, unsigned sl) { if (lane < 2) glds4(ws + (u < NC ? hoff(u) : WS_HOBI) + lane * 4, gsl + sl * 8u); };
  auto rd_granule = [&](unsigned sl) { const LAS unsigned* q = (const LAS unsigned*)(size_t)(gsl + sl * 8u); return ((unsigned long long)rfl(q[1]) << 32) | rfl(q[0]); };
  int c = v0 + R0 * nv, c1 = c + nv, c2 = c1 + nv, c3 = c2 + nv;
  unsigned long long gc = P_LDG(ws, hoff(c)), g1 = P_LDG(ws, hoff(c1));
  dma_granule(c2, 0u);
  DRAIN_ALL();
  gc = uni64(gc); g1 = uni64(g1);
  if (!tag_ok(gc, cx.call)) {
    unsigned pc = atomic_add_lane0_nowait(pcnt + (prev >> 3)), dummy = 0;
    DRAIN_PIN2(pc, dummy);
    if (rfl(pc) == 8u * cx.call - 1u) p_publish(cx, ws, wd, 0xFFFFFFFFu, prev_hdst, 0ull, lane);
    prev = -1;
    if (!poll_granule(cx, ws, hoff(c), gc)) return;
  }
  if (prev >= 0) prev_hsrc = 0xFFFFFFFFu;
  unsigned sc = 0, gp = 0;
  bool have1 = tag_ok(g1, cx.call);
  dma_rec(c, gc, !hp_raw(gc) && !tail(c), 0u);
  dma_rec(c1, g1, have1 && !hp_raw(g1) && !tail(c1), 1u);
  DRAIN_ALL();
  while (c < NC) {
    const int w = c & 7;
    const PItem cur = p_item(p.rank, d, c >> 3);
    unsigned tk = atomic_add_lane0_nowait(tkc);
    unsigned pc = atomic_add_lane0_nowait(prev >= 0 ? pcnt + (prev >> 3) : dummy_cnt);
    u32x2 hl = ld64_nowait(ws + ((prev >= 0 && prev_hsrc != 0xFFFFFFFFu) ? prev_hsrc : WS_HOBI) + (lane & 7) * 8);
    dma_granule(c3, gp ^ 1u);
    const bool rawc = hp_raw(gc) || tail(c);
    if (!rawc) {
      const LAS u32x4* q = (const LAS u32x4*)(size_t)(ring + sc * 3072u + lo);
      const u32x4 d0 = q[0], d1 = q[64], d2 = q[128];
      char* o = wd + (size_t)(cur.ddst + (unsigned)w * 3072u + lo); const unsigned len = hp_len(gc);
      st_peer(d0, o);
      if (1024u + lo < len) st_peer(d1, o + 1024);
      if (2048u + lo < len) st_peer(d2, o + 2048);
    } else {
      u32x4 q[4];
      if (cur.kind == 2) {
        const rsrc_t so = mkr(p.out[p.rank], OUT_BYTES), dd = mkr(p.out[d], OUT_BYTES); const MapO map{cur.rs};
#pragma unroll
        for (int j = 0; j < 4; ++j) q[j] = ld128s(so, map(w, j, lane));
#pragma unroll
        for (int j = 0; j < 4; ++j) st128s(q[j], dd, map(w, j, lane));
      } else {
#pragma unroll
        for (int j = 0; j < 4; ++j) q[j] = ld128s(lw, cur.rs + (unsigned)(w * 4096 + j * 1024) + lo);
#pragma unroll
        for (int j = 0; j < 4; ++j) st128s(q[j], dw, cur.kind ? MapW{cur.rd}(w, j, lane) : MapX{cur.rd}(w, j, lane));
      }
      VM0();
    }
    const unsigned long long g2 = rd_granule(gp);
    const bool have2 = c2 < NC && tag_ok(g2, cx.call);
    const unsigned s2 = sc >= 1u ? sc - 1u : 2u;
    dma_rec(c2 < NC ? c2 : 0, g2, have2 && !hp_raw(g2) && !tail(c2), s2);
    asm volatile("s_waitcnt vmcnt(3)" : "+v"(tk), "+v"(pc), "+v"(hl) :: "memory");
    if (prev >= 0 && rfl(pc) == 8u * cx.call - 1u) p_publish(cx, ws, wd, prev_hsrc, prev_hdst, u64_of(hl), lane);
    const int c4 = (int)(rfl(tk) - tkbase);
    prev = c; prev_hsrc = tail(c) ? 0xFFFFFFFFu : cur.hsrc; prev_hdst = cur.hdst;
    const unsigned s1 = sc == 2u ? 0u : sc + 1u;
    if (c1 < NC && !have1) {
      VM0();
      unsigned pc2 = atomic_add_lane0_nowait(pcnt + (c >> 3));
      u32x2 hl2 = ld64_nowait(ws + (prev_hsrc != 0xFFFFFFFFu ? prev_hsrc : WS_HOBI) + (lane & 7) * 8);
      asm volatile("s_waitcnt vmcnt(0)" : "+v"(pc2), "+v"(hl2) :: "memory");
      if (rfl(pc2) == 8u * cx.call - 1u) p_publish(cx, ws, wd, prev_hsrc, prev_hdst, u64_of(hl2), lane);
      prev = -1;
      g1 = 0;
      if (!poll_granule(cx, ws, hoff(c1), g1)) return;
      dma_rec(c1, g1, !hp_raw(g1) && !tail(c1), s1);
      VM0();
    }
    c = c1; c1 = c2; c2 = c3; c3 = c4;
    gc = g1; g1 = g2; have1 = have2; sc = s1; gp ^= 1u;
  }
  if (prev >= 0) {
    VM0();
    unsigned pc = atomic_add_lane0_nowait(pcnt + (prev >> 3));
    u32x2 hl2 = ld64_nowait(ws + (prev_hsrc != 0xFFFFFFFFu ? prev_hsrc : WS_HOBI) + (lane & 7) * 8);
    asm volatile("s_waitcnt vmcnt(0)" : "+v"(pc), "+v"(hl2) :: "memory");
    if (rfl(pc) == 8u * cx.call - 1u) p_publish(cx, ws, wd, prev_hsrc, prev_hdst, u64_of(hl2), lane);
  }
  STAMPL(p, 2);
}

struct DTask { unsigned hl, pk, dst; int kind; int sig; };
__device__ __forceinline__ unsigned d_map(const DTask& t, int w, int j, int l) {
  return t.kind == 0 ? MapX{t.dst}(w, j, l) : t.kind == 1 ? MapW{t.dst}(w, j, l) : MapO{t.dst}(w, j, l);
}
template <bool OUT> __device__ __forceinline__ DTask d_task(int rank, int i) {
  DTask t;
  if (!OUT) {
    int kt, sl, k, isw;
    if (i < 7 * 256) { kt = i / 112; const int u = i - kt * 112; sl = u >> 4; isw = u & 1; k = (u & 15) >> 1; }
    else { const int u = i - 7 * 256; kt = (u / 56) & 15; const int u2 = u - (u / 56) * 56; sl = u2 >> 3; isw = 0; k = 8 + (u2 & 7); }
    int src = rank + 1 + sl; if (src >= NRANK) src -= NRANK;
    const int T = kt * 8 + src, img = isw ? 2048 + k * 128 + T : k * 128 + T;
    const unsigned slot = CFG_SEQIN ? (unsigned)(src * 384 + (k < 8 ? kt * 16 + k * 2 + isw : 256 + kt * 8 + (k - 8))) : (unsigned)img;
    t.hl = WS_HIN + slot * 64u; t.pk = WS_PIN + slot * PKT;
    t.dst = WS_IMG + (unsigned)img * IMG; t.kind = isw; t.sig = img;
  } else {
    const int task = (i / (NRANK - 1)) & (NOT - 1), k = i % (NRANK - 1);
    int s = rank + 1 + k; if (s >= NRANK) s -= NRANK;
    const unsigned tidx = (unsigned)(s * NOT + task);
    t.hl = WS_HOUT + tidx * 64u; t.pk = WS_POUT + tidx * PKT; t.dst = out_task_base(s, task); t.kind = 2; t.sig = 0;
  }
  return t;
}
template <bool OUT>
__device__ __forceinline__ void role_decode(const Params& p, const Ctx& cx, unsigned lds_ring, int dw, int nw) {
  constexpr int TOTAL = OUT ? NOT * (NRANK - 1) : 7 * 384;
  const int tid = threadIdx.x, wid = (int)rfl(tid >> 6), lane = tid & 63;
  char* ws = p.ws[p.rank];
  char* fin = OUT ? p.out[p.rank] : ws;
  gu32* cnt = (gu32*)(ws + WS_CNT);
  const unsigned co = (unsigned)wid * 3072u, ho = (unsigned)wid * 8u, lo = (unsigned)lane * 16u;
  const unsigned ring = lds_ring, gsl = lds_ring + 3u * 3072u;
  int i = dw;
  if (i >= TOTAL) return;
  auto dma_rec = [&](const DTask& t, unsigned long long g, bool on, unsigned sl) {
    const unsigned len = on ? hp_len(g) : 0u;
    const char* q = ws + (on ? t.pk + co : WS_PIN);
#pragma unroll
    for (int k = 0; k < 3; ++k) glds16(q + ((unsigned)k * 1024u + lo < len ? (unsigned)k * 1024u + lo : lo), ring + sl * 3072u + (unsigned)k * 1024u);
  };
  auto dma_granule = [&](int u, unsigned sl) { if (lane < 2) glds4(ws + (u < TOTAL ? d_task<OUT>(p.rank, u).hl + ho : WS_HIN) + lane * 4, gsl + sl * 8u); };
  auto rd_granule = [&](unsigned sl) { const LAS unsigned* q = (const LAS unsigned*)(size_t)(gsl + sl * 8u); return ((unsigned long long)rfl(q[1]) << 32) | rfl(q[0]); };
  auto signal = [&](int img) { if (!OUT && img >= 0) (void)atomic_add_lane0_nowait_noret(cnt + img * 32); };
  DTask cur = d_task<OUT>(p.rank, i), t1 = cur;
  unsigned long long gc = ld_granule(ws, cur.hl + ho), g1 = 0;
  bool ok = poll_granule(cx, ws, cur.hl + ho, gc);
  bool have1 = false;
  if (i + nw < TOTAL) { t1 = d_task<OUT>(p.rank, i + nw); g1 = uni64(ld_granule(ws, t1.hl + ho)); have1 = tag_ok(g1, cx.call); }
  dma_rec(cur, gc, ok && !hp_raw(gc), 0u);
  dma_rec(t1, g1, have1 && !hp_raw(g1), 1u);
  dma_granule(i + 2 * nw, 0u);
  asm volatile("s_waitcnt vmcnt(0)" ::: "memory");
  unsigned sc = 0, gp = 0;
  int prev = -1;
  while (ok) {
    const int n1 = i + nw, n2 = i + 2 * nw, n3 = i + 3 * nw;
    dma_granule(n3, gp ^ 1u);
    const unsigned long long g2 = rd_granule(gp);
    const bool have2 = n2 < TOTAL && tag_ok(g2, cx.call);
    const DTask t2 = d_task<OUT>(p.rank, n2 < TOTAL ? n2 : i);
    dma_rec(t2, g2, have2 && !hp_raw(g2), sc >= 1u ? sc - 1u : 2u);
    if (!hp_raw(gc)) {
      unsigned v[16];
      decode_rec<OUT>((const LAS unsigned*)(size_t)(ring + sc * 3072u), gc, lane, v);
#pragma unroll
      for (int j = 0; j < 4; ++j) st_peer((u32x4){v[4 * j], v[4 * j + 1], v[4 * j + 2], v[4 * j + 3]}, fin + d_map(cur, wid, j, lane));
      asm volatile("s_waitcnt vmcnt(7)" ::: "memory");
    } else {
      asm volatile("s_waitcnt vmcnt(0)" ::: "memory");
    }
    signal(prev);
    prev = cur.sig;
    if (n1 >= TOTAL) break;
    const unsigned s1 = sc == 2u ? 0u : sc + 1u;
    if (!have1) {
      asm volatile("s_waitcnt vmcnt(0)" ::: "memory");
      signal(prev); prev = -1;
      g1 = 0;
      ok = poll_granule(cx, ws, t1.hl + ho, g1);
      dma_rec(t1, g1, ok && !hp_raw(g1), s1);
      asm volatile("s_waitcnt vmcnt(0)" ::: "memory");
    }
    cur = t1; gc = g1; t1 = t2; g1 = g2; have1 = have2; sc = s1; gp ^= 1u; i = n1;
  }
  asm volatile("s_waitcnt vmcnt(0)" ::: "memory");
  if (!OUT) {
    if (ok) signal(prev);
  } else if (!ok) {
    for (; i < TOTAL; i += nw) {
      const DTask t = d_task<OUT>(p.rank, i);
#pragma unroll
      for (int j = 0; j < 4; ++j) st_peer((u32x4){0x7FC07FC0u, 0x7FC07FC0u, 0x7FC07FC0u, 0x7FC07FC0u}, fin + d_map(t, wid, j, lane));
    }
  }
}

__host__ __device__ __forceinline__ int lds_byte(int r, int c) { const int st = (r >> 4) * 2 + (c >> 5), rr = r & 15, cc = c & 31, ob = rr * 64 + cc * 2; return st * 1024 + (ob ^ (((ob >> 9) & 1) << 5)); }
__device__ __forceinline__ unsigned cvt_pk_bf16(float lo, float hi) { unsigned r; asm volatile("v_cvt_pk_bf16_f32 %0, %1, %2" : "=v"(r) : "v"(lo), "v"(hi)); return r; }
__device__ __forceinline__ float gelu1(float y) {
  const float k1 = -2.0f * 0.7978845608028654f * 1.4426950408889634f, k3 = k1 * 0.044715f;
  const float t = y * (k3 * (y * y) + k1);
  return y * __builtin_amdgcn_rcpf(1.0f + __builtin_amdgcn_exp2f(t));
}

__device__ __forceinline__ void gemm_unit(const Params& p, const Ctx& cx, LAS unsigned char* lds, int pm, int pn, int h, int ph) {
  constexpr int NTH = NT / 2;
  const int tid = threadIdx.x, wid = (int)rfl(tid >> 6), lane = tid & 63, wr = wid >> 2, wc = wid & 3, fr = lane & 15, fq = lane >> 4;
  char* ws = p.ws[p.rank];
  const char* cA = ws + WS_IMG + (size_t)(pm * 128 + h) * IMG;
  const char* cB = ws + WS_IMG + (size_t)(2048 + pn * 128 + h) * IMG;
  const gu32* fl = (const gu32*)(ws + WS_CNT) + (size_t)(((lane & 2) ? 2048 + pn * 128 : pm * 128) + h + 2 * (lane & 1)) * 32;
  const unsigned need = 8u * cx.call;
  const unsigned voff = (unsigned)tid * 16u;
  const unsigned ldsw = (unsigned)wid * 1024u;
  const int aoff = lds_byte(wr * 64 + fr, fq * 8), boff = lds_byte(wc * 32 + fr, fq * 8);
  bool dead = false;
#define KT(t) ((((t) >> 2) << 3) + (((t) & 3) << 1))
  auto wait_tiles = [&](int t) {
    if (wid == 0 && !dead) {
      const gu32* f = fl + KT(t) * 32;
      if (!__all(__hip_atomic_load(f, __ATOMIC_RELAXED, __HIP_MEMORY_SCOPE_AGENT) >= need)) {
        const unsigned long long t0 = rt();
        while (!__all(__hip_atomic_load(f, __ATOMIC_RELAXED, __HIP_MEMORY_SCOPE_AGENT) >= need)) { if (cx.give_up(t0)) { dead = true; break; } }
      }
    }
  };
#define SA(b, h) (((b) * 2 + (h)) * HTB)
#define SB(b, h) ((4 + (b) * 2 + (h)) * HTB)
#define STAGE(bufoff, gbase) do { _Pragma("unroll") for (int _i = 0; _i < 2; ++_i) \
    __builtin_amdgcn_global_load_lds((const GAS unsigned*)((const char*)(gbase) + voff + _i * 8192), (LAS unsigned*)(lds + (bufoff) + ldsw + _i * 8192), 16, 0, CFG_AUX_G); } while (0)
#define LDA(dst, b, h) do { _Pragma("unroll") for (int m = 0; m < 4; ++m) _Pragma("unroll") for (int k = 0; k < 2; ++k) dst[m][k] = *(const LAS bf16x8*)(lds + SA(b, h) + aoff + m * 2048 + k * 1024); } while (0)
#define LDB(dst, b, h) do { _Pragma("unroll") for (int n = 0; n < 2; ++n) _Pragma("unroll") for (int k = 0; k < 2; ++k) dst[n][k] = *(const LAS bf16x8*)(lds + SB(b, h) + boff + n * 2048 + k * 1024); } while (0)
#define MMA(ai, bj, At, Bt) do { __builtin_amdgcn_s_setprio(1); _Pragma("unroll") for (int m = 0; m < 4; ++m) _Pragma("unroll") for (int n = 0; n < 2; ++n) _Pragma("unroll") for (int k = 0; k < 2; ++k) \
    acc[ai][bj][m][n] = __builtin_amdgcn_mfma_f32_16x16x32_bf16(Bt[n][k], At[m][k], acc[ai][bj][m][n], 0, 0, 0); __builtin_amdgcn_s_setprio(0); } while (0)
#define WAIT_V(n) asm volatile("s_waitcnt vmcnt(" #n ")" ::: "memory")
#define WAIT_L(n) asm volatile("s_waitcnt lgkmcnt(" #n ")" ::: "memory")
#define BAR __builtin_amdgcn_s_barrier()
#define SCHED __builtin_amdgcn_sched_barrier(0)
  f32x4 acc[2][2][4][2];
#pragma unroll
  for (int a = 0; a < 2; ++a)
#pragma unroll
    for (int b = 0; b < 2; ++b)
#pragma unroll
      for (int m = 0; m < 4; ++m)
#pragma unroll
        for (int n = 0; n < 2; ++n) acc[a][b][m][n] = (f32x4){0.f, 0.f, 0.f, 0.f};
  bf16x8 At[4][2], B0[2][2], B1[2][2];
  wait_tiles(0); wait_tiles(2);
  BARRIER();
  STAMPL(p, 18 + ph);
  STAGE(SB(0, 0), cB); STAGE(SA(0, 0), cA); STAGE(SB(0, 1), cB + HTB); STAGE(SA(0, 1), cA + HTB);
  if (wr == 1) BAR;
  WAIT_V(4); BAR;
  STAGE(SB(1, 0), cB + 2 * IMG); STAGE(SA(1, 0), cA + 2 * IMG); STAGE(SB(1, 1), cB + 2 * IMG + HTB);
  WAIT_V(6); BAR;
  for (int t = 0; t < NTH; t += 2) {
    const bool last = (t == NTH - 2);
    const char* a1 = cA + (size_t)(KT(t) + 2) * IMG;
    const size_t o2 = last ? 0 : (size_t)KT(t + 2) * IMG;
    const char* a2 = cA + o2; const char* b2 = cB + o2;
    const char* a3 = a2 + 2 * IMG; const char* b3 = b2 + 2 * IMG;
    LAS unsigned* fslot = (LAS unsigned*)(lds + LDS_FLAG + ((t >> 1) & 1) * 16);
    const bool chk = (t + 4 < NTH);
    if (chk && wid == 0 && lane < 4) __builtin_amdgcn_global_load_lds(fl + KT(t + 4) * 32, fslot, 4, 0, AUX_SYS);
    LDB(B0, 0, 0); SCHED; LDA(At, 0, 0); STAGE(SA(1, 1), a1 + HTB);
    WAIT_L(8); BAR; WAIT_L(0); MMA(0, 0, At, B0); BAR; SCHED;
    LDB(B1, 0, 1); STAGE(SB(0, 0), b2);
    BAR; WAIT_L(0); MMA(0, 1, At, B1); BAR;
    LDA(At, 0, 1); STAGE(SA(0, 0), a2);
    BAR; WAIT_L(0); MMA(1, 0, At, B0); BAR; SCHED;
    STAGE(SB(0, 1), b2 + HTB);
    WAIT_V(6); BAR; MMA(1, 1, At, B1); BAR;
    LDB(B0, 1, 0); SCHED; LDA(At, 1, 0); STAGE(SA(0, 1), a2 + HTB);
    WAIT_L(8); BAR; WAIT_L(0); MMA(0, 0, At, B0); BAR; SCHED;
    LDB(B1, 1, 1); STAGE(SB(1, 0), b3);
    BAR; WAIT_L(0); MMA(0, 1, At, B1); BAR;
    LDA(At, 1, 1); STAGE(SA(1, 0), a3);
    BAR; WAIT_L(0); MMA(1, 0, At, B0); BAR; SCHED;
    const u32x4 f = *(const LAS u32x4*)fslot;
    STAGE(SB(1, 1), b3 + HTB);
    WAIT_V(6); BAR; MMA(1, 1, At, B1); BAR;
    if (chk) {
      const unsigned fm = rfl(min(min(f.x, f.y), min(f.z, f.w)));
      if (fm < need) { wait_tiles(t + 4); BARRIER(); }
    }
    if (CFG_BSLEEP > 0 && ph == 1) __builtin_amdgcn_s_sleep(CFG_BSLEEP);
    if (CFG_ASLEEP > 0 && ph == 0) __builtin_amdgcn_s_sleep(CFG_ASLEEP);
  }
  if (wr == 0) BAR;
  STAMPL(p, 20 + 2 * ph);
  const int tile = pm * 8 + pn;
  {
    int tz = threadIdx.x; asm volatile("" : "+v"(tz));
    const int ln = tz & 63;
    const rsrc_t ro = mkr(p.out[p.rank], OUT_BYTES), lw = mkr(ws, WS_END);
    {
      const unsigned po = WS_PART + (unsigned)(tile * 2 + h) * 131072u + (unsigned)(wid * 16384 + ln * 16);
      const unsigned pi = WS_PART + (unsigned)(tile * 2 + 1 - h) * 131072u + (unsigned)(wid * 16384 + ln * 16);
#pragma unroll
      for (int i = 0; i < 4; ++i) {
        unsigned d[12];
#pragma unroll
        for (int k = 0; k < 4; ++k) {
          const int ii = 4 * i + k;
          const u32x4 q = __builtin_bit_cast(u32x4, h ? acc[0][ii >> 3][(ii >> 1) & 3][ii & 1] : acc[1][ii >> 3][(ii >> 1) & 3][ii & 1]);
          d[3 * k] = vperm(q.y, q.x, 0x05030201u); d[3 * k + 1] = vperm(q.z, q.y, 0x06050302u); d[3 * k + 2] = vperm(q.w, q.z, 0x07060503u);
        }
#pragma unroll
        for (int k = 0; k < 3; ++k) st128s((u32x4){d[4 * k], d[4 * k + 1], d[4 * k + 2], d[4 * k + 3]}, lw, po + (3 * i + k) * 1024);
      }
      VM0();
      gu32* fo = (gu32*)(ws + WS_PFLG) + ((tile * 2 + h) * 8 + wid) * 32;
      const gu32* fi = (const gu32*)(ws + WS_PFLG) + ((tile * 2 + 1 - h) * 8 + wid) * 32;
      if (!dead && !cx.timed_out() && ln == 0) __hip_atomic_store(fo, cx.call, __ATOMIC_RELAXED, __HIP_MEMORY_SCOPE_AGENT);
      if (rfl(__hip_atomic_load(fi, __ATOMIC_RELAXED, __HIP_MEMORY_SCOPE_AGENT)) < cx.call) {
        const unsigned long long t0 = rt();
        while (rfl(__hip_atomic_load(fi, __ATOMIC_RELAXED, __HIP_MEMORY_SCOPE_AGENT)) < cx.call) { if (cx.give_up(t0)) break; }
      }
      asm volatile("" ::: "memory");
      u32x4 in[12];
#pragma unroll
      for (int i = 0; i < 12; ++i) in[i] = ld128s(lw, pi + i * 1024);
#pragma unroll
      for (int i = 0; i < 4; ++i) {
        const unsigned d[12] = {in[3 * i].x, in[3 * i].y, in[3 * i].z, in[3 * i].w, in[3 * i + 1].x, in[3 * i + 1].y, in[3 * i + 1].z, in[3 * i + 1].w,
                                in[3 * i + 2].x, in[3 * i + 2].y, in[3 * i + 2].z, in[3 * i + 2].w};
#pragma unroll
        for (int k = 0; k < 4; ++k) {
          const int ii = 4 * i + k;
          const unsigned d0 = d[3 * k], d1 = d[3 * k + 1], d2 = d[3 * k + 2];
          const f32x4 q = __builtin_bit_cast(f32x4, (u32x4){vperm(0u, d0, 0x0201000Cu), vperm(d1, d0, 0x0504030Cu), vperm(d2, d1, 0x0403020Cu), vperm(0u, d2, 0x0302010Cu)});
          if (h) acc[1][ii >> 3][(ii >> 1) & 3][ii & 1] += q; else acc[0][ii >> 3][(ii >> 1) & 3][ii & 1] += q;
        }
      }
    }
    STAMP(p, 12 + 4 * ph);
    const bool bad = cx.timed_out();
#pragma unroll
    for (int qq = 0; qq < 2; ++qq) {
      unsigned u[16];
#pragma unroll
      for (int mm = 0; mm < 2; ++mm) {
        const int m = qq * 2 + mm;
        unsigned d[8];
#pragma unroll
        for (int c = 0; c < 8; ++c) {
          const float y0 = h ? acc[1][c & 1][m][0][c >> 1] : acc[0][c & 1][m][0][c >> 1], y1 = h ? acc[1][c & 1][m][1][c >> 1] : acc[0][c & 1][m][1][c >> 1];
          d[c] = cvt_pk_bf16(gelu1(y0), gelu1(y1)); if (bad) d[c] = 0x7FC07FC0u; }
#pragma unroll
        for (int c = 0; c < 4; ++c) {
          auto r = __builtin_amdgcn_permlane16_swap(d[c], d[c + 4], false, false); d[c] = r[0]; d[c + 4] = r[1];
          auto s2 = __builtin_amdgcn_permlane32_swap(d[c], d[c + 4], false, false); d[c] = s2[0]; d[c + 4] = s2[1];
        }
#pragma unroll
        for (int c = 0; c < 8; ++c) u[8 * mm + c] = d[c];
      }
      const MapO map{out_task_base(p.rank, tile * 4 + h * 2 + qq)};
#pragma unroll
      for (int j = 0; j < 4; ++j) st128s((u32x4){u[4 * j], u[4 * j + 1], u[4 * j + 2], u[4 * j + 3]}, ro, map(wid, j, ln));
    }
    VM0();
    if (!bad && ln < 2)
      __hip_atomic_store((gu32*)(ws + WS_RFLG) + ((tile * 4 + h * 2 + ln) * 8 + wid) * 4, cx.call, __ATOMIC_RELAXED, __HIP_MEMORY_SCOPE_AGENT);
  }
  STAMPL(p, 21 + 2 * ph);
#undef KT
#undef SA
#undef SB
#undef STAGE
#undef LDA
#undef LDB
#undef MMA
#undef WAIT_V
#undef WAIT_L
#undef BAR
#undef SCHED
}

__global__ __launch_bounds__(NTHR, 2) void fused_kernel(Params p) {
  extern __shared__ __attribute__((aligned(16))) unsigned char shm[];
  LAS unsigned char* lds = (LAS unsigned char*)shm;
  const int b = blockIdx.x, xcd = b & 7, slot = b >> 3;
  char* ws = p.ws[p.rank];
  const int wave = (int)rfl(threadIdx.x >> 6);
  unsigned callraw = atomic_add_lane0_nowait((gu32*)(ws + WS_CALL) + (b * 8 + wave) * 4);
  Ctx cx; cx.tmo = (gu32*)(ws + WS_TMO); cx.call = 0;
  LAS unsigned* S = (LAS unsigned*)(lds + LDS_SCR + wave * SCR_BYTES);
  if (b >= N_P) { unsigned dummy = 0; DRAIN_PIN2(callraw, dummy); cx.call = rfl(callraw) + 1u; }
  if (b < N_P) {
    { const int gw = b * 8 + wave; role_push(p, cx, callraw, (unsigned)(size_t)(lds + wave * 9728), gw % 7, gw / 7, PJ * 8); }
  } else if (b < N_P + N_EO) {
    role_encode(p, cx, S, 48 * CFG_EK + (b - N_P), N_EO, NIT);
    role_encode_out(p, cx, S, b - N_P, N_EO);
    STAMP(p, 2);
  } else if (slot < 16) {
    const int dw = b - N_P - N_EO;
    role_decode<false>(p, cx, (unsigned)(size_t)(lds + wave * 9728), dw, N_D);
    STAMPL(p, 1);
    role_decode<true>(p, cx, (unsigned)(size_t)(lds + wave * 9728), dw, N_D + 128);
    STAMP(p, 2);
  } else {
    const int g = slot - 16, h = xcd & 1, q = xcd >> 1;
    const int pm = (q >> 1) * 4 + (g & 3), pn = (q & 1) * 4 + (g >> 2);
    role_encode(p, cx, S, b - 128, 128, 48 * CFG_EK);
    STAMP(p, 9);
#pragma unroll 1
    for (int ph = 0; ph < 2; ++ph) gemm_unit(p, cx, lds, pm + 8 * ph, pn, h, ph);
    BARRIER();
    role_decode<true>(p, cx, (unsigned)(size_t)(lds + wave * 9728), N_D + (b - 128), N_D + 128);
    STAMP(p, 10);
  }
}

}

extern "C" void kernel_launch(int rank, int nranks, ncclComm_t comm, hipStream_t stream,
                              void* const* d_in, const int* in_sizes, int n_in,
                              void* d_out, int out_size,
                              const int* in_dtypes, int out_dtype,
                              void* const* d_out_all, void* const* d_ws_all, size_t ws_bytes) {
  using namespace fk;
  (void)comm; (void)in_dtypes; (void)out_dtype;
  if (nranks != NRANK || n_in < 2 || rank < 0 || rank >= NRANK || ws_bytes < (size_t)WS_END ||
      in_sizes[0] != MD * KS || in_sizes[1] != KS * ND || out_size != MD * ND) {
    fprintf(stderr, "kernel_launch: unexpected problem shape\n");
    return;
  }
  static bool attr_set[64] = {};
  int dev = 0;
  if (hipGetDevice(&dev) != hipSuccess) return;
  if (dev >= 0 && dev < 64 && !attr_set[dev]) {
    hipError_t e = hipFuncSetAttribute((const void*)fused_kernel, hipFuncAttributeMaxDynamicSharedMemorySize, LDS_BYTES);
    if (e != hipSuccess) { fprintf(stderr, "HIP error %s\n", hipGetErrorString(e)); return; }
    attr_set[dev] = true;
  }
  Params p;
  p.x = (const char*)d_in[0]; p.w = (const char*)d_in[1];
  for (int i = 0; i < NRANK; ++i) { p.ws[i] = (char*)d_ws_all[i]; p.out[i] = (char*)d_out_all[i]; }
  p.out[rank] = (char*)d_out;
  p.rank = rank; p.pad = 0;
  hipLaunchKernelGGL(fused_kernel, dim3(GRID), dim3(NTHR), LDS_BYTES, stream, p);
  hipError_t e = hipGetLastError();
  if (e != hipSuccess) fprintf(stderr, "HIP launch error %s\n", hipGetErrorString(e));
}
```

```cpp
#include <hip/hip_runtime.h>
#include <rccl/rccl.h>
#include <cstdio>

namespace fk {

#define LAS __attribute__((address_space(3)))
#define GAS __attribute__((address_space(1)))
typedef unsigned short bf16_t;
typedef short bf16x8 __attribute__((ext_vector_type(8)));
typedef float f32x4 __attribute__((ext_vector_type(4)));
typedef unsigned u32x4 __attribute__((ext_vector_type(4)));
typedef unsigned u32x2 __attribute__((ext_vector_type(2)));
typedef unsigned short u16x2 __attribute__((ext_vector_type(2)));
typedef GAS unsigned gu32;
typedef GAS unsigned long long gu64;
typedef __amdgpu_buffer_rsrc_t rsrc_t;

constexpr int NRANK = 8;
constexpr int MD = 8192, ND = 8192, KD = 8192, KS = KD / NRANK;
constexpr int NT = KD / 64;
constexpr unsigned OUT_BYTES = (unsigned)MD * ND * 2u;
constexpr unsigned IN_BYTES = (unsigned)MD * KS * 2u;

constexpr int GRID = 256, NTHR = 512;
#define CFG_SLEEP 2
#define CFG_PJ 4
#define CFG_AUX_DIN 17
#define CFG_AUX_DOUT 17
#define CFG_SEQIN 1
#define CFG_GMAP 1
#define CFG_RAWTAIL 0
#define CFG_AUX_G 17
#define CFG_RAWHEAD 3
#define CFG_HEADSEQ 1
#define CFG_BSLEEP 0
#define CFG_ASLEEP 0
#define CFG_EK 8
#define CFG_PW 8
constexpr int PJ = CFG_PJ;
constexpr int PW = CFG_PW;
constexpr int PNV = PJ * PW;
#define CFG_NEO 28
constexpr int N_EO = CFG_NEO;
constexpr int N_P = 7 * PJ, N_D = 128 - N_P - N_EO;
constexpr int AUX_SYS = 17;

constexpr int NIMG = 3072;
constexpr unsigned IMG = 32768, PKT = 24576;
constexpr int NIT = 1024;
constexpr int NOT = 512;
constexpr int NPI = 384 + NOT;
constexpr int HCAP = 13;

constexpr unsigned WS_CALL = 0;
constexpr unsigned WS_TMO  = WS_CALL + 256 * 128;
constexpr unsigned WS_TK   = WS_TMO + 1024;
constexpr unsigned WS_CNT  = WS_TMO + 4096;
constexpr unsigned WS_PCNT = WS_CNT + NIMG * 128;
constexpr unsigned WS_HIN  = WS_PCNT + 32768;
constexpr unsigned WS_HOUT = WS_HIN + NIMG * 128;
constexpr unsigned WS_HOBI = WS_HOUT + NRANK * NOT * 128;
constexpr unsigned WS_HOBO = WS_HOBI + NIT * 128;
constexpr unsigned WS_PFLG = WS_HOBO + NOT * 128;
constexpr unsigned WS_RFLG = WS_PFLG + 2048 * 128;
constexpr unsigned WS_DBG  = WS_RFLG + NOT * 8 * 16;
constexpr unsigned WS_PIN  = 4u << 20;
constexpr unsigned WS_POUT = WS_PIN + NIMG * PKT;
constexpr unsigned WS_OBI  = WS_POUT + NRANK * NOT * PKT;
constexpr unsigned WS_OBR  = WS_OBI + NIT * PKT;
constexpr unsigned WS_OBO  = WS_OBR + NIT * IMG;
constexpr unsigned WS_PART = WS_OBO + NOT * PKT;
constexpr unsigned WS_IMG  = WS_PART + 256 * 131072;
constexpr unsigned WS_END  = WS_IMG + NIMG * IMG;
static_assert(WS_DBG + 256 * 64 * 8 <= WS_PIN, "control region overflow");
static_assert((size_t)WS_IMG + (size_t)NIMG * IMG < (1ull << 31), "workspace offsets must fit 31 bits");

constexpr unsigned long long DEADLINE = 40000000ull;

constexpr int HTB = 16384, STAGE_BYTES = 8 * HTB;
constexpr int LDS_FLAG = STAGE_BYTES;
constexpr int LDS_CALL = STAGE_BYTES + 64;
constexpr int LDS_TK = STAGE_BYTES + 96;
constexpr int LDS_DEAD = STAGE_BYTES + 80;
constexpr int LDS_GHDR = STAGE_BYTES + 128;
constexpr int LDS_HDR = 0;
constexpr int LDS_SCR = STAGE_BYTES + 512;
constexpr int LDS_BYTES = STAGE_BYTES + 512 + 8 * 1664;

struct Params {
  const char* x; const char* w;
  char* ws[NRANK]; char* out[NRANK];
  int rank; int pad;
};

__device__ __forceinline__ rsrc_t mkr(const void* p, unsigned bytes) { return __builtin_amdgcn_make_buffer_rsrc((void*)p, 0, (int)bytes, 0x00020000); }
__device__ __forceinline__ u32x4 ld128s(rsrc_t r, unsigned off) { return __builtin_amdgcn_raw_buffer_load_b128(r, (int)off, 0, AUX_SYS); }
__device__ __forceinline__ void st128s(u32x4 v, rsrc_t r, unsigned off) { __builtin_amdgcn_raw_buffer_store_b128(v, r, (int)off, 0, AUX_SYS); }
__device__ __forceinline__ unsigned long long rt() { return __builtin_amdgcn_s_memrealtime(); }
__device__ __forceinline__ unsigned rfl(unsigned v) { return (unsigned)__builtin_amdgcn_readfirstlane((int)v); }
__device__ __forceinline__ unsigned rdl(unsigned v, int l) { return (unsigned)__builtin_amdgcn_readlane((int)v, l); }
#define VM0() do { asm volatile("s_waitcnt vmcnt(0)" ::: "memory"); __builtin_amdgcn_s_waitcnt(0x0F70); } while (0)
#define DRAIN_ALL() do { asm volatile("s_waitcnt vmcnt(0) lgkmcnt(0)" ::: "memory"); __builtin_amdgcn_s_waitcnt(0x0070); } while (0)
#define DRAIN_PIN2(a, b) do { asm volatile("s_waitcnt vmcnt(0) lgkmcnt(0)" : "+v"(a), "+v"(b) :: "memory"); __builtin_amdgcn_s_waitcnt(0x0070); } while (0)
#define BARRIER() do { asm volatile("" ::: "memory"); __builtin_amdgcn_s_barrier(); asm volatile("" ::: "memory"); } while (0)

#define STAMP(p, idx) do { } while (0)
#define STAMPL(p, idx) do { } while (0)

__device__ __forceinline__ unsigned atomic_add_lane0_nowait(gu32* ptr) {
  unsigned r; unsigned long long sv; const unsigned one = 1u;
  asm volatile("s_mov_b64 %1, exec\n\ts_mov_b64 exec, 1\n\tglobal_atomic_add %0, %2, %3, off sc0\n\ts_mov_b64 exec, %1"
               : "=&v"(r), "=&s"(sv) : "v"(ptr), "v"(one) : "memory");
  return r;
}

__device__ __forceinline__ int atomic_add_lane0_nowait_noret(gu32* ptr) {
  unsigned long long sv; const unsigned one = 1u;
  asm volatile("s_mov_b64 %0, exec\n\ts_mov_b64 exec, 1\n\tglobal_atomic_add %1, %2, off\n\ts_mov_b64 exec, %0" : "=&s"(sv) : "v"(ptr), "v"(one) : "memory");
  return 0;
}
struct Ctx {
  gu32* tmo; unsigned call;
  __device__ __forceinline__ bool timed_out() const { return rfl(__hip_atomic_load(tmo, __ATOMIC_RELAXED, __HIP_MEMORY_SCOPE_AGENT)) == call; }
  __device__ __forceinline__ bool give_up(unsigned long long t0) const {
    if (timed_out() || rt() - t0 > DEADLINE) { __hip_atomic_store(tmo, call, __ATOMIC_RELAXED, __HIP_MEMORY_SCOPE_AGENT); return true; }
    __builtin_amdgcn_s_sleep(CFG_SLEEP);
    return false;
  }
};

__device__ __forceinline__ unsigned dpp_shr_add(unsigned x) {
  x += (unsigned)__builtin_amdgcn_update_dpp(0, (int)x, 0x111, 0xf, 0xf, false);
  x += (unsigned)__builtin_amdgcn_update_dpp(0, (int)x, 0x112, 0xf, 0xf, false);
  x += (unsigned)__builtin_amdgcn_update_dpp(0, (int)x, 0x114, 0xf, 0xf, false);
  x += (unsigned)__builtin_amdgcn_update_dpp(0, (int)x, 0x118, 0xf, 0xf, false);
  x += (unsigned)__builtin_amdgcn_update_dpp(0, (int)x, 0x142, 0xa, 0xf, false);
  x += (unsigned)__builtin_amdgcn_update_dpp(0, (int)x, 0x143, 0xc, 0xf, false);
  return x;
}
__device__ __forceinline__ unsigned row_max(unsigned x) {
  x = max(x, (unsigned)__builtin_amdgcn_update_dpp(0, (int)x, 0x111, 0xf, 0xf, false));
  x = max(x, (unsigned)__builtin_amdgcn_update_dpp(0, (int)x, 0x112, 0xf, 0xf, false));
  x = max(x, (unsigned)__builtin_amdgcn_update_dpp(0, (int)x, 0x114, 0xf, 0xf, false));
  x = max(x, (unsigned)__builtin_amdgcn_update_dpp(0, (int)x, 0x118, 0xf, 0xf, false));
  return x;
}
__device__ __forceinline__ unsigned wave_max_u32(unsigned x) {
  x = row_max(x);
  x = max(x, (unsigned)__builtin_amdgcn_update_dpp(0, (int)x, 0x142, 0xa, 0xf, false));
  x = max(x, (unsigned)__builtin_amdgcn_update_dpp(0, (int)x, 0x143, 0xc, 0xf, false));
  return rdl(x, 63);
}
__device__ __forceinline__ unsigned vperm(unsigned a, unsigned b, unsigned sel) { return __builtin_amdgcn_perm(a, b, sel); }
__device__ __forceinline__ unsigned lut4(unsigned tab, unsigned sel) { return __builtin_amdgcn_perm(0u, tab, sel); }
__device__ __forceinline__ unsigned bmask(unsigned e) { return __builtin_amdgcn_perm(0u, 0u, e | 0x0C0C0C0Cu); }
__device__ __forceinline__ unsigned bfi(unsigned m, unsigned a, unsigned b) { return (m & a) | (~m & b); }
__device__ __forceinline__ unsigned mul24(unsigned a, unsigned k) { unsigned r; asm("v_mul_u32_u24 %0, %1, %2" : "=v"(r) : "s"(k), "v"(a)); return r; }
__device__ __forceinline__ unsigned rank4(unsigned e) { return mul24(e, 0x010101u) << 8; }
__device__ __forceinline__ unsigned pc(unsigned e) { return (unsigned)__builtin_popcount(e); }
__device__ __forceinline__ unsigned nz4(unsigned x) { return ((x + 0x0F0F0F0Fu) >> 4) & 0x01010101u; }

constexpr int SCR_BYTES = 1664;

template <int W> __device__ __forceinline__ unsigned pack_bits(unsigned c, unsigned e) {
  const unsigned pre = rank4(e), em = bmask(e);
  if (W == 1) return __builtin_amdgcn_udot4(c, lut4(0x08040201u, pre) & em, 0u, false);
  if (W == 2) return __builtin_amdgcn_udot4(c, lut4(0x40100401u, pre) & em, 0u, false);
  return __builtin_amdgcn_udot4(c, lut4(0x00001001u, pre) & em, 0u, false) | (__builtin_amdgcn_udot4(c, lut4(0x10010000u, pre) & em, 0u, false) << 8);
}
template <int W> __device__ __forceinline__ unsigned unpack_bits(unsigned w, unsigned e) {
  unsigned s;
  if (W == 1) s = mul24(w & 0xFu, 0x00204081u) & 0x01010101u;
  else if (W == 2) { w &= 0xFFu; const unsigned t = (w | (w << 12)) & 0x000F000Fu; s = (t | (t << 6)) & 0x03030303u; }
  else if (W == 4) { w &= 0xFFFFu; const unsigned t = (w | (w << 8)) & 0x00FF00FFu; s = (t | (t << 4)) & 0x0F0F0F0Fu; }
  else s = w;
  return lut4(s, rank4(e)) & bmask(e);
}
__device__ __forceinline__ void lds_or(LAS unsigned* q, unsigned x) { if (x != 0u) __hip_atomic_fetch_or(q, x, __ATOMIC_RELAXED, __HIP_MEMORY_SCOPE_WAVEFRONT); }
__device__ __forceinline__ void flush32(LAS unsigned* S, unsigned off, unsigned acc) {
  const unsigned long long x = (unsigned long long)acc << (off & 31u);
  LAS unsigned* q = S + (off >> 5);
  lds_or(q, (unsigned)x); lds_or(q + 1, (unsigned)(x >> 32));
}
__device__ __forceinline__ void flush64(LAS unsigned* S, unsigned off, unsigned long long acc) {
  const unsigned sh = off & 31u;
  const unsigned long long x = acc << sh;
  LAS unsigned* q = S + (off >> 5);
  lds_or(q, (unsigned)x); lds_or(q + 1, (unsigned)(x >> 32)); lds_or(q + 2, ((unsigned)(acc >> 32) >> 1) >> (31u - sh));
}
__device__ __forceinline__ unsigned fetch32(const LAS unsigned* S, unsigned off) {
  const LAS unsigned* q = S + (off >> 5);
  return __builtin_amdgcn_alignbit(q[1], q[0], off & 31u);
}
__device__ __forceinline__ unsigned long long fetch64(const LAS unsigned* S, unsigned off) {
  const LAS unsigned* q = S + (off >> 5);
  const unsigned a = q[0], b = q[1], c = q[2];
  return ((unsigned long long)__builtin_amdgcn_alignbit(c, b, off & 31u) << 32) | __builtin_amdgcn_alignbit(b, a, off & 31u);
}
__device__ __forceinline__ unsigned wave_sum_lane63(unsigned x) { return dpp_shr_add(x); }
__device__ __forceinline__ unsigned sym_of(bool out, unsigned base, unsigned t) { return out ? ((base - (t & 7u)) | ((t & 8u) << 4)) & 0xFFu : (base - t) & 0xFFu; }

template <bool OUT>
__device__ __forceinline__ unsigned long long encode_chunk(const unsigned (&v)[16], LAS unsigned* S, int lane, u32x4& P0, u32x4& P1, u32x2& F) {
  u16x2 mx = __builtin_bit_cast(u16x2, v[0] & 0x7FFF7FFFu);
#pragma unroll
  for (int i = 1; i < 16; ++i) mx = __builtin_elementwise_max(mx, __builtin_bit_cast(u16x2, v[i] & 0x7FFF7FFFu));
  const unsigned base = wave_max_u32(max((unsigned)mx.x, (unsigned)mx.y)) >> (OUT ? 8 : 7);
  const unsigned B4 = base * 0x01010101u;
  *(LAS u32x4*)(S + lane * 4) = (u32x4){0u, 0u, 0u, 0u};
  if (lane < (OUT ? 34 : 1)) *(LAS u32x4*)(S + 256 + lane * 4) = (u32x4){0u, 0u, 0u, 0u};
  unsigned H[8], L[8], R[8], oor[8], anyo = 0;
#pragma unroll
  for (int g = 0; g < 8; ++g) {
    const unsigned Hs = vperm(v[2 * g + 1], v[2 * g], 0x07050301u), Ls = vperm(v[2 * g + 1], v[2 * g], 0x06040200u);
    if (OUT) { H[g] = Hs; L[g] = Ls; const unsigned D = B4 - (Hs & 0x7F7F7F7Fu); R[g] = (D & 0x07070707u) | ((Hs >> 4) & 0x08080808u); oor[g] = D & 0x78787878u; }
    else { H[g] = bfi(0x01010101u, Ls >> 7, Hs << 1); L[g] = bfi(0x80808080u, Hs, Ls); const unsigned D = B4 - H[g]; R[g] = D & 0x0F0F0F0Fu; oor[g] = D & 0xF0F0F0F0u; }
    anyo |= oor[g];
  }
  unsigned key;
  {
    unsigned long long hc = 0;
#pragma unroll
    for (int b = 0; b < 8; ++b) hc += 1ull << (((R[b >> 2] >> (8 * (b & 3))) & 15u) * 4u);
    const unsigned lo = (unsigned)hc, hi = (unsigned)(hc >> 32);
    unsigned q[8];
#pragma unroll
    for (int k = 0; k < 8; ++k) q[k] = rdl(dpp_shr_add(((lo >> (4 * k)) & 15u) | (((hi >> (4 * k)) & 15u) << 16)), 63);
    const int k7 = lane & 7;
    const unsigned qa = (k7 & 1) ? ((k7 & 2) ? q[3] : q[1]) : ((k7 & 2) ? q[2] : q[0]), qb = (k7 & 1) ? ((k7 & 2) ? q[7] : q[5]) : ((k7 & 2) ? q[6] : q[4]);
    const unsigned cnt = (((k7 & 4) ? qb : qa) >> ((lane & 8) * 2)) & 0xFFFFu;
    key = (lane < 16) ? (((cnt + 1u) << 4) | (unsigned)lane) : 0u;
  }
  unsigned tt = 0, cl = 0x1Fu;
#pragma unroll
  for (int i = 0; i < 7; ++i) {
    const unsigned bias = (OUT && i == 0 && lane >= 8 && lane < 16) ? (32u << 4) : 0u;
    const unsigned m = rdl(row_max(key + bias), 15);
    tt |= (m & 15u) << (4 * i);
    if (key + bias == m) { key = 0u; cl = (i < 3) ? (unsigned)i : (i == 3) ? 3u : (7u | ((unsigned)(i - 4) << 3)); }
  }
  cl <<= 8 * (lane & 3);
  cl |= (unsigned)__builtin_amdgcn_update_dpp(0, (int)cl, 0xB1, 0xf, 0xf, false);
  cl |= (unsigned)__builtin_amdgcn_update_dpp(0, (int)cl, 0x4E, 0xf, 0xf, false);
  const unsigned tb0 = rdl(cl, 0), tb1 = rdl(cl, 4), tb2 = rdl(cl, 8), tb3 = rdl(cl, 12);
  unsigned cls[8];
#pragma unroll
  for (int g = 0; g < 8; ++g) {
    const unsigned r7 = R[g] & 0x07070707u;
    cls[g] = bfi(bmask((R[g] >> 3) & 0x01010101u), vperm(tb3, tb2, r7), vperm(tb1, tb0, r7));
  }
  const bool has_oor = __any(anyo != 0u);
  unsigned e4[8];
#pragma unroll
  for (int g = 0; g < 8; ++g) e4[g] = 0u;
  if (has_oor) {
    const unsigned T1 = ((tt >> 4) & 15u) * 0x01010101u;
#pragma unroll
    for (int g = 0; g < 8; ++g) {
      e4[g] = nz4(OUT ? (oor[g] >> 3) : (oor[g] >> 4));
      const unsigned m = bmask(e4[g]);
      cls[g] |= m & 0x1F1F1F1Fu; R[g] = bfi(m, T1, R[g]);
    }
  }
  bool modeB = false; unsigned n0 = 0;
  if (OUT) {
    unsigned z[8], mxn = 0;
#pragma unroll
    for (int g = 0; g < 8; ++g) z[g] = lut4(0x000000FFu, cls[g] & 0x03030303u);
#pragma unroll
    for (int g = 0; g < 2; ++g) {
      const unsigned hn = (L[g] >> 4) & z[g] & 0x0F0F0F0Fu;
      mxn = max(mxn, max(max(hn & 0xFFu, (hn >> 8) & 0xFFu), max((hn >> 16) & 0xFFu, hn >> 24)));
    }
    n0 = wave_max_u32(mxn);
    unsigned viol[8], nv = 0;
    const unsigned N4 = (n0 | 0x80u) * 0x01010101u;
#pragma unroll
    for (int g = 0; g < 8; ++g) {
      const unsigned dn = (N4 - ((L[g] >> 4) & 0x0F0F0F0Fu)) & 0x7F7F7F7Fu;
      viol[g] = z[g] & bmask(((dn + 0x7D7D7D7Du) >> 7) & 0x01010101u);
      nv += pc(viol[g] & 0x01010101u);
    }
    modeB = n0 < 8u && rdl(dpp_shr_add(nv), 63) < 24u;
    if (modeB) {
#pragma unroll
      for (int g = 0; g < 8; ++g) cls[g] |= viol[g] & 0x1F1F1F1Fu;
    }
  }
  unsigned e2[8], e2b[8], e3[8], e0[8], m2[8], n2 = 0, n2b = 0, n3 = 0, n4 = 0, c0 = 0, cm2 = 0;
  const unsigned N0 = n0 * 0x01010101u;
#pragma unroll
  for (int g = 0; g < 8; ++g) {
    const unsigned c1 = cls[g] & 0x03030303u;
    e2[g] = lut4(0x01000000u, c1); e2b[g] = (cls[g] >> 2) & 0x01010101u; e3[g] = lut4(0x01000000u, (cls[g] >> 3) & 0x03030303u);
    n2 += pc(e2[g]); n2b += pc(e2b[g]); n3 += pc(e3[g]); n4 += pc(e4[g]);
    if (OUT) {
      e0[g] = modeB ? lut4(0x00000001u, c1) : 0u;
      m2[g] = e0[g] & nz4(((L[g] >> 4) & 0x0F0F0F0Fu) ^ N0);
      c0 += pc(e0[g]); cm2 += pc(m2[g]);
    } else { e0[g] = 0u; m2[g] = 0u; }
  }
  const unsigned x1 = c0 | (n2 << 16), x2 = cm2 | (n2b << 16), x3 = n3 | (n4 << 16);
  const unsigned s1 = dpp_shr_add(x1), s2 = dpp_shr_add(x2), s3 = dpp_shr_add(x3);
  const unsigned t1 = rdl(s1, 63), t2 = rdl(s2, 63), t3 = rdl(s3, 63);
  const unsigned C0 = t1 & 0xFFFFu, N2 = t1 >> 16, CM2 = t2 & 0xFFFFu, N2B = t2 >> 16, N3 = t3 & 0xFFFFu, N4c = t3 >> 16;
  const unsigned bSN = OUT ? 4u * (2048u - C0) : 0u;
  const unsigned aM1 = bSN, a2a = aM1 + C0, aM2 = a2a + N2, a2b = aM2 + CM2, a3 = a2b + 2u * N2B, a4 = a3 + 4u * N3, bits = a4 + 8u * N4c;
  const bool raw = bits > (OUT ? 12288u : 4096u);
  const unsigned len = ((OUT ? 12288u : 20480u) + bits + 255u) >> 8;
  if (!raw) {
    const unsigned p1 = s1 - x1, p2 = s2 - x2, p3 = s3 - x3;
    const unsigned oN = 4u * (32u * (unsigned)lane - (p1 & 0xFFFFu)), oM1 = aM1 + (p1 & 0xFFFFu), o2a = a2a + (p1 >> 16), oM2 = aM2 + (p2 & 0xFFFFu),
                   o2b = a2b + 2u * (p2 >> 16), o3 = a3 + 4u * (p3 & 0xFFFFu);
    unsigned o4 = a4 + 8u * (p3 >> 16);
    unsigned wM1 = 0, wM2 = 0, w2a = 0, kM1 = 0, kM2 = 0, k2a = 0, k2b = 0, kN = 0, k3 = 0, kNa = 0, k3a = 0;
    unsigned long long w2b = 0, wN = 0, w3 = 0;
#pragma unroll
    for (int g = 0; g < 8; ++g) {
      if (g == 4) {
        if (OUT) { flush64(S, oN, wN); wN = 0; kNa = kN; kN = 0; }
        flush64(S, o3, w3); w3 = 0; k3a = k3; k3 = 0;
      }
      if (OUT) {
        const unsigned hn = (L[g] >> 4) & 0x0F0F0F0Fu, eN = e0[g] ^ 0x01010101u;
        wN |= (unsigned long long)pack_bits<4>(hn, eN) << kN; kN += 4u * pc(eN);
        wM1 |= pack_bits<1>(m2[g], e0[g]) << kM1; kM1 += pc(e0[g]);
        wM2 |= pack_bits<1>(nz4(((N0 | 0x10101010u) - 0x01010101u - hn) & 0x0F0F0F0Fu), m2[g]) << kM2; kM2 += pc(m2[g]);
      }
      w2a |= pack_bits<1>(e2b[g], e2[g]) << k2a; k2a += pc(e2[g]);
      w2b |= (unsigned long long)pack_bits<2>((cls[g] >> 3) & 0x03030303u, e2b[g]) << k2b; k2b += 2u * pc(e2b[g]);
      w3 |= (unsigned long long)pack_bits<4>(R[g], e3[g]) << k3; k3 += 4u * pc(e3[g]);
    }
    if (OUT) { flush64(S, oN + kNa, wN); flush32(S, oM1, wM1); flush32(S, oM2, wM2); }
    flush32(S, o2a, w2a); flush64(S, o2b, w2b); flush64(S, o3 + k3a, w3);
    if (has_oor) {
#pragma unroll
      for (int g = 0; g < 8; ++g)
#pragma unroll
        for (int b = 0; b < 4; ++b)
          if ((e4[g] >> (8 * b)) & 1u) {
            const unsigned long long x = (unsigned long long)((H[g] >> (8 * b)) & 0xFFu) << (o4 & 31u);
            __hip_atomic_fetch_or(S + (o4 >> 5), (unsigned)x, __ATOMIC_RELAXED, __HIP_MEMORY_SCOPE_WAVEFRONT);
            __hip_atomic_fetch_or(S + (o4 >> 5) + 1, (unsigned)(x >> 32), __ATOMIC_RELAXED, __HIP_MEMORY_SCOPE_WAVEFRONT);
            o4 += 8u;
          }
    }
  }
  unsigned Fw[2];
#pragma unroll
  for (int i = 0; i < 2; ++i)
    Fw[i] = (cls[4 * i] & 0x03030303u) | ((cls[4 * i + 1] & 0x03030303u) << 2) | ((cls[4 * i + 2] & 0x03030303u) << 4) | ((cls[4 * i + 3] & 0x03030303u) << 6);
  F = (u32x2){Fw[0], Fw[1]};
  if (OUT) {
    unsigned p4[4];
#pragma unroll
    for (int i = 0; i < 4; ++i) p4[i] = (L[2 * i] & 0x0F0F0F0Fu) | ((L[2 * i + 1] & 0x0F0F0F0Fu) << 4);
    P0 = (u32x4){p4[0], p4[1], p4[2], p4[3]}; P1 = P0;
  } else {
    P0 = (u32x4){L[0], L[1], L[2], L[3]}; P1 = (u32x4){L[4], L[5], L[6], L[7]};
  }
  return (unsigned long long)(base | (raw ? 0x8000u : (len << 8))) | ((unsigned long long)tt << 16) | (modeB ? (1ull << 44) | ((unsigned long long)n0 << 45) : 0ull);
}
__device__ __forceinline__ bool hp_raw(unsigned long long hp) { return ((unsigned)hp & 0x8000u) != 0u; }
__device__ __forceinline__ unsigned hp_len(unsigned long long hp) { return (((unsigned)hp >> 8) & 127u) * 32u; }

template <bool OUT>
__device__ __forceinline__ void store_chunk(rsrc_t r, unsigned o, const LAS unsigned* S, unsigned len, int lane, u32x4 P0, u32x4 P1, u32x2 F) {
  const unsigned l16 = (unsigned)lane * 16u;
  st128s(P0, r, o + l16);
  const u32x4 sv = *(const LAS u32x4*)(S + lane * 4);
  if (OUT) {
    const u32x4 sw = *(const LAS u32x4*)(S + 256 + (lane & 31) * 4);
    __builtin_amdgcn_raw_buffer_store_b64(F, r, (int)(o + 1024 + lane * 8), 0, AUX_SYS);
    if (1536u + l16 < len) st128s(sv, r, o + 1536 + l16);
    if (lane < 32 && 2560u + l16 < len) st128s(sw, r, o + 2560 + l16);
  } else {
    st128s(P1, r, o + 1024 + l16);
    __builtin_amdgcn_raw_buffer_store_b64(F, r, (int)(o + 2048 + lane * 8), 0, AUX_SYS);
    if (lane < 32 && 2560u + l16 < len) st128s(sv, r, o + 2560 + l16);
  }
}

template <bool OUT>
__device__ __forceinline__ void decode_rec(const LAS unsigned* R, unsigned long long hp, int lane, unsigned (&v)[16]) {
  const unsigned base = (unsigned)hp & 0xFFu, tt = (unsigned)(hp >> 16) & 0xFFFFFFFu;
  const bool modeB = ((hp >> 44) & 1u) != 0u;
  const unsigned n0 = (unsigned)(hp >> 45) & 7u;
  const u32x4 P0 = *(const LAS u32x4*)(R + lane * 4), P1 = OUT ? P0 : *(const LAS u32x4*)(R + 256 + lane * 4);
  const u32x2 F = *(const LAS u32x2*)(R + (OUT ? 256 : 512) + lane * 2);
  const LAS unsigned* S = R + (OUT ? 384 : 640);
  unsigned tab1 = 0, tab2 = 0;
#pragma unroll
  for (int i = 0; i < 4; ++i) tab1 |= sym_of(OUT, base, (tt >> (4 * i)) & 15u) << (8 * i);
#pragma unroll
  for (int i = 0; i < 3; ++i) tab2 |= sym_of(OUT, base, (tt >> (16 + 4 * i)) & 15u) << (8 * i);
  const unsigned B4 = base * 0x01010101u, T1 = ((tt >> 4) & 15u) * 0x01010101u, N0 = n0 * 0x01010101u;
  unsigned c1[8], e2[8], e0[8], n2 = 0, c0 = 0;
#pragma unroll
  for (int g = 0; g < 8; ++g) {
    c1[g] = (((g < 4) ? F.x : F.y) >> (2 * (g & 3))) & 0x03030303u;
    e2[g] = lut4(0x01000000u, c1[g]); n2 += pc(e2[g]);
    if (OUT) { e0[g] = modeB ? lut4(0x00000001u, c1[g]) : 0u; c0 += pc(e0[g]); } else e0[g] = 0u;
  }
  const unsigned x1 = c0 | (n2 << 16), s1 = dpp_shr_add(x1), t1 = rdl(s1, 63), p1 = s1 - x1;
  const unsigned C0 = t1 & 0xFFFFu, N2 = t1 >> 16;
  const unsigned aM1 = OUT ? 4u * (2048u - C0) : 0u, a2a = aM1 + C0, aM2 = a2a + N2;
  const unsigned oN = 4u * (32u * (unsigned)lane - (p1 & 0xFFFFu));
  unsigned hn[8], m2[8], e2b[8], cm2 = 0, n2b = 0;
  {
    unsigned w2a = fetch32(S, a2a + (p1 >> 16)), wM1 = 0; unsigned long long wN = 0, wN2 = 0;
    if (OUT) {
      wM1 = fetch32(S, aM1 + (p1 & 0xFFFFu)); wN = fetch64(S, oN);
      wN2 = fetch64(S, oN + 4u * (16u - pc(e0[0]) - pc(e0[1]) - pc(e0[2]) - pc(e0[3])));
    }
#pragma unroll
    for (int g = 0; g < 8; ++g) {
      e2b[g] = unpack_bits<1>(w2a, e2[g]); w2a >>= pc(e2[g]); n2b += pc(e2b[g]);
      if (OUT) {
        if (g == 4) wN = wN2;
        const unsigned eN = e0[g] ^ 0x01010101u;
        hn[g] = unpack_bits<4>((unsigned)wN, eN); wN >>= 4u * pc(eN);
        m2[g] = unpack_bits<1>(wM1, e0[g]); wM1 >>= pc(e0[g]); cm2 += pc(m2[g]);
      }
    }
  }
  const unsigned x2 = cm2 | (n2b << 16), s2 = dpp_shr_add(x2), t2 = rdl(s2, 63), p2 = s2 - x2;
  const unsigned CM2 = t2 & 0xFFFFu, N2B = t2 >> 16, a2b = aM2 + CM2, a3 = a2b + 2u * N2B;
  unsigned wM2 = OUT ? fetch32(S, aM2 + (p2 & 0xFFFFu)) : 0u;
  unsigned long long w2b = fetch64(S, a2b + 2u * (p2 >> 16));
  unsigned H[8], L[8], e3[8], n3 = 0;
#pragma unroll
  for (int g = 0; g < 8; ++g) {
    const unsigned c2 = unpack_bits<2>((unsigned)w2b, e2b[g]); w2b >>= 2u * pc(e2b[g]);
    e3[g] = lut4(0x01000000u, c2); n3 += pc(e3[g]);
    H[g] = bfi(bmask(e2b[g]), lut4(tab2, c2), lut4(tab1, c1[g]));
    if (OUT) {
      const unsigned b2 = unpack_bits<1>(wM2, m2[g]); wM2 >>= pc(m2[g]);
      const unsigned nib = hn[g] | ((N0 - m2[g] - b2) & bmask(e0[g]));
      const unsigned p4 = (g >> 1) == 0 ? P0.x : (g >> 1) == 1 ? P0.y : (g >> 1) == 2 ? P0.z : P0.w;
      L[g] = ((p4 >> (4 * (g & 1))) & 0x0F0F0F0Fu) | (nib << 4);
    } else {
      L[g] = g == 0 ? P0.x : g == 1 ? P0.y : g == 2 ? P0.z : g == 3 ? P0.w : g == 4 ? P1.x : g == 5 ? P1.y : g == 6 ? P1.z : P1.w;
    }
  }
  const unsigned s3 = dpp_shr_add(n3), N3 = rdl(s3, 63);
  const unsigned o3 = a3 + 4u * (s3 - n3);
  unsigned long long w3 = fetch64(S, o3);
  const unsigned long long w3b = fetch64(S, o3 + 4u * (pc(e3[0]) + pc(e3[1]) + pc(e3[2]) + pc(e3[3])));
  unsigned e4[8], any4 = 0;
#pragma unroll
  for (int g = 0; g < 8; ++g) {
    if (g == 4) w3 = w3b;
    unsigned R = unpack_bits<4>((unsigned)w3, e3[g]); w3 >>= 4u * pc(e3[g]);
    e4[g] = e3[g] & ~nz4(R ^ T1);
    R &= ~bmask(e4[g]);
    const unsigned H3 = OUT ? ((B4 - (R & 0x07070707u)) | ((R & 0x08080808u) << 4)) : (B4 - R);
    H[g] = bfi(bmask(e3[g]), H3, H[g]);
    any4 |= e4[g];
  }
  if (__any(any4 != 0u)) {
    unsigned n4 = 0;
#pragma unroll
    for (int g = 0; g < 8; ++g) n4 += pc(e4[g]);
    unsigned o4 = a3 + 4u * N3 + 8u * (dpp_shr_add(n4) - n4);
#pragma unroll
    for (int g = 0; g < 8; ++g) { H[g] = bfi(bmask(e4[g]), unpack_bits<8>(fetch32(S, o4), e4[g]), H[g]); o4 += 8u * pc(e4[g]); }
  }
#pragma unroll
  for (int g = 0; g < 8; ++g) {
    const unsigned Hs = OUT ? H[g] : bfi(0x80808080u, L[g], H[g] >> 1), Ls = OUT ? L[g] : bfi(0x80808080u, H[g] << 7, L[g]);
    v[2 * g] = vperm(Hs, Ls, 0x05010400u);
    v[2 * g + 1] = vperm(Hs, Ls, 0x07030602u);
  }
}

struct MapX { unsigned base; __device__ __forceinline__ unsigned operator()(int w, int j, int l) const { return base + (unsigned)(w * 4096 + j * 1024 + l * 16); } };
struct MapW { unsigned base; __device__ __forceinline__ unsigned operator()(int w, int j, int l) const { return base + (unsigned)((j >> 1) * 16384 + ((w >> 1) * 2 + (j & 1)) * 2048 + (w & 1) * 1024 + l * 16); } };
struct MapO { unsigned base; __device__ __forceinline__ unsigned operator()(int w, int j, int l) const { return base + (unsigned)((64 * (w >> 2) + 16 * (j >> 1) + (l & 15)) * (ND * 2) + (64 * (w & 3) + 32 * (j & 1) + 8 * (l >> 4)) * 2); } };
__device__ __forceinline__ unsigned out_task_base(int rank, int task) {
  const int tile = task >> 2, q = task & 3, pm = tile >> 3, pn = tile & 7;
  return (unsigned)(((rank >> 2) * 4096 + pm * 256 + (q >> 1) * 128 + (q & 1) * 32) * (ND * 2) + ((rank & 3) * 2048 + pn * 256) * 2);
}

__device__ __forceinline__ unsigned long long mk_granule(unsigned long long payload, unsigned call) { return payload | ((unsigned long long)(call & 0xFFFFu) << 48); }
__device__ __forceinline__ unsigned long long uni64(unsigned long long g) { return ((unsigned long long)rfl((unsigned)(g >> 32)) << 32) | rfl((unsigned)g); }
__device__ __forceinline__ unsigned long long ld_granule(const char* ws, unsigned off) {
  return __hip_atomic_load((const gu64*)(ws + off), __ATOMIC_RELAXED, __HIP_MEMORY_SCOPE_SYSTEM);
}
__device__ __forceinline__ bool tag_ok(unsigned long long g, unsigned call) { return (unsigned)(g >> 48) == (call & 0xFFFFu); }
__device__ __forceinline__ bool poll_granule(const Ctx& cx, const char* ws, unsigned off, unsigned long long& g) {
  g = uni64(g);
  if (!tag_ok(g, cx.call)) {
    const unsigned long long t0 = rt();
    do {
      if (cx.give_up(t0)) return false;
      g = uni64(ld_granule(ws, off));
    } while (!tag_ok(g, cx.call));
  }
  asm volatile("" ::: "memory");
  return true;
}

template <class Map>
__device__ __forceinline__ void nan_chunk(rsrc_t fin, const Map& map, int wid, int lane) {
#pragma unroll
  for (int j = 0; j < 4; ++j) st128s((u32x4){0x7FC07FC0u, 0x7FC07FC0u, 0x7FC07FC0u, 0x7FC07FC0u}, fin, map(wid, j, lane));
}

struct InTask { int kt, isw, idx; };
__device__ __forceinline__ InTask in_task(int i) {
  InTask t;
  if (i < 768) { t.kt = i / 48; const int u = i - t.kt * 48, u3 = u / 3, r3 = u - u3 * 3;
    if (r3 == 0) { t.isw = 0; t.idx = (u3 & 1) * 16 + (u3 >> 1); } else { t.isw = 1; const int c = u3 * 2 + r3 - 1; t.idx = (c & 3) * 8 + (c >> 2); } }
  else { const int u = i - 768; t.kt = (u >> 4) & 15; t.isw = 0; t.idx = (u & 1) * 16 + 8 + ((u & 15) >> 1); }
  return t;
}
__device__ __forceinline__ int otask_of(int isw, int idx, int kt) { return isw * 512 + idx * 16 + kt; }
__device__ __forceinline__ void in_load(const Params& p, const InTask& t, u32x4 (&r)[4], int wid, int lane) {
  const int rr = lane >> 2, cp = (lane & 3) ^ ((rr >> 3) << 1);
  if (!t.isw) {
    const rsrc_t rx = mkr(p.x, IN_BYTES);
#pragma unroll
    for (int j = 0; j < 4; ++j) {
      const unsigned off = (unsigned)((256 * t.idx + 32 * wid + 16 * (j >> 1) + rr) * (KS * 2) + (64 * t.kt + 32 * (j & 1) + 8 * cp) * 2);
      r[j] = __builtin_amdgcn_raw_buffer_load_b128(rx, (int)off, 0, 0);
    }
  } else {
    const rsrc_t rw = mkr(p.w, IN_BYTES);
    const unsigned off0 = (unsigned)((64 * t.kt + 32 * (wid & 1) + 8 * cp) * (ND * 2) + (256 * t.idx + 64 * (wid >> 1) + 4 * rr) * 2);
#pragma unroll
    for (int d = 0; d < 4; ++d) {
      const u32x2 a = __builtin_amdgcn_raw_buffer_load_b64(rw, (int)(off0 + (unsigned)(2 * d) * (ND * 2)), 0, 0);
      const u32x2 b = __builtin_amdgcn_raw_buffer_load_b64(rw, (int)(off0 + (unsigned)(2 * d + 1) * (ND * 2)), 0, 0);
      r[d] = __builtin_shufflevector(a, b, 0, 1, 2, 3);
    }
  }
}
__device__ __forceinline__ void in_prep(int isw, const u32x4 (&r)[4], unsigned (&v)[16]) {
  if (isw) {
#pragma unroll
    for (int d = 0; d < 4; ++d) {
      v[0 + d] = __builtin_amdgcn_perm(r[d].z, r[d].x, 0x05040100u);
      v[4 + d] = __builtin_amdgcn_perm(r[d].z, r[d].x, 0x07060302u);
      v[8 + d] = __builtin_amdgcn_perm(r[d].w, r[d].y, 0x05040100u);
      v[12 + d] = __builtin_amdgcn_perm(r[d].w, r[d].y, 0x07060302u);
    }
  } else {
#pragma unroll
    for (int j = 0; j < 4; ++j) { v[4 * j] = r[j].x; v[4 * j + 1] = r[j].y; v[4 * j + 2] = r[j].z; v[4 * j + 3] = r[j].w; }
  }
}

__device__ __forceinline__ void role_encode(const Params& p, const Ctx& cx, LAS unsigned* S, int e, int ne, int iend) {
  const int tid = threadIdx.x, wid = (int)rfl(tid >> 6), lane = tid & 63;
  char* ws = p.ws[p.rank];
  const rsrc_t lw = mkr(ws, WS_END);
  u32x4 A[4];
  if (e >= iend) return;
  in_load(p, in_task(e), A, wid, lane);
  DRAIN_ALL();
  for (int i = e; i < iend; i += ne) {
    const InTask t = in_task(i);
    unsigned v[16];
    in_prep(t.isw, A, v);
    if (i + ne < iend) in_load(p, in_task(i + ne), A, wid, lane);
    u32x4 P0, P1; u32x2 F;
    const unsigned long long hp = encode_chunk<false>(v, S, lane, P0, P1, F);
    const int ot = otask_of(t.isw, t.idx, t.kt);
    const bool mine = t.isw ? ((t.idx >> 3) == (p.rank & 3)) : ((t.idx >> 4) == (p.rank >> 2));
    const int img = t.isw ? 2048 + (t.idx & 7) * 128 + t.kt * 8 + p.rank : (t.idx & 15) * 128 + t.kt * 8 + p.rank;
    if (!hp_raw(hp)) {
      store_chunk<false>(lw, WS_OBI + (unsigned)ot * PKT + (unsigned)wid * 3072u, S, hp_len(hp), lane, P0, P1, F);
    } else {
#pragma unroll
      for (int j = 0; j < 4; ++j) st128s((u32x4){v[4 * j], v[4 * j + 1], v[4 * j + 2], v[4 * j + 3]}, lw, WS_OBR + (unsigned)ot * IMG + (unsigned)(wid * 4096 + j * 1024 + lane * 16));
    }
    if (mine) {
      const unsigned ib = WS_IMG + (unsigned)img * IMG;
#pragma unroll
      for (int j = 0; j < 4; ++j) st128s((u32x4){v[4 * j], v[4 * j + 1], v[4 * j + 2], v[4 * j + 3]}, lw, t.isw ? MapW{ib}(wid, j, lane) : MapX{ib}(wid, j, lane));
    }
    DRAIN_ALL();
    if (lane == 0) {
      __hip_atomic_store((gu64*)(ws + WS_HOBI + (unsigned)ot * 64u) + wid, mk_granule(hp, cx.call), __ATOMIC_RELAXED, __HIP_MEMORY_SCOPE_SYSTEM);
      if (mine) __hip_atomic_fetch_add((gu32*)(ws + WS_CNT) + img * 32, 1u, __ATOMIC_RELAXED, __HIP_MEMORY_SCOPE_AGENT);
    }
  }
}

__device__ __forceinline__ bool eo_ready(const Ctx& cx, const gu32* f, bool block) {
  if (rfl(__hip_atomic_load(f, __ATOMIC_RELAXED, __HIP_MEMORY_SCOPE_AGENT)) >= cx.call) return true;
  if (!block) return false;
  const unsigned long long t0 = rt();
  while (rfl(__hip_atomic_load(f, __ATOMIC_RELAXED, __HIP_MEMORY_SCOPE_AGENT)) < cx.call) { if (cx.give_up(t0)) return false; }
  return true;
}
__device__ __forceinline__ void role_encode_out(const Params& p, const Ctx& cx, LAS unsigned* S, int e, int ne) {
  const int tid = threadIdx.x, wid = (int)rfl(tid >> 6), lane = tid & 63;
  char* ws = p.ws[p.rank];
  const rsrc_t lw = mkr(ws, WS_END), ro = mkr(p.out[p.rank], OUT_BYTES);
  const gu32* rf = (const gu32*)(ws + WS_RFLG) + wid * 4;
  auto load = [&](int task, u32x4 (&A)[4]) { const MapO map{out_task_base(p.rank, task)};
#pragma unroll
    for (int j = 0; j < 4; ++j) A[j] = ld128s(ro, map(wid, j, lane)); };
  u32x4 A[4];
#pragma unroll
  for (int j = 0; j < 4; ++j) {
    unsigned t[4];
#pragma unroll
    for (int k = 0; k < 4; ++k) t[k] = ((0x3FBE3EBDu >> (((lane + 4 * j + k) & 3) * 8)) & 0xFFu) * 0x01000100u + ((unsigned)(lane * 29 + (4 * j + k) * 7) & 0x1Fu) * 0x00010003u;
    A[j] = (u32x4){t[0], t[1], t[2], t[3]};
  }
#pragma unroll 1
  for (int i = e - ne; i < NOT; i += ne) {
    unsigned v[16];
#pragma unroll
    for (int j = 0; j < 4; ++j) { v[4 * j] = A[j].x; v[4 * j + 1] = A[j].y; v[4 * j + 2] = A[j].z; v[4 * j + 3] = A[j].w; }
    const int nx = i + ne;
    bool have = false;
    if (nx < NOT) { have = eo_ready(cx, rf + nx * 32, false); if (have) load(nx, A); }
    u32x4 P0, P1; u32x2 F;
    const unsigned long long hp = encode_chunk<true>(v, S, lane, P0, P1, F);
    if (!hp_raw(hp)) store_chunk<true>(lw, i < 0 ? 0xFFFF0000u : WS_OBO + (unsigned)i * PKT + (unsigned)wid * 3072u, S, hp_len(hp), lane, P0, P1, F);
    DRAIN_ALL();
    if (i >= 0 && lane == 0) __hip_atomic_store((gu64*)(ws + WS_HOBO + (unsigned)i * 64u) + wid, mk_granule(hp, cx.call), __ATOMIC_RELAXED, __HIP_MEMORY_SCOPE_SYSTEM);
    if (nx < NOT && !have) {
      if (!eo_ready(cx, rf + nx * 32, true)) return;
      load(nx, A);
      DRAIN_ALL();
    }
  }
}

struct PItem { unsigned hsrc, dsrc, hdst, ddst, rs, rd; int kind; };
__device__ __forceinline__ PItem p_item(int rank, int d, int i) {
  PItem r;
  if (i < 384) {
    int kt, isw, k;
    if (i < 256) { kt = i >> 4; isw = i & 1; k = (i & 15) >> 1; } else { kt = (i - 256) >> 3; isw = 0; k = 8 + ((i - 256) & 7); }
    const int idx = isw ? (d & 3) * 8 + k : (d >> 2) * 16 + k;
    const int ot = otask_of(isw, idx, kt), T = kt * 8 + rank, img = isw ? 2048 + k * 128 + T : k * 128 + T;
    const unsigned slot = CFG_SEQIN ? (unsigned)(rank * 384 + i) : (unsigned)img;
    r.hsrc = WS_HOBI + (unsigned)ot * 64u; r.dsrc = WS_OBI + (unsigned)ot * PKT;
    r.hdst = WS_HIN + slot * 64u; r.ddst = WS_PIN + slot * PKT;
    r.rs = WS_OBR + (unsigned)ot * IMG; r.rd = WS_IMG + (unsigned)img * IMG; r.kind = isw;
  } else {
    const int task = (i - 384) & (NOT - 1);
    r.hsrc = WS_HOBO + (unsigned)task * 64u; r.dsrc = WS_OBO + (unsigned)task * PKT;
    r.hdst = WS_HOUT + (unsigned)(rank * NOT + task) * 64u; r.ddst = WS_POUT + (unsigned)(rank * NOT + task) * PKT;
    r.rs = r.rd = out_task_base(rank, task); r.kind = 2;
  }
  return r;
}

#define P_LDG(ws, off) ld_granule(ws, off)

__device__ __forceinline__ void p_publish(const Ctx& cx, const char* ws, char* wd, unsigned hsrc, unsigned hdst, unsigned long long hl, int lane) {
  if (hsrc == 0xFFFFFFFFu) hl = mk_granule(0x8000ull, cx.call);
  else {
    const char* src = ws + hsrc + (lane & 7) * 8;
    while (!__all(tag_ok(hl, cx.call))) {
      u32x2 r; asm volatile("global_load_dwordx2 %0, %1, off sc0 sc1\n\ts_waitcnt vmcnt(0)" : "=&v"(r) : "v"(src) : "memory");
      hl = ((unsigned long long)r.y << 32) | r.x;
    }
  }
  if (lane < 8) {
    const u32x2 v = (u32x2){(unsigned)hl, (unsigned)(hl >> 32)};
    asm volatile("global_store_dwordx2 %0, %1, off sc0 sc1\n\ts_nop 1" :: "v"(wd + hdst + lane * 8), "v"(v) : "memory");
  }
}

__device__ __forceinline__ void st_peer(u32x4 v, char* ptr) { asm volatile("global_store_dwordx4 %0, %1, off sc0 sc1\n\ts_nop 1" :: "v"(ptr), "v"(v) : "memory"); }
__device__ __forceinline__ void glds16(const char* src, unsigned dst) {
  unsigned keep; asm volatile("s_mov_b32 %0, m0\n\ts_mov_b32 m0, %2\n\ts_nop 0\n\tglobal_load_lds_dwordx4 %1, off sc0 sc1\n\ts_mov_b32 m0, %0" : "=&s"(keep) : "v"(src), "s"(rfl(dst)) : "memory");
}
__device__ __forceinline__ void glds4(const char* src, unsigned dst) {
  unsigned keep; asm volatile("s_mov_b32 %0, m0\n\ts_mov_b32 m0, %2\n\ts_nop 0\n\tglobal_load_lds_dword %1, off sc0 sc1\n\ts_mov_b32 m0, %0" : "=&s"(keep) : "v"(src), "s"(rfl(dst)) : "memory");
}
__device__ __forceinline__ u32x2 ld64_nowait(const char* src) {
  u32x2 r; asm volatile("global_load_dwordx2 %0, %1, off sc0 sc1" : "=&v"(r) : "v"(src) : "memory");
  return r;
}
__device__ __forceinline__ unsigned long long u64_of(u32x2 v) { return ((unsigned long long)v.y << 32) | v.x; }
__device__ __forceinline__ void role_push(const Params& p, Ctx& cx, unsigned callraw, unsigned lds_ring, int link, int v0, int nv) {
  constexpr int NC = NPI * 8;
  const int lane = threadIdx.x & 63;
  int d = p.rank + 1 + link; if (d >= NRANK) d -= NRANK;
  if ((unsigned)d >= (unsigned)NRANK) return;
  char* ws = p.ws[p.rank]; char* wd = p.ws[d];
  const rsrc_t lw = mkr(ws, WS_END), dw = mkr(wd, WS_END);
  gu32* tkc = (gu32*)(ws + WS_TK) + link * 32;
  gu32* pcnt = (gu32*)(ws + WS_PCNT) + link * NPI;
  const unsigned lo = (unsigned)lane * 16u;
  constexpr int R0 = CFG_RAWHEAD;
  static_assert(R0 * PNV <= 128 && (R0 * PNV) % 8 == 0, "raw head chunks must be whole K-tile 0 items");
  int prev; unsigned prev_hsrc = 0, prev_hdst;
  {
    auto head_task = [&](int u) { const int i0 = u >> 3; InTask t; t.kt = 0; t.isw = i0 & 1; t.idx = t.isw ? (d & 3) * 8 + (i0 >> 1) : (d >> 2) * 16 + (i0 >> 1); return t; };
    u32x4 A[R0][4];
#pragma unroll
    for (int r = 0; r < R0; ++r) in_load(p, head_task(v0 + r * nv), A[r], (v0 + r * nv) & 7, lane);
    unsigned dummy = 0;
    DRAIN_PIN2(callraw, dummy);
    cx.call = rfl(callraw) + 1u;
#pragma unroll
    for (int r = 0; r < R0; ++r) {
      const int u = v0 + r * nv, w = u & 7;
      const InTask t = head_task(u);
      unsigned v[16];
      in_prep(t.isw, A[r], v);
      const PItem f = p_item(p.rank, d, u >> 3);
      unsigned pc = 0;
      if (CFG_HEADSEQ && r > 0) { VM0(); pc = atomic_add_lane0_nowait(pcnt + (prev >> 3)); }
#pragma unroll
      for (int j = 0; j < 4; ++j) st128s((u32x4){v[4 * j], v[4 * j + 1], v[4 * j + 2], v[4 * j + 3]}, dw, t.isw ? MapW{f.rd}(w, j, lane) : MapX{f.rd}(w, j, lane));
      if (CFG_HEADSEQ && r > 0) {
        asm volatile("s_waitcnt vmcnt(4)" : "+v"(pc) :: "memory");
        if (rfl(pc) == 8u * cx.call - 1u) p_publish(cx, ws, wd, 0xFFFFFFFFu, prev_hdst, 0ull, lane);
      }
      if (r == 0) STAMP(p, 3);
      prev = u; prev_hdst = f.hdst;
    }
    if (!CFG_HEADSEQ && R0 > 1) {
      VM0();
#pragma unroll
      for (int r = 0; r + 1 < R0; ++r) {
        const int u = v0 + r * nv;
        unsigned pc = atomic_add_lane0_nowait(pcnt + (u >> 3)), dummy2 = 0;
        DRAIN_PIN2(pc, dummy2);
        if (rfl(pc) == 8u * cx.call - 1u) p_publish(cx, ws, wd, 0xFFFFFFFFu, p_item(p.rank, d, u >> 3).hdst, 0ull, lane);
      }
    }
  }
  const unsigned tkbase = (cx.call - 1u) * (unsigned)(NC - R0 * nv) - (unsigned)(R0 + 4) * (unsigned)nv;
  gu32* dummy_cnt = (gu32*)(ws + WS_TK) + 8 * 32 + (v0 & 7);
  const unsigned ring = lds_ring, gsl = lds_ring + 3u * 3072u;
  auto hoff = [&](int u) { return p_item(p.rank, d, u >> 3).hsrc + (unsigned)(u & 7) * 8u; };
  auto tail = [&](int u) { return CFG_RAWTAIL > 0 && u >= NC - 8 * CFG_RAWTAIL; };
  auto dma_rec = [&](int u, unsigned long long g, bool on, unsigned sl) {
    const unsigned base = on ? p_item(p.rank, d, u >> 3).dsrc + (unsigned)(u & 7) * 3072u : WS_OBI, len = on ? hp_len(g) : 0u;
    const char* q = ws + base;
#pragma unroll
    for (int k = 0; k < 3; ++k)
      if ((unsigned)k * 1024u + lo < len || lane == 0) glds16(q + ((unsigned)k * 1024u + lo < len ? (unsigned)k * 1024u + lo : 0u), ring + sl * 3072u + (unsigned)k * 1024u);
  };
  auto dma_granule = [&](int u, unsigned sl) { if (lane < 2) glds4(ws + (u < NC ? hoff(u) : WS_HOBI) + lane * 4, gsl + sl * 8u); };
  auto rd_granule = [&](unsigned sl) { const LAS unsigned* q = (const LAS unsigned*)(size_t)(gsl + sl * 8u); return ((unsigned long long)rfl(q[1]) << 32) | rfl(q[0]); };
  int c = v0 + R0 * nv, c1 = c + nv, c2 = c1 + nv, c3 = c2 + nv;
  unsigned long long gc = P_LDG(ws, hoff(c)), g1 = P_LDG(ws, hoff(c1));
  dma_granule(c2, 0u);
  DRAIN_ALL();
  gc = uni64(gc); g1 = uni64(g1);
  if (!tag_ok(gc, cx.call)) {
    unsigned pc = atomic_add_lane0_nowait(pcnt + (prev >> 3)), dummy = 0;
    DRAIN_PIN2(pc, dummy);
    if (rfl(pc) == 8u * cx.call - 1u) p_publish(cx, ws, wd, 0xFFFFFFFFu, prev_hdst, 0ull, lane);
    prev = -1;
    if (!poll_granule(cx, ws, hoff(c), gc)) return;
  }
  if (prev >= 0) prev_hsrc = 0xFFFFFFFFu;
  unsigned sc = 0, gp = 0;
  bool have1 = tag_ok(g1, cx.call);
  dma_rec(c, gc, !hp_raw(gc) && !tail(c), 0u);
  dma_rec(c1, g1, have1 && !hp_raw(g1) && !tail(c1), 1u);
  DRAIN_ALL();
  while (c < NC) {
    const int w = c & 7;
    const PItem cur = p_item(p.rank, d, c >> 3);
    unsigned tk = atomic_add_lane0_nowait(tkc);
    unsigned pc = atomic_add_lane0_nowait(prev >= 0 ? pcnt + (prev >> 3) : dummy_cnt);
    u32x2 hl = ld64_nowait(ws + ((prev >= 0 && prev_hsrc != 0xFFFFFFFFu) ? prev_hsrc : WS_HOBI) + (lane & 7) * 8);
    dma_granule(c3, gp ^ 1u);
    const bool rawc = hp_raw(gc) || tail(c);
    if (!rawc) {
      const LAS u32x4* q = (const LAS u32x4*)(size_t)(ring + sc * 3072u + lo);
      const u32x4 d0 = q[0], d1 = q[64], d2 = q[128];
      char* o = wd + (size_t)(cur.ddst + (unsigned)w * 3072u + lo); const unsigned len = hp_len(gc);
      st_peer(d0, o);
      if (1024u + lo < len) st_peer(d1, o + 1024);
      if (2048u + lo < len) st_peer(d2, o + 2048);
    } else {
      u32x4 q[4];
      if (cur.kind == 2) {
        const rsrc_t so = mkr(p.out[p.rank], OUT_BYTES), dd = mkr(p.out[d], OUT_BYTES); const MapO map{cur.rs};
#pragma unroll
        for (int j = 0; j < 4; ++j) q[j] = ld128s(so, map(w, j, lane));
#pragma unroll
        for (int j = 0; j < 4; ++j) st128s(q[j], dd, map(w, j, lane));
      } else {
#pragma unroll
        for (int j = 0; j < 4; ++j) q[j] = ld128s(lw, cur.rs + (unsigned)(w * 4096 + j * 1024) + lo);
#pragma unroll
        for (int j = 0; j < 4; ++j) st128s(q[j], dw, cur.kind ? MapW{cur.rd}(w, j, lane) : MapX{cur.rd}(w, j, lane));
      }
      VM0();
    }
    const unsigned long long g2 = rd_granule(gp);
    const bool have2 = c2 < NC && tag_ok(g2, cx.call);
    const unsigned s2 = sc >= 1u ? sc - 1u : 2u;
    dma_rec(c2 < NC ? c2 : 0, g2, have2 && !hp_raw(g2) && !tail(c2), s2);
    asm volatile("s_waitcnt vmcnt(3)" : "+v"(tk), "+v"(pc), "+v"(hl) :: "memory");
    if (prev >= 0 && rfl(pc) == 8u * cx.call - 1u) p_publish(cx, ws, wd, prev_hsrc, prev_hdst, u64_of(hl), lane);
    const int c4 = (int)(rfl(tk) - tkbase);
    prev = c; prev_hsrc = tail(c) ? 0xFFFFFFFFu : cur.hsrc; prev_hdst = cur.hdst;
    const unsigned s1 = sc == 2u ? 0u : sc + 1u;
    if (c1 < NC && !have1) {
      VM0();
      unsigned pc2 = atomic_add_lane0_nowait(pcnt + (c >> 3));
      u32x2 hl2 = ld64_nowait(ws + (prev_hsrc != 0xFFFFFFFFu ? prev_hsrc : WS_HOBI) + (lane & 7) * 8);
      asm volatile("s_waitcnt vmcnt(0)" : "+v"(pc2), "+v"(hl2) :: "memory");
      if (rfl(pc2) == 8u * cx.call - 1u) p_publish(cx, ws, wd, prev_hsrc, prev_hdst, u64_of(hl2), lane);
      prev = -1;
      g1 = 0;
      if (!poll_granule(cx, ws, hoff(c1), g1)) return;
      dma_rec(c1, g1, !hp_raw(g1) && !tail(c1), s1);
      VM0();
    }
    c = c1; c1 = c2; c2 = c3; c3 = c4;
    gc = g1; g1 = g2; have1 = have2; sc = s1; gp ^= 1u;
  }
  if (prev >= 0) {
    VM0();
    unsigned pc = atomic_add_lane0_nowait(pcnt + (prev >> 3));
    u32x2 hl2 = ld64_nowait(ws + (prev_hsrc != 0xFFFFFFFFu ? prev_hsrc : WS_HOBI) + (lane & 7) * 8);
    asm volatile("s_waitcnt vmcnt(0)" : "+v"(pc), "+v"(hl2) :: "memory");
    if (rfl(pc) == 8u * cx.call - 1u) p_publish(cx, ws, wd, prev_hsrc, prev_hdst, u64_of(hl2), lane);
  }
  STAMPL(p, 2);
}

struct DTask { unsigned hl, pk, dst; int kind; int sig; };
__device__ __forceinline__ unsigned d_map(const DTask& t, int w, int j, int l) {
  return t.kind == 0 ? MapX{t.dst}(w, j, l) : t.kind == 1 ? MapW{t.dst}(w, j, l) : MapO{t.dst}(w, j, l);
}
template <bool OUT> __device__ __forceinline__ DTask d_task(int rank, int i) {
  DTask t;
  if (!OUT) {
    int kt, sl, k, isw;
    if (i < 7 * 256) { kt = i / 112; const int u = i - kt * 112; sl = u >> 4; isw = u & 1; k = (u & 15) >> 1; }
    else { const int u = i - 7 * 256; kt = (u / 56) & 15; const int u2 = u - (u / 56) * 56; sl = u2 >> 3; isw = 0; k = 8 + (u2 & 7); }
    int src = rank + 1 + sl; if (src >= NRANK) src -= NRANK;
    const int T = kt * 8 + src, img = isw ? 2048 + k * 128 + T : k * 128 + T;
    const unsigned slot = CFG_SEQIN ? (unsigned)(src * 384 + (k < 8 ? kt * 16 + k * 2 + isw : 256 + kt * 8 + (k - 8))) : (unsigned)img;
    t.hl = WS_HIN + slot * 64u; t.pk = WS_PIN + slot * PKT;
    t.dst = WS_IMG + (unsigned)img * IMG; t.kind = isw; t.sig = img;
  } else {
    const int task = (i / (NRANK - 1)) & (NOT - 1), k = i % (NRANK - 1);
    int s = rank + 1 + k; if (s >= NRANK) s -= NRANK;
    const unsigned tidx = (unsigned)(s * NOT + task);
    t.hl = WS_HOUT + tidx * 64u; t.pk = WS_POUT + tidx * PKT; t.dst = out_task_base(s, task); t.kind = 2; t.sig = 0;
  }
  return t;
}
template <bool OUT>
__device__ __forceinline__ void role_decode(const Params& p, const Ctx& cx, unsigned lds_ring, int dw, int nw) {
  constexpr int TOTAL = OUT ? NOT * (NRANK - 1) : 7 * 384;
  const int tid = threadIdx.x, wid = (int)rfl(tid >> 6), lane = tid & 63;
  char* ws = p.ws[p.rank];
  char* fin = OUT ? p.out[p.rank] : ws;
  gu32* cnt = (gu32*)(ws + WS_CNT);
  const unsigned co = (unsigned)wid * 3072u, ho = (unsigned)wid * 8u, lo = (unsigned)lane * 16u;
  const unsigned ring = lds_ring, gsl = lds_ring + 3u * 3072u;
  int i = dw;
  if (i >= TOTAL) return;
  auto dma_rec = [&](const DTask& t, unsigned long long g, bool on, unsigned sl) {
    const unsigned len = on ? hp_len(g) : 0u;
    const char* q = ws + (on ? t.pk + co : WS_PIN);
#pragma unroll
    for (int k = 0; k < 3; ++k)
      if ((unsigned)k * 1024u + lo < len || lane == 0) glds16(q + ((unsigned)k * 1024u + lo < len ? (unsigned)k * 1024u + lo : 0u), ring + sl * 3072u + (unsigned)k * 1024u);
  };
  auto dma_granule = [&](int u, unsigned sl) { if (lane < 2) glds4(ws + (u < TOTAL ? d_task<OUT>(p.rank, u).hl + ho : WS_HIN) + lane * 4, gsl + sl * 8u); };
  auto rd_granule = [&](unsigned sl) { const LAS unsigned* q = (const LAS unsigned*)(size_t)(gsl + sl * 8u); return ((unsigned long long)rfl(q[1]) << 32) | rfl(q[0]); };
  auto signal = [&](int img) { if (!OUT && img >= 0) (void)atomic_add_lane0_nowait_noret(cnt + img * 32); };
  DTask cur = d_task<OUT>(p.rank, i), t1 = cur;
  unsigned long long gc = ld_granule(ws, cur.hl + ho), g1 = 0;
  bool ok = poll_granule(cx, ws, cur.hl + ho, gc);
  bool have1 = false;
  if (i + nw < TOTAL) { t1 = d_task<OUT>(p.rank, i + nw); g1 = uni64(ld_granule(ws, t1.hl + ho)); have1 = tag_ok(g1, cx.call); }
  dma_rec(cur, gc, ok && !hp_raw(gc), 0u);
  dma_rec(t1, g1, have1 && !hp_raw(g1), 1u);
  dma_granule(i + 2 * nw, 0u);
  asm volatile("s_waitcnt vmcnt(0)" ::: "memory");
  unsigned sc = 0, gp = 0;
  int prev = -1;
  while (ok) {
    const int n1 = i + nw, n2 = i + 2 * nw, n3 = i + 3 * nw;
    dma_granule(n3, gp ^ 1u);
    const unsigned long long g2 = rd_granule(gp);
    const bool have2 = n2 < TOTAL && tag_ok(g2, cx.call);
    const DTask t2 = d_task<OUT>(p.rank, n2 < TOTAL ? n2 : i);
    dma_rec(t2, g2, have2 && !hp_raw(g2), sc >= 1u ? sc - 1u : 2u);
    if (!hp_raw(gc)) {
      unsigned v[16];
      decode_rec<OUT>((const LAS unsigned*)(size_t)(ring + sc * 3072u), gc, lane, v);
#pragma unroll
      for (int j = 0; j < 4; ++j) st_peer((u32x4){v[4 * j], v[4 * j + 1], v[4 * j + 2], v[4 * j + 3]}, fin + d_map(cur, wid, j, lane));
      asm volatile("s_waitcnt vmcnt(7)" ::: "memory");
    } else {
      asm volatile("s_waitcnt vmcnt(0)" ::: "memory");
    }
    signal(prev);
    prev = cur.sig;
    if (n1 >= TOTAL) break;
    const unsigned s1 = sc == 2u ? 0u : sc + 1u;
    if (!have1) {
      asm volatile("s_waitcnt vmcnt(0)" ::: "memory");
      signal(prev); prev = -1;
      g1 = 0;
      ok = poll_granule(cx, ws, t1.hl + ho, g1);
      dma_rec(t1, g1, ok && !hp_raw(g1), s1);
      asm volatile("s_waitcnt vmcnt(0)" ::: "memory");
    }
    cur = t1; gc = g1; t1 = t2; g1 = g2; have1 = have2; sc = s1; gp ^= 1u; i = n1;
  }
  asm volatile("s_waitcnt vmcnt(0)" ::: "memory");
  if (!OUT) {
    if (ok) signal(prev);
  } else if (!ok) {
    for (; i < TOTAL; i += nw) {
      const DTask t = d_task<OUT>(p.rank, i);
#pragma unroll
      for (int j = 0; j < 4; ++j) st_peer((u32x4){0x7FC07FC0u, 0x7FC07FC0u, 0x7FC07FC0u, 0x7FC07FC0u}, fin + d_map(t, wid, j, lane));
    }
  }
}

__host__ __device__ __forceinline__ int lds_byte(int r, int c) { const int st = (r >> 4) * 2 + (c >> 5), rr = r & 15, cc = c & 31, ob = rr * 64 + cc * 2; return st * 1024 + (ob ^ (((ob >> 9) & 1) << 5)); }
__device__ __forceinline__ unsigned cvt_pk_bf16(float lo, float hi) { unsigned r; asm volatile("v_cvt_pk_bf16_f32 %0, %1, %2" : "=v"(r) : "v"(lo), "v"(hi)); return r; }
__device__ __forceinline__ float gelu1(float y) {
  const float k1 = -2.0f * 0.7978845608028654f * 1.4426950408889634f, k3 = k1 * 0.044715f;
  const float t = y * (k3 * (y * y) + k1);
  return y * __builtin_amdgcn_rcpf(1.0f + __builtin_amdgcn_exp2f(t));
}

__device__ __forceinline__ void gemm_unit(const Params& p, const Ctx& cx, LAS unsigned char* lds, int pm, int pn, int h, int ph) {
  constexpr int NTH = NT / 2;
  const int tid = threadIdx.x, wid = (int)rfl(tid >> 6), lane = tid & 63, wr = wid >> 2, wc = wid & 3, fr = lane & 15, fq = lane >> 4;
  char* ws = p.ws[p.rank];
  const char* cA = ws + WS_IMG + (size_t)(pm * 128 + h) * IMG;
  const char* cB = ws + WS_IMG + (size_t)(2048 + pn * 128 + h) * IMG;
  const gu32* fl = (const gu32*)(ws + WS_CNT) + (size_t)(((lane & 2) ? 2048 + pn * 128 : pm * 128) + h + 2 * (lane & 1)) * 32;
  const unsigned need = 8u * cx.call;
  const unsigned voff = (unsigned)tid * 16u;
  const unsigned ldsw = (unsigned)wid * 1024u;
  const int aoff = lds_byte(wr * 64 + fr, fq * 8), boff = lds_byte(wc * 32 + fr, fq * 8);
  bool dead = false;
#define KT(t) ((((t) >> 2) << 3) + (((t) & 3) << 1))
  auto wait_tiles = [&](int t) {
    if (wid == 0 && !dead) {
      const gu32* f = fl + KT(t) * 32;
      if (!__all(__hip_atomic_load(f, __ATOMIC_RELAXED, __HIP_MEMORY_SCOPE_AGENT) >= need)) {
        const unsigned long long t0 = rt();
        while (!__all(__hip_atomic_load(f, __ATOMIC_RELAXED, __HIP_MEMORY_SCOPE_AGENT) >= need)) { if (cx.give_up(t0)) { dead = true; break; } }
      }
    }
  };
#define SA(b, h) (((b) * 2 + (h)) * HTB)
#define SB(b, h) ((4 + (b) * 2 + (h)) * HTB)
#define STAGE(bufoff, gbase) do { _Pragma("unroll") for (int _i = 0; _i < 2; ++_i) \
    __builtin_amdgcn_global_load_lds((const GAS unsigned*)((const char*)(gbase) + voff + _i * 8192), (LAS unsigned*)(lds + (bufoff) + ldsw + _i * 8192), 16, 0, CFG_AUX_G); } while (0)
#define LDA(dst, b, h) do { _Pragma("unroll") for (int m = 0; m < 4; ++m) _Pragma("unroll") for (int k = 0; k < 2; ++k) dst[m][k] = *(const LAS bf16x8*)(lds + SA(b, h) + aoff + m * 2048 + k * 1024); } while (0)
#define LDB(dst, b, h) do { _Pragma("unroll") for (int n = 0; n < 2; ++n) _Pragma("unroll") for (int k = 0; k < 2; ++k) dst[n][k] = *(const LAS bf16x8*)(lds + SB(b, h) + boff + n * 2048 + k * 1024); } while (0)
#define MMA(ai, bj, At, Bt) do { __builtin_amdgcn_s_setprio(1); _Pragma("unroll") for (int m = 0; m < 4; ++m) _Pragma("unroll") for (int n = 0; n < 2; ++n) _Pragma("unroll") for (int k = 0; k < 2; ++k) \
    acc[ai][bj][m][n] = __builtin_amdgcn_mfma_f32_16x16x32_bf16(Bt[n][k], At[m][k], acc[ai][bj][m][n], 0, 0, 0); __builtin_amdgcn_s_setprio(0); } while (0)
#define WAIT_V(n) asm volatile("s_waitcnt vmcnt(" #n ")" ::: "memory")
#define WAIT_L(n) asm volatile("s_waitcnt lgkmcnt(" #n ")" ::: "memory")
#define BAR __builtin_amdgcn_s_barrier()
#define SCHED __builtin_amdgcn_sched_barrier(0)
  f32x4 acc[2][2][4][2];
#pragma unroll
  for (int a = 0; a < 2; ++a)
#pragma unroll
    for (int b = 0; b < 2; ++b)
#pragma unroll
      for (int m = 0; m < 4; ++m)
#pragma unroll
        for (int n = 0; n < 2; ++n) acc[a][b][m][n] = (f32x4){0.f, 0.f, 0.f, 0.f};
  bf16x8 At[4][2], B0[2][2], B1[2][2];
  wait_tiles(0); wait_tiles(2);
  BARRIER();
  STAMPL(p, 18 + ph);
  STAGE(SB(0, 0), cB); STAGE(SA(0, 0), cA); STAGE(SB(0, 1), cB + HTB); STAGE(SA(0, 1), cA + HTB);
  if (wr == 1) BAR;
  WAIT_V(4); BAR;
  STAGE(SB(1, 0), cB + 2 * IMG); STAGE(SA(1, 0), cA + 2 * IMG); STAGE(SB(1, 1), cB + 2 * IMG + HTB);
  WAIT_V(6); BAR;
  for (int t = 0; t < NTH; t += 2) {
    const bool last = (t == NTH - 2);
    const char* a1 = cA + (size_t)(KT(t) + 2) * IMG;
    const size_t o2 = last ? 0 : (size_t)KT(t + 2) * IMG;
    const char* a2 = cA + o2; const char* b2 = cB + o2;
    const char* a3 = a2 + 2 * IMG; const char* b3 = b2 + 2 * IMG;
    LAS unsigned* fslot = (LAS unsigned*)(lds + LDS_FLAG + ((t >> 1) & 1) * 16);
    const bool chk = (t + 4 < NTH);
    if (chk && wid == 0 && lane < 4) __builtin_amdgcn_global_load_lds(fl + KT(t + 4) * 32, fslot, 4, 0, AUX_SYS);
    LDB(B0, 0, 0); SCHED; LDA(At, 0, 0); STAGE(SA(1, 1), a1 + HTB);
    WAIT_L(8); BAR; WAIT_L(0); MMA(0, 0, At, B0); BAR; SCHED;
    LDB(B1, 0, 1); STAGE(SB(0, 0), b2);
    BAR; WAIT_L(0); MMA(0, 1, At, B1); BAR;
    LDA(At, 0, 1); STAGE(SA(0, 0), a2);
    BAR; WAIT_L(0); MMA(1, 0, At, B0); BAR; SCHED;
    STAGE(SB(0, 1), b2 + HTB);
    WAIT_V(6); BAR; MMA(1, 1, At, B1); BAR;
    LDB(B0, 1, 0); SCHED; LDA(At, 1, 0); STAGE(SA(0, 1), a2 + HTB);
    WAIT_L(8); BAR; WAIT_L(0); MMA(0, 0, At, B0); BAR; SCHED;
    LDB(B1, 1, 1); STAGE(SB(1, 0), b3);
    BAR; WAIT_L(0); MMA(0, 1, At, B1); BAR;
    LDA(At, 1, 1); STAGE(SA(1, 0), a3);
    BAR; WAIT_L(0); MMA(1, 0, At, B0); BAR; SCHED;
    const u32x4 f = *(const LAS u32x4*)fslot;
    STAGE(SB(1, 1), b3 + HTB);
    WAIT_V(6); BAR; MMA(1, 1, At, B1); BAR;
    if (chk) {
      const unsigned fm = rfl(min(min(f.x, f.y), min(f.z, f.w)));
      if (fm < need) { wait_tiles(t + 4); BARRIER(); }
    }
    if (CFG_BSLEEP > 0 && ph == 1) __builtin_amdgcn_s_sleep(CFG_BSLEEP);
    if (CFG_ASLEEP > 0 && ph == 0) __builtin_amdgcn_s_sleep(CFG_ASLEEP);
  }
  if (wr == 0) BAR;
  STAMPL(p, 20 + 2 * ph);
  const int tile = pm * 8 + pn;
  {
    int tz = threadIdx.x; asm volatile("" : "+v"(tz));
    const int ln = tz & 63;
    const rsrc_t ro = mkr(p.out[p.rank], OUT_BYTES), lw = mkr(ws, WS_END);
    {
      const unsigned po = WS_PART + (unsigned)(tile * 2 + h) * 131072u + (unsigned)(wid * 16384 + ln * 16);
      const unsigned pi = WS_PART + (unsigned)(tile * 2 + 1 - h) * 131072u + (unsigned)(wid * 16384 + ln * 16);
#pragma unroll
      for (int i = 0; i < 4; ++i) {
        unsigned d[12];
#pragma unroll
        for (int k = 0; k < 4; ++k) {
          const int ii = 4 * i + k;
          const u32x4 q = __builtin_bit_cast(u32x4, h ? acc[0][ii >> 3][(ii >> 1) & 3][ii & 1] : acc[1][ii >> 3][(ii >> 1) & 3][ii & 1]);
          d[3 * k] = vperm(q.y, q.x, 0x05030201u); d[3 * k + 1] = vperm(q.z, q.y, 0x06050302u); d[3 * k + 2] = vperm(q.w, q.z, 0x07060503u);
        }
#pragma unroll
        for (int k = 0; k < 3; ++k) st128s((u32x4){d[4 * k], d[4 * k + 1], d[4 * k + 2], d[4 * k + 3]}, lw, po + (3 * i + k) * 1024);
      }
      VM0();
      gu32* fo = (gu32*)(ws + WS_PFLG) + ((tile * 2 + h) * 8 + wid) * 32;
      const gu32* fi = (const gu32*)(ws + WS_PFLG) + ((tile * 2 + 1 - h) * 8 + wid) * 32;
      if (!dead && !cx.timed_out() && ln == 0) __hip_atomic_store(fo, cx.call, __ATOMIC_RELAXED, __HIP_MEMORY_SCOPE_AGENT);
      if (rfl(__hip_atomic_load(fi, __ATOMIC_RELAXED, __HIP_MEMORY_SCOPE_AGENT)) < cx.call) {
        const unsigned long long t0 = rt();
        while (rfl(__hip_atomic_load(fi, __ATOMIC_RELAXED, __HIP_MEMORY_SCOPE_AGENT)) < cx.call) { if (cx.give_up(t0)) break; }
      }
      asm volatile("" ::: "memory");
      u32x4 in[12];
#pragma unroll
      for (int i = 0; i < 12; ++i) in[i] = ld128s(lw, pi + i * 1024);
#pragma unroll
      for (int i = 0; i < 4; ++i) {
        const unsigned d[12] = {in[3 * i].x, in[3 * i].y, in[3 * i].z, in[3 * i].w, in[3 * i + 1].x, in[3 * i + 1].y, in[3 * i + 1].z, in[3 * i + 1].w,
                                in[3 * i + 2].x, in[3 * i + 2].y, in[3 * i + 2].z, in[3 * i + 2].w};
#pragma unroll
        for (int k = 0; k < 4; ++k) {
          const int ii = 4 * i + k;
          const unsigned d0 = d[3 * k], d1 = d[3 * k + 1], d2 = d[3 * k + 2];
          const f32x4 q = __builtin_bit_cast(f32x4, (u32x4){vperm(0u, d0, 0x0201000Cu), vperm(d1, d0, 0x0504030Cu), vperm(d2, d1, 0x0403020Cu), vperm(0u, d2, 0x0302010Cu)});
          if (h) acc[1][ii >> 3][(ii >> 1) & 3][ii & 1] += q; else acc[0][ii >> 3][(ii >> 1) & 3][ii & 1] += q;
        }
      }
    }
    STAMP(p, 12 + 4 * ph);
    const bool bad = cx.timed_out();
#pragma unroll
    for (int qq = 0; qq < 2; ++qq) {
      unsigned u[16];
#pragma unroll
      for (int mm = 0; mm < 2; ++mm) {
        const int m = qq * 2 + mm;
        unsigned d[8];
#pragma unroll
        for (int c = 0; c < 8; ++c) {
          const float y0 = h ? acc[1][c & 1][m][0][c >> 1] : acc[0][c & 1][m][0][c >> 1], y1 = h ? acc[1][c & 1][m][1][c >> 1] : acc[0][c & 1][m][1][c >> 1];
          d[c] = cvt_pk_bf16(gelu1(y0), gelu1(y1)); if (bad) d[c] = 0x7FC07FC0u; }
#pragma unroll
        for (int c = 0; c < 4; ++c) {
          auto r = __builtin_amdgcn_permlane16_swap(d[c], d[c + 4], false, false); d[c] = r[0]; d[c + 4] = r[1];
          auto s2 = __builtin_amdgcn_permlane32_swap(d[c], d[c + 4], false, false); d[c] = s2[0]; d[c + 4] = s2[1];
        }
#pragma unroll
        for (int c = 0; c < 8; ++c) u[8 * mm + c] = d[c];
      }
      const MapO map{out_task_base(p.rank, tile * 4 + h * 2 + qq)};
#pragma unroll
      for (int j = 0; j < 4; ++j) st128s((u32x4){u[4 * j], u[4 * j + 1], u[4 * j + 2], u[4 * j + 3]}, ro, map(wid, j, ln));
    }
    VM0();
    if (!bad && ln < 2)
      __hip_atomic_store((gu32*)(ws + WS_RFLG) + ((tile * 4 + h * 2 + ln) * 8 + wid) * 4, cx.call, __ATOMIC_RELAXED, __HIP_MEMORY_SCOPE_AGENT);
  }
  STAMPL(p, 21 + 2 * ph);
#undef KT
#undef SA
#undef SB
#undef STAGE
#undef LDA
#undef LDB
#undef MMA
#undef WAIT_V
#undef WAIT_L
#undef BAR
#undef SCHED
}

__global__ __launch_bounds__(NTHR, 2) void fused_kernel(Params p) {
  extern __shared__ __attribute__((aligned(16))) unsigned char shm[];
  LAS unsigned char* lds = (LAS unsigned char*)shm;
  const int b = blockIdx.x, xcd = b & 7, slot = b >> 3;
  char* ws = p.ws[p.rank];
  const int wave = (int)rfl(threadIdx.x >> 6);
  unsigned callraw = atomic_add_lane0_nowait((gu32*)(ws + WS_CALL) + (b * 8 + wave) * 4);
  Ctx cx; cx.tmo = (gu32*)(ws + WS_TMO); cx.call = 0;
  LAS unsigned* S = (LAS unsigned*)(lds + LDS_SCR + wave * SCR_BYTES);
  if (b >= N_P || wave >= PW) { unsigned dummy = 0; DRAIN_PIN2(callraw, dummy); cx.call = rfl(callraw) + 1u; }
  if (b < N_P) {
    if (wave < PW) { const int gw = b * PW + wave; role_push(p, cx, callraw, (unsigned)(size_t)(lds + wave * 9728), gw % 7, gw / 7, PNV); }
  } else if (b < N_P + N_EO) {
    role_encode(p, cx, S, 48 * CFG_EK + (b - N_P), N_EO, NIT);
    role_encode_out(p, cx, S, b - N_P, N_EO);
    STAMP(p, 2);
  } else if (slot < 16) {
    const int dw = b - N_P - N_EO;
    role_decode<false>(p, cx, (unsigned)(size_t)(lds + wave * 9728), dw, N_D);
    STAMPL(p, 1);
    role_decode<true>(p, cx, (unsigned)(size_t)(lds + wave * 9728), dw, N_D + 128);
    STAMP(p, 2);
  } else {
    const int g = slot - 16, h = xcd & 1, q = xcd >> 1;
    const int pm = (q >> 1) * 4 + (g & 3), pn = (q & 1) * 4 + (g >> 2);
    role_encode(p, cx, S, b - 128, 128, 48 * CFG_EK);
    STAMP(p, 9);
#pragma unroll 1
    for (int ph = 0; ph < 2; ++ph) gemm_unit(p, cx, lds, pm + 8 * ph, pn, h, ph);
    BARRIER();
    role_decode<true>(p, cx, (unsigned)(size_t)(lds + wave * 9728), N_D + (b - 128), N_D + 128);
    STAMP(p, 10);
  }
}

}

extern "C" void kernel_launch(int rank, int nranks, ncclComm_t comm, hipStream_t stream,
                              void* const* d_in, const int* in_sizes, int n_in,
                              void* d_out, int out_size,
                              const int* in_dtypes, int out_dtype,
                              void* const* d_out_all, void* const* d_ws_all, size_t ws_bytes) {
  using namespace fk;
  (void)comm; (void)in_dtypes; (void)out_dtype;
  if (nranks != NRANK || n_in < 2 || rank < 0 || rank >= NRANK || ws_bytes < (size_t)WS_END ||
      in_sizes[0] != MD * KS || in_sizes[1] != KS * ND || out_size != MD * ND) {
    fprintf(stderr, "kernel_launch: unexpected problem shape\n");
    return;
  }
  static bool attr_set[64] = {};
  int dev = 0;
  if (hipGetDevice(&dev) != hipSuccess) return;
  if (dev >= 0 && dev < 64 && !attr_set[dev]) {
    hipError_t e = hipFuncSetAttribute((const void*)fused_kernel, hipFuncAttributeMaxDynamicSharedMemorySize, LDS_BYTES);
    if (e != hipSuccess) { fprintf(stderr, "HIP error %s\n", hipGetErrorString(e)); return; }
    attr_set[dev] = true;
  }
  Params p;
  p.x = (const char*)d_in[0]; p.w = (const char*)d_in[1];
  for (int i = 0; i < NRANK; ++i) { p.ws[i] = (char*)d_ws_all[i]; p.out[i] = (char*)d_out_all[i]; }
  p.out[rank] = (char*)d_out;
  p.rank = rank; p.pad = 0;
  hipLaunchKernelGGL(fused_kernel, dim3(GRID), dim3(NTHR), LDS_BYTES, stream, p);
  hipError_t e = hipGetLastError();
  if (e != hipSuccess) fprintf(stderr, "HIP launch error %s\n", hipGetErrorString(e));
}
```

```cpp
#include <hip/hip_runtime.h>
#include <rccl/rccl.h>
#include <cstdio>

namespace fk {

#define LAS __attribute__((address_space(3)))
#define GAS __attribute__((address_space(1)))
typedef unsigned short bf16_t;
typedef short bf16x8 __attribute__((ext_vector_type(8)));
typedef float f32x4 __attribute__((ext_vector_type(4)));
typedef unsigned u32x4 __attribute__((ext_vector_type(4)));
typedef unsigned u32x2 __attribute__((ext_vector_type(2)));
typedef unsigned short u16x2 __attribute__((ext_vector_type(2)));
typedef GAS unsigned gu32;
typedef GAS unsigned long long gu64;
typedef __amdgpu_buffer_rsrc_t rsrc_t;

constexpr int NRANK = 8;
constexpr int MD = 8192, ND = 8192, KD = 8192, KS = KD / NRANK;
constexpr int NT = KD / 64;
constexpr unsigned OUT_BYTES = (unsigned)MD * ND * 2u;
constexpr unsigned IN_BYTES = (unsigned)MD * KS * 2u;

constexpr int GRID = 256, NTHR = 512;
#define CFG_SLEEP 2
#define CFG_PJ 4
#define CFG_RAWHEAD 3
#define CFG_EK 8
#define CFG_PW 8
constexpr int PJ = CFG_PJ;
constexpr int PW = CFG_PW;
constexpr int PNV = PJ * PW;
#define CFG_NEO 28
constexpr int N_EO = CFG_NEO;
constexpr int N_P = 7 * PJ, N_D = 128 - N_P - N_EO;
constexpr int AUX_SYS = 17;

constexpr int NIMG = 3072;
constexpr unsigned IMG = 32768, PKT = 24576;
constexpr int NIT = 1024;
constexpr int NOT = 512;
constexpr int NPI = 384 + NOT;

constexpr unsigned WS_CALL = 0;
constexpr unsigned WS_TMO  = WS_CALL + 256 * 128;
constexpr unsigned WS_TK   = WS_TMO + 1024;
constexpr unsigned WS_CNT  = WS_TMO + 4096;
constexpr unsigned WS_PCNT = WS_CNT + NIMG * 128;
constexpr unsigned WS_HIN  = WS_PCNT + 32768;
constexpr unsigned WS_HOUT = WS_HIN + NIMG * 128;
constexpr unsigned WS_HOBI = WS_HOUT + NRANK * NOT * 128;
constexpr unsigned WS_HOBO = WS_HOBI + NIT * 128;
constexpr unsigned WS_PFLG = WS_HOBO + NOT * 128;
constexpr unsigned WS_RFLG = WS_PFLG + 2048 * 128;
constexpr unsigned WS_DBG  = WS_RFLG + NOT * 8 * 16;
constexpr unsigned WS_PIN  = 4u << 20;
constexpr unsigned WS_POUT = WS_PIN + NIMG * PKT;
constexpr unsigned WS_OBI  = WS_POUT + NRANK * NOT * PKT;
constexpr unsigned WS_OBR  = WS_OBI + NIT * PKT;
constexpr unsigned WS_OBO  = WS_OBR + NIT * IMG;
constexpr unsigned WS_PART = WS_OBO + NOT * PKT;
constexpr unsigned WS_IMG  = WS_PART + 256 * 131072;
constexpr unsigned WS_END  = WS_IMG + NIMG * IMG;
static_assert(WS_DBG + 256 * 64 * 8 <= WS_PIN, "control region overflow");
static_assert((size_t)WS_IMG + (size_t)NIMG * IMG < (1ull << 31), "workspace offsets must fit 31 bits");

constexpr unsigned long long DEADLINE = 40000000ull;

constexpr int HTB = 16384, STAGE_BYTES = 8 * HTB;
constexpr int LDS_FLAG = STAGE_BYTES;
constexpr int LDS_SCR = STAGE_BYTES + 512;
constexpr int LDS_BYTES = STAGE_BYTES + 512 + 8 * 1664;

struct Params {
  const char* x; const char* w;
  char* ws[NRANK]; char* out[NRANK];
  int rank; int pad;
};

__device__ __forceinline__ rsrc_t mkr(const void* p, unsigned bytes) { return __builtin_amdgcn_make_buffer_rsrc((void*)p, 0, (int)bytes, 0x00020000); }
__device__ __forceinline__ u32x4 ld128s(rsrc_t r, unsigned off) { return __builtin_amdgcn_raw_buffer_load_b128(r, (int)off, 0, AUX_SYS); }
__device__ __forceinline__ void st128s(u32x4 v, rsrc_t r, unsigned off) { __builtin_amdgcn_raw_buffer_store_b128(v, r, (int)off, 0, AUX_SYS); }
__device__ __forceinline__ unsigned long long rt() { return __builtin_amdgcn_s_memrealtime(); }
__device__ __forceinline__ unsigned rfl(unsigned v) { return (unsigned)__builtin_amdgcn_readfirstlane((int)v); }
__device__ __forceinline__ unsigned rdl(unsigned v, int l) { return (unsigned)__builtin_amdgcn_readlane((int)v, l); }
#define VM0() do { asm volatile("s_waitcnt vmcnt(0)" ::: "memory"); __builtin_amdgcn_s_waitcnt(0x0F70); } while (0)
#define DRAIN_ALL() do { asm volatile("s_waitcnt vmcnt(0) lgkmcnt(0)" ::: "memory"); __builtin_amdgcn_s_waitcnt(0x0070); } while (0)
#define DRAIN_PIN2(a, b) do { asm volatile("s_waitcnt vmcnt(0) lgkmcnt(0)" : "+v"(a), "+v"(b) :: "memory"); __builtin_amdgcn_s_waitcnt(0x0070); } while (0)
#define BARRIER() do { asm volatile("" ::: "memory"); __builtin_amdgcn_s_barrier(); asm volatile("" ::: "memory"); } while (0)

#define STAMP(p, idx) do { } while (0)
#define STAMPL(p, idx) do { } while (0)

__device__ __forceinline__ unsigned atomic_add_lane0_nowait(gu32* ptr) {
  unsigned r; unsigned long long sv; const unsigned one = 1u;
  asm volatile("s_mov_b64 %1, exec\n\ts_mov_b64 exec, 1\n\tglobal_atomic_add %0, %2, %3, off sc0\n\ts_mov_b64 exec, %1"
               : "=&v"(r), "=&s"(sv) : "v"(ptr), "v"(one) : "memory");
  return r;
}

__device__ __forceinline__ int atomic_add_lane0_nowait_noret(gu32* ptr) {
  unsigned long long sv; const unsigned one = 1u;
  asm volatile("s_mov_b64 %0, exec\n\ts_mov_b64 exec, 1\n\tglobal_atomic_add %1, %2, off\n\ts_mov_b64 exec, %0" : "=&s"(sv) : "v"(ptr), "v"(one) : "memory");
  return 0;
}
struct Ctx {
  gu32* tmo; unsigned call;
  __device__ __forceinline__ bool timed_out() const { return rfl(__hip_atomic_load(tmo, __ATOMIC_RELAXED, __HIP_MEMORY_SCOPE_AGENT)) == call; }
  __device__ __forceinline__ bool give_up(unsigned long long t0) const {
    if (timed_out() || rt() - t0 > DEADLINE) { __hip_atomic_store(tmo, call, __ATOMIC_RELAXED, __HIP_MEMORY_SCOPE_AGENT); return true; }
    __builtin_amdgcn_s_sleep(CFG_SLEEP);
    return false;
  }
};

__device__ __forceinline__ unsigned dpp_shr_add(unsigned x) {
  x += (unsigned)__builtin_amdgcn_update_dpp(0, (int)x, 0x111, 0xf, 0xf, false);
  x += (unsigned)__builtin_amdgcn_update_dpp(0, (int)x, 0x112, 0xf, 0xf, false);
  x += (unsigned)__builtin_amdgcn_update_dpp(0, (int)x, 0x114, 0xf, 0xf, false);
  x += (unsigned)__builtin_amdgcn_update_dpp(0, (int)x, 0x118, 0xf, 0xf, false);
  x += (unsigned)__builtin_amdgcn_update_dpp(0, (int)x, 0x142, 0xa, 0xf, false);
  x += (unsigned)__builtin_amdgcn_update_dpp(0, (int)x, 0x143, 0xc, 0xf, false);
  return x;
}
__device__ __forceinline__ unsigned row_max(unsigned x) {
  x = max(x, (unsigned)__builtin_amdgcn_update_dpp(0, (int)x, 0x111, 0xf, 0xf, false));
  x = max(x, (unsigned)__builtin_amdgcn_update_dpp(0, (int)x, 0x112, 0xf, 0xf, false));
  x = max(x, (unsigned)__builtin_amdgcn_update_dpp(0, (int)x, 0x114, 0xf, 0xf, false));
  x = max(x, (unsigned)__builtin_amdgcn_update_dpp(0, (int)x, 0x118, 0xf, 0xf, false));
  return x;
}
__device__ __forceinline__ unsigned wave_max_u32(unsigned x) {
  x = row_max(x);
  x = max(x, (unsigned)__builtin_amdgcn_update_dpp(0, (int)x, 0x142, 0xa, 0xf, false));
  x = max(x, (unsigned)__builtin_amdgcn_update_dpp(0, (int)x, 0x143, 0xc, 0xf, false));
  return rdl(x, 63);
}
__device__ __forceinline__ unsigned vperm(unsigned a, unsigned b, unsigned sel) { return __builtin_amdgcn_perm(a, b, sel); }
__device__ __forceinline__ unsigned lut4(unsigned tab, unsigned sel) { return __builtin_amdgcn_perm(0u, tab, sel); }
__device__ __forceinline__ unsigned bmask(unsigned e) { return __builtin_amdgcn_perm(0u, 0u, e | 0x0C0C0C0Cu); }
__device__ __forceinline__ unsigned bfi(unsigned m, unsigned a, unsigned b) { return (m & a) | (~m & b); }
__device__ __forceinline__ unsigned mul24(unsigned a, unsigned k) { unsigned r; asm("v_mul_u32_u24 %0, %1, %2" : "=v"(r) : "s"(k), "v"(a)); return r; }
__device__ __forceinline__ unsigned rank4(unsigned e) { return mul24(e, 0x010101u) << 8; }
__device__ __forceinline__ unsigned pc(unsigned e) { return (unsigned)__builtin_popcount(e); }
__device__ __forceinline__ unsigned nz4(unsigned x) { return ((x + 0x0F0F0F0Fu) >> 4) & 0x01010101u; }

constexpr int SCR_BYTES = 1664;

template <int W> __device__ __forceinline__ unsigned pack_bits(unsigned c, unsigned e) {
  const unsigned pre = rank4(e), em = bmask(e);
  if (W == 1) return __builtin_amdgcn_udot4(c, lut4(0x08040201u, pre) & em, 0u, false);
  if (W == 2) return __builtin_amdgcn_udot4(c, lut4(0x40100401u, pre) & em, 0u, false);
  return __builtin_amdgcn_udot4(c, lut4(0x00001001u, pre) & em, 0u, false) | (__builtin_amdgcn_udot4(c, lut4(0x10010000u, pre) & em, 0u, false) << 8);
}
template <int W> __device__ __forceinline__ unsigned unpack_bits(unsigned w, unsigned e) {
  unsigned s;
  if (W == 1) s = mul24(w & 0xFu, 0x00204081u) & 0x01010101u;
  else if (W == 2) { w &= 0xFFu; const unsigned t = (w | (w << 12)) & 0x000F000Fu; s = (t | (t << 6)) & 0x03030303u; }
  else if (W == 4) { w &= 0xFFFFu; const unsigned t = (w | (w << 8)) & 0x00FF00FFu; s = (t | (t << 4)) & 0x0F0F0F0Fu; }
  else s = w;
  return lut4(s, rank4(e)) & bmask(e);
}
__device__ __forceinline__ void lds_or(LAS unsigned* q, unsigned x) { if (x != 0u) __hip_atomic_fetch_or(q, x, __ATOMIC_RELAXED, __HIP_MEMORY_SCOPE_WAVEFRONT); }
__device__ __forceinline__ void flush32(LAS unsigned* S, unsigned off, unsigned acc) {
  const unsigned long long x = (unsigned long long)acc << (off & 31u);
  LAS unsigned* q = S + (off >> 5);
  lds_or(q, (unsigned)x); lds_or(q + 1, (unsigned)(x >> 32));
}
__device__ __forceinline__ void flush64(LAS unsigned* S, unsigned off, unsigned long long acc) {
  const unsigned sh = off & 31u;
  const unsigned long long x = acc << sh;
  LAS unsigned* q = S + (off >> 5);
  lds_or(q, (unsigned)x); lds_or(q + 1, (unsigned)(x >> 32)); lds_or(q + 2, ((unsigned)(acc >> 32) >> 1) >> (31u - sh));
}
__device__ __forceinline__ unsigned fetch32(const LAS unsigned* S, unsigned off) {
  const LAS unsigned* q = S + (off >> 5);
  return __builtin_amdgcn_alignbit(q[1], q[0], off & 31u);
}
__device__ __forceinline__ unsigned long long fetch64(const LAS unsigned* S, unsigned off) {
  const LAS unsigned* q = S + (off >> 5);
  const unsigned a = q[0], b = q[1], c = q[2];
  return ((unsigned long long)__builtin_amdgcn_alignbit(c, b, off & 31u) << 32) | __builtin_amdgcn_alignbit(b, a, off & 31u);
}
__device__ __forceinline__ unsigned wave_sum_lane63(unsigned x) { return dpp_shr_add(x); }
__device__ __forceinline__ unsigned sym_of(bool out, unsigned base, unsigned t) { return out ? ((base - (t & 7u)) | ((t & 8u) << 4)) & 0xFFu : (base - t) & 0xFFu; }

template <bool OUT>
__device__ __forceinline__ unsigned long long encode_chunk(const unsigned (&v)[16], LAS unsigned* S, int lane, u32x4& P0, u32x4& P1, u32x2& F) {
  u16x2 mx = __builtin_bit_cast(u16x2, v[0] & 0x7FFF7FFFu);
#pragma unroll
  for (int i = 1; i < 16; ++i) mx = __builtin_elementwise_max(mx, __builtin_bit_cast(u16x2, v[i] & 0x7FFF7FFFu));
  const unsigned base = wave_max_u32(max((unsigned)mx.x, (unsigned)mx.y)) >> (OUT ? 8 : 7);
  const unsigned B4 = base * 0x01010101u;
  *(LAS u32x4*)(S + lane * 4) = (u32x4){0u, 0u, 0u, 0u};
  if (lane < (OUT ? 34 : 1)) *(LAS u32x4*)(S + 256 + lane * 4) = (u32x4){0u, 0u, 0u, 0u};
  unsigned H[8], L[8], R[8], oor[8], anyo = 0;
#pragma unroll
  for (int g = 0; g < 8; ++g) {
    const unsigned Hs = vperm(v[2 * g + 1], v[2 * g], 0x07050301u), Ls = vperm(v[2 * g + 1], v[2 * g], 0x06040200u);
    if (OUT) { H[g] = Hs; L[g] = Ls; const unsigned D = B4 - (Hs & 0x7F7F7F7Fu); R[g] = (D & 0x07070707u) | ((Hs >> 4) & 0x08080808u); oor[g] = D & 0x78787878u; }
    else { H[g] = bfi(0x01010101u, Ls >> 7, Hs << 1); L[g] = bfi(0x80808080u, Hs, Ls); const unsigned D = B4 - H[g]; R[g] = D & 0x0F0F0F0Fu; oor[g] = D & 0xF0F0F0F0u; }
    anyo |= oor[g];
  }
  unsigned key;
  {
    unsigned long long hc = 0;
#pragma unroll
    for (int b = 0; b < 8; ++b) hc += 1ull << (((R[b >> 2] >> (8 * (b & 3))) & 15u) * 4u);
    const unsigned lo = (unsigned)hc, hi = (unsigned)(hc >> 32);
    unsigned q[8];
#pragma unroll
    for (int k = 0; k < 8; ++k) q[k] = rdl(dpp_shr_add(((lo >> (4 * k)) & 15u) | (((hi >> (4 * k)) & 15u) << 16)), 63);
    const int k7 = lane & 7;
    const unsigned qa = (k7 & 1) ? ((k7 & 2) ? q[3] : q[1]) : ((k7 & 2) ? q[2] : q[0]), qb = (k7 & 1) ? ((k7 & 2) ? q[7] : q[5]) : ((k7 & 2) ? q[6] : q[4]);
    const unsigned cnt = (((k7 & 4) ? qb : qa) >> ((lane & 8) * 2)) & 0xFFFFu;
    key = (lane < 16) ? (((cnt + 1u) << 4) | (unsigned)lane) : 0u;
  }
  unsigned tt = 0, cl = 0x1Fu;
#pragma unroll
  for (int i = 0; i < 7; ++i) {
    const unsigned bias = (OUT && i == 0 && lane >= 8 && lane < 16) ? (32u << 4) : 0u;
    const unsigned m = rdl(row_max(key + bias), 15);
    tt |= (m & 15u) << (4 * i);
    if (key + bias == m) { key = 0u; cl = (i < 3) ? (unsigned)i : (i == 3) ? 3u : (7u | ((unsigned)(i - 4) << 3)); }
  }
  cl <<= 8 * (lane & 3);
  cl |= (unsigned)__builtin_amdgcn_update_dpp(0, (int)cl, 0xB1, 0xf, 0xf, false);
  cl |= (unsigned)__builtin_amdgcn_update_dpp(0, (int)cl, 0x4E, 0xf, 0xf, false);
  const unsigned tb0 = rdl(cl, 0), tb1 = rdl(cl, 4), tb2 = rdl(cl, 8), tb3 = rdl(cl, 12);
  unsigned cls[8];
#pragma unroll
  for (int g = 0; g < 8; ++g) {
    const unsigned r7 = R[g] & 0x07070707u;
    cls[g] = bfi(bmask((R[g] >> 3) & 0x01010101u), vperm(tb3, tb2, r7), vperm(tb1, tb0, r7));
  }
  const bool has_oor = __any(anyo != 0u);
  unsigned e4[8];
#pragma unroll
  for (int g = 0; g < 8; ++g) e4[g] = 0u;
  if (has_oor) {
    const unsigned T1 = ((tt >> 4) & 15u) * 0x01010101u;
#pragma unroll
    for (int g = 0; g < 8; ++g) {
      e4[g] = nz4(OUT ? (oor[g] >> 3) : (oor[g] >> 4));
      const unsigned m = bmask(e4[g]);
      cls[g] |= m & 0x1F1F1F1Fu; R[g] = bfi(m, T1, R[g]);
    }
  }
  bool modeB = false; unsigned n0 = 0;
  if (OUT) {
    unsigned z[8], mxn = 0;
#pragma unroll
    for (int g = 0; g < 8; ++g) z[g] = lut4(0x000000FFu, cls[g] & 0x03030303u);
#pragma unroll
    for (int g = 0; g < 2; ++g) {
      const unsigned hn = (L[g] >> 4) & z[g] & 0x0F0F0F0Fu;
      mxn = max(mxn, max(max(hn & 0xFFu, (hn >> 8) & 0xFFu), max((hn >> 16) & 0xFFu, hn >> 24)));
    }
    n0 = wave_max_u32(mxn);
    unsigned viol[8], nv = 0;
    const unsigned N4 = (n0 | 0x80u) * 0x01010101u;
#pragma unroll
    for (int g = 0; g < 8; ++g) {
      const unsigned dn = (N4 - ((L[g] >> 4) & 0x0F0F0F0Fu)) & 0x7F7F7F7Fu;
      viol[g] = z[g] & bmask(((dn + 0x7D7D7D7Du) >> 7) & 0x01010101u);
      nv += pc(viol[g] & 0x01010101u);
    }
    modeB = n0 < 8u && rdl(dpp_shr_add(nv), 63) < 24u;
    if (modeB) {
#pragma unroll
      for (int g = 0; g < 8; ++g) cls[g] |= viol[g] & 0x1F1F1F1Fu;
    }
  }
  unsigned e2[8], e2b[8], e3[8], e0[8], m2[8], n2 = 0, n2b = 0, n3 = 0, n4 = 0, c0 = 0, cm2 = 0;
  const unsigned N0 = n0 * 0x01010101u;
#pragma unroll
  for (int g = 0; g < 8; ++g) {
    const unsigned c1 = cls[g] & 0x03030303u;
    e2[g] = lut4(0x01000000u, c1); e2b[g] = (cls[g] >> 2) & 0x01010101u; e3[g] = lut4(0x01000000u, (cls[g] >> 3) & 0x03030303u);
    n2 += pc(e2[g]); n2b += pc(e2b[g]); n3 += pc(e3[g]); n4 += pc(e4[g]);
    if (OUT) {
      e0[g] = modeB ? lut4(0x00000001u, c1) : 0u;
      m2[g] = e0[g] & nz4(((L[g] >> 4) & 0x0F0F0F0Fu) ^ N0);
      c0 += pc(e0[g]); cm2 += pc(m2[g]);
    } else { e0[g] = 0u; m2[g] = 0u; }
  }
  const unsigned x1 = c0 | (n2 << 16), x2 = cm2 | (n2b << 16), x3 = n3 | (n4 << 16);
  const unsigned s1 = dpp_shr_add(x1), s2 = dpp_shr_add(x2), s3 = dpp_shr_add(x3);
  const unsigned t1 = rdl(s1, 63), t2 = rdl(s2, 63), t3 = rdl(s3, 63);
  const unsigned C0 = t1 & 0xFFFFu, N2 = t1 >> 16, CM2 = t2 & 0xFFFFu, N2B = t2 >> 16, N3 = t3 & 0xFFFFu, N4c = t3 >> 16;
  const unsigned bSN = OUT ? 4u * (2048u - C0) : 0u;
  const unsigned aM1 = bSN, a2a = aM1 + C0, aM2 = a2a + N2, a2b = aM2 + CM2, a3 = a2b + 2u * N2B, a4 = a3 + 4u * N3, bits = a4 + 8u * N4c;
  const bool raw = bits > (OUT ? 12288u : 4096u);
  const unsigned len = ((OUT ? 12288u : 20480u) + bits + 255u) >> 8;
  if (!raw) {
    const unsigned p1 = s1 - x1, p2 = s2 - x2, p3 = s3 - x3;
    const unsigned oN = 4u * (32u * (unsigned)lane - (p1 & 0xFFFFu)), oM1 = aM1 + (p1 & 0xFFFFu), o2a = a2a + (p1 >> 16), oM2 = aM2 + (p2 & 0xFFFFu),
                   o2b = a2b + 2u * (p2 >> 16), o3 = a3 + 4u * (p3 & 0xFFFFu);
    unsigned o4 = a4 + 8u * (p3 >> 16);
    unsigned wM1 = 0, wM2 = 0, w2a = 0, kM1 = 0, kM2 = 0, k2a = 0, k2b = 0, kN = 0, k3 = 0, kNa = 0, k3a = 0;
    unsigned long long w2b = 0, wN = 0, w3 = 0;
#pragma unroll
    for (int g = 0; g < 8; ++g) {
      if (g == 4) {
        if (OUT) { flush64(S, oN, wN); wN = 0; kNa = kN; kN = 0; }
        flush64(S, o3, w3); w3 = 0; k3a = k3; k3 = 0;
      }
      if (OUT) {
        const unsigned hn = (L[g] >> 4) & 0x0F0F0F0Fu, eN = e0[g] ^ 0x01010101u;
        wN |= (unsigned long long)pack_bits<4>(hn, eN) << kN; kN += 4u * pc(eN);
        wM1 |= pack_bits<1>(m2[g], e0[g]) << kM1; kM1 += pc(e0[g]);
        wM2 |= pack_bits<1>(nz4(((N0 | 0x10101010u) - 0x01010101u - hn) & 0x0F0F0F0Fu), m2[g]) << kM2; kM2 += pc(m2[g]);
      }
      w2a |= pack_bits<1>(e2b[g], e2[g]) << k2a; k2a += pc(e2[g]);
      w2b |= (unsigned long long)pack_bits<2>((cls[g] >> 3) & 0x03030303u, e2b[g]) << k2b; k2b += 2u * pc(e2b[g]);
      w3 |= (unsigned long long)pack_bits<4>(R[g], e3[g]) << k3; k3 += 4u * pc(e3[g]);
    }
    if (OUT) { flush64(S, oN + kNa, wN); flush32(S, oM1, wM1); flush32(S, oM2, wM2); }
    flush32(S, o2a, w2a); flush64(S, o2b, w2b); flush64(S, o3 + k3a, w3);
    if (has_oor) {
#pragma unroll
      for (int g = 0; g < 8; ++g)
#pragma unroll
        for (int b = 0; b < 4; ++b)
          if ((e4[g] >> (8 * b)) & 1u) {
            const unsigned long long x = (unsigned long long)((H[g] >> (8 * b)) & 0xFFu) << (o4 & 31u);
            __hip_atomic_fetch_or(S + (o4 >> 5), (unsigned)x, __ATOMIC_RELAXED, __HIP_MEMORY_SCOPE_WAVEFRONT);
            __hip_atomic_fetch_or(S + (o4 >> 5) + 1, (unsigned)(x >> 32), __ATOMIC_RELAXED, __HIP_MEMORY_SCOPE_WAVEFRONT);
            o4 += 8u;
          }
    }
  }
  unsigned Fw[2];
#pragma unroll
  for (int i = 0; i < 2; ++i)
    Fw[i] = (cls[4 * i] & 0x03030303u) | ((cls[4 * i + 1] & 0x03030303u) << 2) | ((cls[4 * i + 2] & 0x03030303u) << 4) | ((cls[4 * i + 3] & 0x03030303u) << 6);
  F = (u32x2){Fw[0], Fw[1]};
  if (OUT) {
    unsigned p4[4];
#pragma unroll
    for (int i = 0; i < 4; ++i) p4[i] = (L[2 * i] & 0x0F0F0F0Fu) | ((L[2 * i + 1] & 0x0F0F0F0Fu) << 4);
    P0 = (u32x4){p4[0], p4[1], p4[2], p4[3]}; P1 = P0;
  } else {
    P0 = (u32x4){L[0], L[1], L[2], L[3]}; P1 = (u32x4){L[4], L[5], L[6], L[7]};
  }
  return (unsigned long long)(base | (raw ? 0x8000u : (len << 8))) | ((unsigned long long)tt << 16) | (modeB ? (1ull << 44) | ((unsigned long long)n0 << 45) : 0ull);
}
__device__ __forceinline__ bool hp_raw(unsigned long long hp) { return ((unsigned)hp & 0x8000u) != 0u; }
__device__ __forceinline__ unsigned hp_len(unsigned long long hp) { return (((unsigned)hp >> 8) & 127u) * 32u; }

template <bool OUT>
__device__ __forceinline__ void store_chunk(rsrc_t r, unsigned o, const LAS unsigned* S, unsigned len, int lane, u32x4 P0, u32x4 P1, u32x2 F) {
  const unsigned l16 = (unsigned)lane * 16u;
  st128s(P0, r, o + l16);
  const u32x4 sv = *(const LAS u32x4*)(S + lane * 4);
  if (OUT) {
    const u32x4 sw = *(const LAS u32x4*)(S + 256 + (lane & 31) * 4);
    __builtin_amdgcn_raw_buffer_store_b64(F, r, (int)(o + 1024 + lane * 8), 0, AUX_SYS);
    if (1536u + l16 < len) st128s(sv, r, o + 1536 + l16);
    if (lane < 32 && 2560u + l16 < len) st128s(sw, r, o + 2560 + l16);
  } else {
    st128s(P1, r, o + 1024 + l16);
    __builtin_amdgcn_raw_buffer_store_b64(F, r, (int)(o + 2048 + lane * 8), 0, AUX_SYS);
    if (lane < 32 && 2560u + l16 < len) st128s(sv, r, o + 2560 + l16);
  }
}

template <bool OUT>
__device__ __forceinline__ void decode_rec(const LAS unsigned* R, unsigned long long hp, int lane, unsigned (&v)[16]) {
  const unsigned base = (unsigned)hp & 0xFFu, tt = (unsigned)(hp >> 16) & 0xFFFFFFFu;
  const bool modeB = ((hp >> 44) & 1u) != 0u;
  const unsigned n0 = (unsigned)(hp >> 45) & 7u;
  const u32x4 P0 = *(const LAS u32x4*)(R + lane * 4), P1 = OUT ? P0 : *(const LAS u32x4*)(R + 256 + lane * 4);
  const u32x2 F = *(const LAS u32x2*)(R + (OUT ? 256 : 512) + lane * 2);
  const LAS unsigned* S = R + (OUT ? 384 : 640);
  unsigned tab1 = 0, tab2 = 0;
#pragma unroll
  for (int i = 0; i < 4; ++i) tab1 |= sym_of(OUT, base, (tt >> (4 * i)) & 15u) << (8 * i);
#pragma unroll
  for (int i = 0; i < 3; ++i) tab2 |= sym_of(OUT, base, (tt >> (16 + 4 * i)) & 15u) << (8 * i);
  const unsigned B4 = base * 0x01010101u, T1 = ((tt >> 4) & 15u) * 0x01010101u, N0 = n0 * 0x01010101u;
  unsigned c1[8], e2[8], e0[8], n2 = 0, c0 = 0;
#pragma unroll
  for (int g = 0; g < 8; ++g) {
    c1[g] = (((g < 4) ? F.x : F.y) >> (2 * (g & 3))) & 0x03030303u;
    e2[g] = lut4(0x01000000u, c1[g]); n2 += pc(e2[g]);
    if (OUT) { e0[g] = modeB ? lut4(0x00000001u, c1[g]) : 0u; c0 += pc(e0[g]); } else e0[g] = 0u;
  }
  const unsigned x1 = c0 | (n2 << 16), s1 = dpp_shr_add(x1), t1 = rdl(s1, 63), p1 = s1 - x1;
  const unsigned C0 = t1 & 0xFFFFu, N2 = t1 >> 16;
  const unsigned aM1 = OUT ? 4u * (2048u - C0) : 0u, a2a = aM1 + C0, aM2 = a2a + N2;
  const unsigned oN = 4u * (32u * (unsigned)lane - (p1 & 0xFFFFu));
  unsigned hn[8], m2[8], e2b[8], cm2 = 0, n2b = 0;
  {
    unsigned w2a = fetch32(S, a2a + (p1 >> 16)), wM1 = 0; unsigned long long wN = 0, wN2 = 0;
    if (OUT) {
      wM1 = fetch32(S, aM1 + (p1 & 0xFFFFu)); wN = fetch64(S, oN);
      wN2 = fetch64(S, oN + 4u * (16u - pc(e0[0]) - pc(e0[1]) - pc(e0[2]) - pc(e0[3])));
    }
#pragma unroll
    for (int g = 0; g < 8; ++g) {
      e2b[g] = unpack_bits<1>(w2a, e2[g]); w2a >>= pc(e2[g]); n2b += pc(e2b[g]);
      if (OUT) {
        if (g == 4) wN = wN2;
        const unsigned eN = e0[g] ^ 0x01010101u;
        hn[g] = unpack_bits<4>((unsigned)wN, eN); wN >>= 4u * pc(eN);
        m2[g] = unpack_bits<1>(wM1, e0[g]); wM1 >>= pc(e0[g]); cm2 += pc(m2[g]);
      }
    }
  }
  const unsigned x2 = cm2 | (n2b << 16), s2 = dpp_shr_add(x2), t2 = rdl(s2, 63), p2 = s2 - x2;
  const unsigned CM2 = t2 & 0xFFFFu, N2B = t2 >> 16, a2b = aM2 + CM2, a3 = a2b + 2u * N2B;
  unsigned wM2 = OUT ? fetch32(S, aM2 + (p2 & 0xFFFFu)) : 0u;
  unsigned long long w2b = fetch64(S, a2b + 2u * (p2 >> 16));
  unsigned H[8], L[8], e3[8], n3 = 0;
#pragma unroll
  for (int g = 0; g < 8; ++g) {
    const unsigned c2 = unpack_bits<2>((unsigned)w2b, e2b[g]); w2b >>= 2u * pc(e2b[g]);
    e3[g] = lut4(0x01000000u, c2); n3 += pc(e3[g]);
    H[g] = bfi(bmask(e2b[g]), lut4(tab2, c2), lut4(tab1, c1[g]));
    if (OUT) {
      const unsigned b2 = unpack_bits<1>(wM2, m2[g]); wM2 >>= pc(m2[g]);
      const unsigned nib = hn[g] | ((N0 - m2[g] - b2) & bmask(e0[g]));
      const unsigned p4 = (g >> 1) == 0 ? P0.x : (g >> 1) == 1 ? P0.y : (g >> 1) == 2 ? P0.z : P0.w;
      L[g] = ((p4 >> (4 * (g & 1))) & 0x0F0F0F0Fu) | (nib << 4);
    } else {
      L[g] = g == 0 ? P0.x : g == 1 ? P0.y : g == 2 ? P0.z : g == 3 ? P0.w : g == 4 ? P1.x : g == 5 ? P1.y : g == 6 ? P1.z : P1.w;
    }
  }
  const unsigned s3 = dpp_shr_add(n3), N3 = rdl(s3, 63);
  const unsigned o3 = a3 + 4u * (s3 - n3);
  unsigned long long w3 = fetch64(S, o3);
  const unsigned long long w3b = fetch64(S, o3 + 4u * (pc(e3[0]) + pc(e3[1]) + pc(e3[2]) + pc(e3[3])));
  unsigned e4[8], any4 = 0;
#pragma unroll
  for (int g = 0; g < 8; ++g) {
    if (g == 4) w3 = w3b;
    unsigned R = unpack_bits<4>((unsigned)w3, e3[g]); w3 >>= 4u * pc(e3[g]);
    e4[g] = e3[g] & ~nz4(R ^ T1);
    R &= ~bmask(e4[g]);
    const unsigned H3 = OUT ? ((B4 - (R & 0x07070707u)) | ((R & 0x08080808u) << 4)) : (B4 - R);
    H[g] = bfi(bmask(e3[g]), H3, H[g]);
    any4 |= e4[g];
  }
  if (__any(any4 != 0u)) {
    unsigned n4 = 0;
#pragma unroll
    for (int g = 0; g < 8; ++g) n4 += pc(e4[g]);
    unsigned o4 = a3 + 4u * N3 + 8u * (dpp_shr_add(n4) - n4);
#pragma unroll
    for (int g = 0; g < 8; ++g) { H[g] = bfi(bmask(e4[g]), unpack_bits<8>(fetch32(S, o4), e4[g]), H[g]); o4 += 8u * pc(e4[g]); }
  }
#pragma unroll
  for (int g = 0; g < 8; ++g) {
    const unsigned Hs = OUT ? H[g] : bfi(0x80808080u, L[g], H[g] >> 1), Ls = OUT ? L[g] : bfi(0x80808080u, H[g] << 7, L[g]);
    v[2 * g] = vperm(Hs, Ls, 0x05010400u);
    v[2 * g + 1] = vperm(Hs, Ls, 0x07030602u);
  }
}

struct MapX { unsigned base; __device__ __forceinline__ unsigned operator()(int w, int j, int l) const { return base + (unsigned)(w * 4096 + j * 1024 + l * 16); } };
struct MapW { unsigned base; __device__ __forceinline__ unsigned operator()(int w, int j, int l) const { return base + (unsigned)((j >> 1) * 16384 + ((w >> 1) * 2 + (j & 1)) * 2048 + (w & 1) * 1024 + l * 16); } };
struct MapO { unsigned base; __device__ __forceinline__ unsigned operator()(int w, int j, int l) const { return base + (unsigned)((64 * (w >> 2) + 16 * (j >> 1) + (l & 15)) * (ND * 2) + (64 * (w & 3) + 32 * (j & 1) + 8 * (l >> 4)) * 2); } };
__device__ __forceinline__ unsigned out_task_base(int rank, int task) {
  const int tile = task >> 2, q = task & 3, pm = tile >> 3, pn = tile & 7;
  return (unsigned)(((rank >> 2) * 4096 + pm * 256 + (q >> 1) * 128 + (q & 1) * 32) * (ND * 2) + ((rank & 3) * 2048 + pn * 256) * 2);
}

__device__ __forceinline__ unsigned long long mk_granule(unsigned long long payload, unsigned call) { return payload | ((unsigned long long)(call & 0xFFFFu) << 48); }
__device__ __forceinline__ unsigned long long uni64(unsigned long long g) { return ((unsigned long long)rfl((unsigned)(g >> 32)) << 32) | rfl((unsigned)g); }
__device__ __forceinline__ unsigned long long ld_granule(const char* ws, unsigned off) {
  return __hip_atomic_load((const gu64*)(ws + off), __ATOMIC_RELAXED, __HIP_MEMORY_SCOPE_SYSTEM);
}
__device__ __forceinline__ bool tag_ok(unsigned long long g, unsigned call) { return (unsigned)(g >> 48) == (call & 0xFFFFu); }
__device__ __forceinline__ bool poll_granule(const Ctx& cx, const char* ws, unsigned off, unsigned long long& g) {
  g = uni64(g);
  if (!tag_ok(g, cx.call)) {
    const unsigned long long t0 = rt();
    do {
      if (cx.give_up(t0)) return false;
      g = uni64(ld_granule(ws, off));
    } while (!tag_ok(g, cx.call));
  }
  asm volatile("" ::: "memory");
  return true;
}

struct InTask { int kt, isw, idx; };
__device__ __forceinline__ InTask in_task(int i) {
  InTask t;
  if (i < 768) { t.kt = i / 48; const int u = i - t.kt * 48, u3 = u / 3, r3 = u - u3 * 3;
    if (r3 == 0) { t.isw = 0; t.idx = (u3 & 1) * 16 + (u3 >> 1); } else { t.isw = 1; const int c = u3 * 2 + r3 - 1; t.idx = (c & 3) * 8 + (c >> 2); } }
  else { const int u = i - 768; t.kt = (u >> 4) & 15; t.isw = 0; t.idx = (u & 1) * 16 + 8 + ((u & 15) >> 1); }
  return t;
}
__device__ __forceinline__ int otask_of(int isw, int idx, int kt) { return isw * 512 + idx * 16 + kt; }
__device__ __forceinline__ void in_load(const Params& p, const InTask& t, u32x4 (&r)[4], int wid, int lane) {
  const int rr = lane >> 2, cp = (lane & 3) ^ ((rr >> 3) << 1);
  if (!t.isw) {
    const rsrc_t rx = mkr(p.x, IN_BYTES);
#pragma unroll
    for (int j = 0; j < 4; ++j) {
      const unsigned off = (unsigned)((256 * t.idx + 32 * wid + 16 * (j >> 1) + rr) * (KS * 2) + (64 * t.kt + 32 * (j & 1) + 8 * cp) * 2);
      r[j] = __builtin_amdgcn_raw_buffer_load_b128(rx, (int)off, 0, 0);
    }
  } else {
    const rsrc_t rw = mkr(p.w, IN_BYTES);
    const unsigned off0 = (unsigned)((64 * t.kt + 32 * (wid & 1) + 8 * cp) * (ND * 2) + (256 * t.idx + 64 * (wid >> 1) + 4 * rr) * 2);
#pragma unroll
    for (int d = 0; d < 4; ++d) {
      const u32x2 a = __builtin_amdgcn_raw_buffer_load_b64(rw, (int)(off0 + (unsigned)(2 * d) * (ND * 2)), 0, 0);
      const u32x2 b = __builtin_amdgcn_raw_buffer_load_b64(rw, (int)(off0 + (unsigned)(2 * d + 1) * (ND * 2)), 0, 0);
      r[d] = __builtin_shufflevector(a, b, 0, 1, 2, 3);
    }
  }
}
__device__ __forceinline__ void in_prep(int isw, const u32x4 (&r)[4], unsigned (&v)[16]) {
  if (isw) {
#pragma unroll
    for (int d = 0; d < 4; ++d) {
      v[0 + d] = __builtin_amdgcn_perm(r[d].z, r[d].x, 0x05040100u);
      v[4 + d] = __builtin_amdgcn_perm(r[d].z, r[d].x, 0x07060302u);
      v[8 + d] = __builtin_amdgcn_perm(r[d].w, r[d].y, 0x05040100u);
      v[12 + d] = __builtin_amdgcn_perm(r[d].w, r[d].y, 0x07060302u);
    }
  } else {
#pragma unroll
    for (int j = 0; j < 4; ++j) { v[4 * j] = r[j].x; v[4 * j + 1] = r[j].y; v[4 * j + 2] = r[j].z; v[4 * j + 3] = r[j].w; }
  }
}

__device__ __forceinline__ void role_encode(const Params& p, const Ctx& cx, LAS unsigned* S, int e, int ne, int iend) {
  const int tid = threadIdx.x, wid = (int)rfl(tid >> 6), lane = tid & 63;
  char* ws = p.ws[p.rank];
  const rsrc_t lw = mkr(ws, WS_END);
  u32x4 A[4];
  if (e >= iend) return;
  in_load(p, in_task(e), A, wid, lane);
  DRAIN_ALL();
  for (int i = e; i < iend; i += ne) {
    const InTask t = in_task(i);
    unsigned v[16];
    in_prep(t.isw, A, v);
    if (i + ne < iend) in_load(p, in_task(i + ne), A, wid, lane);
    u32x4 P0, P1; u32x2 F;
    const unsigned long long hp = encode_chunk<false>(v, S, lane, P0, P1, F);
    const int ot = otask_of(t.isw, t.idx, t.kt);
    const bool mine = t.isw ? ((t.idx >> 3) == (p.rank & 3)) : ((t.idx >> 4) == (p.rank >> 2));
    const int img = t.isw ? 2048 + (t.idx & 7) * 128 + t.kt * 8 + p.rank : (t.idx & 15) * 128 + t.kt * 8 + p.rank;
    if (!hp_raw(hp)) {
      store_chunk<false>(lw, WS_OBI + (unsigned)ot * PKT + (unsigned)wid * 3072u, S, hp_len(hp), lane, P0, P1, F);
    } else {
#pragma unroll
      for (int j = 0; j < 4; ++j) st128s((u32x4){v[4 * j], v[4 * j + 1], v[4 * j + 2], v[4 * j + 3]}, lw, WS_OBR + (unsigned)ot * IMG + (unsigned)(wid * 4096 + j * 1024 + lane * 16));
    }
    if (mine) {
      const unsigned ib = WS_IMG + (unsigned)img * IMG;
#pragma unroll
      for (int j = 0; j < 4; ++j) st128s((u32x4){v[4 * j], v[4 * j + 1], v[4 * j + 2], v[4 * j + 3]}, lw, t.isw ? MapW{ib}(wid, j, lane) : MapX{ib}(wid, j, lane));
    }
    DRAIN_ALL();
    if (lane == 0) {
      __hip_atomic_store((gu64*)(ws + WS_HOBI + (unsigned)ot * 64u) + wid, mk_granule(hp, cx.call), __ATOMIC_RELAXED, __HIP_MEMORY_SCOPE_SYSTEM);
      if (mine) __hip_atomic_fetch_add((gu32*)(ws + WS_CNT) + img * 32, 1u, __ATOMIC_RELAXED, __HIP_MEMORY_SCOPE_AGENT);
    }
  }
}

__device__ __forceinline__ bool eo_ready(const Ctx& cx, const gu32* f, bool block) {
  if (rfl(__hip_atomic_load(f, __ATOMIC_RELAXED, __HIP_MEMORY_SCOPE_AGENT)) >= cx.call) return true;
  if (!block) return false;
  const unsigned long long t0 = rt();
  while (rfl(__hip_atomic_load(f, __ATOMIC_RELAXED, __HIP_MEMORY_SCOPE_AGENT)) < cx.call) { if (cx.give_up(t0)) return false; }
  return true;
}
__device__ __forceinline__ void role_encode_out(const Params& p, const Ctx& cx, LAS unsigned* S, int e, int ne) {
  const int tid = threadIdx.x, wid = (int)rfl(tid >> 6), lane = tid & 63;
  char* ws = p.ws[p.rank];
  const rsrc_t lw = mkr(ws, WS_END), ro = mkr(p.out[p.rank], OUT_BYTES);
  const gu32* rf = (const gu32*)(ws + WS_RFLG) + wid * 4;
  auto load = [&](int task, u32x4 (&A)[4]) { const MapO map{out_task_base(p.rank, task)};
#pragma unroll
    for (int j = 0; j < 4; ++j) A[j] = ld128s(ro, map(wid, j, lane)); };
  u32x4 A[4];
#pragma unroll
  for (int j = 0; j < 4; ++j) {
    unsigned t[4];
#pragma unroll
    for (int k = 0; k < 4; ++k) t[k] = ((0x3FBE3EBDu >> (((lane + 4 * j + k) & 3) * 8)) & 0xFFu) * 0x01000100u + ((unsigned)(lane * 29 + (4 * j + k) * 7) & 0x1Fu) * 0x00010003u;
    A[j] = (u32x4){t[0], t[1], t[2], t[3]};
  }
#pragma unroll 1
  for (int i = e - ne; i < NOT; i += ne) {
    unsigned v[16];
#pragma unroll
    for (int j = 0; j < 4; ++j) { v[4 * j] = A[j].x; v[4 * j + 1] = A[j].y; v[4 * j + 2] = A[j].z; v[4 * j + 3] = A[j].w; }
    const int nx = i + ne;
    bool have = false;
    if (nx < NOT) { have = eo_ready(cx, rf + nx * 32, false); if (have) load(nx, A); }
    u32x4 P0, P1; u32x2 F;
    const unsigned long long hp = encode_chunk<true>(v, S, lane, P0, P1, F);
    if (!hp_raw(hp)) store_chunk<true>(lw, i < 0 ? 0xFFFF0000u : WS_OBO + (unsigned)i * PKT + (unsigned)wid * 3072u, S, hp_len(hp), lane, P0, P1, F);
    DRAIN_ALL();
    if (i >= 0 && lane == 0) __hip_atomic_store((gu64*)(ws + WS_HOBO + (unsigned)i * 64u) + wid, mk_granule(hp, cx.call), __ATOMIC_RELAXED, __HIP_MEMORY_SCOPE_SYSTEM);
    if (nx < NOT && !have) {
      if (!eo_ready(cx, rf + nx * 32, true)) return;
      load(nx, A);
      DRAIN_ALL();
    }
  }
}

struct PItem { unsigned hsrc, dsrc, hdst, ddst, rs, rd; int kind; };
__device__ __forceinline__ PItem p_item(int rank, int d, int i) {
  PItem r;
  if (i < 384) {
    int kt, isw, k;
    if (i < 256) { kt = i >> 4; isw = i & 1; k = (i & 15) >> 1; } else { kt = (i - 256) >> 3; isw = 0; k = 8 + ((i - 256) & 7); }
    const int idx = isw ? (d & 3) * 8 + k : (d >> 2) * 16 + k;
    const int ot = otask_of(isw, idx, kt), T = kt * 8 + rank, img = isw ? 2048 + k * 128 + T : k * 128 + T;
    const unsigned slot = (unsigned)(rank * 384 + i);
    r.hsrc = WS_HOBI + (unsigned)ot * 64u; r.dsrc = WS_OBI + (unsigned)ot * PKT;
    r.hdst = WS_HIN + slot * 64u; r.ddst = WS_PIN + slot * PKT;
    r.rs = WS_OBR + (unsigned)ot * IMG; r.rd = WS_IMG + (unsigned)img * IMG; r.kind = isw;
  } else {
    const int task = (i - 384) & (NOT - 1);
    r.hsrc = WS_HOBO + (unsigned)task * 64u; r.dsrc = WS_OBO + (unsigned)task * PKT;
    r.hdst = WS_HOUT + (unsigned)(rank * NOT + task) * 64u; r.ddst = WS_POUT + (unsigned)(rank * NOT + task) * PKT;
    r.rs = r.rd = out_task_base(rank, task); r.kind = 2;
  }
  return r;
}

__device__ __forceinline__ void p_publish(const Ctx& cx, const char* ws, char* wd, unsigned hsrc, unsigned hdst, unsigned long long hl, int lane) {
  if (hsrc == 0xFFFFFFFFu) hl = mk_granule(0x8000ull, cx.call);
  else {
    const char* src = ws + hsrc + (lane & 7) * 8;
    while (!__all(tag_ok(hl, cx.call))) {
      u32x2 r; asm volatile("global_load_dwordx2 %0, %1, off sc0 sc1\n\ts_waitcnt vmcnt(0)" : "=&v"(r) : "v"(src) : "memory");
      hl = ((unsigned long long)r.y << 32) | r.x;
    }
  }
  if (lane < 8) {
    const u32x2 v = (u32x2){(unsigned)hl, (unsigned)(hl >> 32)};
    asm volatile("global_store_dwordx2 %0, %1, off sc0 sc1\n\ts_nop 1" :: "v"(wd + hdst + lane * 8), "v"(v) : "memory");
  }
}

__device__ __forceinline__ void st_peer(u32x4 v, char* ptr) { asm volatile("global_store_dwordx4 %0, %1, off sc0 sc1\n\ts_nop 1" :: "v"(ptr), "v"(v) : "memory"); }
__device__ __forceinline__ void glds16(const char* src, unsigned dst) {
  unsigned keep; asm volatile("s_mov_b32 %0, m0\n\ts_mov_b32 m0, %2\n\ts_nop 0\n\tglobal_load_lds_dwordx4 %1, off sc0 sc1\n\ts_mov_b32 m0, %0" : "=&s"(keep) : "v"(src), "s"(rfl(dst)) : "memory");
}
__device__ __forceinline__ void glds4(const char* src, unsigned dst) {
  unsigned keep; asm volatile("s_mov_b32 %0, m0\n\ts_mov_b32 m0, %2\n\ts_nop 0\n\tglobal_load_lds_dword %1, off sc0 sc1\n\ts_mov_b32 m0, %0" : "=&s"(keep) : "v"(src), "s"(rfl(dst)) : "memory");
}
__device__ __forceinline__ u32x2 ld64_nowait(const char* src) {
  u32x2 r; asm volatile("global_load_dwordx2 %0, %1, off sc0 sc1" : "=&v"(r) : "v"(src) : "memory");
  return r;
}
__device__ __forceinline__ unsigned long long u64_of(u32x2 v) { return ((unsigned long long)v.y << 32) | v.x; }
__device__ __forceinline__ void role_push(const Params& p, Ctx& cx, unsigned callraw, unsigned lds_ring, int link, int v0, int nv) {
  constexpr int NC = NPI * 8;
  const int lane = threadIdx.x & 63;
  int d = p.rank + 1 + link; if (d >= NRANK) d -= NRANK;
  if ((unsigned)d >= (unsigned)NRANK) return;
  char* ws = p.ws[p.rank]; char* wd = p.ws[d];
  const rsrc_t lw = mkr(ws, WS_END), dw = mkr(wd, WS_END);
  gu32* tkc = (gu32*)(ws + WS_TK) + link * 32;
  gu32* pcnt = (gu32*)(ws + WS_PCNT) + link * NPI;
  const unsigned lo = (unsigned)lane * 16u;
  constexpr int R0 = CFG_RAWHEAD;
  static_assert(R0 * PNV <= 128 && (R0 * PNV) % 8 == 0, "raw head chunks must be whole K-tile 0 items");
  int prev; unsigned prev_hsrc = 0, prev_hdst;
  {
    auto head_task = [&](int u) { const int i0 = u >> 3; InTask t; t.kt = 0; t.isw = i0 & 1; t.idx = t.isw ? (d & 3) * 8 + (i0 >> 1) : (d >> 2) * 16 + (i0 >> 1); return t; };
    u32x4 A[R0][4];
#pragma unroll
    for (int r = 0; r < R0; ++r) in_load(p, head_task(v0 + r * nv), A[r], (v0 + r * nv) & 7, lane);
    unsigned dummy = 0;
    DRAIN_PIN2(callraw, dummy);
    cx.call = rfl(callraw) + 1u;
#pragma unroll
    for (int r = 0; r < R0; ++r) {
      const int u = v0 + r * nv, w = u & 7;
      const InTask t = head_task(u);
      unsigned v[16];
      in_prep(t.isw, A[r], v);
      const PItem f = p_item(p.rank, d, u >> 3);
      unsigned pc = 0;
      if (r > 0) { VM0(); pc = atomic_add_lane0_nowait(pcnt + (prev >> 3)); }
#pragma unroll
      for (int j = 0; j < 4; ++j) st128s((u32x4){v[4 * j], v[4 * j + 1], v[4 * j + 2], v[4 * j + 3]}, dw, t.isw ? MapW{f.rd}(w, j, lane) : MapX{f.rd}(w, j, lane));
      if (r > 0) {
        asm volatile("s_waitcnt vmcnt(4)" : "+v"(pc) :: "memory");
        if (rfl(pc) == 8u * cx.call - 1u) p_publish(cx, ws, wd, 0xFFFFFFFFu, prev_hdst, 0ull, lane);
      }
      if (r == 0) STAMP(p, 3);
      prev = u; prev_hdst = f.hdst;
    }
  }
  const unsigned tkbase = (cx.call - 1u) * (unsigned)(NC - R0 * nv) - (unsigned)(R0 + 4) * (unsigned)nv;
  gu32* dummy_cnt = (gu32*)(ws + WS_TK) + 8 * 32 + (v0 & 7);
  const unsigned ring = lds_ring, gsl = lds_ring + 3u * 3072u;
  auto hoff = [&](int u) { return p_item(p.rank, d, u >> 3).hsrc + (unsigned)(u & 7) * 8u; };
  auto dma_rec = [&](int u, unsigned long long g, bool on, unsigned sl) {
    const unsigned base = on ? p_item(p.rank, d, u >> 3).dsrc + (unsigned)(u & 7) * 3072u : WS_OBI, len = on ? hp_len(g) : 0u;
    const char* q = ws + base;
#pragma unroll
    for (int k = 0; k < 3; ++k)
      if ((unsigned)k * 1024u + lo < len || lane == 0) glds16(q + ((unsigned)k * 1024u + lo < len ? (unsigned)k * 1024u + lo : 0u), ring + sl * 3072u + (unsigned)k * 1024u);
  };
  auto dma_granule = [&](int u, unsigned sl) { if (lane < 2) glds4(ws + (u < NC ? hoff(u) : WS_HOBI) + lane * 4, gsl + sl * 8u); };
  auto rd_granule = [&](unsigned sl) { const LAS unsigned* q = (const LAS unsigned*)(size_t)(gsl + sl * 8u); return ((unsigned long long)rfl(q[1]) << 32) | rfl(q[0]); };
  int c = v0 + R0 * nv, c1 = c + nv, c2 = c1 + nv, c3 = c2 + nv;
  unsigned long long gc = ld_granule(ws, hoff(c)), g1 = ld_granule(ws, hoff(c1));
  dma_granule(c2, 0u);
  DRAIN_ALL();
  gc = uni64(gc); g1 = uni64(g1);
  if (!tag_ok(gc, cx.call)) {
    unsigned pc = atomic_add_lane0_nowait(pcnt + (prev >> 3)), dummy = 0;
    DRAIN_PIN2(pc, dummy);
    if (rfl(pc) == 8u * cx.call - 1u) p_publish(cx, ws, wd, 0xFFFFFFFFu, prev_hdst, 0ull, lane);
    prev = -1;
    if (!poll_granule(cx, ws, hoff(c), gc)) return;
  }
  if (prev >= 0) prev_hsrc = 0xFFFFFFFFu;
  unsigned sc = 0, gp = 0;
  bool have1 = tag_ok(g1, cx.call);
  dma_rec(c, gc, !hp_raw(gc), 0u);
  dma_rec(c1, g1, have1 && !hp_raw(g1), 1u);
  DRAIN_ALL();
  while (c < NC) {
    const int w = c & 7;
    const PItem cur = p_item(p.rank, d, c >> 3);
    unsigned tk = atomic_add_lane0_nowait(tkc);
    unsigned pc = atomic_add_lane0_nowait(prev >= 0 ? pcnt + (prev >> 3) : dummy_cnt);
    u32x2 hl = ld64_nowait(ws + ((prev >= 0 && prev_hsrc != 0xFFFFFFFFu) ? prev_hsrc : WS_HOBI) + (lane & 7) * 8);
    dma_granule(c3, gp ^ 1u);
    if (!hp_raw(gc)) {
      const LAS u32x4* q = (const LAS u32x4*)(size_t)(ring + sc * 3072u + lo);
      const u32x4 d0 = q[0], d1 = q[64], d2 = q[128];
      char* o = wd + (size_t)(cur.ddst + (unsigned)w * 3072u + lo); const unsigned len = hp_len(gc);
      st_peer(d0, o);
      if (1024u + lo < len) st_peer(d1, o + 1024);
      if (2048u + lo < len) st_peer(d2, o + 2048);
    } else {
      u32x4 q[4];
      if (cur.kind == 2) {
        const rsrc_t so = mkr(p.out[p.rank], OUT_BYTES), dd = mkr(p.out[d], OUT_BYTES); const MapO map{cur.rs};
#pragma unroll
        for (int j = 0; j < 4; ++j) q[j] = ld128s(so, map(w, j, lane));
#pragma unroll
        for (int j = 0; j < 4; ++j) st128s(q[j], dd, map(w, j, lane));
      } else {
#pragma unroll
        for (int j = 0; j < 4; ++j) q[j] = ld128s(lw, cur.rs + (unsigned)(w * 4096 + j * 1024) + lo);
#pragma unroll
        for (int j = 0; j < 4; ++j) st128s(q[j], dw, cur.kind ? MapW{cur.rd}(w, j, lane) : MapX{cur.rd}(w, j, lane));
      }
      VM0();
    }
    const unsigned long long g2 = rd_granule(gp);
    const bool have2 = c2 < NC && tag_ok(g2, cx.call);
    const unsigned s2 = sc >= 1u ? sc - 1u : 2u;
    dma_rec(c2 < NC ? c2 : 0, g2, have2 && !hp_raw(g2), s2);
    asm volatile("s_waitcnt vmcnt(3)" : "+v"(tk), "+v"(pc), "+v"(hl) :: "memory");
    if (prev >= 0 && rfl(pc) == 8u * cx.call - 1u) p_publish(cx, ws, wd, prev_hsrc, prev_hdst, u64_of(hl), lane);
    const int c4 = (int)(rfl(tk) - tkbase);
    prev = c; prev_hsrc = cur.hsrc; prev_hdst = cur.hdst;
    const unsigned s1 = sc == 2u ? 0u : sc + 1u;
    if (c1 < NC && !have1) {
      VM0();
      unsigned pc2 = atomic_add_lane0_nowait(pcnt + (c >> 3));
      u32x2 hl2 = ld64_nowait(ws + (prev_hsrc != 0xFFFFFFFFu ? prev_hsrc : WS_HOBI) + (lane & 7) * 8);
      asm volatile("s_waitcnt vmcnt(0)" : "+v"(pc2), "+v"(hl2) :: "memory");
      if (rfl(pc2) == 8u * cx.call - 1u) p_publish(cx, ws, wd, prev_hsrc, prev_hdst, u64_of(hl2), lane);
      prev = -1;
      g1 = 0;
      if (!poll_granule(cx, ws, hoff(c1), g1)) return;
      dma_rec(c1, g1, !hp_raw(g1), s1);
      VM0();
    }
    c = c1; c1 = c2; c2 = c3; c3 = c4;
    gc = g1; g1 = g2; have1 = have2; sc = s1; gp ^= 1u;
  }
  if (prev >= 0) {
    VM0();
    unsigned pc = atomic_add_lane0_nowait(pcnt + (prev >> 3));
    u32x2 hl2 = ld64_nowait(ws + (prev_hsrc != 0xFFFFFFFFu ? prev_hsrc : WS_HOBI) + (lane & 7) * 8);
    asm volatile("s_waitcnt vmcnt(0)" : "+v"(pc), "+v"(hl2) :: "memory");
    if (rfl(pc) == 8u * cx.call - 1u) p_publish(cx, ws, wd, prev_hsrc, prev_hdst, u64_of(hl2), lane);
  }
  STAMPL(p, 2);
}

struct DTask { unsigned hl, pk, dst; int kind; int sig; };
__device__ __forceinline__ unsigned d_map(const DTask& t, int w, int j, int l) {
  return t.kind == 0 ? MapX{t.dst}(w, j, l) : t.kind == 1 ? MapW{t.dst}(w, j, l) : MapO{t.dst}(w, j, l);
}
template <bool OUT> __device__ __forceinline__ DTask d_task(int rank, int i) {
  DTask t;
  if (!OUT) {
    int kt, sl, k, isw;
    if (i < 7 * 256) { kt = i / 112; const int u = i - kt * 112; sl = u >> 4; isw = u & 1; k = (u & 15) >> 1; }
    else { const int u = i - 7 * 256; kt = (u / 56) & 15; const int u2 = u - (u / 56) * 56; sl = u2 >> 3; isw = 0; k = 8 + (u2 & 7); }
    int src = rank + 1 + sl; if (src >= NRANK) src -= NRANK;
    const int T = kt * 8 + src, img = isw ? 2048 + k * 128 + T : k * 128 + T;
    const unsigned slot = (unsigned)(src * 384 + (k < 8 ? kt * 16 + k * 2 + isw : 256 + kt * 8 + (k - 8)));
    t.hl = WS_HIN + slot * 64u; t.pk = WS_PIN + slot * PKT;
    t.dst = WS_IMG + (unsigned)img * IMG; t.kind = isw; t.sig = img;
  } else {
    const int task = (i / (NRANK - 1)) & (NOT - 1), k = i % (NRANK - 1);
    int s = rank + 1 + k; if (s >= NRANK) s -= NRANK;
    const unsigned tidx = (unsigned)(s * NOT + task);
    t.hl = WS_HOUT + tidx * 64u; t.pk = WS_POUT + tidx * PKT; t.dst = out_task_base(s, task); t.kind = 2; t.sig = 0;
  }
  return t;
}
template <bool OUT>
__device__ __forceinline__ void role_decode(const Params& p, const Ctx& cx, unsigned lds_ring, int dw, int nw) {
  constexpr int TOTAL = OUT ? NOT * (NRANK - 1) : 7 * 384;
  const int tid = threadIdx.x, wid = (int)rfl(tid >> 6), lane = tid & 63;
  char* ws = p.ws[p.rank];
  char* fin = OUT ? p.out[p.rank] : ws;
  gu32* cnt = (gu32*)(ws + WS_CNT);
  const unsigned co = (unsigned)wid * 3072u, ho = (unsigned)wid * 8u, lo = (unsigned)lane * 16u;
  const unsigned ring = lds_ring, gsl = lds_ring + 3u * 3072u;
  int i = dw;
  if (i >= TOTAL) return;
  auto dma_rec = [&](const DTask& t, unsigned long long g, bool on, unsigned sl) {
    const unsigned len = on ? hp_len(g) : 0u;
    const char* q = ws + (on ? t.pk + co : WS_PIN);
#pragma unroll
    for (int k = 0; k < 3; ++k)
      if ((unsigned)k * 1024u + lo < len || lane == 0) glds16(q + ((unsigned)k * 1024u + lo < len ? (unsigned)k * 1024u + lo : 0u), ring + sl * 3072u + (unsigned)k * 1024u);
  };
  auto dma_granule = [&](int u, unsigned sl) { if (lane < 2) glds4(ws + (u < TOTAL ? d_task<OUT>(p.rank, u).hl + ho : WS_HIN) + lane * 4, gsl + sl * 8u); };
  auto rd_granule = [&](unsigned sl) { const LAS unsigned* q = (const LAS unsigned*)(size_t)(gsl + sl * 8u); return ((unsigned long long)rfl(q[1]) << 32) | rfl(q[0]); };
  auto signal = [&](int img) { if (!OUT && img >= 0) (void)atomic_add_lane0_nowait_noret(cnt + img * 32); };
  DTask cur = d_task<OUT>(p.rank, i), t1 = cur;
  unsigned long long gc = ld_granule(ws, cur.hl + ho), g1 = 0;
  bool ok = poll_granule(cx, ws, cur.hl + ho, gc);
  bool have1 = false;
  if (i + nw < TOTAL) { t1 = d_task<OUT>(p.rank, i + nw); g1 = uni64(ld_granule(ws, t1.hl + ho)); have1 = tag_ok(g1, cx.call); }
  dma_rec(cur, gc, ok && !hp_raw(gc), 0u);
  dma_rec(t1, g1, have1 && !hp_raw(g1), 1u);
  dma_granule(i + 2 * nw, 0u);
  asm volatile("s_waitcnt vmcnt(0)" ::: "memory");
  unsigned sc = 0, gp = 0;
  int prev = -1;
  while (ok) {
    const int n1 = i + nw, n2 = i + 2 * nw, n3 = i + 3 * nw;
    dma_granule(n3, gp ^ 1u);
    const unsigned long long g2 = rd_granule(gp);
    const bool have2 = n2 < TOTAL && tag_ok(g2, cx.call);
    const DTask t2 = d_task<OUT>(p.rank, n2 < TOTAL ? n2 : i);
    dma_rec(t2, g2, have2 && !hp_raw(g2), sc >= 1u ? sc - 1u : 2u);
    if (!hp_raw(gc)) {
      unsigned v[16];
      decode_rec<OUT>((const LAS unsigned*)(size_t)(ring + sc * 3072u), gc, lane, v);
#pragma unroll
      for (int j = 0; j < 4; ++j) st_peer((u32x4){v[4 * j], v[4 * j + 1], v[4 * j + 2], v[4 * j + 3]}, fin + d_map(cur, wid, j, lane));
      asm volatile("s_waitcnt vmcnt(7)" ::: "memory");
    } else {
      asm volatile("s_waitcnt vmcnt(0)" ::: "memory");
    }
    signal(prev);
    prev = cur.sig;
    if (n1 >= TOTAL) break;
    const unsigned s1 = sc == 2u ? 0u : sc + 1u;
    if (!have1) {
      asm volatile("s_waitcnt vmcnt(0)" ::: "memory");
      signal(prev); prev = -1;
      g1 = 0;
      ok = poll_granule(cx, ws, t1.hl + ho, g1);
      dma_rec(t1, g1, ok && !hp_raw(g1), s1);
      asm volatile("s_waitcnt vmcnt(0)" ::: "memory");
    }
    cur = t1; gc = g1; t1 = t2; g1 = g2; have1 = have2; sc = s1; gp ^= 1u; i = n1;
  }
  asm volatile("s_waitcnt vmcnt(0)" ::: "memory");
  if (!OUT) {
    if (ok) signal(prev);
  } else if (!ok) {
    for (; i < TOTAL; i += nw) {
      const DTask t = d_task<OUT>(p.rank, i);
#pragma unroll
      for (int j = 0; j < 4; ++j) st_peer((u32x4){0x7FC07FC0u, 0x7FC07FC0u, 0x7FC07FC0u, 0x7FC07FC0u}, fin + d_map(t, wid, j, lane));
    }
  }
}

__host__ __device__ __forceinline__ int lds_byte(int r, int c) { const int st = (r >> 4) * 2 + (c >> 5), rr = r & 15, cc = c & 31, ob = rr * 64 + cc * 2; return st * 1024 + (ob ^ (((ob >> 9) & 1) << 5)); }
__device__ __forceinline__ unsigned cvt_pk_bf16(float lo, float hi) { unsigned r; asm volatile("v_cvt_pk_bf16_f32 %0, %1, %2" : "=v"(r) : "v"(lo), "v"(hi)); return r; }
__device__ __forceinline__ float gelu1(float y) {
  const float k1 = -2.0f * 0.7978845608028654f * 1.4426950408889634f, k3 = k1 * 0.044715f;
  const float t = y * (k3 * (y * y) + k1);
  return y * __builtin_amdgcn_rcpf(1.0f + __builtin_amdgcn_exp2f(t));
}

__device__ __forceinline__ void gemm_unit(const Params& p, const Ctx& cx, LAS unsigned char* lds, int pm, int pn, int h, int ph) {
  constexpr int NTH = NT / 2;
  const int tid = threadIdx.x, wid = (int)rfl(tid >> 6), lane = tid & 63, wr = wid >> 2, wc = wid & 3, fr = lane & 15, fq = lane >> 4;
  char* ws = p.ws[p.rank];
  const char* cA = ws + WS_IMG + (size_t)(pm * 128 + h) * IMG;
  const char* cB = ws + WS_IMG + (size_t)(2048 + pn * 128 + h) * IMG;
  const gu32* fl = (const gu32*)(ws + WS_CNT) + (size_t)(((lane & 2) ? 2048 + pn * 128 : pm * 128) + h + 2 * (lane & 1)) * 32;
  const unsigned need = 8u * cx.call;
  const unsigned voff = (unsigned)tid * 16u;
  const unsigned ldsw = (unsigned)wid * 1024u;
  const int aoff = lds_byte(wr * 64 + fr, fq * 8), boff = lds_byte(wc * 32 + fr, fq * 8);
  bool dead = false;
#define KT(t) ((((t) >> 2) << 3) + (((t) & 3) << 1))
  auto wait_tiles = [&](int t) {
    if (wid == 0 && !dead) {
      const gu32* f = fl + KT(t) * 32;
      if (!__all(__hip_atomic_load(f, __ATOMIC_RELAXED, __HIP_MEMORY_SCOPE_AGENT) >= need)) {
        const unsigned long long t0 = rt();
        while (!__all(__hip_atomic_load(f, __ATOMIC_RELAXED, __HIP_MEMORY_SCOPE_AGENT) >= need)) { if (cx.give_up(t0)) { dead = true; break; } }
      }
    }
  };
#define SA(b, h) (((b) * 2 + (h)) * HTB)
#define SB(b, h) ((4 + (b) * 2 + (h)) * HTB)
#define STAGE(bufoff, gbase) do { _Pragma("unroll") for (int _i = 0; _i < 2; ++_i) \
    __builtin_amdgcn_global_load_lds((const GAS unsigned*)((const char*)(gbase) + voff + _i * 8192), (LAS unsigned*)(lds + (bufoff) + ldsw + _i * 8192), 16, 0, AUX_SYS); } while (0)
#define LDA(dst, b, h) do { _Pragma("unroll") for (int m = 0; m < 4; ++m) _Pragma("unroll") for (int k = 0; k < 2; ++k) dst[m][k] = *(const LAS bf16x8*)(lds + SA(b, h) + aoff + m * 2048 + k * 1024); } while (0)
#define LDB(dst, b, h) do { _Pragma("unroll") for (int n = 0; n < 2; ++n) _Pragma("unroll") for (int k = 0; k < 2; ++k) dst[n][k] = *(const LAS bf16x8*)(lds + SB(b, h) + boff + n * 2048 + k * 1024); } while (0)
#define MMA(ai, bj, At, Bt) do { __builtin_amdgcn_s_setprio(1); _Pragma("unroll") for (int m = 0; m < 4; ++m) _Pragma("unroll") for (int n = 0; n < 2; ++n) _Pragma("unroll") for (int k = 0; k < 2; ++k) \
    acc[ai][bj][m][n] = __builtin_amdgcn_mfma_f32_16x16x32_bf16(Bt[n][k], At[m][k], acc[ai][bj][m][n], 0, 0, 0); __builtin_amdgcn_s_setprio(0); } while (0)
#define WAIT_V(n) asm volatile("s_waitcnt vmcnt(" #n ")" ::: "memory")
#define WAIT_L(n) asm volatile("s_waitcnt lgkmcnt(" #n ")" ::: "memory")
#define BAR __builtin_amdgcn_s_barrier()
#define SCHED __builtin_amdgcn_sched_barrier(0)
  f32x4 acc[2][2][4][2];
#pragma unroll
  for (int a = 0; a < 2; ++a)
#pragma unroll
    for (int b = 0; b < 2; ++b)
#pragma unroll
      for (int m = 0; m < 4; ++m)
#pragma unroll
        for (int n = 0; n < 2; ++n) acc[a][b][m][n] = (f32x4){0.f, 0.f, 0.f, 0.f};
  bf16x8 At[4][2], B0[2][2], B1[2][2];
  wait_tiles(0); wait_tiles(2);
  BARRIER();
  STAMPL(p, 18 + ph);
  STAGE(SB(0, 0), cB); STAGE(SA(0, 0), cA); STAGE(SB(0, 1), cB + HTB); STAGE(SA(0, 1), cA + HTB);
  if (wr == 1) BAR;
  WAIT_V(4); BAR;
  STAGE(SB(1, 0), cB + 2 * IMG); STAGE(SA(1, 0), cA + 2 * IMG); STAGE(SB(1, 1), cB + 2 * IMG + HTB);
  WAIT_V(6); BAR;
  for (int t = 0; t < NTH; t += 2) {
    const bool last = (t == NTH - 2);
    const char* a1 = cA + (size_t)(KT(t) + 2) * IMG;
    const size_t o2 = last ? 0 : (size_t)KT(t + 2) * IMG;
    const char* a2 = cA + o2; const char* b2 = cB + o2;
    const char* a3 = a2 + 2 * IMG; const char* b3 = b2 + 2 * IMG;
    LAS unsigned* fslot = (LAS unsigned*)(lds + LDS_FLAG + ((t >> 1) & 1) * 16);
    const bool chk = (t + 4 < NTH);
    if (chk && wid == 0 && lane < 4) __builtin_amdgcn_global_load_lds(fl + KT(t + 4) * 32, fslot, 4, 0, AUX_SYS);
    LDB(B0, 0, 0); SCHED; LDA(At, 0, 0); STAGE(SA(1, 1), a1 + HTB);
    WAIT_L(8); BAR; WAIT_L(0); MMA(0, 0, At, B0); BAR; SCHED;
    LDB(B1, 0, 1); STAGE(SB(0, 0), b2);
    BAR; WAIT_L(0); MMA(0, 1, At, B1); BAR;
    LDA(At, 0, 1); STAGE(SA(0, 0), a2);
    BAR; WAIT_L(0); MMA(1, 0, At, B0); BAR; SCHED;
    STAGE(SB(0, 1), b2 + HTB);
    WAIT_V(6); BAR; MMA(1, 1, At, B1); BAR;
    LDB(B0, 1, 0); SCHED; LDA(At, 1, 0); STAGE(SA(0, 1), a2 + HTB);
    WAIT_L(8); BAR; WAIT_L(0); MMA(0, 0, At, B0); BAR; SCHED;
    LDB(B1, 1, 1); STAGE(SB(1, 0), b3);
    BAR; WAIT_L(0); MMA(0, 1, At, B1); BAR;
    LDA(At, 1, 1); STAGE(SA(1, 0), a3);
    BAR; WAIT_L(0); MMA(1, 0, At, B0); BAR; SCHED;
    const u32x4 f = *(const LAS u32x4*)fslot;
    STAGE(SB(1, 1), b3 + HTB);
    WAIT_V(6); BAR; MMA(1, 1, At, B1); BAR;
    if (chk) {
      const unsigned fm = rfl(min(min(f.x, f.y), min(f.z, f.w)));
      if (fm < need) { wait_tiles(t + 4); BARRIER(); }
    }
  }
  if (wr == 0) BAR;
  STAMPL(p, 20 + 2 * ph);
  const int tile = pm * 8 + pn;
  {
    int tz = threadIdx.x; asm volatile("" : "+v"(tz));
    const int ln = tz & 63;
    const rsrc_t ro = mkr(p.out[p.rank], OUT_BYTES), lw = mkr(ws, WS_END);
    {
      const unsigned po = WS_PART + (unsigned)(tile * 2 + h) * 131072u + (unsigned)(wid * 16384 + ln * 16);
      const unsigned pi = WS_PART + (unsigned)(tile * 2 + 1 - h) * 131072u + (unsigned)(wid * 16384 + ln * 16);
#pragma unroll
      for (int i = 0; i < 4; ++i) {
        unsigned d[12];
#pragma unroll
        for (int k = 0; k < 4; ++k) {
          const int ii = 4 * i + k;
          const u32x4 q = __builtin_bit_cast(u32x4, h ? acc[0][ii >> 3][(ii >> 1) & 3][ii & 1] : acc[1][ii >> 3][(ii >> 1) & 3][ii & 1]);
          d[3 * k] = vperm(q.y, q.x, 0x05030201u); d[3 * k + 1] = vperm(q.z, q.y, 0x06050302u); d[3 * k + 2] = vperm(q.w, q.z, 0x07060503u);
        }
#pragma unroll
        for (int k = 0; k < 3; ++k) st128s((u32x4){d[4 * k], d[4 * k + 1], d[4 * k + 2], d[4 * k + 3]}, lw, po + (3 * i + k) * 1024);
      }
      VM0();
      gu32* fo = (gu32*)(ws + WS_PFLG) + ((tile * 2 + h) * 8 + wid) * 32;
      const gu32* fi = (const gu32*)(ws + WS_PFLG) + ((tile * 2 + 1 - h) * 8 + wid) * 32;
      if (!dead && !cx.timed_out() && ln == 0) __hip_atomic_store(fo, cx.call, __ATOMIC_RELAXED, __HIP_MEMORY_SCOPE_AGENT);
      if (rfl(__hip_atomic_load(fi, __ATOMIC_RELAXED, __HIP_MEMORY_SCOPE_AGENT)) < cx.call) {
        const unsigned long long t0 = rt();
        while (rfl(__hip_atomic_load(fi, __ATOMIC_RELAXED, __HIP_MEMORY_SCOPE_AGENT)) < cx.call) { if (cx.give_up(t0)) break; }
      }
      asm volatile("" ::: "memory");
      u32x4 in[12];
#pragma unroll
      for (int i = 0; i < 12; ++i) in[i] = ld128s(lw, pi + i * 1024);
#pragma unroll
      for (int i = 0; i < 4; ++i) {
        const unsigned d[12] = {in[3 * i].x, in[3 * i].y, in[3 * i].z, in[3 * i].w, in[3 * i + 1].x, in[3 * i + 1].y, in[3 * i + 1].z, in[3 * i + 1].w,
                                in[3 * i + 2].x, in[3 * i + 2].y, in[3 * i + 2].z, in[3 * i + 2].w};
#pragma unroll
        for (int k = 0; k < 4; ++k) {
          const int ii = 4 * i + k;
          const unsigned d0 = d[3 * k], d1 = d[3 * k + 1], d2 = d[3 * k + 2];
          const f32x4 q = __builtin_bit_cast(f32x4, (u32x4){vperm(0u, d0, 0x0201000Cu), vperm(d1, d0, 0x0504030Cu), vperm(d2, d1, 0x0403020Cu), vperm(0u, d2, 0x0302010Cu)});
          if (h) acc[1][ii >> 3][(ii >> 1) & 3][ii & 1] += q; else acc[0][ii >> 3][(ii >> 1) & 3][ii & 1] += q;
        }
      }
    }
    STAMP(p, 12 + 4 * ph);
    const bool bad = cx.timed_out();
#pragma unroll
    for (int qq = 0; qq < 2; ++qq) {
      unsigned u[16];
#pragma unroll
      for (int mm = 0; mm < 2; ++mm) {
        const int m = qq * 2 + mm;
        unsigned d[8];
#pragma unroll
        for (int c = 0; c < 8; ++c) {
          const float y0 = h ? acc[1][c & 1][m][0][c >> 1] : acc[0][c & 1][m][0][c >> 1], y1 = h ? acc[1][c & 1][m][1][c >> 1] : acc[0][c & 1][m][1][c >> 1];
          d[c] = cvt_pk_bf16(gelu1(y0), gelu1(y1)); if (bad) d[c] = 0x7FC07FC0u; }
#pragma unroll
        for (int c = 0; c < 4; ++c) {
          auto r = __builtin_amdgcn_permlane16_swap(d[c], d[c + 4], false, false); d[c] = r[0]; d[c + 4] = r[1];
          auto s2 = __builtin_amdgcn_permlane32_swap(d[c], d[c + 4], false, false); d[c] = s2[0]; d[c + 4] = s2[1];
        }
#pragma unroll
        for (int c = 0; c < 8; ++c) u[8 * mm + c] = d[c];
      }
      const MapO map{out_task_base(p.rank, tile * 4 + h * 2 + qq)};
#pragma unroll
      for (int j = 0; j < 4; ++j) st128s((u32x4){u[4 * j], u[4 * j + 1], u[4 * j + 2], u[4 * j + 3]}, ro, map(wid, j, ln));
    }
    VM0();
    if (!bad && ln < 2)
      __hip_atomic_store((gu32*)(ws + WS_RFLG) + ((tile * 4 + h * 2 + ln) * 8 + wid) * 4, cx.call, __ATOMIC_RELAXED, __HIP_MEMORY_SCOPE_AGENT);
  }
  STAMPL(p, 21 + 2 * ph);
#undef KT
#undef SA
#undef SB
#undef STAGE
#undef LDA
#undef LDB
#undef MMA
#undef WAIT_V
#undef WAIT_L
#undef BAR
#undef SCHED
}

__global__ __launch_bounds__(NTHR, 2) void fused_kernel(Params p) {
  extern __shared__ __attribute__((aligned(16))) unsigned char shm[];
  LAS unsigned char* lds = (LAS unsigned char*)shm;
  const int b = blockIdx.x, xcd = b & 7, slot = b >> 3;
  char* ws = p.ws[p.rank];
  const int wave = (int)rfl(threadIdx.x >> 6);
  unsigned callraw = atomic_add_lane0_nowait((gu32*)(ws + WS_CALL) + (b * 8 + wave) * 4);
  Ctx cx; cx.tmo = (gu32*)(ws + WS_TMO); cx.call = 0;
  LAS unsigned* S = (LAS unsigned*)(lds + LDS_SCR + wave * SCR_BYTES);
  if (b >= N_P || wave >= PW) { unsigned dummy = 0; DRAIN_PIN2(callraw, dummy); cx.call = rfl(callraw) + 1u; }
  if (b < N_P) {
    if (wave < PW) { const int gw = b * PW + wave; role_push(p, cx, callraw, (unsigned)(size_t)(lds + wave * 9728), gw % 7, gw / 7, PNV); }
  } else if (b < N_P + N_EO) {
    role_encode(p, cx, S, 48 * CFG_EK + (b - N_P), N_EO, NIT);
    role_encode_out(p, cx, S, b - N_P, N_EO);
    STAMP(p, 2);
  } else if (slot < 16) {
    const int dw = b - N_P - N_EO;
    role_decode<false>(p, cx, (unsigned)(size_t)(lds + wave * 9728), dw, N_D);
    STAMPL(p, 1);
    role_decode<true>(p, cx, (unsigned)(size_t)(lds + wave * 9728), dw, N_D + 128);
    STAMP(p, 2);
  } else {
    const int g = slot - 16, h = xcd & 1, q = xcd >> 1;
    const int pm = (q >> 1) * 4 + (g & 3), pn = (q & 1) * 4 + (g >> 2);
    role_encode(p, cx, S, b - 128, 128, 48 * CFG_EK);
    STAMP(p, 9);
#pragma unroll 1
    for (int ph = 0; ph < 2; ++ph) gemm_unit(p, cx, lds, pm + 8 * ph, pn, h, ph);
    BARRIER();
    role_decode<true>(p, cx, (unsigned)(size_t)(lds + wave * 9728), N_D + (b - 128), N_D + 128);
    STAMP(p, 10);
  }
}

}

extern "C" void kernel_launch(int rank, int nranks, ncclComm_t comm, hipStream_t stream,
                              void* const* d_in, const int* in_sizes, int n_in,
                              void* d_out, int out_size,
                              const int* in_dtypes, int out_dtype,
                              void* const* d_out_all, void* const* d_ws_all, size_t ws_bytes) {
  using namespace fk;
  (void)comm; (void)in_dtypes; (void)out_dtype;
  if (nranks != NRANK || n_in < 2 || rank < 0 || rank >= NRANK || ws_bytes < (size_t)WS_END ||
      in_sizes[0] != MD * KS || in_sizes[1] != KS * ND || out_size != MD * ND) {
    fprintf(stderr, "kernel_launch: unexpected problem shape\n");
    return;
  }
  static bool attr_set[64] = {};
  int dev = 0;
  if (hipGetDevice(&dev) != hipSuccess) return;
  if (dev >= 0 && dev < 64 && !attr_set[dev]) {
    hipError_t e = hipFuncSetAttribute((const void*)fused_kernel, hipFuncAttributeMaxDynamicSharedMemorySize, LDS_BYTES);
    if (e != hipSuccess) { fprintf(stderr, "HIP error %s\n", hipGetErrorString(e)); return; }
    attr_set[dev] = true;
  }
  Params p;
  p.x = (const char*)d_in[0]; p.w = (const char*)d_in[1];
  for (int i = 0; i < NRANK; ++i) { p.ws[i] = (char*)d_ws_all[i]; p.out[i] = (char*)d_out_all[i]; }
  p.out[rank] = (char*)d_out;
  p.rank = rank; p.pad = 0;
  hipLaunchKernelGGL(fused_kernel, dim3(GRID), dim3(NTHR), LDS_BYTES, stream, p);
  hipError_t e = hipGetLastError();
  if (e != hipSuccess) fprintf(stderr, "HIP launch error %s\n", hipGetErrorString(e));
}
```
